```python
import numpy as np

kernel_name = "gpu_gemm_rs_m4096_k4096_n32768_bf16_none"


ABSMAX_RTOL = 0.008

COMPUTE_DTYPE = "bfloat16"
EPILOGUE = "none"
IN_QUANT = None
OUT_QUANT = None
IN_DTYPES = {"x": "bfloat16", "w_mat": "bfloat16"}
WORLD_SIZE = 8

def _to_bf16(x):
    u = np.ascontiguousarray(x, dtype=np.float32).view(np.uint32)
    return ((u + 0x7FFF + ((u >> 16) & 1)) & 0xFFFF0000).view(np.float32)

def setup_inputs():

    x = _to_bf16(np.random.randn(4096, 4096).astype(np.float32))
    w_mat = _to_bf16((np.random.randn(4096, 32768) / np.sqrt(4096)).astype(np.float32))
    return {"x": x, "w_mat": w_mat}

def _epilogue(y):
    return y

def reference(x, w_mat):
    y = (x @ w_mat).astype(np.float32)
    return _epilogue(y)
```

```cpp
#include <hip/hip_runtime.h>
#include <rccl/rccl.h>
#include <cstdio>

#define LAS __attribute__((address_space(3)))
typedef short bf16x8 __attribute__((ext_vector_type(8)));
typedef float f32x4 __attribute__((ext_vector_type(4)));
typedef unsigned u32x4 __attribute__((ext_vector_type(4)));
typedef unsigned u32x2 __attribute__((ext_vector_type(2)));
typedef unsigned long long u64;

namespace {

constexpr int NR = 8;
constexpr int NSLOT = 128;
constexpr int LSLOT = 36 * 1024;
constexpr int LDS_BYTES = 4 * LSLOT;
constexpr int NCOMM = 48;
constexpr int PW = 48;
constexpr int NPIECE = 2304;
constexpr u64 DEADLINE = 100000000ull;

constexpr size_t OFF_CNT = 0;
constexpr size_t OFF_TMO = 16384;
constexpr size_t OFF_READY = 32768;
constexpr size_t OFF_REC = 131072;
constexpr size_t REC_PAR = (size_t)NR * 3 * 1024 * 8;
constexpr size_t OFF_DUMMY = 589824;
constexpr size_t OFF_DBG = 655360;
constexpr size_t OFF_A = 1u << 20;
constexpr size_t OFF_B = OFF_A + (32u << 20);
constexpr int NUNIT = 1024 + 2 * NR * 1024;
constexpr size_t PK_PAR = (size_t)NUNIT * 3072;
constexpr size_t ESC_PAR = (size_t)NUNIT * 1024;
constexpr size_t OFF_PK = OFF_B + (32u << 20);
constexpr size_t OFF_ESC = OFF_PK + 2 * PK_PAR;
constexpr size_t WS_NEED = OFF_ESC + 2 * ESC_PAR;
static_assert(OFF_REC + 2 * REC_PAR <= OFF_DUMMY && OFF_DUMMY + 7 * PW * 128 <= OFF_DBG, "layout");

struct Params {
    const char* x;
    const char* w;
    char* out;
    char* ws[NR];
    int rank;
    int pad;
    unsigned short order[512];
    __attribute__((aligned(4))) unsigned char worder[256];
};

#define STAMP(ws, idx) do { } while (0)

__device__ __forceinline__ unsigned rfl(unsigned v) { return __builtin_amdgcn_readfirstlane(v); }
__device__ __forceinline__ unsigned perm(unsigned hi, unsigned lo, unsigned sel) { return __builtin_amdgcn_perm(hi, lo, sel); }
__device__ __forceinline__ unsigned bfi(unsigned m, unsigned a, unsigned b) { return (a & m) | (b & ~m); }
__device__ __forceinline__ unsigned rotl4(unsigned v) { return __builtin_amdgcn_alignbit(v, v, 28); }
__device__ __forceinline__ unsigned rotr4(unsigned v) { return __builtin_amdgcn_alignbit(v, v, 4); }
__device__ __forceinline__ unsigned cvt_pk(float lo, float hi) { unsigned r; asm("v_cvt_pk_bf16_f32 %0, %1, %2" : "=v"(r) : "v"(lo), "v"(hi)); return r; }

template <int OFF> __device__ __forceinline__ void st_wt(const char* sbase, unsigned voff, u32x4 v) {
    asm volatile("global_store_dwordx4 %0, %1, %2 offset:%3 sc0 sc1\n\ts_nop 1" :: "v"(voff), "v"(v), "s"(sbase), "i"(OFF) : "memory");
}
__device__ __forceinline__ void wait_vm0() { asm volatile("s_waitcnt vmcnt(0)" ::: "memory"); }

__device__ __forceinline__ bool timed_out(unsigned* tmo, unsigned call, u64 t0, unsigned& spins) {
    if ((++spins & 63u) != 0u) return false;
    if (rfl(__hip_atomic_load(tmo, __ATOMIC_RELAXED, __HIP_MEMORY_SCOPE_AGENT)) == call) return true;
    if (__builtin_amdgcn_s_memrealtime() - t0 > DEADLINE) {
        __hip_atomic_store(tmo, call, __ATOMIC_RELAXED, __HIP_MEMORY_SCOPE_AGENT);
        return true;
    }
    return false;
}

__device__ __forceinline__ unsigned pack_store(const unsigned (&L)[8], const unsigned (&H)[8], const char* pk, char* esc, int lane) {
    unsigned m = max(max(H[0] & 0x7f7f7f7fu, H[2] & 0x7f7f7f7fu), max(H[5] & 0x7f7f7f7fu, H[7] & 0x7f7f7f7fu));
    m = max(m, (unsigned)__builtin_amdgcn_update_dpp(0, (int)m, 0x111, 0xf, 0xf, false));
    m = max(m, (unsigned)__builtin_amdgcn_update_dpp(0, (int)m, 0x112, 0xf, 0xf, false));
    m = max(m, (unsigned)__builtin_amdgcn_update_dpp(0, (int)m, 0x114, 0xf, 0xf, false));
    m = max(m, (unsigned)__builtin_amdgcn_update_dpp(0, (int)m, 0x118, 0xf, 0xf, false));
    const unsigned m0 = __builtin_amdgcn_readlane(m, 15), m1 = __builtin_amdgcn_readlane(m, 31);
    const unsigned m2 = __builtin_amdgcn_readlane(m, 47), m3 = __builtin_amdgcn_readlane(m, 63);
    const unsigned top = max(max(m0, m1), max(m2, m3)) >> 24;
    const unsigned lo = top > 7u ? top - 7u : 0u;
    const unsigned lo4 = lo * 0x01010101u;
    u32x4 N, X;
#pragma unroll
    for (int a = 0; a < 4; ++a) {
        const unsigned d0 = H[a] - lo4, d1 = rotl4(H[4 + a] - lo4);
        N[a] = bfi(0x87878787u, d0, d1);
        X[a] = bfi(0x87878787u, d1, d0);
    }
    const unsigned voff = lane * 16;
    st_wt<0>(pk, voff, (u32x4){L[0], L[1], L[2], L[3]});
    st_wt<1024>(pk, voff, (u32x4){L[4], L[5], L[6], L[7]});
    st_wt<2048>(pk, voff, N);
    const u64 em = __builtin_amdgcn_ballot_w64((X[0] | X[1] | X[2] | X[3]) != 0u);
    unsigned m8 = 0;
    if (em != 0ull) {
        m8 = (unsigned)__builtin_amdgcn_ballot_w64(lane < 8 && ((em >> (lane * 8)) & 0xffull) != 0ull);
        if ((m8 >> (lane >> 3)) & 1u) st_wt<0>(esc, voff, X);
    }
    return (m8 << 8) | lo;
}
__device__ __forceinline__ void join_bytes(const unsigned (&L)[8], const unsigned (&H)[8], u32x4 (&o)[4]) {
#pragma unroll
    for (int hf = 0; hf < 2; ++hf)
#pragma unroll
        for (int a = 0; a < 4; ++a) {
            o[2 * hf][a] = perm(H[4 * hf + a], L[4 * hf + a], 0x05010400u);
            o[2 * hf + 1][a] = perm(H[4 * hf + a], L[4 * hf + a], 0x07030602u);
        }
}
__device__ __forceinline__ void split_bytes(const u32x4 (&d)[4], unsigned (&L)[8], unsigned (&H)[8]) {
#pragma unroll
    for (int hf = 0; hf < 2; ++hf)
#pragma unroll
        for (int a = 0; a < 4; ++a) {
            L[4 * hf + a] = perm(d[2 * hf + 1][a], d[2 * hf][a], 0x06040200u);
            H[4 * hf + a] = perm(d[2 * hf + 1][a], d[2 * hf][a], 0x07050301u);
        }
}
__device__ __forceinline__ void store_stage(const char* dst, unsigned voff, const u32x4 (&o)[4]) {
    st_wt<0>(dst, voff, o[0]); st_wt<1024>(dst, voff, o[1]); st_wt<2048>(dst, voff, o[2]); st_wt<3072>(dst, voff, o[3]);
}
__device__ __forceinline__ void store_out(const char* dst, unsigned voff, const u32x4 (&o)[4]) {
    st_wt<0>(dst, voff, o[0]); st_wt<64>(dst, voff, o[1]);
    st_wt<0>(dst + 16 * 65536, voff, o[2]); st_wt<64>(dst + 16 * 65536, voff, o[3]);
}

__device__ __forceinline__ void pack_piece(const Params& P, int piece, unsigned cntv, int lane) {
    char* ws = P.ws[P.rank];
    const int li = lane & 15, lg = lane >> 4;
    unsigned* ready = (unsigned*)(ws + OFF_READY);
    unsigned rec = 0;
    if (piece < 256) {
        const int h = piece >> 4, q = piece & 15;
        const char* src = P.x + (size_t)(256 * q + li) * 1024 + h * 64 + lg * 16;
        u32x4 d[4][4];
#pragma unroll
        for (int u = 0; u < 4; ++u)
#pragma unroll
            for (int c = 0; c < 4; ++c) d[u][c] = *(const u32x4*)(src + (64 * u + 16 * c) * 1024);
        const unsigned dest = (unsigned)lane >> 2;
        char* peer = P.ws[dest & 7u];
        __builtin_amdgcn_sched_barrier(0);
#pragma unroll
        for (int u = 0; u < 4; ++u) asm volatile("" : "+v"(d[u][0]), "+v"(d[u][1]), "+v"(d[u][2]), "+v"(d[u][3]));
        asm volatile("" : "+v"(peer));
        const unsigned call = rfl(cntv) + 1u, par = call & 1u;
#pragma unroll
        for (int u = 0; u < 4; ++u) {
            unsigned L[8], H[8];
            split_bytes(d[u], L, H);
            const int g = h * 64 + 4 * q + u;
            const unsigned r = pack_store(L, H, ws + OFF_PK + par * PK_PAR + (size_t)g * 3072, ws + OFF_ESC + par * ESC_PAR + (size_t)g * 1024, lane);
            if ((lane & 3) == u) rec = r;
        }
        wait_vm0();
        if (dest < (unsigned)NR && dest != (unsigned)P.rank) {
            u64* rp = (u64*)(peer + OFF_REC + par * REC_PAR) + (P.rank * 3 + 0) * 1024 + h * 64 + 4 * q + (lane & 3);
            __hip_atomic_store(rp, ((u64)call << 32) | rec, __ATOMIC_RELAXED, __HIP_MEMORY_SCOPE_SYSTEM);
        }
#pragma unroll
        for (int u = 0; u < 4; ++u) store_stage(ws + OFF_A + (size_t)(h * 256 + 16 * q + 4 * u) * 1024, lane * 16, d[u]);
        wait_vm0();
        if (lane < 4) __hip_atomic_store(ready + (4 * q + lane) * NSLOT + h, call, __ATOMIC_RELAXED, __HIP_MEMORY_SCOPE_SYSTEM);
    } else {
        const int idx = piece - 256;
        const int e = (((const unsigned*)P.worder)[idx >> 5] >> (idx & 24)) & 255;
        const int h = e >> 4, pn = e & 15;
        const unsigned dest = (unsigned)(P.rank + 1 + (idx & 7)) & 7u;
        const int cp = 32 * ((li >> 1) & 1) + 8 * (li >> 2) + 4 * (li & 1);
        const char* src = P.w + (size_t)(32 * h + 8 * lg) * 65536 + (size_t)(dest * 4096 + 256 * pn + cp) * 2;
        u32x2 v[4][8];
#pragma unroll
        for (int j = 0; j < 8; ++j)
#pragma unroll
            for (int u = 0; u < 4; ++u) v[u][j] = *(const u32x2*)(src + (size_t)j * 65536 + u * 128);
        const bool own = dest == (unsigned)P.rank;
        __builtin_amdgcn_sched_barrier(0);
#pragma unroll
        for (int u = 0; u < 4; ++u)
            asm volatile("" : "+v"(v[u][0]), "+v"(v[u][1]), "+v"(v[u][2]), "+v"(v[u][3]), "+v"(v[u][4]), "+v"(v[u][5]), "+v"(v[u][6]), "+v"(v[u][7]));
        const unsigned call = rfl(cntv) + 1u, par = call & 1u;
#pragma unroll
        for (int u = 0; u < 4; ++u) {
            unsigned L[8], H[8];
#pragma unroll
            for (int hf = 0; hf < 2; ++hf)
#pragma unroll
                for (int a = 0; a < 4; ++a) {
                    L[4 * hf + a] = perm(v[u][2 * a + 1][hf], v[u][2 * a][hf], 0x06020400u);
                    H[4 * hf + a] = perm(v[u][2 * a + 1][hf], v[u][2 * a][hf], 0x07030501u);
                }
            if (own) {
                u32x4 o[4];
                join_bytes(L, H, o);
                store_stage(ws + OFF_B + (size_t)(h * 256 + 16 * pn + 4 * u) * 1024, lane * 16, o);
            } else {
                const int g = 1024 + (int)dest * 1024 + h * 64 + 4 * pn + u;
                const unsigned r = pack_store(L, H, ws + OFF_PK + par * PK_PAR + (size_t)g * 3072, ws + OFF_ESC + par * ESC_PAR + (size_t)g * 1024, lane);
                if (lane == u) rec = r;
            }
        }
        wait_vm0();
        if (lane < 4) {
            if (own) {
                __hip_atomic_store(ready + (64 + 4 * pn + lane) * NSLOT + h, call, __ATOMIC_RELAXED, __HIP_MEMORY_SCOPE_SYSTEM);
            } else {
                u64* rp = (u64*)(P.ws[dest] + OFF_REC + par * REC_PAR) + (P.rank * 3 + 1) * 1024 + h * 64 + 4 * pn + lane;
                __hip_atomic_store(rp, ((u64)call << 32) | rec, __ATOMIC_RELAXED, __HIP_MEMORY_SCOPE_SYSTEM);
            }
        }
    }
}

__device__ __forceinline__ void pull_unit(u32x4& p0, u32x4& p1, u32x4& p2, u32x4& p3, u32x2& snap, const char* pk, u32x4 srd, unsigned esc_soff,
                                          unsigned voff, unsigned esc_voff, const u64* recp, const char* ws, unsigned flag_off, unsigned call) {
    asm volatile("s_nop 4\n\t"
                 "global_load_dwordx4 %0, %5, %6 sc0 sc1\n\t"
                 "global_load_dwordx4 %1, %5, %6 offset:1024 sc0 sc1\n\t"
                 "global_load_dwordx4 %2, %5, %6 offset:2048 sc0 sc1\n\t"
                 "buffer_load_dwordx4 %3, %7, %8, %9 offen sc0 sc1\n\t"
                 "global_load_dwordx2 %4, %10, off sc0 sc1\n\t"
                 "s_waitcnt vmcnt(5)\n\t"
                 "s_mov_b64 exec, 1\n\t"
                 "global_store_dword %12, %13, %11 sc0 sc1\n\t"
                 "s_mov_b64 exec, -1\n\t"
                 "s_waitcnt vmcnt(1)"
                 : "=&v"(p0), "=&v"(p1), "=&v"(p2), "=&v"(p3), "=&v"(snap)
                 : "v"(voff), "s"(pk), "v"(esc_voff), "s"(srd), "s"(esc_soff), "v"(recp), "s"(ws), "v"(flag_off), "v"(call)
                 : "memory");
}

__device__ __forceinline__ void pull_link(const Params& P, int q, int w, unsigned call, int lane) {
    char* ws = P.ws[P.rank];
    const unsigned par = call & 1u;
    const unsigned p = (unsigned)(P.rank + 1 + q) & 7u;
    const char* peer = P.ws[p];
    unsigned* tmo = (unsigned*)(ws + OFF_TMO);
    const int n = w + PW * lane;
    unsigned vg, vdst, vrdy, ridx;
    if (n < 2048) {
        const int e = P.order[n >> 2];
        const int type = e >> 8, h = (e >> 4) & 15, idx = 4 * (e & 15) + (n & 3), id = h * 64 + idx, slot = 16 + 7 * h + q;
        vg = type ? 1024 + P.rank * 1024 + id : id;
        vdst = (unsigned)(type ? OFF_B : OFF_A) + (unsigned)(slot * 256 + 4 * idx) * 1024u;
        vrdy = (unsigned)OFF_READY + (unsigned)((type * 64 + idx) * NSLOT + slot) * 4u;
        ridx = type * 1024 + id;
    } else {
        const int id = n - 2048;
        vg = 9216 + P.rank * 1024 + id;
        vdst = (unsigned)(id & 15) * (32u * 65536u) + (p * 4096u + 64u * (unsigned)(id >> 4)) * 2u;
        vrdy = 0;
        ridx = 2048 + id;
    }
    const u64* recp = (const u64*)(ws + OFF_REC + par * REC_PAR) + p * 3072 + ridx;
    const char* pkb = peer + OFF_PK + par * PK_PAR;
    u32x4 srd;
    {
        const u64 eb = (u64)(peer + OFF_ESC + par * ESC_PAR);
        srd[0] = rfl((unsigned)eb); srd[1] = rfl((unsigned)(eb >> 32)) & 0xffffu; srd[2] = (unsigned)ESC_PAR; srd[3] = 0x00020000u;
    }
    const unsigned voff = lane * 16, voff_out = (lane & 15) * 65536 + (lane >> 4) * 16;
    const unsigned dummy = (unsigned)OFF_DUMMY + (unsigned)(q * PW + w) * 128u;
    u32x2 rec = {0u, 0u};
    bool dead = false;
    constexpr int CNT = 3072 / PW;
    static_assert(CNT * PW == 3072 && CNT <= 64, "units per wave");
    unsigned pend = dummy;
    for (int i = 0; i < CNT; ++i) {
        if ((unsigned)__builtin_amdgcn_readlane(rec[1], i) != call) {
            if (pend != dummy) {
                wait_vm0();
                if (lane == 0) __hip_atomic_store((unsigned*)(ws + pend), call, __ATOMIC_RELAXED, __HIP_MEMORY_SCOPE_SYSTEM);
                pend = dummy;
            }
            const u64 t0 = __builtin_amdgcn_s_memrealtime();
            unsigned spins = 0;
            while (!dead) {
                const u64 v = __hip_atomic_load(recp, __ATOMIC_RELAXED, __HIP_MEMORY_SCOPE_SYSTEM);
                rec[0] = (unsigned)v; rec[1] = (unsigned)(v >> 32);
                if ((unsigned)__builtin_amdgcn_readlane(rec[1], i) == call) break;
                if (timed_out(tmo, call, t0, spins)) dead = true;
                __builtin_amdgcn_s_sleep(1);
            }
        }
        const unsigned info = dead ? 0u : (unsigned)__builtin_amdgcn_readlane(rec[0], i);
        const unsigned g = __builtin_amdgcn_readlane(vg, i);
        const unsigned ev = ((info >> (8 + (lane >> 3))) & 1u) ? voff : 0x80000000u;
        u32x4 p0, p1, p2, p3;
        pull_unit(p0, p1, p2, p3, rec, pkb + (size_t)g * 3072, srd, g * 1024u, voff, ev, recp, ws, pend, call);
        const unsigned lo4 = (info & 0xffu) * 0x01010101u;
        unsigned L[8], H[8];
#pragma unroll
        for (int a = 0; a < 4; ++a) {
            L[a] = p0[a]; L[4 + a] = p1[a];
            H[a] = bfi(0x87878787u, p2[a], p3[a]) + lo4;
            H[4 + a] = rotr4(bfi(0x87878787u, p3[a], p2[a])) + lo4;
        }
        u32x4 o[4];
        join_bytes(L, H, o);
        const unsigned dst = __builtin_amdgcn_readlane(vdst, i), rdy = __builtin_amdgcn_readlane(vrdy, i);
        if (rdy) {
            store_stage(ws + dst, voff, o);
            pend = rdy;
        } else {
            if (dead) {
#pragma unroll
                for (int c = 0; c < 4; ++c) o[c] = (u32x4){0x7fc07fc0u, 0x7fc07fc0u, 0x7fc07fc0u, 0x7fc07fc0u};
            }
            store_out(P.out + dst, voff_out, o);
            pend = dummy;
        }
    }
}

__device__ __forceinline__ void glds4(unsigned m0v, unsigned voff, const char* sb) {
    asm volatile("s_mov_b32 m0, %0\n\ts_nop 0\n\t"
                 "global_load_lds_dwordx4 %1, %2 sc0 sc1\n\t"
                 "global_load_lds_dwordx4 %1, %2 offset:1024 sc0 sc1\n\t"
                 "global_load_lds_dwordx4 %1, %2 offset:2048 sc0 sc1\n\t"
                 "global_load_lds_dwordx4 %1, %2 offset:3072 sc0 sc1" :: "s"(m0v), "v"(voff), "s"(sb) : "memory");
}
__device__ __forceinline__ void glds1(unsigned m0v, unsigned voff, const char* sb) {
    asm volatile("s_mov_b32 m0, %0\n\ts_nop 0\n\tglobal_load_lds_dwordx4 %1, %2 sc0 sc1" :: "s"(m0v), "v"(voff), "s"(sb) : "memory");
}

__device__ __forceinline__ void gemm_tile(const Params& P, LAS char* lds, int pm, int pn, unsigned call, int lane, int wid) {
    char* ws = P.ws[P.rank];
    unsigned* tmo = (unsigned*)(ws + OFF_TMO);
    const unsigned* ready = (const unsigned*)(ws + OFF_READY);
    const int wr = wid >> 2, wc = wid & 3;
    const unsigned ldsbase = (unsigned)(size_t)lds;
    const char* g1; const char* g2; unsigned l1, l2; const unsigned* r1; const unsigned* r2;
    if (wr == 0) {
        const int u1 = 5 * pm + wc; const bool has5 = pm < 12;
        g1 = ws + OFF_A + (size_t)u1 * 4096; g2 = has5 ? ws + OFF_A + (size_t)(20 * pm + 16 + wc) * 1024 : g1;
        l1 = ldsbase + wc * 4096; l2 = ldsbase + (16 + wc) * 1024;
        r1 = ready + u1 * NSLOT; r2 = has5 ? ready + (5 * pm + 4) * NSLOT : r1;
    } else {
        const int u1 = 4 * pn + wc;
        g1 = ws + OFF_B + (size_t)u1 * 4096; g2 = g1; l1 = ldsbase + 20480 + wc * 4096; l2 = l1;
        r1 = ready + (64 + u1) * NSLOT; r2 = r1;
    }
    const unsigned voff = lane * 16;
    u64 mask = 0;
#define ENSURE(t) do { if (((t) & 63) == 0) mask = 0; \
    if (!((mask >> ((t) & 63)) & 1ull)) { const u64 t0_ = __builtin_amdgcn_s_memrealtime(); unsigned sp_ = 0; \
        for (;;) { \
            const unsigned a_ = __hip_atomic_load(r1 + ((t) & 64) + lane, __ATOMIC_RELAXED, __HIP_MEMORY_SCOPE_SYSTEM); \
            const unsigned b_ = __hip_atomic_load(r2 + ((t) & 64) + lane, __ATOMIC_RELAXED, __HIP_MEMORY_SCOPE_SYSTEM); \
            mask = __builtin_amdgcn_ballot_w64(a_ >= call && b_ >= call); \
            if ((mask >> ((t) & 63)) & 1ull) break; \
            if (timed_out(tmo, call, t0_, sp_)) { mask = ~0ull; break; } \
            __builtin_amdgcn_s_sleep(2); } } } while (0)
#define STAGE(t, bt) do { const size_t go_ = (size_t)(t) * 262144; const unsigned lo_ = (bt) * LSLOT; \
    glds4(l1 + lo_, voff, g1 + go_); if (wr == 0) glds1(l2 + lo_, voff, g2 + go_); } while (0)
#define WAITV() do { if (wr == 0) asm volatile("s_waitcnt vmcnt(10)" ::: "memory"); else asm volatile("s_waitcnt vmcnt(8)" ::: "memory"); } while (0)
    f32x4 acc[10][4];
#pragma unroll
    for (int m = 0; m < 10; ++m)
#pragma unroll
        for (int n = 0; n < 4; ++n) acc[m][n] = (f32x4){0.f, 0.f, 0.f, 0.f};
    for (int t = 0; t < 3; ++t) { ENSURE(t); STAGE(t, t); }
    WAITV();
    __builtin_amdgcn_s_barrier();
    STAMP(ws, 2);
    if (wr == 1) __builtin_amdgcn_s_barrier();
    const unsigned aoff = wr * 10240 + lane * 16, boff = 20480 + wc * 4096 + lane * 16;
    for (int s = 0; s < NSLOT; ++s) {
        const unsigned so = (s & 3) * LSLOT;
        bf16x8 A[10], B[4];
        LAS const char* pa = lds + aoff + so; LAS const char* pb = lds + boff + so;
#pragma unroll
        for (int n = 0; n < 4; ++n) B[n] = *(LAS const bf16x8*)(pb + n * 1024);
#pragma unroll
        for (int m = 0; m < 10; ++m) A[m] = *(LAS const bf16x8*)(pa + m * 1024);
        __builtin_amdgcn_sched_barrier(0);
        const int t = s + 3 < NSLOT ? s + 3 : NSLOT - 1;
        if (s + 3 < NSLOT) ENSURE(t);
        STAGE(t, (s + 3) & 3);
        WAITV();
        asm volatile("s_waitcnt lgkmcnt(0)" ::: "memory");
        __builtin_amdgcn_sched_barrier(0);
        __builtin_amdgcn_s_barrier();
        __builtin_amdgcn_sched_barrier(0);
        __builtin_amdgcn_s_setprio(1);
#pragma unroll
        for (int m = 0; m < 10; ++m)
#pragma unroll
            for (int n = 0; n < 4; ++n) acc[m][n] = __builtin_amdgcn_mfma_f32_16x16x32_bf16(B[n], A[m], acc[m][n], 0, 0, 0);
        __builtin_amdgcn_s_setprio(0);
        __builtin_amdgcn_sched_barrier(0);
        __builtin_amdgcn_s_barrier();
        __builtin_amdgcn_sched_barrier(0);
    }
    if (wr == 0) __builtin_amdgcn_s_barrier();
#undef ENSURE
#undef STAGE
#undef WAITV
    wait_vm0();
    STAMP(ws, 3);
    const bool bad = rfl(__hip_atomic_load(tmo, __ATOMIC_RELAXED, __HIP_MEMORY_SCOPE_AGENT)) == call;
    const unsigned par = call & 1u;
    const int cb = 4 * pn + wc, rb0 = 10 * pm + 5 * wr;
    const unsigned voff_out = (lane & 15) * 65536 + (lane >> 4) * 16;
    unsigned rec = 0;
#pragma unroll
    for (int u = 0; u < 5; ++u) {
        const int rb = rb0 + u;
        if (rb < 128) {
            u32x4 d[4];
#pragma unroll
            for (int c = 0; c < 4; ++c) {
                const int mi = 2 * u + (c >> 1), b = c & 1;
                d[c][0] = cvt_pk(acc[mi][0][2 * b], acc[mi][1][2 * b]);
                d[c][1] = cvt_pk(acc[mi][2][2 * b], acc[mi][3][2 * b]);
                d[c][2] = cvt_pk(acc[mi][0][2 * b + 1], acc[mi][1][2 * b + 1]);
                d[c][3] = cvt_pk(acc[mi][2][2 * b + 1], acc[mi][3][2 * b + 1]);
                if (bad) d[c] = (u32x4){0x7fc07fc0u, 0x7fc07fc0u, 0x7fc07fc0u, 0x7fc07fc0u};
            }
            const int dest = rb >> 4;
            if (dest == P.rank) {
                store_out(P.out + (size_t)(rb & 15) * (32 * 65536) + (size_t)(P.rank * 4096 + 64 * cb) * 2, voff_out, d);
            } else {
                unsigned L[8], H[8];
                split_bytes(d, L, H);
                const int g = 9216 + dest * 1024 + cb * 16 + (rb & 15);
                const unsigned r = pack_store(L, H, ws + OFF_PK + par * PK_PAR + (size_t)g * 3072, ws + OFF_ESC + par * ESC_PAR + (size_t)g * 1024, lane);
                if (lane == u) rec = r;
            }
        }
    }
    const unsigned rbl = (unsigned)(rb0 + lane), dl = rbl >> 4;
    char* peer = P.ws[dl & 7u];
    wait_vm0();
    if (lane < 5 && dl < (unsigned)NR && dl != (unsigned)P.rank) {
        u64* rp = (u64*)(peer + OFF_REC + par * REC_PAR) + (P.rank * 3 + 2) * 1024 + cb * 16 + (rbl & 15u);
        __hip_atomic_store(rp, ((u64)call << 32) | rec, __ATOMIC_RELAXED, __HIP_MEMORY_SCOPE_SYSTEM);
    }
}

__global__ void __launch_bounds__(512, 2) fused_gemm_rs(const Params P) {
    __shared__ __attribute__((aligned(16))) char smem[LDS_BYTES];
    const int bid = blockIdx.x, xcd = bid & 7, j = bid >> 3;
    const int lane = threadIdx.x & 63, wid = rfl(threadIdx.x >> 6);
    char* ws = P.ws[P.rank];
    STAMP(ws, 0);
    unsigned* cnt = (unsigned*)(ws + OFF_CNT) + bid * 8 + wid;
    const unsigned cntv = __hip_atomic_load(cnt, __ATOMIC_RELAXED, __HIP_MEMORY_SCOPE_AGENT);
    const bool comm = j < NCOMM / 8;
    int pm = 0, pn = 0, piece, pstep;
    if (comm) {
        piece = bid * 8 + wid;
        pstep = bid >= 42 ? 48 : NPIECE;
    } else {
        const int jj = j - NCOMM / 8;
        pm = jj % 13; pn = 2 * xcd + jj / 13;
        piece = NCOMM * 8 + (pn * 13 + pm) * 8 + wid;
        pstep = NPIECE;
        const u64 t0 = __builtin_amdgcn_s_memrealtime();
        const u64 dly = (u64)(piece >> 2);
        while (__builtin_amdgcn_s_memrealtime() - t0 < dly) __builtin_amdgcn_s_sleep(1);
    }
    for (;;) {
        pack_piece(P, piece, cntv, lane);
        piece = piece < 2048 && pstep < NPIECE ? 2048 + (piece - 42 * 8) : piece + pstep;
        if (piece >= NPIECE) break;
    }
    const unsigned call = rfl(cntv) + 1u;
    if (lane == 0) __hip_atomic_store(cnt, call, __ATOMIC_RELAXED, __HIP_MEMORY_SCOPE_AGENT);
    STAMP(ws, 1);
    if (comm) {
        if (bid < 42) pull_link(P, bid % 7, (bid / 7) * 8 + wid, call, lane);
    } else {
        gemm_tile(P, (LAS char*)smem, pm, pn, call, lane, wid);
    }
    STAMP(ws, 4);
}

}

#define LAG_NUM 23

extern "C" void kernel_launch(int rank, int nranks, ncclComm_t comm, hipStream_t stream,
                              void* const* d_in, const int* in_sizes, int n_in,
                              void* d_out, int out_size, const int* in_dtypes, int out_dtype,
                              void* const* d_out_all, void* const* d_ws_all, size_t ws_bytes) {
    (void)comm; (void)in_dtypes; (void)out_dtype; (void)d_out_all;
    if (nranks != NR || (unsigned)rank >= (unsigned)NR || n_in < 2 || ws_bytes < WS_NEED || in_sizes[0] != 4096 * 512 ||
        in_sizes[1] != 512 * 32768 || out_size != 512 * 32768) {
        fprintf(stderr, "kernel_launch: unexpected problem shape\n");
        return;
    }
    Params P;
    P.x = (const char*)d_in[0];
    P.w = (const char*)d_in[1];
    P.out = (char*)d_out;
    for (int p = 0; p < NR; ++p) P.ws[p] = (char*)d_ws_all[p];
    P.rank = rank;
    P.pad = 0;
    int no = 0, nw = 0;
    for (int ph = 0; ph < 64; ++ph) {
        if (ph < 16)
            for (int q = 0; q < 16; ++q) P.order[no++] = (unsigned short)(ph << 4 | q);
        for (int pn = 0; pn < 16; ++pn) {
            const int h = ph - pn * LAG_NUM / 32;
            if (h >= 0 && h < 16) {
                P.order[no++] = (unsigned short)(256 | h << 4 | pn);
                P.worder[nw++] = (unsigned char)(h << 4 | pn);
            }
        }
    }
    hipLaunchKernelGGL(fused_gemm_rs, dim3(256), dim3(512), 0, stream, P);
    hipError_t e = hipGetLastError();
    if (e != hipSuccess) fprintf(stderr, "launch failed: %s\n", hipGetErrorString(e));
}
```

```cpp
#include <hip/hip_runtime.h>
#include <rccl/rccl.h>
#include <cstdio>

#define LAS __attribute__((address_space(3)))
typedef short bf16x8 __attribute__((ext_vector_type(8)));
typedef float f32x4 __attribute__((ext_vector_type(4)));
typedef unsigned u32x4 __attribute__((ext_vector_type(4)));
typedef unsigned u32x2 __attribute__((ext_vector_type(2)));
typedef unsigned long long u64;

namespace {

constexpr int NR = 8;
constexpr int NSLOT = 128;
constexpr int LSLOT = 36 * 1024;
constexpr int LDS_BYTES = 4 * LSLOT;
constexpr int NCOMM = 48;
#define PWG 6
constexpr int PW = PWG * 8;
constexpr int NPIECE = 1152;
constexpr unsigned NTK = 3072;
constexpr u64 DEADLINE = 100000000ull;

constexpr size_t OFF_CNT = 0;
constexpr size_t OFF_TMO = 16384;
constexpr size_t OFF_TKT = 20480;
constexpr size_t OFF_READY = 32768;
constexpr size_t OFF_REC = 131072;
constexpr size_t REC_PAR = (size_t)NR * 3 * 1024 * 8;
constexpr size_t OFF_DUMMY = 589824;
constexpr size_t OFF_DBG = 655360;
constexpr size_t OFF_A = 1u << 20;
constexpr size_t OFF_B = OFF_A + (32u << 20);
constexpr int NUNIT = 1024 + 2 * NR * 1024;
constexpr size_t PK_PAR = (size_t)NUNIT * 3072;
constexpr size_t ESC_PAR = (size_t)NUNIT * 1024;
constexpr size_t OFF_PK = OFF_B + (32u << 20);
constexpr size_t OFF_ESC = OFF_PK + 2 * PK_PAR;
constexpr size_t WS_NEED = OFF_ESC + 2 * ESC_PAR;
static_assert(OFF_REC + 2 * REC_PAR <= OFF_DUMMY && OFF_DUMMY + 7 * PW * 128 <= OFF_DBG, "layout");

struct Params {
    const char* x;
    const char* w;
    char* out;
    char* me;
    char* ws[NR];
    int rank;
    int pad;
    unsigned order[64];
    unsigned plist[NPIECE / 2];
};

#define STAMP(ws, idx) do { } while (0)

__device__ __forceinline__ unsigned rfl(unsigned v) { return __builtin_amdgcn_readfirstlane(v); }
__device__ __forceinline__ unsigned perm(unsigned hi, unsigned lo, unsigned sel) { return __builtin_amdgcn_perm(hi, lo, sel); }
__device__ __forceinline__ unsigned bfi(unsigned m, unsigned a, unsigned b) { return (a & m) | (b & ~m); }
__device__ __forceinline__ unsigned rotl4(unsigned v) { return __builtin_amdgcn_alignbit(v, v, 28); }
__device__ __forceinline__ unsigned rotr4(unsigned v) { return __builtin_amdgcn_alignbit(v, v, 4); }
__device__ __forceinline__ unsigned cvt_pk(float lo, float hi) { unsigned r; asm("v_cvt_pk_bf16_f32 %0, %1, %2" : "=v"(r) : "v"(lo), "v"(hi)); return r; }

template <int OFF> __device__ __forceinline__ void st_wt(const char* sbase, unsigned voff, u32x4 v) {
    asm volatile("global_store_dwordx4 %0, %1, %2 offset:%3 sc0 sc1\n\ts_nop 1" :: "v"(voff), "v"(v), "s"(sbase), "i"(OFF) : "memory");
}
__device__ __forceinline__ void wait_vm0() { asm volatile("s_waitcnt vmcnt(0)" ::: "memory"); }

__device__ __forceinline__ bool timed_out(unsigned* tmo, unsigned call, u64 t0, unsigned& spins) {
    if ((++spins & 63u) != 0u) return false;
    if (rfl(__hip_atomic_load(tmo, __ATOMIC_RELAXED, __HIP_MEMORY_SCOPE_AGENT)) == call) return true;
    if (__builtin_amdgcn_s_memrealtime() - t0 > DEADLINE) {
        __hip_atomic_store(tmo, call, __ATOMIC_RELAXED, __HIP_MEMORY_SCOPE_AGENT);
        return true;
    }
    return false;
}

__device__ __forceinline__ unsigned pack_store(const unsigned (&L)[8], const unsigned (&H)[8], const char* pk, char* esc, int lane) {
    unsigned m = max(max(H[0] & 0x7f7f7f7fu, H[2] & 0x7f7f7f7fu), max(H[5] & 0x7f7f7f7fu, H[7] & 0x7f7f7f7fu));
    m = max(m, (unsigned)__builtin_amdgcn_update_dpp(0, (int)m, 0x111, 0xf, 0xf, false));
    m = max(m, (unsigned)__builtin_amdgcn_update_dpp(0, (int)m, 0x112, 0xf, 0xf, false));
    m = max(m, (unsigned)__builtin_amdgcn_update_dpp(0, (int)m, 0x114, 0xf, 0xf, false));
    m = max(m, (unsigned)__builtin_amdgcn_update_dpp(0, (int)m, 0x118, 0xf, 0xf, false));
    const unsigned m0 = __builtin_amdgcn_readlane(m, 15), m1 = __builtin_amdgcn_readlane(m, 31);
    const unsigned m2 = __builtin_amdgcn_readlane(m, 47), m3 = __builtin_amdgcn_readlane(m, 63);
    const unsigned top = max(max(m0, m1), max(m2, m3)) >> 24;
    const unsigned lo = top > 7u ? top - 7u : 0u;
    const unsigned lo4 = lo * 0x01010101u;
    u32x4 N, X;
#pragma unroll
    for (int a = 0; a < 4; ++a) {
        const unsigned d0 = H[a] - lo4, d1 = rotl4(H[4 + a] - lo4);
        N[a] = bfi(0x87878787u, d0, d1);
        X[a] = bfi(0x87878787u, d1, d0);
    }
    const unsigned voff = lane * 16;
    st_wt<0>(pk, voff, (u32x4){L[0], L[1], L[2], L[3]});
    st_wt<1024>(pk, voff, (u32x4){L[4], L[5], L[6], L[7]});
    st_wt<2048>(pk, voff, N);
    const u64 em = __builtin_amdgcn_ballot_w64((X[0] | X[1] | X[2] | X[3]) != 0u);
    unsigned m8 = 0;
    if (em != 0ull) {
        m8 = (unsigned)__builtin_amdgcn_ballot_w64(lane < 8 && ((em >> (lane * 8)) & 0xffull) != 0ull);
        if ((m8 >> (lane >> 3)) & 1u) st_wt<0>(esc, voff, X);
    }
    return (m8 << 8) | lo;
}
__device__ __forceinline__ void join_bytes(const unsigned (&L)[8], const unsigned (&H)[8], u32x4 (&o)[4]) {
#pragma unroll
    for (int hf = 0; hf < 2; ++hf)
#pragma unroll
        for (int a = 0; a < 4; ++a) {
            o[2 * hf][a] = perm(H[4 * hf + a], L[4 * hf + a], 0x05010400u);
            o[2 * hf + 1][a] = perm(H[4 * hf + a], L[4 * hf + a], 0x07030602u);
        }
}
__device__ __forceinline__ void split_bytes(const u32x4 (&d)[4], unsigned (&L)[8], unsigned (&H)[8]) {
#pragma unroll
    for (int hf = 0; hf < 2; ++hf)
#pragma unroll
        for (int a = 0; a < 4; ++a) {
            L[4 * hf + a] = perm(d[2 * hf + 1][a], d[2 * hf][a], 0x06040200u);
            H[4 * hf + a] = perm(d[2 * hf + 1][a], d[2 * hf][a], 0x07050301u);
        }
}
__device__ __forceinline__ void store_stage(const char* dst, unsigned voff, const u32x4 (&o)[4]) {
    st_wt<0>(dst, voff, o[0]); st_wt<1024>(dst, voff, o[1]); st_wt<2048>(dst, voff, o[2]); st_wt<3072>(dst, voff, o[3]);
}
__device__ __forceinline__ void store_out(const char* dst, unsigned voff, const u32x4 (&o)[4]) {
    st_wt<0>(dst, voff, o[0]); st_wt<64>(dst, voff, o[1]);
    st_wt<0>(dst + 16 * 65536, voff, o[2]); st_wt<64>(dst + 16 * 65536, voff, o[3]);
}

__device__ __forceinline__ void pack_piece(const Params& P, char* ws, const char* px, const char* pw, LAS unsigned* lrec, unsigned e,
                                           unsigned cntv, int lane, int wid) {
    const int li = lane & 15, lg = lane >> 4;
    unsigned* ready = (unsigned*)(ws + OFF_READY);
    const int h = (e >> 3) & 15, idx = (e & 7) * 8 + wid, id = h * 64 + idx;
    const bool isx = (e >> 7) == 0u;
    const unsigned dest = ((e >> 7) - 1u) & 7u;
    const bool own = !isx && dest == (unsigned)P.rank;
    const unsigned rdest = isx ? (unsigned)lane >> 3 : dest;
    char* peer = P.ws[rdest];
    unsigned L[8], H[8];
    u32x4 d[4];
    if (isx) {
        const char* src = px + (size_t)(64 * idx + li) * 1024 + h * 64 + lg * 16;
#pragma unroll
        for (int c = 0; c < 4; ++c) d[c] = *(const u32x4*)(src + 16 * c * 1024);
        __builtin_amdgcn_sched_barrier(0);
        split_bytes(d, L, H);
    } else {
        const int cp = 32 * ((li >> 1) & 1) + 8 * (li >> 2) + 4 * (li & 1);
        const char* src = pw + (size_t)(32 * h + 8 * lg) * 65536 + (size_t)(dest * 4096 + 64 * idx + cp) * 2;
        u32x2 v[8];
#pragma unroll
        for (int j = 0; j < 8; ++j) v[j] = *(const u32x2*)(src + (size_t)j * 65536);
        __builtin_amdgcn_sched_barrier(0);
#pragma unroll
        for (int hf = 0; hf < 2; ++hf)
#pragma unroll
            for (int a = 0; a < 4; ++a) {
                L[4 * hf + a] = perm(v[2 * a + 1][hf], v[2 * a][hf], 0x06020400u);
                H[4 * hf + a] = perm(v[2 * a + 1][hf], v[2 * a][hf], 0x07030501u);
            }
        join_bytes(L, H, d);
    }
    asm volatile("" :: "v"(peer));
    const unsigned call = rfl(cntv) + 1u, par = call & 1u;
    if (!own) {
        const int g = isx ? id : 1024 + (int)dest * 1024 + id;
        const unsigned r = pack_store(L, H, ws + OFF_PK + par * PK_PAR + (size_t)g * 3072, ws + OFF_ESC + par * ESC_PAR + (size_t)g * 1024, lane);
        if (lane == 0) lrec[wid] = r;
    }
    if (isx || own) store_stage(ws + (isx ? OFF_A : OFF_B) + (size_t)(h * 256 + 4 * idx) * 1024, lane * 16, d);
    wait_vm0();
    if ((isx || own) && lane == 0)
        __hip_atomic_store(ready + ((isx ? 0 : 64) + idx) * NSLOT + h, call, __ATOMIC_RELAXED, __HIP_MEMORY_SCOPE_SYSTEM);
    asm volatile("s_waitcnt lgkmcnt(0)" ::: "memory");
    __builtin_amdgcn_s_barrier();
    asm volatile("" ::: "memory");
    if (wid == 0 && !own && (isx || lane < 8) && rdest != (unsigned)P.rank) {
        u64* rp = (u64*)(peer + OFF_REC + par * REC_PAR) + (P.rank * 3 + (isx ? 0 : 1)) * 1024 + h * 64 + (e & 7) * 8 + (lane & 7);
        __hip_atomic_store(rp, ((u64)call << 32) | lrec[lane & 7], __ATOMIC_RELAXED, __HIP_MEMORY_SCOPE_SYSTEM);
    }
}

__device__ __forceinline__ void pull_unit(u32x4& p0, u32x4& p1, u32x4& p2, u32x4& p3, u32x2& rec, unsigned& tk, const char* pk, u32x4 srd,
                                          unsigned esc_soff, unsigned voff, unsigned esc_voff, const char* ws, unsigned rec_off,
                                          unsigned tk_off, unsigned tk_add, unsigned flag_off, unsigned call) {
    asm volatile("s_nop 4\n\t"
                 "global_load_dwordx4 %0, %6, %7 sc0 sc1\n\t"
                 "global_load_dwordx4 %1, %6, %7 offset:1024 sc0 sc1\n\t"
                 "global_load_dwordx4 %2, %6, %7 offset:2048 sc0 sc1\n\t"
                 "buffer_load_dwordx4 %3, %8, %9, %10 offen sc0 sc1\n\t"
                 "s_mov_b64 exec, 1\n\t"
                 "global_load_dwordx2 %4, %12, %11 sc0 sc1\n\t"
                 "global_atomic_add %5, %13, %14, %11 sc0\n\t"
                 "s_waitcnt vmcnt(6)\n\t"
                 "global_store_dword %15, %16, %11 sc0 sc1\n\t"
                 "s_mov_b64 exec, -1\n\t"
                 "s_waitcnt vmcnt(1)"
                 : "=&v"(p0), "=&v"(p1), "=&v"(p2), "=&v"(p3), "=&v"(rec), "=&v"(tk)
                 : "v"(voff), "s"(pk), "v"(esc_voff), "s"(srd), "s"(esc_soff), "s"(ws), "v"(rec_off), "v"(tk_off), "v"(tk_add),
                   "v"(flag_off), "v"(call)
                 : "memory");
}

struct Desc { unsigned g, dst, rdy, rec; };

__device__ __forceinline__ void pull_link(const Params& P, int q, int w, unsigned call, int lane) {
    char* ws = P.me;
    const unsigned par = call & 1u;
    const unsigned p = (unsigned)(P.rank + 1 + q) & 7u;
    const char* peer = P.ws[p];
    unsigned* tmo = (unsigned*)(ws + OFF_TMO);
    unsigned vorder = P.order[lane];
    asm volatile("" : "+v"(vorder));
    const unsigned rec_base = (unsigned)(OFF_REC + par * REC_PAR) + p * 3072u * 8u;
    const unsigned dummy = (unsigned)OFF_DUMMY + (unsigned)(q * PW + w) * 128u;
    auto describe = [&](unsigned n) {
        Desc d;
        if (n < 2048u) {
            const unsigned e = ((unsigned)__builtin_amdgcn_readlane(vorder, n >> 5) >> (n & 24u)) & 255u;
            const unsigned type = e >> 7, h = (e >> 3) & 15u, idx = (e & 7u) * 8u + (n & 7u), id = h * 64u + idx, slot = 16u + 7u * h + q;
            d.g = type ? 1024u + P.rank * 1024u + id : id;
            d.dst = (unsigned)(type ? OFF_B : OFF_A) + (slot * 256u + 4u * idx) * 1024u;
            d.rdy = (unsigned)OFF_READY + ((type * 64u + idx) * NSLOT + slot) * 4u;
            d.rec = rec_base + (type * 1024u + id) * 8u;
        } else {
            const unsigned id = (n - 2048u) & 1023u;
            d.g = 9216u + P.rank * 1024u + id;
            d.dst = (id & 15u) * (32u * 65536u) + (p * 4096u + 64u * (id >> 4)) * 2u;
            d.rdy = 0;
            d.rec = rec_base + (2048u + id) * 8u;
        }
        return d;
    };
    const char* pkb = peer + OFF_PK + par * PK_PAR;
    u32x4 srd;
    {
        const u64 eb = (u64)(peer + OFF_ESC + par * ESC_PAR);
        srd[0] = rfl((unsigned)eb); srd[1] = rfl((unsigned)(eb >> 32)) & 0xffffu; srd[2] = (unsigned)ESC_PAR; srd[3] = 0x00020000u;
    }
    const unsigned voff = lane * 16, voff_out = (lane & 15) * 65536 + (lane >> 4) * 16;
    const unsigned tk_off = (unsigned)OFF_TKT + q * 128u;
    const unsigned tk_base = (call - 1u) * (NTK - PW) - 2u * PW;
    unsigned n1 = PW + w, info = 0, icall = 0, pend = dummy;
    Desc d0 = describe(w), d1 = describe(n1);
    bool dead = false;
    for (;;) {
        if (icall != call) {
            if (pend != dummy) {
                wait_vm0();
                if (lane == 0) __hip_atomic_store((unsigned*)(ws + pend), call, __ATOMIC_RELAXED, __HIP_MEMORY_SCOPE_SYSTEM);
                pend = dummy;
            }
            const u64 t0 = __builtin_amdgcn_s_memrealtime();
            unsigned spins = 0;
            info = 0;
            while (!dead) {
                const u64 v = __hip_atomic_load((const u64*)(ws + d0.rec), __ATOMIC_RELAXED, __HIP_MEMORY_SCOPE_SYSTEM);
                if (rfl((unsigned)(v >> 32)) == call) { info = rfl((unsigned)v); break; }
                if (timed_out(tmo, call, t0, spins)) dead = true;
                __builtin_amdgcn_s_sleep(1);
            }
        }
        const unsigned ev = ((info >> (8 + (lane >> 3))) & 1u) ? voff : 0x80000000u;
        u32x4 p0, p1, p2, p3; u32x2 rec; unsigned tk;
        pull_unit(p0, p1, p2, p3, rec, tk, pkb + (size_t)d0.g * 3072, srd, d0.g * 1024u, voff, ev, ws, d1.rec, tk_off, n1 < NTK ? 1u : 0u, pend, call);
        const unsigned lo4 = (info & 0xffu) * 0x01010101u;
        unsigned L[8], H[8];
#pragma unroll
        for (int a = 0; a < 4; ++a) {
            L[a] = p0[a]; L[4 + a] = p1[a];
            H[a] = bfi(0x87878787u, p2[a], p3[a]) + lo4;
            H[4 + a] = rotr4(bfi(0x87878787u, p3[a], p2[a])) + lo4;
        }
        u32x4 o[4];
        join_bytes(L, H, o);
        if (d0.rdy) {
            store_stage(ws + d0.dst, voff, o);
            pend = d0.rdy;
        } else {
            if (dead) {
#pragma unroll
                for (int c = 0; c < 4; ++c) o[c] = (u32x4){0x7fc07fc0u, 0x7fc07fc0u, 0x7fc07fc0u, 0x7fc07fc0u};
            }
            store_out(P.out + d0.dst, voff_out, o);
            pend = dummy;
        }
        if (n1 >= NTK) break;
        info = rfl(rec[0]); icall = rfl(rec[1]);
        d0 = d1;
        n1 = rfl(tk) - tk_base;
        d1 = describe(n1);
    }
    if (pend != dummy) {
        wait_vm0();
        if (lane == 0) __hip_atomic_store((unsigned*)(ws + pend), call, __ATOMIC_RELAXED, __HIP_MEMORY_SCOPE_SYSTEM);
    }
}

__device__ __forceinline__ void glds4(unsigned m0v, unsigned voff, const char* sb) {
    asm volatile("s_mov_b32 m0, %0\n\ts_nop 0\n\t"
                 "global_load_lds_dwordx4 %1, %2 sc0 sc1\n\t"
                 "global_load_lds_dwordx4 %1, %2 offset:1024 sc0 sc1\n\t"
                 "global_load_lds_dwordx4 %1, %2 offset:2048 sc0 sc1\n\t"
                 "global_load_lds_dwordx4 %1, %2 offset:3072 sc0 sc1" :: "s"(m0v), "v"(voff), "s"(sb) : "memory");
}
__device__ __forceinline__ void glds1(unsigned m0v, unsigned voff, const char* sb) {
    asm volatile("s_mov_b32 m0, %0\n\ts_nop 0\n\tglobal_load_lds_dwordx4 %1, %2 sc0 sc1" :: "s"(m0v), "v"(voff), "s"(sb) : "memory");
}

__device__ __forceinline__ void gemm_tile(const Params& P, LAS char* lds, int pm, int pn, unsigned call, int lane, int wid) {
    char* ws = P.me;
    unsigned* tmo = (unsigned*)(ws + OFF_TMO);
    const unsigned* ready = (const unsigned*)(ws + OFF_READY);
    const int wr = wid >> 2, wc = wid & 3;
    const unsigned ldsbase = (unsigned)(size_t)lds;
    const char* g1; const char* g2; unsigned l1, l2; const unsigned* r1; const unsigned* r2;
    if (wr == 0) {
        const int u1 = 5 * pm + wc; const bool has5 = pm < 12;
        g1 = ws + OFF_A + (size_t)u1 * 4096; g2 = has5 ? ws + OFF_A + (size_t)(20 * pm + 16 + wc) * 1024 : g1;
        l1 = ldsbase + wc * 4096; l2 = ldsbase + (16 + wc) * 1024;
        r1 = ready + u1 * NSLOT; r2 = has5 ? ready + (5 * pm + 4) * NSLOT : r1;
    } else {
        const int u1 = 4 * pn + wc;
        g1 = ws + OFF_B + (size_t)u1 * 4096; g2 = g1; l1 = ldsbase + 20480 + wc * 4096; l2 = l1;
        r1 = ready + (64 + u1) * NSLOT; r2 = r1;
    }
    const unsigned voff = lane * 16;
    u64 mask = 0;
#define ENSURE(t) do { if (((t) & 63) == 0) mask = 0; \
    if (!((mask >> ((t) & 63)) & 1ull)) { const u64 t0_ = __builtin_amdgcn_s_memrealtime(); unsigned sp_ = 0; \
        for (;;) { \
            const unsigned a_ = __hip_atomic_load(r1 + ((t) & 64) + lane, __ATOMIC_RELAXED, __HIP_MEMORY_SCOPE_SYSTEM); \
            const unsigned b_ = __hip_atomic_load(r2 + ((t) & 64) + lane, __ATOMIC_RELAXED, __HIP_MEMORY_SCOPE_SYSTEM); \
            mask = __builtin_amdgcn_ballot_w64(a_ >= call && b_ >= call); \
            if ((mask >> ((t) & 63)) & 1ull) break; \
            if (timed_out(tmo, call, t0_, sp_)) { mask = ~0ull; break; } \
            __builtin_amdgcn_s_sleep(2); } } } while (0)
#define STAGE(t, bt) do { const size_t go_ = (size_t)(t) * 262144; const unsigned lo_ = (bt) * LSLOT; \
    glds4(l1 + lo_, voff, g1 + go_); if (wr == 0) glds1(l2 + lo_, voff, g2 + go_); } while (0)
#define WAITV() do { if (wr == 0) asm volatile("s_waitcnt vmcnt(10)" ::: "memory"); else asm volatile("s_waitcnt vmcnt(8)" ::: "memory"); } while (0)
    f32x4 acc[10][4];
#pragma unroll
    for (int m = 0; m < 10; ++m)
#pragma unroll
        for (int n = 0; n < 4; ++n) acc[m][n] = (f32x4){0.f, 0.f, 0.f, 0.f};
    for (int t = 0; t < 3; ++t) { ENSURE(t); STAGE(t, t); }
    WAITV();
    __builtin_amdgcn_s_barrier();
    STAMP(ws, 2);
    if (wr == 1) __builtin_amdgcn_s_barrier();
    const unsigned aoff = wr * 10240 + lane * 16, boff = 20480 + wc * 4096 + lane * 16;
    for (int s = 0; s < NSLOT; ++s) {
        const unsigned so = (s & 3) * LSLOT;
        bf16x8 A[10], B[4];
        LAS const char* pa = lds + aoff + so; LAS const char* pb = lds + boff + so;
#pragma unroll
        for (int n = 0; n < 4; ++n) B[n] = *(LAS const bf16x8*)(pb + n * 1024);
#pragma unroll
        for (int m = 0; m < 10; ++m) A[m] = *(LAS const bf16x8*)(pa + m * 1024);
        __builtin_amdgcn_sched_barrier(0);
        const int t = s + 3 < NSLOT ? s + 3 : NSLOT - 1;
        if (s + 3 < NSLOT) ENSURE(t);
        STAGE(t, (s + 3) & 3);
        WAITV();
        asm volatile("s_waitcnt lgkmcnt(0)" ::: "memory");
        __builtin_amdgcn_sched_barrier(0);
        __builtin_amdgcn_s_barrier();
        __builtin_amdgcn_sched_barrier(0);
        __builtin_amdgcn_s_setprio(1);
#pragma unroll
        for (int m = 0; m < 10; ++m)
#pragma unroll
            for (int n = 0; n < 4; ++n) acc[m][n] = __builtin_amdgcn_mfma_f32_16x16x32_bf16(B[n], A[m], acc[m][n], 0, 0, 0);
        __builtin_amdgcn_s_setprio(0);
        __builtin_amdgcn_sched_barrier(0);
        __builtin_amdgcn_s_barrier();
        __builtin_amdgcn_sched_barrier(0);
    }
    if (wr == 0) __builtin_amdgcn_s_barrier();
#undef ENSURE
#undef STAGE
#undef WAITV
    wait_vm0();
    STAMP(ws, 3);
    const bool bad = rfl(__hip_atomic_load(tmo, __ATOMIC_RELAXED, __HIP_MEMORY_SCOPE_AGENT)) == call;
    const unsigned par = call & 1u;
    const int cb = 4 * pn + wc, rb0 = 10 * pm + 5 * wr;
    const unsigned voff_out = (lane & 15) * 65536 + (lane >> 4) * 16;
    unsigned rec = 0;
#pragma unroll
    for (int u = 0; u < 5; ++u) {
        const int rb = rb0 + u;
        if (rb < 128) {
            u32x4 d[4];
#pragma unroll
            for (int c = 0; c < 4; ++c) {
                const int mi = 2 * u + (c >> 1), b = c & 1;
                d[c][0] = cvt_pk(acc[mi][0][2 * b], acc[mi][1][2 * b]);
                d[c][1] = cvt_pk(acc[mi][2][2 * b], acc[mi][3][2 * b]);
                d[c][2] = cvt_pk(acc[mi][0][2 * b + 1], acc[mi][1][2 * b + 1]);
                d[c][3] = cvt_pk(acc[mi][2][2 * b + 1], acc[mi][3][2 * b + 1]);
                if (bad) d[c] = (u32x4){0x7fc07fc0u, 0x7fc07fc0u, 0x7fc07fc0u, 0x7fc07fc0u};
            }
            const int dest = rb >> 4;
            if (dest == P.rank) {
                store_out(P.out + (size_t)(rb & 15) * (32 * 65536) + (size_t)(P.rank * 4096 + 64 * cb) * 2, voff_out, d);
            } else {
                unsigned L[8], H[8];
                split_bytes(d, L, H);
                const int g = 9216 + dest * 1024 + cb * 16 + (rb & 15);
                const unsigned r = pack_store(L, H, ws + OFF_PK + par * PK_PAR + (size_t)g * 3072, ws + OFF_ESC + par * ESC_PAR + (size_t)g * 1024, lane);
                if (lane == u) rec = r;
            }
        }
    }
    const unsigned rbl = (unsigned)(rb0 + lane), dl = rbl >> 4;
    char* peer = P.ws[dl & 7u];
    wait_vm0();
    if (lane < 5 && dl < (unsigned)NR && dl != (unsigned)P.rank) {
        u64* rp = (u64*)(peer + OFF_REC + par * REC_PAR) + (P.rank * 3 + 2) * 1024 + cb * 16 + (rbl & 15u);
        __hip_atomic_store(rp, ((u64)call << 32) | rec, __ATOMIC_RELAXED, __HIP_MEMORY_SCOPE_SYSTEM);
    }
}

#define PK_R1 52
#define PK_R2 104
constexpr int NPACKERS = 208 + NCOMM - 7 * PWG;

__global__ void __launch_bounds__(512, 2) fused_gemm_rs(const Params P) {
    __shared__ __attribute__((aligned(16))) char smem[LDS_BYTES];
    const int bid = blockIdx.x, xcd = bid & 7, j = bid >> 3;
    const int lane = threadIdx.x & 63, wid = rfl(threadIdx.x >> 6);
    const bool comm = j < NCOMM / 8;
    const int jj = j - NCOMM / 8;
    const int pm = comm ? 0 : jj % 13, pn = comm ? 0 : 2 * xcd + jj / 13;
    const int o = comm ? 208 + bid - 7 * PWG : pn * 13 + pm;
    char* ws = P.me; const char* px = P.x; const char* pw = P.w; unsigned ew = P.plist[o >> 1];
    asm volatile("" :: "s"(ws), "s"(px), "s"(pw), "s"(ew), "s"(P.rank));
    STAMP(ws, 0);
    unsigned* cnt = (unsigned*)(ws + OFF_CNT) + bid * 8 + wid;
    const unsigned cntv = __hip_atomic_load(cnt, __ATOMIC_RELAXED, __HIP_MEMORY_SCOPE_AGENT);
    if (bid >= 7 * PWG) {
        int piece = o;
        for (int r = 0;; ++r) {
            const unsigned e = (ew >> ((piece & 1) * 16)) & 0xffffu;
            pack_piece(P, ws, px, pw, (LAS unsigned*)smem + (r & 1) * 8, e, cntv, lane, wid);
            if (r == 0) { if (o < PK_R1) break; piece = NPACKERS + o - PK_R1; }
            else if (r == 1) { if (o < PK_R2) break; piece = 2 * NPACKERS - PK_R1 + o - PK_R2; }
            else piece += NPACKERS - PK_R2;
            if (piece >= NPIECE) break;
            ew = P.plist[piece >> 1];
        }
        asm volatile("s_waitcnt lgkmcnt(0)" ::: "memory");
        __builtin_amdgcn_s_barrier();
    }
    const unsigned call = rfl(cntv) + 1u;
    if (lane == 0) __hip_atomic_store(cnt, call, __ATOMIC_RELAXED, __HIP_MEMORY_SCOPE_AGENT);
    STAMP(ws, 1);
    if (comm) {
        if (bid < 7 * PWG) pull_link(P, bid % 7, (bid / 7) * 8 + wid, call, lane);
    } else {
        gemm_tile(P, (LAS char*)smem, pm, pn, call, lane, wid);
    }
    STAMP(ws, 4);
}

}

#define LAG_NUM 23

extern "C" void kernel_launch(int rank, int nranks, ncclComm_t comm, hipStream_t stream,
                              void* const* d_in, const int* in_sizes, int n_in,
                              void* d_out, int out_size, const int* in_dtypes, int out_dtype,
                              void* const* d_out_all, void* const* d_ws_all, size_t ws_bytes) {
    (void)comm; (void)in_dtypes; (void)out_dtype; (void)d_out_all;
    if (nranks != NR || (unsigned)rank >= (unsigned)NR || n_in < 2 || ws_bytes < WS_NEED || in_sizes[0] != 4096 * 512 ||
        in_sizes[1] != 512 * 32768 || out_size != 512 * 32768) {
        fprintf(stderr, "kernel_launch: unexpected problem shape\n");
        return;
    }
    Params P;
    P.x = (const char*)d_in[0];
    P.w = (const char*)d_in[1];
    P.out = (char*)d_out;
    for (int p = 0; p < NR; ++p) P.ws[p] = (char*)d_ws_all[p];
    P.me = P.ws[rank];
    P.rank = rank;
    P.pad = 0;
    unsigned char* order = (unsigned char*)P.order;
    unsigned char worder[128];
    int no = 0, nw = 0;
    for (int ph = 0; ph < 64; ++ph) {
        if (ph < 16)
            for (int q = 0; q < 8; ++q) order[no++] = (unsigned char)(ph << 3 | q);
        for (int pr = 0; pr < 8; ++pr) {
            const int h = ph - pr * LAG_NUM / 16;
            if (h >= 0 && h < 16) {
                order[no++] = (unsigned char)(128 | h << 3 | pr);
                worder[nw++] = (unsigned char)(h << 3 | pr);
            }
        }
    }
    unsigned short* plist = (unsigned short*)P.plist;
    int np = 0, nr = 0;
    for (int r = 0; r < 3; ++r) {
        if (r < 2)
            for (int i = 0; i < 64; ++i) {
                plist[np++] = (unsigned short)(r * 64 + i);
                plist[np++] = (unsigned short)((rank + 1) << 7 | (r * 64 + i));
            }
        const int end = r == 0 ? NPACKERS : r == 1 ? 2 * NPACKERS - PK_R1 : NPIECE;
        for (; np < end; ++nr) plist[np++] = (unsigned short)((((rank + 1 + nr % 7) & 7) + 1) << 7 | worder[nr / 7]);
    }
    hipLaunchKernelGGL(fused_gemm_rs, dim3(256), dim3(512), 0, stream, P);
    hipError_t e = hipGetLastError();
    if (e != hipSuccess) fprintf(stderr, "launch failed: %s\n", hipGetErrorString(e));
}
```

```cpp
#include <hip/hip_runtime.h>
#include <rccl/rccl.h>
#include <cstdio>

#define LAS __attribute__((address_space(3)))
typedef short bf16x8 __attribute__((ext_vector_type(8)));
typedef float f32x4 __attribute__((ext_vector_type(4)));
typedef unsigned u32x4 __attribute__((ext_vector_type(4)));
typedef unsigned u32x2 __attribute__((ext_vector_type(2)));
typedef unsigned long long u64;

namespace {

constexpr int NR = 8;
constexpr int NSLOT = 128;
constexpr int LSLOT = 36 * 1024;
constexpr int LDS_LUT = 4 * LSLOT;
constexpr int LDS_REC = LDS_LUT + 2048;
constexpr int LDS_BYTES = LDS_REC + 128;
constexpr int SCR_WAVE = 9216;
constexpr int SCR_UNIT = 4096;
constexpr int SCR_V = 1152;
constexpr int NCOMM = 48;
#define PWG 6
#define PWAVES 30
#define PULLMAP 0
#define COLMAP 4
constexpr int PW = PWAVES;
constexpr int NPIECE = 1152;
constexpr unsigned NTK = 1536;
constexpr u64 DEADLINE = 100000000ull;

constexpr size_t OFF_CNT = 0;
constexpr size_t OFF_TMO = 16384;
constexpr size_t OFF_TKT = 20480;
constexpr size_t OFF_READY = 32768;
constexpr size_t OFF_REC = 131072;
constexpr size_t REC_PAR = (size_t)NR * 3 * 1024 * 8;
constexpr size_t OFF_DUMMY = 589824;
constexpr size_t OFF_DBG = 655360;
constexpr size_t OFF_A = 1u << 20;
constexpr size_t OFF_B = OFF_A + (32u << 20);
constexpr int NUNIT = 1024 + 2 * NR * 1024;
constexpr int FIXB = 2560;
constexpr int VCAP = 2624;
constexpr int PAIRB = 2 * FIXB + 2 * VCAP;
constexpr int VMID = 2 * FIXB + VCAP;
constexpr size_t PK_PAR = (size_t)(NUNIT / 2) * PAIRB;
constexpr size_t OFF_PK = OFF_B + (32u << 20);
constexpr size_t WS_NEED = OFF_PK + 2 * PK_PAR;
static_assert(OFF_REC + 2 * REC_PAR <= OFF_DUMMY && OFF_DUMMY + 7 * PW * 128 <= OFF_DBG, "layout");

struct Params {
    const char* x;
    const char* w;
    char* out;
    char* me;
    char* ws[NR];
    int rank;
    unsigned rbord;
    unsigned order[64];
    unsigned plist[NPIECE / 2];
};

#define STAMP(ws, idx) do { } while (0)

__device__ __forceinline__ unsigned rfl(unsigned v) { return __builtin_amdgcn_readfirstlane(v); }
__device__ __forceinline__ unsigned perm(unsigned hi, unsigned lo, unsigned sel) { return __builtin_amdgcn_perm(hi, lo, sel); }
__device__ __forceinline__ unsigned bfi(unsigned m, unsigned a, unsigned b) { return (a & m) | (b & ~m); }
__device__ __forceinline__ unsigned rotl4(unsigned v) { return __builtin_amdgcn_alignbit(v, v, 28); }
__device__ __forceinline__ unsigned rotr4(unsigned v) { return __builtin_amdgcn_alignbit(v, v, 4); }
__device__ __forceinline__ unsigned cvt_pk(float lo, float hi) { unsigned r; asm("v_cvt_pk_bf16_f32 %0, %1, %2" : "=v"(r) : "v"(lo), "v"(hi)); return r; }

template <int OFF> __device__ __forceinline__ void st_wt(const char* sbase, unsigned voff, u32x4 v) {
    asm volatile("global_store_dwordx4 %0, %1, %2 offset:%3 sc0 sc1\n\ts_nop 1" :: "v"(voff), "v"(v), "s"(sbase), "i"(OFF) : "memory");
}
#define OUTBITS " sc0 sc1"
template <int OFF> __device__ __forceinline__ void st_out(const char* sbase, unsigned voff, u32x4 v) {
    asm volatile("global_store_dwordx4 %0, %1, %2 offset:%3" OUTBITS "\n\ts_nop 1" :: "v"(voff), "v"(v), "s"(sbase), "i"(OFF) : "memory");
}
__device__ __forceinline__ void wait_vm0() { asm volatile("s_waitcnt vmcnt(0)" ::: "memory"); }

__device__ __forceinline__ bool timed_out(unsigned* tmo, unsigned call, u64 t0, unsigned& spins) {
    if ((++spins & 63u) != 0u) return false;
    if (rfl(__hip_atomic_load(tmo, __ATOMIC_RELAXED, __HIP_MEMORY_SCOPE_AGENT)) == call) return true;
    if (__builtin_amdgcn_s_memrealtime() - t0 > DEADLINE) {
        __hip_atomic_store(tmo, call, __ATOMIC_RELAXED, __HIP_MEMORY_SCOPE_AGENT);
        return true;
    }
    return false;
}

#define LDS_FENCE() asm volatile("" ::: "memory")
template <int CTRL, int ROWMASK> __device__ __forceinline__ unsigned dpp0(unsigned v) {
    return (unsigned)__builtin_amdgcn_update_dpp(0, (int)v, CTRL, ROWMASK, 0xf, false);
}
__device__ __forceinline__ unsigned wave_scan(unsigned v) {
    v += dpp0<0x111, 0xf>(v); v += dpp0<0x112, 0xf>(v); v += dpp0<0x114, 0xf>(v); v += dpp0<0x118, 0xf>(v);
    v += dpp0<0x142, 0xa>(v); v += dpp0<0x143, 0xc>(v);
    return v;
}
__device__ __forceinline__ unsigned lane_rank(u64 m) {
    return __builtin_amdgcn_mbcnt_hi((unsigned)(m >> 32), __builtin_amdgcn_mbcnt_lo((unsigned)m, 0u));
}
__device__ __forceinline__ void build_lut(LAS char* lds, bool dec, int tid) {
    if (tid < 256) {
        u64 e = 0x0c0c0c0c0c0c0c0cull;
        unsigned t = 0;
#pragma unroll
        for (unsigned j = 0; j < 8; ++j)
            if ((tid >> j) & 1) {
                const unsigned slot = (t & 1u) * 4u + (t >> 1);
                const unsigned at = dec ? j : slot, val = dec ? slot : j;
                e = (e & ~(0xffull << (8 * at))) | ((u64)val << (8 * at));
                ++t;
            }
        *(LAS u64*)(lds + LDS_LUT + tid * 8) = e;
    }
}
__device__ __forceinline__ u32x2 lut_at(LAS char* lds, unsigned f) {
    return *(LAS const u32x2*)(lds + __builtin_amdgcn_udot4(f, 0x40201008u, (unsigned)LDS_LUT, false));
}
__device__ __forceinline__ unsigned shuffle8(unsigned w, u32x2 sel) {
    const unsigned a = w & 0x0f0f0f0fu, b = (w >> 4) & 0x0f0f0f0fu;
    return perm(b, a, sel[0]) | (perm(b, a, sel[1]) << 4);
}
__device__ __forceinline__ void lds_or(LAS char* p, unsigned v) {
    __hip_atomic_fetch_or((LAS unsigned*)p, v, __ATOMIC_RELAXED, __HIP_MEMORY_SCOPE_WAVEFRONT);
}
__device__ __forceinline__ void put_bits(LAS char* base, unsigned pos, unsigned v) {
    const u64 sh = (u64)v << (pos & 31u);
    LAS char* p = base + ((pos >> 5) << 2);
    lds_or(p, (unsigned)sh); lds_or(p + 4, (unsigned)(sh >> 32));
}
__device__ __forceinline__ unsigned get_bits(LAS char* base, unsigned pos) {
    LAS const unsigned* p = (LAS const unsigned*)(base + ((pos >> 5) << 2));
    return __builtin_amdgcn_alignbit(p[1], p[0], pos);
}

template <int NU, class Mid>
__device__ __forceinline__ void pack_store(const u32x4 (&d)[NU][4], char* pk, const unsigned (&g)[NU], LAS char* lds, LAS char* scr, int lane,
                                           unsigned (&info)[NU], Mid mid) {
    const unsigned voff = lane * 16;
    unsigned E[NU][8];
#pragma unroll
    for (int u = 0; u < NU; ++u) {
        u32x4 S[2];
#pragma unroll
        for (int hf = 0; hf < 2; ++hf)
#pragma unroll
            for (int a = 0; a < 4; ++a) {
                const unsigned p = __builtin_amdgcn_alignbit(d[u][2 * hf][a], d[u][2 * hf][a], 31);
                const unsigned r = __builtin_amdgcn_alignbit(d[u][2 * hf + 1][a], d[u][2 * hf + 1][a], 31);
                S[hf][a] = perm(r, p, 0x06040200u);
                E[u][4 * hf + a] = perm(r, p, 0x07050301u);
            }
        const char* fix = pk + (size_t)(g[u] >> 1) * PAIRB + (g[u] & 1u) * FIXB;
        st_wt<0>(fix, voff, S[0]);
        st_wt<1024>(fix, voff, S[1]);
    }
    unsigned sum = __builtin_amdgcn_sad_u8(E[0][0], 0u, __builtin_amdgcn_sad_u8(E[0][2], 0u, __builtin_amdgcn_sad_u8(E[0][5], 0u, __builtin_amdgcn_sad_u8(E[0][7], 0u, 0u))));
    sum = __builtin_amdgcn_readlane(wave_scan(sum), 63);
    const unsigned top = min(((sum + 983u) >> 10) + 2u, 255u), top4 = top * 0x01010101u;
    unsigned N[NU][4], F[NU][4], cnt = 0;
    u32x4 X[NU];
    u32x2 sel[NU][4];
#pragma unroll
    for (int u = 0; u < NU; ++u) {
        unsigned C[4], c = 0;
#pragma unroll
        for (int a = 0; a < 4; ++a) {
            const unsigned d0 = top4 - E[u][a], d1 = rotl4(top4 - E[u][4 + a]);
            N[u][a] = bfi(0x0f0f0f0fu, d0, d1);
            X[u][a] = bfi(0x0f0f0f0fu, d1, d0);
            const unsigned big = (N[u][a] | (N[u][a] >> 1)) & 0x44444444u;
            C[a] = N[u][a] & 0x33333333u & ~(big - (big >> 2));
            F[u][a] = ~(C[a] | (C[a] >> 1)) & 0x11111111u;
            c += __builtin_popcount(F[u][a]);
            sel[u][a] = lut_at(lds, F[u][a]);
        }
        cnt |= c << (16 * u);
        const u32x2 cc = {C[0] | (C[1] << 2), C[2] | (C[3] << 2)};
        const char* fix = pk + (size_t)(g[u] >> 1) * PAIRB + (g[u] & 1u) * FIXB;
        asm volatile("global_store_dwordx2 %0, %1, %2 offset:2048 sc0 sc1" :: "v"(lane * 8), "v"(cc), "s"(fix) : "memory");
    }
    const unsigned incl = wave_scan(cnt), tot = __builtin_amdgcn_readlane(incl, 63);
    unsigned a16[NU];
#pragma unroll
    for (int u = 0; u < NU; ++u) {
        LAS char* V = scr + u * SCR_UNIT + SCR_V;
        info[u] = top; a16[u] = 0;
        const u64 em = __builtin_amdgcn_ballot_w64((X[u][0] | X[u][1] | X[u][2] | X[u][3]) != 0u);
        if (em != 0ull) {
            const unsigned l = __builtin_ctzll(em);
            const unsigned x0 = __builtin_amdgcn_readlane(X[u][0], l), x1 = __builtin_amdgcn_readlane(X[u][1], l);
            const unsigned x2 = __builtin_amdgcn_readlane(X[u][2], l), x3 = __builtin_amdgcn_readlane(X[u][3], l);
            const unsigned any = x0 | x1 | x2 | x3, nz = (x0 != 0u) + (x1 != 0u) + (x2 != 0u) + (x3 != 0u);
            const unsigned nib = __builtin_ctz(any) >> 2, val = any >> (4u * nib);
            if ((em & (em - 1ull)) == 0ull && nz == 1u && val < 16u) {
                info[u] |= (l | (x0 ? 0u : x1 ? 1u : x2 ? 2u : 3u) << 6 | nib << 8 | val << 11) << 17;
            } else {
                info[u] |= 1u << 16;
                a16[u] = 1u + __builtin_popcountll(em);
                if (lane == 0) *(LAS u32x4*)V = (u32x4){(unsigned)em, (unsigned)(em >> 32), 0u, 0u};
                if ((em >> lane) & 1ull) *(LAS u32x4*)(V + 16u * (1u + lane_rank(em))) = X[u];
            }
        }
        if (((tot >> (16 * u)) & 0xffffu) > 640u) {
            *(LAS u32x2*)(scr + u * SCR_UNIT + 512 + lane * 8) = (u32x2){0u, 0u};
            *(LAS u32x4*)(V + 16u * a16[u] + 512 + voff) = (u32x4){0u, 0u, 0u, 0u};
            if (lane < 2) *(LAS u32x4*)(V + 16u * a16[u] + 1536 + voff) = (u32x4){0u, 0u, 0u, 0u};
        }
    }
#pragma unroll
    for (int u = 0; u < NU; ++u) {
        *(LAS u32x2*)(scr + u * SCR_UNIT + lane * 8) = (u32x2){0u, 0u};
        *(LAS u32x2*)(scr + u * SCR_UNIT + SCR_V + 16u * a16[u] + lane * 8) = (u32x2){0u, 0u};
    }
    LDS_FENCE();
#pragma unroll
    for (int u = 0; u < NU; ++u) {
        unsigned pos = (((incl - cnt) >> (16 * u)) & 0xffffu) * 4u;
#pragma unroll
        for (int a = 0; a < 4; ++a) {
            put_bits(scr + u * SCR_UNIT, pos, shuffle8(N[u][a], sel[u][a]));
            pos += 4u * __builtin_popcount(F[u][a]);
        }
    }
    LDS_FENCE();
    mid();
    unsigned bpos[NU];
#pragma unroll
    for (int u = 0; u < NU; ++u) bpos[u] = ((((tot >> (16 * u)) & 0xffffu) + 15u) >> 4) << 5;
    const unsigned nmax = NU == 1 ? tot : max(tot & 0xffffu, tot >> 16);
    unsigned t = 0;
    do {
        unsigned w[NU], f2[NU], c = 0;
        u32x2 s2[NU];
#pragma unroll
        for (int u = 0; u < NU; ++u) w[u] = *(LAS const unsigned*)(scr + u * SCR_UNIT + (t >> 1) + lane * 4);
#pragma unroll
        for (int u = 0; u < NU; ++u) {
            const bool in = t + lane * 8u < ((tot >> (16 * u)) & 0xffffu);
            w[u] = in ? w[u] : 0u;
            const unsigned e = ((w[u] | (w[u] >> 2)) >> 1) & 0x11111111u;
            const unsigned c2 = (w[u] & 0x55555555u & ~(e << 2)) | e;
            f2[u] = c2 & ~(c2 >> 2) & 0x11111111u;
            s2[u] = lut_at(lds, f2[u]);
            c |= (unsigned)__builtin_popcount(f2[u]) << (16 * u);
            LAS char* dst = in ? scr + u * SCR_UNIT + SCR_V + 16u * a16[u] + (t >> 2) + lane * 2 : scr + u * SCR_UNIT + 1088;
            *(LAS unsigned short*)dst = (unsigned short)(c2 | (c2 >> 15));
        }
        const unsigned in2 = wave_scan(c), t2 = __builtin_amdgcn_readlane(in2, 63);
#pragma unroll
        for (int u = 0; u < NU; ++u) {
            put_bits(scr + u * SCR_UNIT + SCR_V + 16u * a16[u], bpos[u] + (((in2 - c) >> (16 * u)) & 0xffffu) * 4u, shuffle8(w[u], s2[u]));
            bpos[u] += ((t2 >> (16 * u)) & 0xffffu) * 4u;
        }
        t += 512u;
    } while (t < nmax);
    LDS_FENCE();
    u32x4 vv[NU];
#pragma unroll
    for (int u = 0; u < NU; ++u) vv[u] = *(LAS const u32x4*)(scr + u * SCR_UNIT + SCR_V + voff);
#pragma unroll
    for (int u = 0; u < NU; ++u) {
        const unsigned v16 = a16[u] + ((bpos[u] + 127u) >> 7);
        const char* vg = pk + (size_t)(g[u] >> 1) * PAIRB + VMID - ((g[u] & 1u) ? 0u : 16u * v16);
        if ((unsigned)lane < v16) st_wt<0>(vg, voff, vv[u]);
        for (unsigned o = 64u; o < v16; o += 64u)
            if (o + lane < v16) st_wt<0>(vg + 16u * o, voff, *(LAS const u32x4*)(scr + u * SCR_UNIT + SCR_V + 16u * o + voff));
        info[u] |= v16 << 8;
    }
    LDS_FENCE();
}

__device__ __forceinline__ void unpack2(const u32x4 (&p)[2][2], const u32x2 (&cc)[2], const unsigned (&info)[2], LAS char* lds, LAS char* T,
                                        LAS char* const (&V)[2], int lane, u32x4 (&o)[2][4]) {
    u32x4 X[2];
    LAS char* A[2];
#pragma unroll
    for (int u = 0; u < 2; ++u) {
        A[u] = V[u];
        const unsigned xv = (info[u] >> 28) << (((info[u] >> 25) & 7u) * 4u), xa = (info[u] >> 23) & 3u;
        const bool me = (unsigned)lane == ((info[u] >> 17) & 63u);
#pragma unroll
        for (unsigned a = 0; a < 4; ++a) X[u][a] = me ? (xa == a ? xv : 0u) : 0u;
        if (info[u] & 0x10000u) {
            const u32x2 mv = *(LAS const u32x2*)V[u];
            const u64 em = (u64)rfl(mv[0]) | (u64)rfl(mv[1]) << 32;
            if ((em >> lane) & 1ull) X[u] = *(LAS const u32x4*)(V[u] + 16u * (1u + lane_rank(em)));
            A[u] = V[u] + 16u * (1u + __builtin_popcountll(em));
        }
    }
    unsigned z[2];
#pragma unroll
    for (int u = 0; u < 2; ++u) z[u] = *(LAS const unsigned short*)(A[u] + lane * 2);
    unsigned C[2][4], F[2][4], cnt = 0;
    u32x2 sel[2][4];
#pragma unroll
    for (int u = 0; u < 2; ++u) {
        unsigned c = 0;
#pragma unroll
        for (int a = 0; a < 4; ++a) {
            C[u][a] = (cc[u][a >> 1] >> (2 * (a & 1))) & 0x33333333u;
            F[u][a] = ~(C[u][a] | (C[u][a] >> 1)) & 0x11111111u;
            c += __builtin_popcount(F[u][a]);
            sel[u][a] = lut_at(lds, F[u][a]);
        }
        cnt |= c << (16 * u);
    }
    const unsigned incl = wave_scan(cnt), tot = __builtin_amdgcn_readlane(incl, 63);
    const unsigned nmax = max(tot & 0xffffu, tot >> 16);
    unsigned bpos[2] = {(((tot & 0xffffu) + 15u) >> 4) << 5, (((tot >> 16) + 15u) >> 4) << 5};
    for (unsigned t = 0;;) {
        unsigned c2[2], f2[2], c = 0;
        u32x2 s2[2];
#pragma unroll
        for (int u = 0; u < 2; ++u) {
            c2[u] = (z[u] | (z[u] << 15)) & 0x55555555u;
            f2[u] = c2[u] & ~(c2[u] >> 2) & 0x11111111u;
            s2[u] = lut_at(lds, f2[u]);
            c |= (unsigned)__builtin_popcount(f2[u]) << (16 * u);
        }
        const unsigned in2 = wave_scan(c);
        unsigned wv[2];
#pragma unroll
        for (int u = 0; u < 2; ++u) wv[u] = get_bits(A[u], bpos[u] + (((in2 - c) >> (16 * u)) & 0xffffu) * 4u);
#pragma unroll
        for (int u = 0; u < 2; ++u) *(LAS unsigned*)(T + u * SCR_V + (t >> 1) + lane * 4) = (c2[u] ^ f2[u]) | shuffle8(wv[u], s2[u]);
        t += 512u;
        if (t >= nmax) break;
        const unsigned t2 = __builtin_amdgcn_readlane(in2, 63);
#pragma unroll
        for (int u = 0; u < 2; ++u) {
            bpos[u] += ((t2 >> (16 * u)) & 0xffffu) * 4u;
            z[u] = *(LAS const unsigned short*)(A[u] + (t >> 2) + lane * 2);
        }
    }
    LDS_FENCE();
    unsigned w[2][4];
#pragma unroll
    for (int u = 0; u < 2; ++u) {
        unsigned pos = (((incl - cnt) >> (16 * u)) & 0xffffu) * 4u;
#pragma unroll
        for (int a = 0; a < 4; ++a) {
            w[u][a] = get_bits(T + u * SCR_V, pos);
            pos += 4u * __builtin_popcount(F[u][a]);
        }
    }
    LDS_FENCE();
#pragma unroll
    for (int u = 0; u < 2; ++u) {
        const unsigned top4 = (info[u] & 0xffu) * 0x01010101u;
#pragma unroll
        for (int a = 0; a < 4; ++a) {
            const unsigned n = C[u][a] | shuffle8(w[u][a], sel[u][a]);
            const unsigned e0 = top4 - bfi(0x0f0f0f0fu, n, X[u][a]), e1 = top4 - rotr4(bfi(0x0f0f0f0fu, X[u][a], n));
            const unsigned q0 = perm(e0, p[u][0][a], 0x05010400u), q1 = perm(e0, p[u][0][a], 0x07030602u);
            const unsigned q2 = perm(e1, p[u][1][a], 0x05010400u), q3 = perm(e1, p[u][1][a], 0x07030602u);
            o[u][0][a] = __builtin_amdgcn_alignbit(q0, q0, 1); o[u][1][a] = __builtin_amdgcn_alignbit(q1, q1, 1);
            o[u][2][a] = __builtin_amdgcn_alignbit(q2, q2, 1); o[u][3][a] = __builtin_amdgcn_alignbit(q3, q3, 1);
        }
    }
}
__device__ __forceinline__ void store_stage(const char* dst, unsigned voff, const u32x4 (&o)[4]) {
    st_wt<0>(dst, voff, o[0]); st_wt<1024>(dst, voff, o[1]); st_wt<2048>(dst, voff, o[2]); st_wt<3072>(dst, voff, o[3]);
}
__device__ __forceinline__ void store_out(const char* dst, unsigned voff, const u32x4 (&o)[4]) {
    st_out<0>(dst, voff, o[0]); st_out<64>(dst, voff, o[1]);
    st_out<0>(dst + 16 * 65536, voff, o[2]); st_out<64>(dst + 16 * 65536, voff, o[3]);
}

__device__ __forceinline__ void pack_piece(const Params& P, char* ws, const char* px, const char* pw, LAS char* lds, int round, unsigned e,
                                           unsigned cntv, int lane, int wid) {
    LAS unsigned* lrec = (LAS unsigned*)(lds + LDS_REC) + (round & 1) * 16;
    const int li = lane & 15, lg = lane >> 4;
    unsigned* ready = (unsigned*)(ws + OFF_READY);
    const int h = (e >> 3) & 15, idx = (e & 7) * 8 + wid, id = h * 64 + idx;
    const bool isx = (e >> 7) == 0u;
    const unsigned dest = ((e >> 7) - 1u) & 7u;
    const bool own = !isx && dest == (unsigned)P.rank;
    const unsigned rdest = isx ? (unsigned)lane >> 3 : dest;
    char* peer = P.ws[rdest];
    u32x4 dd[1][4];
    u32x4 (&d)[4] = dd[0];
    u32x2 v[8];
    if (isx) {
        const char* src = px + (size_t)(64 * idx + li) * 1024 + h * 64 + lg * 16;
#pragma unroll
        for (int c = 0; c < 4; ++c) {
            const u32x4 t = *(const u32x4*)(src + 16 * c * 1024);
            v[2 * c] = (u32x2){t[0], t[1]}; v[2 * c + 1] = (u32x2){t[2], t[3]};
        }
    } else {
        const int cp = 32 * ((li >> 1) & 1) + 8 * (li >> 2) + 4 * (li & 1);
        const char* src = pw + (size_t)(32 * h + 8 * lg) * 65536 + (size_t)(dest * 4096 + 64 * idx + cp) * 2;
#pragma unroll
        for (int j = 0; j < 8; ++j) v[j] = *(const u32x2*)(src + (size_t)j * 65536);
    }
    __builtin_amdgcn_sched_barrier(0);
    if (round == 0) {
        build_lut(lds, false, threadIdx.x);
        asm volatile("s_waitcnt lgkmcnt(0)" ::: "memory");
        __builtin_amdgcn_s_barrier();
        asm volatile("" ::: "memory");
    }
    if (isx) {
#pragma unroll
        for (int c = 0; c < 4; ++c) d[c] = (u32x4){v[2 * c][0], v[2 * c][1], v[2 * c + 1][0], v[2 * c + 1][1]};
    } else {
#pragma unroll
        for (int hf = 0; hf < 2; ++hf)
#pragma unroll
            for (int a = 0; a < 4; ++a) {
                d[2 * hf][a] = perm(v[2 * a + 1][hf], v[2 * a][hf], 0x05040100u);
                d[2 * hf + 1][a] = perm(v[2 * a + 1][hf], v[2 * a][hf], 0x07060302u);
            }
    }
    asm volatile("" :: "v"(peer));
    const unsigned call = rfl(cntv) + 1u, par = call & 1u;
    if (!own) {
        const unsigned g[1] = {(unsigned)(isx ? id : 1024 + (int)dest * 1024 + id)};
        unsigned r[1];
        pack_store<1>(dd, ws + OFF_PK + par * PK_PAR, g, lds, lds + wid * SCR_WAVE, lane, r, [] {});
        if (lane == 0) lrec[wid] = r[0];
    }
    if (isx || own) store_stage(ws + (isx ? OFF_A : OFF_B) + (size_t)(h * 256 + 4 * idx) * 1024, lane * 16, d);
    wait_vm0();
    if ((isx || own) && lane == 0)
        __hip_atomic_store(ready + ((isx ? 0 : 64) + idx) * NSLOT + h, call, __ATOMIC_RELAXED, __HIP_MEMORY_SCOPE_SYSTEM);
    asm volatile("s_waitcnt lgkmcnt(0)" ::: "memory");
    __builtin_amdgcn_s_barrier();
    asm volatile("" ::: "memory");
    if (wid == 0 && !own && (isx || lane < 8) && rdest != (unsigned)P.rank) {
        u64* rp = (u64*)(peer + OFF_REC + par * REC_PAR) + (P.rank * 3 + (isx ? 0 : 1)) * 1024 + h * 64 + (e & 7) * 8 + (lane & 7);
        __hip_atomic_store(rp, ((u64)call << 32) | lrec[lane & 7], __ATOMIC_RELAXED, __HIP_MEMORY_SCOPE_SYSTEM);
    }
}

__device__ __forceinline__ void pack_two(const Params& P, char* ws, const char* pw, LAS char* lds, int round, unsigned ew, unsigned call,
                                         int lane, int wid) {
    LAS unsigned* lrec = (LAS unsigned*)(lds + LDS_REC) + (round & 1) * 16;
    const int li = lane & 15, lg = lane >> 4;
    const unsigned par = call & 1u;
    const unsigned el = (lane & 8) ? ew >> 16 : ew & 0xffffu;
    char* peer = P.ws[((el >> 7) - 1u) & 7u];
    const int cp = 32 * ((li >> 1) & 1) + 8 * (li >> 2) + 4 * (li & 1);
    u32x2 v[2][8];
    unsigned g[2];
#pragma unroll
    for (int u = 0; u < 2; ++u) {
        const unsigned e = (ew >> (16 * u)) & 0xffffu, h = (e >> 3) & 15u, idx = (e & 7u) * 8u + wid, dest = ((e >> 7) - 1u) & 7u;
        g[u] = 1024u + dest * 1024u + h * 64u + idx;
        const char* src = pw + (size_t)(32 * h + 8 * lg) * 65536 + (size_t)(dest * 4096 + 64 * idx + cp) * 2;
#pragma unroll
        for (int j = 0; j < 8; ++j) v[u][j] = *(const u32x2*)(src + (size_t)j * 65536);
    }
    __builtin_amdgcn_sched_barrier(0);
    u32x4 d[2][4];
#pragma unroll
    for (int u = 0; u < 2; ++u)
#pragma unroll
        for (int hf = 0; hf < 2; ++hf)
#pragma unroll
            for (int a = 0; a < 4; ++a) {
                d[u][2 * hf][a] = perm(v[u][2 * a + 1][hf], v[u][2 * a][hf], 0x05040100u);
                d[u][2 * hf + 1][a] = perm(v[u][2 * a + 1][hf], v[u][2 * a][hf], 0x07060302u);
            }
    asm volatile("" :: "v"(peer));
    __builtin_amdgcn_sched_barrier(0);
    unsigned r[2];
    pack_store<2>(d, ws + OFF_PK + par * PK_PAR, g, lds, lds + wid * SCR_WAVE, lane, r, [] {});
    if (lane < 2) lrec[lane * 8 + wid] = lane ? r[1] : r[0];
    wait_vm0();
    asm volatile("s_waitcnt lgkmcnt(0)" ::: "memory");
    __builtin_amdgcn_s_barrier();
    asm volatile("" ::: "memory");
    if (wid == 0 && lane < 16) {
        u64* rp = (u64*)(peer + OFF_REC + par * REC_PAR) + (P.rank * 3 + 1) * 1024 + (el & 127u) * 8u + (lane & 7);
        __hip_atomic_store(rp, ((u64)call << 32) | lrec[lane], __ATOMIC_RELAXED, __HIP_MEMORY_SCOPE_SYSTEM);
    }
}

struct Pulled { u32x4 a0, a1, b0, b1, v, rec; u32x2 ac, bc; unsigned tk; };
__device__ __forceinline__ void pull_pair(Pulled& R, const char* pk, u64 vmask, unsigned voff, unsigned voff8, unsigned voffv, const char* ws,
                                          unsigned rec_off, unsigned tk_off, unsigned tk_add, unsigned flag_off, unsigned call) {
    asm volatile("s_nop 4\n\t"
                 "global_load_dwordx2 %6, %10, %12 offset:-512 sc0 sc1\n\t"
                 "s_mov_b64 exec, %13\n\t"
                 "global_load_dwordx4 %4, %11, %12 sc0 sc1\n\t"
                 "s_mov_b64 exec, -1\n\t"
                 "global_load_dwordx4 %0, %9, %12 offset:-2560 sc0 sc1\n\t"
                 "global_load_dwordx4 %1, %9, %12 offset:-1536 sc0 sc1\n\t"
                 "global_load_dwordx2 %7, %10, %12 offset:2048 sc0 sc1\n\t"
                 "global_load_dwordx4 %2, %9, %12 sc0 sc1\n\t"
                 "global_load_dwordx4 %3, %9, %12 offset:1024 sc0 sc1\n\t"
                 "s_mov_b64 exec, 1\n\t"
                 "global_load_dwordx4 %5, %15, %14 sc0 sc1\n\t"
                 "global_atomic_add %8, %16, %17, %14 sc0\n\t"
                 "s_mov_b64 exec, 3\n\t"
                 "s_waitcnt vmcnt(9)\n\t"
                 "global_store_dword %18, %19, %14 sc0 sc1\n\t"
                 "s_mov_b64 exec, -1\n\t"
                 "s_waitcnt vmcnt(1)"
                 : "=&v"(R.a0), "=&v"(R.a1), "=&v"(R.b0), "=&v"(R.b1), "=&v"(R.v), "=&v"(R.rec), "=&v"(R.ac), "=&v"(R.bc), "=&v"(R.tk)
                 : "v"(voff), "v"(voff8), "v"(voffv), "s"(pk), "s"(vmask), "s"(ws), "v"(rec_off), "v"(tk_off), "v"(tk_add),
                   "v"(flag_off), "v"(call)
                 : "memory");
}

struct Desc { unsigned g, dst, rdy, rec; };

__device__ __forceinline__ void pull_link(const Params& P, LAS char* lds, int q, int w, unsigned call, int lane, int wid) {
    char* ws = P.me;
    const unsigned par = call & 1u;
    const unsigned p = (unsigned)(P.rank + 1 + q) & 7u;
    const char* peer = P.ws[p];
    unsigned* tmo = (unsigned*)(ws + OFF_TMO);
    unsigned vorder = P.order[lane];
    asm volatile("" : "+v"(vorder));
    const unsigned rec_base = (unsigned)(OFF_REC + par * REC_PAR) + p * 3072u * 8u;
    const unsigned dummy = (unsigned)OFF_DUMMY + (unsigned)(q * PW + w) * 128u;
    auto describe = [&](unsigned n) {
        Desc d;
        if (n < 1024u) {
            const unsigned e = ((unsigned)__builtin_amdgcn_readlane(vorder, n >> 4) >> ((n & 12u) * 2u)) & 255u;
            const unsigned type = e >> 7, h = (e >> 3) & 15u, idx = (e & 7u) * 8u + (n & 3u) * 2u, id = h * 64u + idx, slot = 16u + 7u * h + q;
            d.g = type ? 1024u + P.rank * 1024u + id : id;
            d.dst = (unsigned)(type ? OFF_B : OFF_A) + (slot * 256u + 4u * idx) * 1024u;
            d.rdy = (unsigned)OFF_READY + ((type * 64u + idx) * NSLOT + slot) * 4u;
            d.rec = rec_base + (type * 1024u + id) * 8u;
        } else {
            const unsigned id = ((n & 0x1c0u) << 1) | ((n & 7u) << 4) | ((P.rbord >> ((n >> 1) & 28u)) & 14u);
            d.g = 9216u + P.rank * 1024u + id;
            d.dst = (id & 15u) * (32u * 65536u) + (p * 4096u + 64u * (id >> 4)) * 2u;
            d.rdy = 0;
            d.rec = rec_base + (2048u + id) * 8u;
        }
        return d;
    };
    const char* pkb = peer + OFF_PK + par * PK_PAR + FIXB;
    const unsigned voff = lane * 16, voff_out = (lane & 15) * 65536 + (lane >> 4) * 16;
    const unsigned tk_off = (unsigned)OFF_TKT + q * 128u;
    const unsigned tk_base = (call - 1u) * (NTK - PW) - 2u * PW;
    LAS char* T = lds + wid * SCR_WAVE;
    LAS char* M = T + 2 * SCR_V + VCAP;
    unsigned n1 = PW + w, ia = 0, ib = 0, pend = 0;
    bool ok = false, dead = false;
    Desc d0 = describe(w), d1 = describe(n1);
    for (;;) {
        if (!ok) {
            if (pend) {
                wait_vm0();
                if (lane < 2) __hip_atomic_store((unsigned*)(ws + pend) + lane * NSLOT, call, __ATOMIC_RELAXED, __HIP_MEMORY_SCOPE_SYSTEM);
                pend = 0;
            }
            const u64 t0 = __builtin_amdgcn_s_memrealtime();
            unsigned spins = 0;
            ia = ib = 0;
            while (!dead) {
                const u64 v = __hip_atomic_load((const u64*)(ws + d0.rec) + (lane & 1), __ATOMIC_RELAXED, __HIP_MEMORY_SCOPE_SYSTEM);
                if (__builtin_amdgcn_ballot_w64((unsigned)(v >> 32) != call) == 0ull) {
                    ia = __builtin_amdgcn_readlane((unsigned)v, 0); ib = __builtin_amdgcn_readlane((unsigned)v, 1);
                    break;
                }
                if (timed_out(tmo, call, t0, spins)) dead = true;
                __builtin_amdgcn_s_sleep(1);
            }
        }
        const unsigned la = min((ia >> 8) & 255u, (unsigned)VCAP / 16u), lb = min((ib >> 8) & 255u, (unsigned)VCAP / 16u);
        const unsigned mlo = la >= 32u ? ~0u : ~(~0u >> la), mhi = lb >= 32u ? ~0u : ((1u << lb) - 1u) | 1u;
        const char* pk = pkb + (size_t)(d0.g >> 1) * PAIRB;
        Pulled R;
        pull_pair(R, pk, (u64)mlo | (u64)mhi << 32, voff, lane * 8, voff + (VMID - FIXB - 512), ws, d1.rec, tk_off, n1 < NTK ? 1u : 0u,
                  pend ? pend + lane * (NSLOT * 4) : dummy, call);
        *(LAS u32x4*)(M - 512 + voff) = R.v;
        if (la > 32u || lb > 32u) {
            for (int c = -3; c <= 3; ++c) {
                const int off = c * 1024 - 512 + (int)voff;
                if (c != 0 && off >= -16 * (int)la && off < 16 * (int)lb) {
                    u32x4 t;
                    asm volatile("global_load_dwordx4 %0, %1, off sc0 sc1\n\ts_waitcnt vmcnt(0)" : "=&v"(t) : "v"(pk + (VMID - FIXB) + off) : "memory");
                    *(LAS u32x4*)(M + off) = t;
                }
            }
        }
        LDS_FENCE();
        u32x4 o[2][4];
        {
            const u32x4 pp[2][2] = {{R.a0, R.a1}, {R.b0, R.b1}};
            const u32x2 cc[2] = {R.ac, R.bc};
            const unsigned inf[2] = {ia, ib};
            LAS char* const vv[2] = {M - 16u * la, M};
            unpack2(pp, cc, inf, lds, T, vv, lane, o);
        }
        if (d0.rdy) {
            store_stage(ws + d0.dst, voff, o[0]);
            store_stage(ws + d0.dst + 4096, voff, o[1]);
        } else {
            if (dead) {
#pragma unroll
                for (int c = 0; c < 8; ++c) o[c >> 2][c & 3] = (u32x4){0x7fc07fc0u, 0x7fc07fc0u, 0x7fc07fc0u, 0x7fc07fc0u};
            }
            store_out(P.out + d0.dst, voff_out, o[0]);
            store_out(P.out + d0.dst + 32 * 65536, voff_out, o[1]);
        }
        pend = d0.rdy;
        if (n1 >= NTK) break;
        ia = rfl(R.rec[0]); ib = rfl(R.rec[2]);
        ok = rfl(R.rec[1]) == call && rfl(R.rec[3]) == call;
        d0 = d1;
        n1 = rfl(R.tk) - tk_base;
        d1 = describe(n1);
    }
    if (pend) {
        wait_vm0();
        if (lane < 2) __hip_atomic_store((unsigned*)(ws + pend) + lane * NSLOT, call, __ATOMIC_RELAXED, __HIP_MEMORY_SCOPE_SYSTEM);
    }
}

__device__ __forceinline__ void glds4(unsigned m0v, unsigned voff, const char* sb) {
    asm volatile("s_mov_b32 m0, %0\n\ts_nop 0\n\t"
                 "global_load_lds_dwordx4 %1, %2 sc0 sc1\n\t"
                 "global_load_lds_dwordx4 %1, %2 offset:1024 sc0 sc1\n\t"
                 "global_load_lds_dwordx4 %1, %2 offset:2048 sc0 sc1\n\t"
                 "global_load_lds_dwordx4 %1, %2 offset:3072 sc0 sc1" :: "s"(m0v), "v"(voff), "s"(sb) : "memory");
}
__device__ __forceinline__ void glds1(unsigned m0v, unsigned voff, const char* sb) {
    asm volatile("s_mov_b32 m0, %0\n\ts_nop 0\n\tglobal_load_lds_dwordx4 %1, %2 sc0 sc1" :: "s"(m0v), "v"(voff), "s"(sb) : "memory");
}

__device__ __forceinline__ void gemm_tile(const Params& P, LAS char* lds, int pm, int pn, unsigned call, int lane, int wid) {
    char* ws = P.me;
    unsigned* tmo = (unsigned*)(ws + OFF_TMO);
    const unsigned* ready = (const unsigned*)(ws + OFF_READY);
    const int wr = wid >> 2, wc = wid & 3;
    const unsigned ldsbase = (unsigned)(size_t)lds;
    const char* g1; const char* g2; unsigned l1, l2; const unsigned* r1; const unsigned* r2;
    if (wr == 0) {
        const int u1 = 5 * pm + wc; const bool has5 = pm < 12;
        g1 = ws + OFF_A + (size_t)u1 * 4096; g2 = has5 ? ws + OFF_A + (size_t)(20 * pm + 16 + wc) * 1024 : g1;
        l1 = ldsbase + wc * 4096; l2 = ldsbase + (16 + wc) * 1024;
        r1 = ready + u1 * NSLOT; r2 = has5 ? ready + (5 * pm + 4) * NSLOT : r1;
    } else {
        const int u1 = 4 * pn + wc;
        g1 = ws + OFF_B + (size_t)u1 * 4096; g2 = g1; l1 = ldsbase + 20480 + wc * 4096; l2 = l1;
        r1 = ready + (64 + u1) * NSLOT; r2 = r1;
    }
    const unsigned voff = lane * 16;
    u64 mask = 0;
#define ENSURE(t) do { if (((t) & 63) == 0) mask = 0; \
    if (!((mask >> ((t) & 63)) & 1ull)) { const u64 t0_ = __builtin_amdgcn_s_memrealtime(); unsigned sp_ = 0; \
        for (;;) { \
            const unsigned a_ = __hip_atomic_load(r1 + ((t) & 64) + lane, __ATOMIC_RELAXED, __HIP_MEMORY_SCOPE_SYSTEM); \
            const unsigned b_ = __hip_atomic_load(r2 + ((t) & 64) + lane, __ATOMIC_RELAXED, __HIP_MEMORY_SCOPE_SYSTEM); \
            mask = __builtin_amdgcn_ballot_w64(a_ >= call && b_ >= call); \
            if ((mask >> ((t) & 63)) & 1ull) break; \
            if (timed_out(tmo, call, t0_, sp_)) { mask = ~0ull; break; } \
            __builtin_amdgcn_s_sleep(2); } } } while (0)
#define STAGE(t, bt) do { const size_t go_ = (size_t)(t) * 262144; const unsigned lo_ = (bt) * LSLOT; \
    glds4(l1 + lo_, voff, g1 + go_); if (wr == 0) glds1(l2 + lo_, voff, g2 + go_); } while (0)
#define WAITV() do { if (wr == 0) asm volatile("s_waitcnt vmcnt(10)" ::: "memory"); else asm volatile("s_waitcnt vmcnt(8)" ::: "memory"); } while (0)
    f32x4 acc[10][4];
#pragma unroll
    for (int m = 0; m < 10; ++m)
#pragma unroll
        for (int n = 0; n < 4; ++n) acc[m][n] = (f32x4){0.f, 0.f, 0.f, 0.f};
    for (int t = 0; t < 3; ++t) { ENSURE(t); STAGE(t, t); }
    WAITV();
    __builtin_amdgcn_s_barrier();
    STAMP(ws, 2);
    if (wr == 1) __builtin_amdgcn_s_barrier();
    const unsigned aoff = wr * 10240 + lane * 16, boff = 20480 + wc * 4096 + lane * 16;
    for (int s = 0; s < NSLOT; ++s) {
        const unsigned so = (s & 3) * LSLOT;
        bf16x8 A[10], B[4];
        LAS const char* pa = lds + aoff + so; LAS const char* pb = lds + boff + so;
#pragma unroll
        for (int n = 0; n < 4; ++n) B[n] = *(LAS const bf16x8*)(pb + n * 1024);
#pragma unroll
        for (int m = 0; m < 10; ++m) A[m] = *(LAS const bf16x8*)(pa + m * 1024);
        __builtin_amdgcn_sched_barrier(0);
        const int t = s + 3 < NSLOT ? s + 3 : NSLOT - 1;
        if (s + 3 < NSLOT) ENSURE(t);
        STAGE(t, (s + 3) & 3);
        WAITV();
        asm volatile("s_waitcnt lgkmcnt(0)" ::: "memory");
        __builtin_amdgcn_sched_barrier(0);
        __builtin_amdgcn_s_barrier();
        __builtin_amdgcn_sched_barrier(0);
        __builtin_amdgcn_s_setprio(1);
#pragma unroll
        for (int m = 0; m < 10; ++m)
#pragma unroll
            for (int n = 0; n < 4; ++n) acc[m][n] = __builtin_amdgcn_mfma_f32_16x16x32_bf16(B[n], A[m], acc[m][n], 0, 0, 0);
        __builtin_amdgcn_s_setprio(0);
        __builtin_amdgcn_sched_barrier(0);
        __builtin_amdgcn_s_barrier();
        __builtin_amdgcn_sched_barrier(0);
    }
    if (wr == 0) __builtin_amdgcn_s_barrier();
#undef ENSURE
#undef STAGE
#undef WAITV
    wait_vm0();
    __builtin_amdgcn_s_barrier();
    asm volatile("" ::: "memory");
    STAMP(ws, 3);
    const bool bad = rfl(__hip_atomic_load(tmo, __ATOMIC_RELAXED, __HIP_MEMORY_SCOPE_AGENT)) == call;
    const unsigned par = call & 1u;
    const int cb = 4 * pn + wc, rb0 = 10 * pm + 5 * wr;
    const unsigned voff_out = (lane & 15) * 65536 + (lane >> 4) * 16;
    u32x4 d[5][4];
#pragma unroll
    for (int u = 0; u < 5; ++u)
#pragma unroll
        for (int c = 0; c < 4; ++c) {
            const int mi = 2 * u + (c >> 1), b = c & 1;
            d[u][c][0] = cvt_pk(acc[mi][0][2 * b], acc[mi][1][2 * b]);
            d[u][c][1] = cvt_pk(acc[mi][2][2 * b], acc[mi][3][2 * b]);
            d[u][c][2] = cvt_pk(acc[mi][0][2 * b + 1], acc[mi][1][2 * b + 1]);
            d[u][c][3] = cvt_pk(acc[mi][2][2 * b + 1], acc[mi][3][2 * b + 1]);
        }
    if (wr) {
#pragma unroll
        for (int c = 0; c < 4; ++c) {
            const u32x4 t = d[0][c];
            d[0][c] = d[1][c]; d[1][c] = d[2][c]; d[2][c] = d[3][c]; d[3][c] = d[4][c]; d[4][c] = t;
        }
    }
    if (bad) {
#pragma unroll
        for (int c = 0; c < 20; ++c) d[c >> 2][c & 3] = (u32x4){0x7fc07fc0u, 0x7fc07fc0u, 0x7fc07fc0u, 0x7fc07fc0u};
    }
    char* pk = ws + OFF_PK + par * PK_PAR;
    LAS char* scr = lds + wid * SCR_WAVE;
    u64* prec = nullptr;
    unsigned pval = 0, pcnt = 0;
    auto post = [&] {
        if ((unsigned)lane < pcnt) __hip_atomic_store(prec + lane, ((u64)call << 32) | pval, __ATOMIC_RELAXED, __HIP_MEMORY_SCOPE_SYSTEM);
        pcnt = 0;
    };
#pragma unroll
    for (int k = 0; k < 5; k += 2) {
        const unsigned rb = k < 4 ? rb0 + wr + k : rb0 + 4 * (1 - wr), dest = rb >> 4;
        if (dest == (unsigned)P.rank) {
            char* dst = P.out + (size_t)(rb & 15u) * (32 * 65536) + (size_t)(P.rank * 4096 + 64 * cb) * 2;
            store_out(dst, voff_out, d[k]);
            if (k < 4) store_out(dst + 32 * 65536, voff_out, d[k + 1 < 5 ? k + 1 : k]);
        } else if (dest < (unsigned)NR) {
            const unsigned g0 = 9216u + dest * 1024u + cb * 16u + (rb & 15u);
            if (k < 4) {
                const u32x4 dd[2][4] = {{d[k][0], d[k][1], d[k][2], d[k][3]},
                                        {d[k + 1 < 5 ? k + 1 : k][0], d[k + 1 < 5 ? k + 1 : k][1], d[k + 1 < 5 ? k + 1 : k][2], d[k + 1 < 5 ? k + 1 : k][3]}};
                const unsigned g[2] = {g0, g0 + 1u};
                unsigned r[2];
                pack_store<2>(dd, pk, g, lds, scr, lane, r, [&] { if (pcnt) { asm volatile("s_waitcnt vmcnt(6)" ::: "memory"); post(); } });
                pval = lane ? r[1] : r[0]; pcnt = 2;
            } else {
                const u32x4 dd[1][4] = {{d[k][0], d[k][1], d[k][2], d[k][3]}};
                const unsigned g[1] = {g0};
                unsigned r[1];
                pack_store<1>(dd, pk, g, lds, scr, lane, r, [&] { if (pcnt) { asm volatile("s_waitcnt vmcnt(3)" ::: "memory"); post(); } });
                pval = r[0]; pcnt = 1;
            }
            prec = (u64*)(P.ws[dest] + OFF_REC + par * REC_PAR) + (P.rank * 3 + 2) * 1024 + cb * 16 + (rb & 15u);
        }
    }
    wait_vm0();
    post();
}

#define PK_R1 52
#define PK_R2 104
constexpr int NPACKERS = 208 + NCOMM - 7 * PWG;
static_assert(PK_R1 % 2 == 0 && NPACKERS - PK_R1 >= 128 && PK_R2 >= PK_R1, "pack rounds");

__host__ __device__ __forceinline__ void tile_of(int xcd, int jj, int& pm, int& pn) {
    if (COLMAP == 0) { pm = jj % 13; pn = 2 * xcd + jj / 13; }
    else if (COLMAP == 1) { pm = jj % 13; pn = xcd + 8 * (jj / 13); }
    else if (COLMAP == 2) { pm = jj % 13; pn = jj < 13 ? xcd : 15 - xcd; }
    else if (COLMAP == 3) {
        pm = jj >> 1;
        const int first = (pm < 7) == ((xcd & 1) == 0);
        pn = 2 * (first ? xcd >> 1 : 7 - (xcd >> 1)) + (jj & 1);
    } else {
        pm = jj >> 1;
        const int qt = pm < 4 ? 0 : (pm - 1) / 3;
        pn = 2 * ((xcd & 1) + 2 * ((qt + (xcd >> 1)) & 3)) + (jj & 1);
    }
}

__global__ void __launch_bounds__(512, 2) fused_gemm_rs(const Params P) {
    __shared__ __attribute__((aligned(16))) char smem[LDS_BYTES];
    const int bid = blockIdx.x, xcd = bid & 7, j = bid >> 3;
    const int lane = threadIdx.x & 63, wid = rfl(threadIdx.x >> 6);
    const bool comm = j < NCOMM / 8;
    const int jj = j - NCOMM / 8;
    int pm, pn;
    tile_of(xcd, comm ? 0 : jj, pm, pn);
    const int o = comm ? 208 + bid - 7 * PWG : pn * 13 + pm;
    char* ws = P.me; const char* px = P.x; const char* pw = P.w; unsigned ew = P.plist[o >> 1];
    asm volatile("" :: "s"(ws), "s"(px), "s"(pw), "s"(ew), "s"(P.rank));
    STAMP(ws, 0);
    unsigned* cnt = (unsigned*)(ws + OFF_CNT) + bid * 8 + wid;
    const unsigned cntv = __hip_atomic_load(cnt, __ATOMIC_RELAXED, __HIP_MEMORY_SCOPE_AGENT);
    if (bid >= 7 * PWG) {
        int piece = o;
        for (int r = 0; r < 2; ++r) {
            pack_piece(P, ws, px, pw, (LAS char*)smem, r, (ew >> ((piece & 1) * 16)) & 0xffffu, cntv, lane, wid);
            if (o < PK_R1) break;
            piece = NPACKERS + o - PK_R1;
            ew = P.plist[piece >> 1];
        }
        if (o >= PK_R2) {
            int r = 0;
            for (int i = (2 * NPACKERS - PK_R1) / 2 + o - PK_R2; i < NPIECE / 2; i += NPACKERS - PK_R2)
                pack_two(P, ws, pw, (LAS char*)smem, r++, P.plist[i], rfl(cntv) + 1u, lane, wid);
        }
        asm volatile("s_waitcnt lgkmcnt(0)" ::: "memory");
        __builtin_amdgcn_s_barrier();
    } else {
        build_lut((LAS char*)smem, true, threadIdx.x);
        asm volatile("s_waitcnt lgkmcnt(0)" ::: "memory");
        __builtin_amdgcn_s_barrier();
        asm volatile("" ::: "memory");
    }
    const unsigned call = rfl(cntv) + 1u;
    if (lane == 0) __hip_atomic_store(cnt, call, __ATOMIC_RELAXED, __HIP_MEMORY_SCOPE_AGENT);
    STAMP(ws, 1);
    if (comm) {
        if (bid < 7 * PWG && wid < PW / PWG) pull_link(P, (LAS char*)smem, bid % 7, (bid / 7) * (PW / PWG) + wid, call, lane, wid);
    } else {
        gemm_tile(P, (LAS char*)smem, pm, pn, call, lane, wid);
    }
    STAMP(ws, 4);
}

}

#define LAG_NUM 25

extern "C" void kernel_launch(int rank, int nranks, ncclComm_t comm, hipStream_t stream,
                              void* const* d_in, const int* in_sizes, int n_in,
                              void* d_out, int out_size, const int* in_dtypes, int out_dtype,
                              void* const* d_out_all, void* const* d_ws_all, size_t ws_bytes) {
    (void)comm; (void)in_dtypes; (void)out_dtype; (void)d_out_all;
    if (nranks != NR || (unsigned)rank >= (unsigned)NR || n_in < 2 || ws_bytes < WS_NEED || in_sizes[0] != 4096 * 512 ||
        in_sizes[1] != 512 * 32768 || out_size != 512 * 32768) {
        fprintf(stderr, "kernel_launch: unexpected problem shape\n");
        return;
    }
    Params P;
    P.x = (const char*)d_in[0];
    P.w = (const char*)d_in[1];
    P.out = (char*)d_out;
    for (int p = 0; p < NR; ++p) P.ws[p] = (char*)d_ws_all[p];
    P.me = P.ws[rank];
    P.rank = rank;
    P.rbord = 0;
    for (int cls = 0, k = 0; cls < 3; ++cls)
        for (int rp = 0; rp < 8; ++rp) {
            const int m = (16 * rank + 2 * rp) % 10;
            if ((m == 0 || m == 6 ? 0 : m == 4 ? 2 : 1) == cls) P.rbord |= (unsigned)(2 * rp) << (4 * k++);
        }
    unsigned char* order = (unsigned char*)P.order;
    unsigned char worder[128];
    int no = 0, nw = 0;
    for (int ph = 0; ph < 64; ++ph) {
        if (ph < 16)
            for (int q = 0; q < 8; ++q) order[no++] = (unsigned char)(ph << 3 | q);
        for (int pr = 0; pr < 8; ++pr) {
            const int h = ph - pr * LAG_NUM / 16;
            if (h >= 0 && h < 16) {
                order[no++] = (unsigned char)(128 | h << 3 | pr);
                worder[nw++] = (unsigned char)(h << 3 | pr);
            }
        }
    }
    unsigned short* plist = (unsigned short*)P.plist;
    int np = 0, nr = 0;
    for (int r = 0; r < 3; ++r) {
        if (r < 2)
            for (int i = 0; i < 64; ++i) {
                plist[np++] = (unsigned short)(r * 64 + i);
                plist[np++] = (unsigned short)((rank + 1) << 7 | (r * 64 + i));
            }
        const int end = r == 0 ? NPACKERS : r == 1 ? 2 * NPACKERS - PK_R1 : NPIECE;
        for (; np < end; ++nr) plist[np++] = (unsigned short)((((rank + 1 + nr % 7) & 7) + 1) << 7 | worder[nr / 7]);
    }
    hipLaunchKernelGGL(fused_gemm_rs, dim3(256), dim3(512), 0, stream, P);
    hipError_t e = hipGetLastError();
    if (e != hipSuccess) fprintf(stderr, "launch failed: %s\n", hipGetErrorString(e));
}
```

```cpp
#include <hip/hip_runtime.h>
#include <rccl/rccl.h>
#include <cstdio>

#define LAS __attribute__((address_space(3)))
typedef short bf16x8 __attribute__((ext_vector_type(8)));
typedef float f32x4 __attribute__((ext_vector_type(4)));
typedef unsigned u32x4 __attribute__((ext_vector_type(4)));
typedef unsigned u32x2 __attribute__((ext_vector_type(2)));
typedef unsigned long long u64;

namespace {

constexpr int NR = 8;
constexpr int NSLOT = 128;
constexpr int LSLOT = 36 * 1024;
constexpr int LDS_LUT = 4 * LSLOT;
constexpr int LDS_REC = LDS_LUT + 2048;
constexpr int LDS_FLG = LDS_REC + 512;
constexpr int LDS_BYTES = LDS_FLG + 8192;
constexpr int SCR_UNIT = 1408;
constexpr int SCR_WAVE = 5 * SCR_UNIT;
constexpr int SCR_T = 1152;
#define PWG 6
#define NDEC 7
#define LOOKAHEAD 14u
#define FLAG_MASK 3
#define NFLY 10
#define NSMALL 1
constexpr int PW = PWG * NDEC;
constexpr unsigned NTK = 1536;
constexpr u64 DEADLINE = 100000000ull;

constexpr size_t OFF_CNT = 0;
constexpr size_t OFF_TMO = 16384;
constexpr size_t OFF_TKT = 20480;
constexpr size_t OFF_READY = 32768;
constexpr size_t OFF_REC = 131072;
constexpr size_t REC_PAR = (size_t)NR * 3 * 1024 * 8;
constexpr size_t OFF_DBG = 655360;
constexpr size_t OFF_A = 1u << 20;
constexpr size_t OFF_B = OFF_A + (32u << 20);
constexpr int NUNIT = 1024 + 2 * NR * 1024;
constexpr int FIXB = 2560;
constexpr int VCAP = 2624;
constexpr int PAIRB = 2 * FIXB + 2 * VCAP;
constexpr int VMID = 2 * FIXB + VCAP;
constexpr size_t PK_PAR = (size_t)(NUNIT / 2) * PAIRB;
constexpr size_t OFF_PK = OFF_B + (32u << 20);
constexpr size_t WS_NEED = OFF_PK + 2 * PK_PAR;
static_assert(OFF_REC + 2 * REC_PAR <= OFF_DBG, "layout");

struct Params {
    const char* x;
    const char* w;
    char* out;
    char* me;
    char* ws[NR];
    int rank;
    unsigned rbord[2];
    unsigned dstart, dstride;
    unsigned pad;
    unsigned order[64];
    unsigned worder[32];
    u64 job[256];
};

#define STAMP(ws, idx) do { } while (0)

__device__ __forceinline__ unsigned rfl(unsigned v) { return __builtin_amdgcn_readfirstlane(v); }
__device__ __forceinline__ unsigned perm(unsigned hi, unsigned lo, unsigned sel) { return __builtin_amdgcn_perm(hi, lo, sel); }
__device__ __forceinline__ unsigned bfi(unsigned m, unsigned a, unsigned b) { return (a & m) | (b & ~m); }
__device__ __forceinline__ unsigned cvt_pk(float lo, float hi) { unsigned r; asm volatile("v_cvt_pk_bf16_f32 %0, %1, %2" : "=v"(r) : "v"(lo), "v"(hi)); return r; }

template <int OFF> __device__ __forceinline__ void st_wt(const char* sbase, unsigned voff, u32x4 v) {
    asm volatile("global_store_dwordx4 %0, %1, %2 offset:%3 sc0 sc1\n\ts_nop 1" :: "v"(voff), "v"(v), "s"(sbase), "i"(OFF) : "memory");
}
#define OUTBITS " sc0 sc1"
template <int OFF> __device__ __forceinline__ void st_out(const char* sbase, unsigned voff, u32x4 v) {
    asm volatile("global_store_dwordx4 %0, %1, %2 offset:%3" OUTBITS "\n\ts_nop 1" :: "v"(voff), "v"(v), "s"(sbase), "i"(OFF) : "memory");
}
__device__ __forceinline__ void wait_vm0() { asm volatile("s_waitcnt vmcnt(0)" ::: "memory"); }

__device__ __forceinline__ bool timed_out(unsigned* tmo, unsigned call, u64 t0, unsigned& spins) {
    if ((++spins & 63u) != 0u) return false;
    if (rfl(__hip_atomic_load(tmo, __ATOMIC_RELAXED, __HIP_MEMORY_SCOPE_AGENT)) == call) return true;
    if (__builtin_amdgcn_s_memrealtime() - t0 > DEADLINE) {
        __hip_atomic_store(tmo, call, __ATOMIC_RELAXED, __HIP_MEMORY_SCOPE_AGENT);
        return true;
    }
    return false;
}

#define LDS_FENCE() asm volatile("" ::: "memory")
template <int CTRL, int ROWMASK> __device__ __forceinline__ unsigned dpp0(unsigned v) {
    return (unsigned)__builtin_amdgcn_update_dpp(0, (int)v, CTRL, ROWMASK, 0xf, false);
}
__device__ __forceinline__ unsigned wave_scan(unsigned v) {
    v += dpp0<0x111, 0xf>(v); v += dpp0<0x112, 0xf>(v); v += dpp0<0x114, 0xf>(v); v += dpp0<0x118, 0xf>(v);
    v += dpp0<0x142, 0xa>(v); v += dpp0<0x143, 0xc>(v);
    return v;
}
__device__ __forceinline__ unsigned lane_rank(u64 m) {
    return __builtin_amdgcn_mbcnt_hi((unsigned)(m >> 32), __builtin_amdgcn_mbcnt_lo((unsigned)m, 0u));
}
__device__ __forceinline__ void build_lut(LAS char* lds, bool dec, int tid) {
    if (tid < 256) {
        u64 e = 0x0c0c0c0c0c0c0c0cull;
        unsigned t = 0;
#pragma unroll
        for (unsigned j = 0; j < 8; ++j)
            if ((tid >> j) & 1) {
                const unsigned slot = (t & 1u) * 4u + (t >> 1);
                const unsigned at = dec ? j : slot, val = dec ? slot : j;
                e = (e & ~(0xffull << (8 * at))) | ((u64)val << (8 * at));
                ++t;
            }
        *(LAS u64*)(lds + LDS_LUT + tid * 8) = e;
    }
}
__device__ __forceinline__ u32x2 lut_at(LAS char* lds, unsigned f) {
    return *(LAS const u32x2*)(lds + __builtin_amdgcn_udot4(f, 0x40201008u, (unsigned)LDS_LUT, false));
}
__device__ __forceinline__ unsigned shuffle8(unsigned lo, unsigned hi, u32x2 sel) {
    return bfi(0x0f0f0f0fu, perm(hi, lo, sel[0]), perm(hi, lo, sel[1]) << 4);
}
__device__ __forceinline__ unsigned half16(unsigned v, int u) { return (u & 1) ? v >> 16 : v & 0xffffu; }
__device__ __forceinline__ unsigned b_start(unsigned n1) { return n1 <= 512u ? ((n1 + 15u) >> 4) << 5 : ((n1 - 249u) >> 3) << 5; }
__device__ __forceinline__ void lds_or(LAS char* p, unsigned v) {
    __hip_atomic_fetch_or((LAS unsigned*)p, v, __ATOMIC_RELAXED, __HIP_MEMORY_SCOPE_WAVEFRONT);
}
__device__ __forceinline__ unsigned bit_addr(LAS char* p) { return (unsigned)(size_t)p * 8u; }
__device__ __forceinline__ void put_bits(unsigned pos, unsigned v) {
    const u64 sh = (u64)v << (pos & 31u);
    LAS char* p = (LAS char*)(size_t)((pos >> 3) & ~3u);
    lds_or(p, (unsigned)sh); lds_or(p + 4, (unsigned)(sh >> 32));
}
__device__ __forceinline__ unsigned get_bits(unsigned pos) {
    LAS const unsigned* p = (LAS const unsigned*)(size_t)((pos >> 3) & ~3u);
    return __builtin_amdgcn_alignbit(p[1], p[0], pos);
}

template <int NU, class Get, class Mid>
__device__ __forceinline__ void pack_store(Get get, char* pk, const unsigned (&g)[NU], LAS char* lds, LAS char* scr, int lane,
                                           unsigned (&info)[NU], Mid mid) {
    constexpr int NS = (NU + 1) / 2;
    const unsigned voff = lane * 16;
#pragma unroll
    for (int u = 0; u < NU; ++u) *(LAS u32x4*)(scr + u * SCR_UNIT + voff) = (u32x4){0u, 0u, 0u, 0u};
    unsigned D[NU][8], pc[NU][3], cnt[NS], x16[NU], top = 0, top4 = 0;
    u32x2 sel[NU][4];
#pragma unroll
    for (int u = 0; u < NU; ++u) {
        u32x4 d[4], S[2];
        unsigned E[8], C[4], c = 0;
        get(u, d);
#pragma unroll
        for (int hf = 0; hf < 2; ++hf)
#pragma unroll
            for (int a = 0; a < 4; ++a) {
                const unsigned p = __builtin_amdgcn_alignbit(d[2 * hf][a], d[2 * hf][a], 31);
                const unsigned r = __builtin_amdgcn_alignbit(d[2 * hf + 1][a], d[2 * hf + 1][a], 31);
                S[hf][a] = perm(r, p, 0x06040200u);
                E[4 * hf + a] = perm(r, p, 0x07050301u);
            }
        char* pair = pk + (size_t)(g[u] >> 1) * PAIRB;
        const char* fix = pair + (g[u] & 1u) * FIXB;
        if (u == 0) {
            unsigned sum = __builtin_amdgcn_sad_u8(E[0], 0u, __builtin_amdgcn_sad_u8(E[2], 0u, __builtin_amdgcn_sad_u8(E[5], 0u, __builtin_amdgcn_sad_u8(E[7], 0u, 0u))));
            sum = __builtin_amdgcn_readlane(wave_scan(sum), 63);
            top = min(((sum + 983u) >> 10) + 2u, 255u);
            top4 = top * 0x01010101u;
        }
        st_wt<0>(fix, voff, S[0]);
#pragma unroll
        for (int a = 0; a < 4; ++a) {
            D[u][a] = top4 - E[a]; D[u][4 + a] = top4 - E[4 + a];
            unsigned n = bfi(0x0f0f0f0fu, D[u][a], D[u][4 + a] << 4);
            asm("" : "+v"(n));
            unsigned o = n | (n >> 1);
            asm("" : "+v"(o));
            const unsigned f = (~o | (o >> 2)) & 0x11111111u;
            unsigned f3 = (f << 1) + f;
            asm("" : "+v"(f3));
            C[a] = n & 0x33333333u & ~f3;
            c += __builtin_popcount(f);
            if (a < 3) pc[u][a] = c;
            sel[u][a] = lut_at(lds, f);
            if (a == 1) st_wt<1024>(fix, voff, S[1]);
        }
        cnt[u >> 1] = (u & 1) ? cnt[u >> 1] | c << 16 : c;
        const u32x2 cc = {C[0] | (C[1] << 2), C[2] | (C[3] << 2)};
        asm volatile("global_store_dwordx2 %0, %1, %2 offset:2048 sc0 sc1" :: "v"(lane * 8), "v"(cc), "s"(fix) : "memory");
        info[u] = top; x16[u] = 0;
        const u64 em = __builtin_amdgcn_ballot_w64(((D[u][0] | D[u][1] | D[u][2] | D[u][3] | D[u][4] | D[u][5] | D[u][6] | D[u][7]) & 0xf0f0f0f0u) != 0u);
        if (__builtin_expect(em != 0ull, 0)) {
            u32x4 X;
#pragma unroll
            for (int a = 0; a < 4; ++a) X[a] = bfi(0xf0f0f0f0u, D[u][a], D[u][4 + a] >> 4);
            const unsigned l = __builtin_ctzll(em);
            const unsigned x0 = __builtin_amdgcn_readlane(X[0], l), x1 = __builtin_amdgcn_readlane(X[1], l);
            const unsigned x2 = __builtin_amdgcn_readlane(X[2], l), x3 = __builtin_amdgcn_readlane(X[3], l);
            const unsigned any = x0 | x1 | x2 | x3, nz = (x0 != 0u) + (x1 != 0u) + (x2 != 0u) + (x3 != 0u);
            const unsigned nib = __builtin_ctz(any) >> 2, val = any >> (4u * nib);
            if ((em & (em - 1ull)) == 0ull && nz == 1u && val < 16u) {
                info[u] |= (l | (x0 ? 0u : x1 ? 1u : x2 ? 2u : 3u) << 6 | nib << 8 | val << 11) << 17;
            } else {
                info[u] |= 1u << 16;
                x16[u] = 1u + __builtin_popcountll(em);
                const unsigned k = 1u + lane_rank(em);
                if (lane == 0) st_wt<0>(pair, (g[u] & 1u) ? VMID : VMID - 16, (u32x4){(unsigned)em, (unsigned)(em >> 32), 0u, 0u});
                if ((em >> lane) & 1ull) st_wt<0>(pair, (g[u] & 1u) ? VMID + 16u * k : VMID - 16 - 16u * k, X);
            }
        }
    }
    unsigned excl[NS], tot[NS];
#pragma unroll
    for (int k = 0; k < NS; ++k) excl[k] = wave_scan(cnt[k]);
#pragma unroll
    for (int k = 0; k < NS; ++k) { tot[k] = __builtin_amdgcn_readlane(excl[k], 63); excl[k] -= cnt[k]; }
#pragma unroll
    for (int u = 0; u < NU; ++u)
        if (__builtin_expect(half16(tot[u >> 1], u) > 1400u, 0) && lane < 24) *(LAS u32x4*)(scr + u * SCR_UNIT + 1024 + voff) = (u32x4){0u, 0u, 0u, 0u};
    LDS_FENCE();
#pragma unroll
    for (int u = 0; u < NU; ++u) {
        unsigned pos = half16(excl[u >> 1], u) * 4u + bit_addr(scr + u * SCR_UNIT);
        asm("" : "+v"(pos));
#pragma unroll
        for (int a = 0; a < 4; ++a) put_bits(a ? pos + pc[u][a - 1] * 4u : pos, shuffle8(D[u][a], D[u][4 + a], sel[u][a]));
    }
    LDS_FENCE();
    mid();
    unsigned w[NU], c2n[NS];
    u32x2 s2[NU];
#pragma unroll
    for (int u = 0; u < NU; ++u) w[u] = *(LAS const unsigned*)(scr + u * SCR_UNIT + lane * 4);
    LDS_FENCE();
#pragma unroll
    for (int u = 0; u < NU; ++u) {
        const unsigned e = ((w[u] >> 1) | (w[u] >> 3)) & 0x11111111u;
        const unsigned c2 = (w[u] & 0x55555555u & ~(e << 2)) | e;
        s2[u] = lut_at(lds, e);
        const unsigned c = __builtin_popcount(e);
        c2n[u >> 1] = (u & 1) ? c2n[u >> 1] | c << 16 : c;
        *(LAS unsigned short*)(scr + u * SCR_UNIT + 128 + lane * 2) = (unsigned short)(c2 | (c2 >> 15));
    }
    unsigned in2[NS];
#pragma unroll
    for (int k = 0; k < NS; ++k) in2[k] = wave_scan(c2n[k]);
    LDS_FENCE();
#pragma unroll
    for (int u = 0; u < NU; ++u)
        put_bits(bit_addr(scr + u * SCR_UNIT + 128) + b_start(half16(tot[u >> 1], u)) + half16(in2[u >> 1] - c2n[u >> 1], u) * 4u,
                 shuffle8(w[u], w[u] >> 4, s2[u]));
    LDS_FENCE();
    u32x4 vv[NU];
#pragma unroll
    for (int u = 0; u < NU; ++u) vv[u] = *(LAS const u32x4*)(scr + u * SCR_UNIT + 128 + voff);
#pragma unroll
    for (int u = 0; u < NU; ++u) {
        const unsigned c16 = (b_start(half16(tot[u >> 1], u)) + half16(__builtin_amdgcn_readlane(in2[u >> 1], 63), u) * 4u + 127u) >> 7;
        const char* vg = pk + (size_t)(g[u] >> 1) * PAIRB + VMID + ((g[u] & 1u) ? 16 * (int)x16[u] : -16 * (int)(x16[u] + c16));
        if ((unsigned)lane < c16) st_wt<0>(vg, voff, vv[u]);
        if (__builtin_expect(c16 > 64u, 0) && 64u + lane < c16) st_wt<1024>(vg, voff, *(LAS const u32x4*)(scr + u * SCR_UNIT + 1152 + voff));
        info[u] |= (x16[u] + c16) << 8;
    }
    LDS_FENCE();
}

template <class Mid>
__device__ __forceinline__ void unpack2(const u32x4 (&p)[2][2], const u32x2 (&cc)[2], const unsigned (&info)[2], unsigned la, LAS char* lds,
                                        LAS char* T, LAS char* M, int lane, u32x4 (&o)[2][4], Mid mid) {
    u32x4 X[2] = {{0u, 0u, 0u, 0u}, {0u, 0u, 0u, 0u}};
    LAS char* A[2] = {M - 16u * la, M};
    const bool anyx = ((info[0] | info[1]) >> 16) != 0u;
    if (__builtin_expect(anyx, 0)) {
#pragma unroll
        for (int u = 0; u < 2; ++u) {
            if (info[u] & 0x10000u) {
                const u32x2 mv = *(LAS const u32x2*)(u ? M : M - 16);
                const u64 em = (u64)rfl(mv[0]) | (u64)rfl(mv[1]) << 32;
                const unsigned k = 1u + lane_rank(em);
                if ((em >> lane) & 1ull) X[u] = *(LAS const u32x4*)(u ? M + 16u * k : M - 16 - 16u * k);
                if (u) A[1] = M + 16u * (1u + __builtin_popcountll(em));
            } else {
                const unsigned xv = (info[u] >> 28) << (((info[u] >> 25) & 7u) * 4u), xa = (info[u] >> 23) & 3u;
                const bool me = (unsigned)lane == ((info[u] >> 17) & 63u);
#pragma unroll
                for (unsigned a = 0; a < 4; ++a) X[u][a] = me && xa == a ? xv : 0u;
            }
        }
    }
    unsigned z[2], rr[2];
#pragma unroll
    for (int u = 0; u < 2; ++u) {
        z[u] = *(LAS const unsigned short*)(A[u] + lane * 2);
        rr[u] = *(LAS const unsigned*)(A[u] + 128 + lane * 4);
    }
    unsigned pc[2][3], cnt = 0;
    u32x2 sel[2][4];
#pragma unroll
    for (int u = 0; u < 2; ++u) {
        unsigned c = 0;
#pragma unroll
        for (int k = 0; k < 2; ++k) {
            const unsigned h = cc[u][k] >> 1;
            const unsigned f0 = ~(cc[u][k] | h) & 0x11111111u, f1 = ~(cc[u][k] | h) & 0x44444444u;
            c += __builtin_popcount(f0);
            pc[u][2 * k] = c;
            c += __builtin_popcount(f1);
            if (k == 0) pc[u][1] = c;
            sel[u][2 * k] = lut_at(lds, f0);
            sel[u][2 * k + 1] = *(LAS const u32x2*)(lds + __builtin_amdgcn_udot4(f1, 0x10080402u, (unsigned)LDS_LUT, false));
        }
        cnt |= c << (16 * u);
    }
    unsigned c2[2], f2[2], c2n = 0;
    u32x2 s2[2];
#pragma unroll
    for (int u = 0; u < 2; ++u) {
        c2[u] = (z[u] | (z[u] << 15)) & 0x55555555u;
        asm("" : "+v"(c2[u]));
        f2[u] = c2[u] & ~(c2[u] >> 2) & 0x11111111u;
        s2[u] = lut_at(lds, f2[u]);
        c2n |= (unsigned)__builtin_popcount(f2[u]) << (16 * u);
        *(LAS unsigned*)(T + u * SCR_T + 256 + lane * 4) = rr[u];
    }
    const unsigned incl = wave_scan(cnt), in2 = wave_scan(c2n), tot = __builtin_amdgcn_readlane(incl, 63);
    unsigned wv[2];
#pragma unroll
    for (int u = 0; u < 2; ++u) wv[u] = get_bits(bit_addr(A[u]) + b_start(half16(tot, u)) + half16(in2 - c2n, u) * 4u);
#pragma unroll
    for (int u = 0; u < 2; ++u) {
        *(LAS unsigned*)(T + u * SCR_T + lane * 4) = (c2[u] ^ f2[u]) | shuffle8(wv[u], wv[u] >> 4, s2[u]);
        if (__builtin_expect(half16(tot, u) > 1024u, 0)) *(LAS u32x2*)(T + u * SCR_T + 512 + lane * 8) = *(LAS const u32x2*)(A[u] + 384 + lane * 8);
    }
    LDS_FENCE();
    unsigned w[2][4];
#pragma unroll
    for (int u = 0; u < 2; ++u) {
        unsigned pos = half16(incl - cnt, u) * 4u + bit_addr(T + u * SCR_T);
        asm("" : "+v"(pos));
#pragma unroll
        for (int a = 0; a < 4; ++a) w[u][a] = get_bits(a ? pos + pc[u][a - 1] * 4u : pos);
    }
    LDS_FENCE();
    mid();
    unsigned e[2][8];
#pragma unroll
    for (int u = 0; u < 2; ++u) {
        const unsigned top4 = (info[u] & 0xffu) * 0x01010101u;
#pragma unroll
        for (int a = 0; a < 4; ++a) {
            const unsigned lo = w[u][a] & 0x0f0f0f0fu, hi = (w[u][a] >> 4) & 0x0f0f0f0fu;
            e[u][a] = top4 - (perm(hi, lo, sel[u][a][0]) | ((cc[u][a >> 1] >> (2 * (a & 1))) & 0x03030303u));
            e[u][4 + a] = top4 - (perm(hi, lo, sel[u][a][1]) | ((cc[u][a >> 1] >> (2 * (a & 1) + 4)) & 0x03030303u));
        }
    }
    if (__builtin_expect(anyx, 0)) {
#pragma unroll
        for (int u = 0; u < 2; ++u)
#pragma unroll
            for (int a = 0; a < 4; ++a) { e[u][a] -= X[u][a] & 0xf0f0f0f0u; e[u][4 + a] -= (X[u][a] << 4) & 0xf0f0f0f0u; }
    }
#pragma unroll
    for (int u = 0; u < 2; ++u)
#pragma unroll
        for (int a = 0; a < 4; ++a) {
            const unsigned q0 = perm(e[u][a], p[u][0][a], 0x05010400u), q1 = perm(e[u][a], p[u][0][a], 0x07030602u);
            const unsigned q2 = perm(e[u][4 + a], p[u][1][a], 0x05010400u), q3 = perm(e[u][4 + a], p[u][1][a], 0x07030602u);
            o[u][0][a] = __builtin_amdgcn_alignbit(q0, q0, 1); o[u][1][a] = __builtin_amdgcn_alignbit(q1, q1, 1);
            o[u][2][a] = __builtin_amdgcn_alignbit(q2, q2, 1); o[u][3][a] = __builtin_amdgcn_alignbit(q3, q3, 1);
        }
}
__device__ __forceinline__ void store_stage(const char* dst, unsigned voff, const u32x4 (&o)[4]) {
    st_wt<0>(dst, voff, o[0]); st_wt<1024>(dst, voff, o[1]); st_wt<2048>(dst, voff, o[2]); st_wt<3072>(dst, voff, o[3]);
}
__device__ __forceinline__ void store_out(const char* dst, unsigned voff, const u32x4 (&o)[4]) {
    st_out<0>(dst, voff, o[0]); st_out<64>(dst, voff, o[1]);
    st_out<0>(dst + 16 * 65536, voff, o[2]); st_out<64>(dst + 16 * 65536, voff, o[3]);
}

__device__ __forceinline__ void pack_piece(const Params& P, char* ws, const char* px, const char* pw, LAS char* lds, int round, unsigned e,
                                           unsigned cntv, int lane, int wid) {
    LAS unsigned* lrec = (LAS unsigned*)(lds + LDS_REC) + 32 + round * 8;
    const int li = lane & 15, lg = lane >> 4;
    unsigned* ready = (unsigned*)(ws + OFF_READY);
    const int h = (e >> 3) & 15, idx = (e & 7) * 8 + wid, id = h * 64 + idx;
    const unsigned kind = e >> 7;
    const bool isx = kind == 0u || kind == 9u;
    const unsigned dest = (kind - 1u) & 7u;
    const bool own = kind == 9u || kind == (unsigned)P.rank + 1u;
    const unsigned rdest = isx ? (unsigned)lane >> 3 : dest;
    char* peer = P.ws[rdest];
    u32x4 d[4];
    u32x2 v[8];
    if (isx) {
        const char* src = px + (size_t)(64 * idx + li) * 1024 + h * 64 + lg * 16;
#pragma unroll
        for (int c = 0; c < 4; ++c) {
            const u32x4 t = *(const u32x4*)(src + 16 * c * 1024);
            v[2 * c] = (u32x2){t[0], t[1]}; v[2 * c + 1] = (u32x2){t[2], t[3]};
        }
    } else {
        const int cp = 32 * ((li >> 1) & 1) + 8 * (li >> 2) + 4 * (li & 1);
        const char* src = pw + (size_t)(32 * h + 8 * lg) * 65536 + (size_t)(dest * 4096 + 64 * idx + cp) * 2;
#pragma unroll
        for (int j = 0; j < 8; ++j) v[j] = *(const u32x2*)(src + (size_t)j * 65536);
    }
    __builtin_amdgcn_sched_barrier(0);
    if (round == 0) {
        build_lut(lds, false, threadIdx.x);
        if (threadIdx.x >= 448) ((LAS unsigned*)(lds + LDS_FLG))[(threadIdx.x - 448) * 32 + 16] = 0u;
        asm volatile("s_waitcnt lgkmcnt(0)" ::: "memory");
        __builtin_amdgcn_s_barrier();
        asm volatile("" ::: "memory");
    }
    if (isx) {
#pragma unroll
        for (int c = 0; c < 4; ++c) d[c] = (u32x4){v[2 * c][0], v[2 * c][1], v[2 * c + 1][0], v[2 * c + 1][1]};
    } else {
#pragma unroll
        for (int hf = 0; hf < 2; ++hf)
#pragma unroll
            for (int a = 0; a < 4; ++a) {
                d[2 * hf][a] = perm(v[2 * a + 1][hf], v[2 * a][hf], 0x05040100u);
                d[2 * hf + 1][a] = perm(v[2 * a + 1][hf], v[2 * a][hf], 0x07060302u);
            }
    }
    asm volatile("" :: "v"(peer));
    const unsigned call = rfl(cntv) + 1u, par = call & 1u;
    if (own) {
        store_stage(ws + (isx ? OFF_A : OFF_B) + (size_t)(h * 256 + 4 * idx) * 1024, lane * 16, d);
        wait_vm0();
        if (lane == 0) __hip_atomic_store(ready + ((isx ? 0 : 64) + idx) * NSLOT + h, call, __ATOMIC_RELAXED, __HIP_MEMORY_SCOPE_SYSTEM);
        return;
    }
    const unsigned g[1] = {(unsigned)(isx ? id : 1024 + (int)dest * 1024 + id)};
    unsigned r[1];
    pack_store<1>([&](int, u32x4 (&o)[4]) { o[0] = d[0]; o[1] = d[1]; o[2] = d[2]; o[3] = d[3]; }, ws + OFF_PK + par * PK_PAR, g, lds, lds + wid * SCR_WAVE, lane, r, [] {});
    if (lane == 0) lrec[wid] = r[0];
    wait_vm0();
    asm volatile("s_waitcnt lgkmcnt(0)" ::: "memory");
    __builtin_amdgcn_s_barrier();
    asm volatile("" ::: "memory");
    if (wid == 0 && (isx || lane < 8) && rdest != (unsigned)P.rank) {
        u64* rp = (u64*)(peer + OFF_REC + par * REC_PAR) + (P.rank * 3 + (isx ? 0 : 1)) * 1024 + h * 64 + (e & 7) * 8 + (lane & 7);
        __hip_atomic_store(rp, ((u64)call << 32) | lrec[lane & 7], __ATOMIC_RELAXED, __HIP_MEMORY_SCOPE_SYSTEM);
    }
}

__device__ __forceinline__ void pack_bulk(const Params& P, char* ws, const char* pw, LAS char* lds, unsigned i, unsigned call, int lane, int wid) {
    const int li = lane & 15, lg = lane >> 4;
    const unsigned par = call & 1u;
    const unsigned lsrc = (unsigned)(8 * lg) * 65536u + (unsigned)(64 * wid + 32 * ((li >> 1) & 1) + 8 * (li >> 2) + 4 * (li & 1)) * 2u;
    u32x2 v[2][8];
    unsigned en[2];
    auto load = [&](unsigned i, int u) {
        const unsigned k = (i + u) / 7u, dest = (P.rank + 1u + (i + u) % 7u) & 7u, hp = (P.worder[k >> 2] >> ((k & 3u) * 8u)) & 127u;
        en[u] = dest << 7 | hp;
        const char* src = pw + (size_t)(hp >> 3) * (32 * 65536) + (dest * 4096u + (hp & 7u) * 512u) * 2u;
#pragma unroll
        for (int j = 0; j < 8; ++j) v[u][j] = *(const u32x2*)(src + (size_t)j * 65536 + lsrc);
    };
    load(i, 0);
    load(i, 1);
    __builtin_amdgcn_s_waitcnt(0x0F70);
    for (LAS unsigned* lrec = (LAS unsigned*)(lds + LDS_FLG);; lrec += 32) {
        const unsigned el = (lane & 8) ? en[1] : en[0];
        char* peer = P.ws[el >> 7];
        const unsigned g[2] = {1024u + en[0] * 8u + wid, 1024u + en[1] * 8u + wid};
        u32x4 d[2][4];
#pragma unroll
        for (int u = 0; u < 2; ++u)
#pragma unroll
            for (int hf = 0; hf < 2; ++hf)
#pragma unroll
                for (int a = 0; a < 4; ++a) {
                    d[u][2 * hf][a] = perm(v[u][2 * a + 1][hf], v[u][2 * a][hf], 0x05040100u);
                    d[u][2 * hf + 1][a] = perm(v[u][2 * a + 1][hf], v[u][2 * a][hf], 0x07060302u);
                }
        asm volatile("" : "+v"(d[0][0]), "+v"(d[0][1]), "+v"(d[0][2]), "+v"(d[0][3]), "+v"(d[1][0]), "+v"(d[1][1]), "+v"(d[1][2]), "+v"(d[1][3]));
        i += 2u * P.dstride;
        if (i < 896u) load(i, 0);
        __builtin_amdgcn_sched_barrier(0);
        unsigned r[2];
        pack_store<2>([&](int u, u32x4 (&o)[4]) { o[0] = d[u][0]; o[1] = d[u][1]; o[2] = d[u][2]; o[3] = d[u][3]; },
                      ws + OFF_PK + par * PK_PAR, g, lds, lds + wid * SCR_WAVE, lane, r, [&] { if (i < 896u) load(i, 1); });
        if (lane < 2) lrec[lane * 8 + wid] = lane ? r[1] : r[0];
        wait_vm0();
        __builtin_amdgcn_s_waitcnt(0x0F70);
        unsigned arrived = 0;
        if (lane == 0) arrived = __hip_atomic_fetch_add(lrec + 16, 1u, __ATOMIC_RELAXED, __HIP_MEMORY_SCOPE_WORKGROUP);
        LDS_FENCE();
        if (rfl(arrived) == 7u && lane < 16) {
            u64* rp = (u64*)(peer + OFF_REC + par * REC_PAR) + (P.rank * 3 + 1) * 1024 + (el & 127u) * 8u + (lane & 7);
            __hip_atomic_store(rp, ((u64)call << 32) | lrec[lane], __ATOMIC_RELAXED, __HIP_MEMORY_SCOPE_SYSTEM);
        }
        if (i >= 896u) break;
    }
}

constexpr int PSLOT = 13440;
constexpr int PSLOT_M = 2 * FIXB + VCAP;
constexpr int PSLOT_T = PSLOT - 2 * SCR_T;
constexpr int LDS_Q = LDS_REC;
constexpr int LDS_QTAIL = LDS_Q + 256;
constexpr int LDS_FULL = LDS_QTAIL + 16;
constexpr int FLAG_E = (NFLY - 1) / 6 + 1, FLAG_J = (NFLY - 1) % 6;

__device__ __forceinline__ unsigned lds_poll(LAS char* p) {
    return rfl(__hip_atomic_load((LAS unsigned*)p, __ATOMIC_RELAXED, __HIP_MEMORY_SCOPE_WORKGROUP));
}

__device__ __forceinline__ void fetch_link(const Params& P, LAS char* lds, int q, unsigned call, int lane) {
    const unsigned p = (unsigned)(P.rank + 1 + q) & 7u;
    const char* pkb = P.ws[p] + OFF_PK + (call & 1u) * PK_PAR;
    const unsigned ldsbase = (unsigned)(size_t)lds;
    const unsigned v0 = lane * 16, v1 = v0 + 4096, v2 = v0 + (VMID - 512);
    unsigned fa[3] = {0, 0, 0}, fv[3] = {0, 0, 0};
    unsigned ended = 0;
    for (unsigned i = 0;; ++i) {
        LAS char* e = lds + LDS_Q + (i & 15u) * 16u;
        u32x4 ev;
        for (;;) {
            ev = *(volatile LAS u32x4*)e;
            if (rfl(ev[0]) == i + 1u) break;
            if (fa[0] | fa[1] | fa[2]) {
                if (fa[2]) { asm volatile("s_waitcnt vmcnt(12)" ::: "memory"); *(volatile LAS unsigned*)(lds + fa[2]) = fv[2]; fa[2] = 0; }
                else if (fa[1]) { asm volatile("s_waitcnt vmcnt(6)" ::: "memory"); *(volatile LAS unsigned*)(lds + fa[1]) = fv[1]; fa[1] = 0; }
                else { asm volatile("s_waitcnt vmcnt(0)" ::: "memory"); *(volatile LAS unsigned*)(lds + fa[0]) = fv[0]; fa[0] = 0; }
            } else {
                if (ended == NDEC) return;
                __builtin_amdgcn_s_sleep(1);
            }
        }
        const unsigned w1 = rfl(ev[1]);
        if (w1 >> 31) { ++ended; continue; }
        const unsigned owner = w1 >> 24, slot = ldsbase + (owner - 1u) * PSLOT;
        const char* pk = pkb + (size_t)(w1 & 0xffffffu) * PAIRB;
        const u64 vmask = (u64)rfl(ev[2]) | (u64)rfl(ev[3]) << 32;
        const unsigned da = ldsbase + (fa[FLAG_E - 1] ? fa[FLAG_E - 1] : (unsigned)LDS_FULL), dv = fv[FLAG_E - 1];
#define FW(j) "s_waitcnt vmcnt(%10)\n\t.if %11 == " #j "\n\tds_write_b32 %8, %9\n\t.endif\n\t"
        asm volatile("s_mov_b32 m0, %7\n\t"
                     FW(0) "s_mov_b64 exec, %4\n\t"
                     "global_load_lds_dwordx4 %2, %3 sc0 sc1\n\t"
                     "s_mov_b64 exec, -1\n\t"
                     "s_mov_b32 m0, %5\n\t"
                     FW(1) "global_load_lds_dwordx4 %0, %3 offset:2048 sc0 sc1\n\t"
                     FW(2) "global_load_lds_dwordx4 %0, %3 sc0 sc1\n\t"
                     FW(3) "global_load_lds_dwordx4 %0, %3 offset:1024 sc0 sc1\n\t"
                     FW(4) "global_load_lds_dwordx4 %0, %3 offset:3072 sc0 sc1\n\t"
                     "s_mov_b32 m0, %6\n\t"
                     FW(5) "global_load_lds_dwordx4 %1, %3 sc0 sc1"
                     :: "v"(v0), "v"(v1), "v"(v2), "s"(pk), "s"(vmask), "s"(slot), "s"(slot + 4096u), "s"(slot + (PSLOT_M - 512)),
                        "v"(da), "v"(dv), "i"(NFLY - 1), "i"(FLAG_J)
                     : "memory");
#undef FW
        fa[FLAG_E - 1] = 0;
        fa[2] = fa[1]; fv[2] = fv[1]; fa[1] = fa[0]; fv[1] = fv[0];
        fa[0] = LDS_FULL + owner * 4u; fv[0] = i + 1u;
    }
}

struct Desc { unsigned g, dst, rdy, rec; };

__device__ __forceinline__ void decode_link(const Params& P, LAS char* lds, int q, int w, unsigned call, int lane, int wid) {
    char* ws = P.me;
    const unsigned par = call & 1u;
    const unsigned p = (unsigned)(P.rank + 1 + q) & 7u;
    unsigned* tmo = (unsigned*)(ws + OFF_TMO);
    unsigned vorder = P.order[lane];
    asm volatile("" : "+v"(vorder));
    const unsigned rec_base = (unsigned)(OFF_REC + par * REC_PAR) + p * 3072u * 8u;
    auto describe = [&](unsigned n) {
        Desc d;
        if (n < 1024u) {
            const unsigned e = ((unsigned)__builtin_amdgcn_readlane(vorder, n >> 4) >> ((n & 12u) * 2u)) & 255u;
            const unsigned type = e >> 7, h = (e >> 3) & 15u, idx = (e & 7u) * 8u + (n & 3u) * 2u, id = h * 64u + idx, slot = 16u + 7u * h + q;
            d.g = type ? 1024u + P.rank * 1024u + id : id;
            d.dst = (unsigned)(type ? OFF_B : OFF_A) + (slot * 256u + 4u * idx) * 1024u;
            d.rdy = (unsigned)OFF_READY + ((type * 64u + idx) * NSLOT + slot) * 4u;
            d.rec = rec_base + (type * 1024u + id) * 8u;
        } else {
            const unsigned id = ((n & 0x1c0u) << 1) | ((n & 7u) << 4) | ((P.rbord[(n & 0x1c0u) >= NSMALL * 64u] >> ((n >> 1) & 28u)) & 14u);
            d.g = 9216u + P.rank * 1024u + id;
            d.dst = (id & 15u) * (32u * 65536u) + (p * 4096u + 64u * (id >> 4)) * 2u;
            d.rdy = 0;
            d.rec = rec_base + (2048u + id) * 8u;
        }
        return d;
    };
    const char* pkb = P.ws[p] + OFF_PK + par * PK_PAR;
    const unsigned voff = lane * 16, voff_out = (lane & 15) * 65536 + (lane >> 4) * 16;
    const unsigned tk_base = (call - 1u) * (NTK - PW) - 2u * PW;
    LAS char* S = lds + (wid - 1) * PSLOT;
    LAS char* M = S + PSLOT_M;
    bool dead = false;
    auto vlen = [](unsigned i) { return min((i >> 8) & 255u, (unsigned)VCAP / 16u); };
    auto post = [&](unsigned w1, unsigned ia, unsigned ib) {
        const unsigned la = vlen(ia), lb = vlen(ib);
        const unsigned mlo = la >= 32u ? ~0u : ~(~0u >> la), mhi = lb >= 32u ? ~0u : ((1u << lb) - 1u) | 1u;
        unsigned i = 0;
        asm volatile("s_waitcnt lgkmcnt(0)" ::: "memory");
        if (lane == 0) {
            i = __hip_atomic_fetch_add((LAS unsigned*)(lds + LDS_QTAIL), 1u, __ATOMIC_RELAXED, __HIP_MEMORY_SCOPE_WORKGROUP);
            *(volatile LAS u32x4*)(lds + LDS_Q + (i & 15u) * 16u) = (u32x4){i + 1u, w1, mlo, mhi};
        }
        return rfl(i) + 1u;
    };
    auto poll = [&](unsigned rec, unsigned& ia, unsigned& ib) {
        const u64 t0 = __builtin_amdgcn_s_memrealtime();
        unsigned spins = 0;
        ia = ib = 0;
        while (!dead) {
            const u64 v = __hip_atomic_load((const u64*)(ws + rec) + (lane & 1), __ATOMIC_RELAXED, __HIP_MEMORY_SCOPE_SYSTEM);
            if (__builtin_amdgcn_ballot_w64((unsigned)(v >> 32) != call) == 0ull) {
                ia = __builtin_amdgcn_readlane((unsigned)v, 0); ib = __builtin_amdgcn_readlane((unsigned)v, 1);
                break;
            }
            if (timed_out(tmo, call, t0, spins)) dead = true;
            __builtin_amdgcn_s_sleep(1);
        }
    };
    unsigned n1 = PW + w, ia, ib;
    Desc d0 = describe(w), d1 = describe(n1);
    poll(d0.rec, ia, ib);
    STAMP(ws, 2);
    unsigned want = post(d0.g >> 1 | wid << 24, ia, ib);
    for (;;) {
        const bool more = n1 < NTK;
        const u64 rv = __hip_atomic_load((const u64*)(ws + d1.rec) + (lane & 1), __ATOMIC_RELAXED, __HIP_MEMORY_SCOPE_SYSTEM);
        while (lds_poll(lds + LDS_FULL + wid * 4) != want) __builtin_amdgcn_s_sleep(1);
        LDS_FENCE();
        const unsigned la = vlen(ia), lb = vlen(ib);
        if (la > 32u || lb > 32u) {
            const char* mid = pkb + (size_t)(d0.g >> 1) * PAIRB + VMID;
            for (int c = -3; c <= 3; ++c) {
                const int off = c * 1024 - 512 + (int)voff;
                if (c != 0 && off >= -16 * (int)la && off < 16 * (int)lb) {
                    u32x4 t;
                    asm volatile("global_load_dwordx4 %0, %1, off sc0 sc1\n\ts_waitcnt vmcnt(0)" : "=&v"(t) : "v"(mid + off) : "memory");
                    *(LAS u32x4*)(M + off) = t;
                }
            }
            LDS_FENCE();
        }
        u32x4 o[2][4];
        unsigned ja = 0, jb = 0;
        bool posted = false;
        {
            u32x4 pp[2][2];
            u32x2 cc[2];
#pragma unroll
            for (int u = 0; u < 2; ++u) {
                cc[u] = *(LAS const u32x2*)(S + u * FIXB + 2048 + lane * 8);
                pp[u][0] = *(LAS const u32x4*)(S + u * FIXB + voff);
                pp[u][1] = *(LAS const u32x4*)(S + u * FIXB + 1024 + voff);
            }
            const unsigned inf[2] = {ia, ib};
            unpack2(pp, cc, inf, la, lds, S + PSLOT_T, M, lane, o, [&] {
                asm volatile("" : "+v"(pp[0][0]), "+v"(pp[0][1]), "+v"(pp[1][0]), "+v"(pp[1][1]));
                if (more && __builtin_amdgcn_ballot_w64((unsigned)(rv >> 32) != call) == 0ull) {
                    ja = __builtin_amdgcn_readlane((unsigned)rv, 0); jb = __builtin_amdgcn_readlane((unsigned)rv, 1);
                    want = post(d1.g >> 1 | wid << 24, ja, jb);
                    posted = true;
                }
            });
        }
        if (d0.rdy) {
            store_stage(ws + d0.dst, voff, o[0]);
            store_stage(ws + d0.dst + 4096, voff, o[1]);
        } else {
            if (dead) {
#pragma unroll
                for (int c = 0; c < 8; ++c) o[c >> 2][c & 3] = (u32x4){0x7fc07fc0u, 0x7fc07fc0u, 0x7fc07fc0u, 0x7fc07fc0u};
            }
            store_out(P.out + d0.dst, voff_out, o[0]);
            store_out(P.out + d0.dst + 32 * 65536, voff_out, o[1]);
        }
        unsigned tk;
        asm volatile("s_mov_b64 exec, 1\n\tglobal_atomic_add %0, %1, %2, %3 sc0\n\ts_mov_b64 exec, -1\n\ts_waitcnt vmcnt(0)"
                     : "=&v"(tk) : "v"((unsigned)OFF_TKT + q * 128u), "v"(more ? 1u : 0u), "s"(ws) : "memory");
        if (d0.rdy && lane < 2) __hip_atomic_store((unsigned*)(ws + d0.rdy) + lane * NSLOT, call, __ATOMIC_RELAXED, __HIP_MEMORY_SCOPE_SYSTEM);
        const unsigned n2 = rfl(tk) - tk_base;
        if (!more) break;
        if (!posted) {
            poll(d1.rec, ja, jb);
            want = post(d1.g >> 1 | wid << 24, ja, jb);
        }
        ia = ja; ib = jb;
        d0 = d1;
        n1 = n2;
        d1 = describe(n1);
    }
    post(0x80000000u, 0u, 0u);
}

__device__ __forceinline__ void glds4(unsigned m0v, unsigned voff, const char* sb) {
    asm volatile("s_mov_b32 m0, %0\n\ts_nop 0\n\t"
                 "global_load_lds_dwordx4 %1, %2 sc0 sc1\n\t"
                 "global_load_lds_dwordx4 %1, %2 offset:1024 sc0 sc1\n\t"
                 "global_load_lds_dwordx4 %1, %2 offset:2048 sc0 sc1\n\t"
                 "global_load_lds_dwordx4 %1, %2 offset:3072 sc0 sc1" :: "s"(m0v), "v"(voff), "s"(sb) : "memory");
}
__device__ __forceinline__ void glds1(unsigned m0v, unsigned voff, const char* sb) {
    asm volatile("s_mov_b32 m0, %0\n\ts_nop 0\n\tglobal_load_lds_dwordx4 %1, %2 sc0 sc1" :: "s"(m0v), "v"(voff), "s"(sb) : "memory");
}

template <int WR, bool BIG>
__device__ __forceinline__ void k_loop(f32x4 (&acc)[BIG ? 10 : 8][4], char* ws, LAS char* lds, int u0, int pn, unsigned call, int lane, int wid) {
    constexpr int MT = BIG ? 10 : 8;
    constexpr bool has5 = BIG && !WR;
    unsigned* tmo = (unsigned*)(ws + OFF_TMO);
    const int wc = wid & 3;
    const unsigned ldsbase = (unsigned)(size_t)lds;
    const unsigned u1 = WR ? 4 * pn + wc : u0 + wc;
    const char* g1 = ws + (WR ? OFF_B : OFF_A) + (size_t)u1 * 4096;
    const char* g2 = ws + OFF_A + (size_t)(4 * u0 + 16 + wc) * 1024;
    const unsigned l1 = ldsbase + WR * 20480 + wc * 4096, l2 = ldsbase + (16 + wc) * 1024;
    const unsigned voff = lane * 16;
    const unsigned vrow = (unsigned)OFF_READY + (WR * 64 + (lane >= 32 && has5 ? u0 + 4 : u1)) * (NSLOT * 4);
    const unsigned vsnap = vrow + (lane & 31) * 16;
    LAS char* snap = lds + LDS_FLG + wid * 1024;
    *(LAS u32x4*)(snap + voff) = (u32x4){0u, 0u, 0u, 0u};
    asm volatile("s_waitcnt lgkmcnt(0)" ::: "memory");
    unsigned hz = 0;
#define ADVANCE(v) ({ const u64 m_ = __builtin_amdgcn_ballot_w64((v) >= call); \
    (unsigned)__builtin_ctzll(~(u64)((unsigned)m_ & (unsigned)(m_ >> 32))); })
#define ENSURE(t) do { if (__builtin_expect(hz <= (unsigned)(t), 0)) { const u64 t0_ = __builtin_amdgcn_s_memrealtime(); unsigned sp_ = 0; \
        __builtin_amdgcn_s_waitcnt(0xC07F); \
        for (;;) { \
            const unsigned a_ = __hip_atomic_load((const unsigned*)(ws + vrow) + min(hz + (lane & 31u), NSLOT - 1u), __ATOMIC_RELAXED, __HIP_MEMORY_SCOPE_SYSTEM); \
            hz += ADVANCE(a_); \
            if (hz > (unsigned)(t)) break; \
            if (timed_out(tmo, call, t0_, sp_)) { hz = NSLOT; break; } \
            __builtin_amdgcn_s_sleep(2); } } } while (0)
#define STAGE(t, bt) do { const size_t go_ = (size_t)(t) * 262144; const unsigned lo_ = (bt) * LSLOT; \
    glds4(l1 + lo_, voff, g1 + go_); if (has5) glds1(l2 + lo_, voff, g2 + go_); } while (0)
#define WAITV() asm volatile("s_waitcnt vmcnt(%0)" :: "i"(has5 ? 10 : 8) : "memory")
#pragma unroll 1
    for (int t = 0; t < 3; ++t) { ENSURE(t); STAGE(t, t); }
    WAITV();
    __builtin_amdgcn_s_barrier();
    STAMP(ws, 2);
    if (WR == 1) __builtin_amdgcn_s_barrier();
    const unsigned aoff = lane * 16, boff = 20480 + wc * 4096 + lane * 16;
    for (unsigned s = 0; s < NSLOT; ++s) {
        const unsigned t = s + 3 < NSLOT ? s + 3 : NSLOT - 1;
        ENSURE(t);
        STAGE(t, (s + 3) & 3);
        __builtin_amdgcn_sched_barrier(0);
        bf16x8 A[MT], B[4];
        const unsigned so = (s & 3) * LSLOT;
        LAS const char* pa = lds + aoff + so; LAS const char* pb = lds + boff + so;
#pragma unroll
        for (int n = 0; n < 4; ++n) B[n] = *(LAS const bf16x8*)(pb + n * 1024);
#pragma unroll
        for (int m = 0; m < MT; ++m) A[m] = *(LAS const bf16x8*)(pa + (m < 8 ? 8 * WR + m : 8 + 2 * WR + m) * 1024);
        __builtin_amdgcn_sched_barrier(0);
        if (__builtin_expect((s & FLAG_MASK) == 0 && hz < s + LOOKAHEAD, 0)) {
            const unsigned fv = *(volatile LAS const unsigned*)(snap + (lane >> 5) * 512 + min(hz + (lane & 31u), NSLOT - 1u) * 4u);
            hz += ADVANCE(fv);
            glds1(ldsbase + LDS_FLG + wid * 1024, vsnap, ws);
        }
        WAITV();
        __builtin_amdgcn_s_waitcnt(0xC07F);
        __builtin_amdgcn_sched_barrier(0);
        __builtin_amdgcn_s_barrier();
        __builtin_amdgcn_sched_barrier(0);
        __builtin_amdgcn_s_setprio(1);
#pragma unroll
        for (int m = 0; m < MT; ++m)
#pragma unroll
            for (int n = 0; n < 4; ++n) acc[m][n] = __builtin_amdgcn_mfma_f32_16x16x32_bf16(B[n], A[m], acc[m][n], 0, 0, 0);
        __builtin_amdgcn_s_setprio(0);
        __builtin_amdgcn_sched_barrier(0);
        __builtin_amdgcn_s_barrier();
        __builtin_amdgcn_sched_barrier(0);
    }
    if (WR == 0) __builtin_amdgcn_s_barrier();
#undef ENSURE
#undef ADVANCE
#undef STAGE
#undef WAITV
}

template <bool BIG>
__device__ __forceinline__ void gemm_tile(const Params& P, LAS char* lds, int u0, int pn, unsigned call, int lane, int wid) {
    constexpr int NU = BIG ? 5 : 4;
    char* ws = P.me;
    unsigned* tmo = (unsigned*)(ws + OFF_TMO);
    const int wr = wid >> 2, wc = wid & 3;
    f32x4 acc[2 * NU][4];
#pragma unroll
    for (int m = 0; m < 2 * NU; ++m)
#pragma unroll
        for (int n = 0; n < 4; ++n) acc[m][n] = (f32x4){0.f, 0.f, 0.f, 0.f};
    if (wr == 0) k_loop<0, BIG>(acc, ws, lds, u0, pn, call, lane, wid);
    else k_loop<1, BIG>(acc, ws, lds, u0, pn, call, lane, wid);
    const unsigned tmov = __hip_atomic_load(tmo, __ATOMIC_RELAXED, __HIP_MEMORY_SCOPE_AGENT);
    const unsigned rbl = (lane < 4 ? 2 * u0 + 4 * wr + lane : 2 * u0 + 8 + wr) & 127u;
    char* peer = P.ws[rbl >> 4];
    asm volatile("" :: "v"(peer), "v"(tmov));
    wait_vm0();
    __builtin_amdgcn_s_barrier();
    asm volatile("" ::: "memory");
    STAMP(ws, 3);
    const unsigned par = call & 1u;
    const int cb = 4 * pn + wc;
    const unsigned voff_out = (lane & 15) * 65536 + (lane >> 4) * 16;
    if (__builtin_expect(rfl(tmov) == call, 0)) {
#pragma unroll
        for (int m = 0; m < 2 * NU; ++m)
#pragma unroll
            for (int n = 0; n < 4; ++n) acc[m][n] = (f32x4){__builtin_nanf(""), __builtin_nanf(""), __builtin_nanf(""), __builtin_nanf("")};
    }
    unsigned rb[NU], g[NU], nown = 0;
#pragma unroll
    for (int u = 0; u < NU; ++u) {
        rb[u] = (u < 4 ? 2 * u0 + 4 * wr + u : 2 * u0 + 8 + wr) & 127u;
        g[u] = 9216u + (rb[u] >> 4) * 1024u + cb * 16u + (rb[u] & 15u);
        nown += (rb[u] >> 4) == (unsigned)P.rank;
    }
    auto get = [&](int u, u32x4 (&d)[4]) {
#pragma unroll
        for (int c = 0; c < 4; ++c) {
            const int mi = 2 * u + (c >> 1), b = c & 1;
            d[c][0] = cvt_pk(acc[mi][0][2 * b], acc[mi][1][2 * b]);
            d[c][1] = cvt_pk(acc[mi][2][2 * b], acc[mi][3][2 * b]);
            d[c][2] = cvt_pk(acc[mi][0][2 * b + 1], acc[mi][1][2 * b + 1]);
            d[c][3] = cvt_pk(acc[mi][2][2 * b + 1], acc[mi][3][2 * b + 1]);
        }
    };
    if (nown) {
#pragma unroll
        for (int u = 0; u < NU; ++u)
            if ((rb[u] >> 4) == (unsigned)P.rank) {
                u32x4 d[4];
                get(u, d);
                store_out(P.out + (size_t)(rb[u] & 15u) * (32 * 65536) + (size_t)(P.rank * 4096 + 64 * cb) * 2, voff_out, d);
            }
    }
    if (nown < NU) {
        unsigned r[NU];
        pack_store<NU>(get, ws + OFF_PK + par * PK_PAR, g, lds, lds + wid * SCR_WAVE, lane, r, [] {});
        const unsigned rec = lane == 0 ? r[0] : lane == 1 ? r[1] : lane == 2 ? r[2] : lane == 3 ? r[3] : r[NU - 1];
        wait_vm0();
        if (lane < NU && (rbl >> 4) != (unsigned)P.rank) {
            u64* rp = (u64*)(peer + OFF_REC + par * REC_PAR) + (P.rank * 3 + 2) * 1024 + cb * 16 + (rbl & 15u);
            __hip_atomic_store(rp, ((u64)call << 32) | rec, __ATOMIC_RELAXED, __HIP_MEMORY_SCOPE_SYSTEM);
        }
    }
}

__global__ void __launch_bounds__(512, 2) fused_gemm_rs(const Params P) {
    __shared__ __attribute__((aligned(16))) char smem[LDS_BYTES];
    const int bid = blockIdx.x;
    const int lane = threadIdx.x & 63, wid = rfl(threadIdx.x >> 6);
    char* ws = P.me; const char* px = P.x; const char* pw = P.w; const u64 job = P.job[bid];
    asm volatile("" :: "s"(ws), "s"(px), "s"(pw), "s"(job), "s"(P.rank));
    STAMP(ws, 0);
    unsigned* cnt = (unsigned*)(ws + OFF_CNT) + bid * 8 + wid;
    const unsigned cntv = __hip_atomic_load(cnt, __ATOMIC_RELAXED, __HIP_MEMORY_SCOPE_AGENT);
    if (bid >= 7 * PWG) {
#pragma unroll 1
        for (int r = 0; r < 3; ++r) {
            const unsigned e = (unsigned)(job >> (11 + 11 * r)) & 0x7ffu;
            if ((e >> 7) == 15u) break;
            pack_piece(P, ws, px, pw, (LAS char*)smem, r, e, cntv, lane, wid);
        }
        const unsigned i0 = P.dstart + 2u * (unsigned)(job >> 44);
        if (i0 < 896u) pack_bulk(P, ws, pw, (LAS char*)smem, i0, rfl(cntv) + 1u, lane, wid);
        asm volatile("s_waitcnt lgkmcnt(0)" ::: "memory");
        __builtin_amdgcn_s_barrier();
    } else {
        build_lut((LAS char*)smem, true, threadIdx.x);
        if (threadIdx.x >= 384) ((LAS unsigned*)(smem + LDS_Q))[threadIdx.x - 384] = 0u;
        asm volatile("s_waitcnt lgkmcnt(0)" ::: "memory");
        __builtin_amdgcn_s_barrier();
        asm volatile("" ::: "memory");
    }
    const unsigned call = rfl(cntv) + 1u;
    if (lane == 0) __hip_atomic_store(cnt, call, __ATOMIC_RELAXED, __HIP_MEMORY_SCOPE_AGENT);
    STAMP(ws, 1);
    if (bid < 7 * PWG) {
        if (wid == 0) fetch_link(P, (LAS char*)smem, bid % 7, call, lane);
        else if (wid <= NDEC) decode_link(P, (LAS char*)smem, bid % 7, (wid - 1) * PWG + bid / 7, call, lane, wid);
    } else if (((unsigned)job & 63u) != 63u) {
        if (job & 64u) gemm_tile<true>(P, (LAS char*)smem, (unsigned)job & 63u, ((unsigned)job >> 7) & 15u, call, lane, wid);
        else gemm_tile<false>(P, (LAS char*)smem, (unsigned)job & 63u, ((unsigned)job >> 7) & 15u, call, lane, wid);
    }
    STAMP(ws, 4);
}

#define LAGS {0, 0, 1, 3, 5, 6, 8, 9}
#ifndef DBL_PAIR
#define DBL_PAIR 5
#endif

void make_plan(Params& P, int rank) {
    const int lag[8] = LAGS;
    unsigned char* order = (unsigned char*)P.order;
    unsigned char* worder = (unsigned char*)P.worder;
    int no = 0, nw = 0;
    for (int ph = 0; ph < 64; ++ph)
        for (int pr = 0; pr < 8; ++pr) {
            const int h = ph - lag[pr];
            if (h >= 0 && h < 16) {
                order[no++] = (unsigned char)(128 | h << 3 | pr);
                worder[nw++] = (unsigned char)(h << 3 | pr);
            }
            if (pr == 0 && ph < 16)
                for (int q = 0; q < 8; ++q) order[no++] = (unsigned char)(ph << 3 | q);
        }
    for (int big = 0; big < 2; ++big) {
        P.rbord[big] = 0;
        const int cls_of[5] = {0, 1, 2, 3, 4};
        for (int cls = 0, k = 0; cls < 5; ++cls)
            for (int rp = 0; rp < 8; ++rp) {
                const int g = 8 * rank + rp;
                if (cls_of[big && g < 60 ? g % 5 : g % 4] == cls) P.rbord[big] |= (unsigned)(2 * rp) << (4 * k++);
            }
    }
    int seq[256], nseq = 0;
    const int mix[8] = {0, 4, 2, 6, 1, 5, 3, 7};
    for (int qt = 0; qt < 4; ++qt)
        for (int i = 0; i < 8; ++i) {
            const int pr = mix[(i + (qt & 1) * 4) & 7];
            const int r0 = pr < NSMALL ? 4 * qt : (13 * ((qt + pr) & 3) + 2) / 4, r1 = pr < NSMALL ? r0 + 4 : (13 * (((qt + pr) & 3) + 1) + 2) / 4;
            for (int r = r0; r < r1; ++r)
                for (int c = 0; c < 2; ++c) seq[nseq++] = (pr < NSMALL ? 4 * r : 5 * r | (r < 12) << 6) | (2 * pr + c) << 7;
        }
    int tile[256], byp[256], nby = 0;
    {
        int spare = 256 - 7 * PWG - nseq, k = 0;
        for (int x = 0; x < 8; ++x) {
            bool gave = false;
            for (int b = 255 - (7 - x); b >= 7 * PWG; b -= 8) {
                if (spare > 0 && !gave && (b - 7 * PWG) / 8 + 1 > nseq / 8) { tile[b] = 63; --spare; gave = spare < 8 - x ? true : false; continue; }
                tile[b] = k < nseq ? seq[k++] : 63;
            }
        }
    }
    for (int pr = 0; pr <= 8; ++pr)
        for (int b = 7 * PWG; b < 256; ++b)
            if ((tile[b] == 63 ? 8 : tile[b] >> 8) == pr) byp[nby++] = b;
    int first[10];
    for (int pr = 0, i = 0; pr <= 9; ++pr) {
        while (i < nby && (tile[byp[i]] == 63 ? 8 : tile[byp[i]] >> 8) < pr) ++i;
        first[pr] = i;
    }
    int nr = 0;
    for (int b = 0; b < 256; ++b) P.job[b] = (u64)(b < 7 * PWG ? 63 : tile[b]) | 0x1ffffffffull << 11 | 511ull << 44;
    for (int r = 0; r < 3; ++r) {
        int list[256], n = 0;
        const int own = (rank + 1) << 7, from = r == 0 ? 0 : first[2 * r];
        if (r < 2) {
            for (int h = 8 * r; h < 8 * r + 8; ++h) {
                for (int q = 0; q < 8; ++q) list[n++] = 9 << 7 | h << 3 | q;
                list[n++] = own | h << 3; list[n++] = own | h << 3 | 1;
            }
            for (int h = 2 * r; h < (r ? 16 : 2); ++h) { list[n++] = own | h << 3 | 2; list[n++] = own | h << 3 | 3; }
        } else {
            for (int h = 0; h < 16; ++h)
                for (int q = 4; q < 8; ++q) list[n++] = own | h << 3 | q;
        }
        for (int i = 48 * r; i < (r < 2 ? 48 * r + 48 : 128); ++i) list[n++] = i;
        while (n < nby - from - (r == 2 ? (nr & 1) != ((nby - from - n) & 1) : 0)) { list[n++] = (((rank + 1 + nr % 7) & 7) + 1) << 7 | worder[nr / 7]; ++nr; }
        for (int i = 0; i < n; ++i) {
            u64& j = P.job[byp[from + i]];
            j = (j & ~(0x7ffull << (11 + 11 * r))) | (u64)list[i] << (11 + 11 * r);
        }
    }
    P.dstart = nr;
    P.dstride = nby - first[DBL_PAIR];
    for (int i = first[DBL_PAIR]; i < nby; ++i) P.job[byp[i]] = (P.job[byp[i]] & ~(511ull << 44)) | (u64)(nby - 1 - i) << 44;
    P.pad = 0;
}

}

extern "C" void kernel_launch(int rank, int nranks, ncclComm_t comm, hipStream_t stream,
                              void* const* d_in, const int* in_sizes, int n_in,
                              void* d_out, int out_size, const int* in_dtypes, int out_dtype,
                              void* const* d_out_all, void* const* d_ws_all, size_t ws_bytes) {
    (void)comm; (void)in_dtypes; (void)out_dtype; (void)d_out_all;
    if (nranks != NR || (unsigned)rank >= (unsigned)NR || n_in < 2 || ws_bytes < WS_NEED || in_sizes[0] != 4096 * 512 ||
        in_sizes[1] != 512 * 32768 || out_size != 512 * 32768) {
        fprintf(stderr, "kernel_launch: unexpected problem shape\n");
        return;
    }
    Params P;
    P.x = (const char*)d_in[0];
    P.w = (const char*)d_in[1];
    P.out = (char*)d_out;
    for (int p = 0; p < NR; ++p) P.ws[p] = (char*)d_ws_all[p];
    P.me = P.ws[rank];
    P.rank = rank;
    make_plan(P, rank);
    hipLaunchKernelGGL(fused_gemm_rs, dim3(256), dim3(512), 0, stream, P);
    hipError_t e = hipGetLastError();
    if (e != hipSuccess) fprintf(stderr, "launch failed: %s\n", hipGetErrorString(e));
}
```

```cpp
#include <hip/hip_runtime.h>
#include <rccl/rccl.h>
#include <cstdio>

#define LAS __attribute__((address_space(3)))
typedef short bf16x8 __attribute__((ext_vector_type(8)));
typedef float f32x4 __attribute__((ext_vector_type(4)));
typedef unsigned u32x4 __attribute__((ext_vector_type(4)));
typedef unsigned u32x2 __attribute__((ext_vector_type(2)));
typedef unsigned long long u64;

namespace {

constexpr int NR = 8;
constexpr int NSLOT = 128;
constexpr int LSLOT = 36 * 1024;
constexpr int LDS_LUT = 4 * LSLOT;
constexpr int LDS_REC = LDS_LUT + 2048;
constexpr int LDS_FLG = LDS_REC + 512;
constexpr int LDS_BYTES = LDS_FLG + 8192;
constexpr int SCR_UNIT = 1408;
constexpr int SCR_WAVE = 5 * SCR_UNIT;
constexpr int SCR_T = 1152;
#define PWG 6
#define NDEC 7
#define LOOKAHEAD 14u
#define FLAG_MASK 3
#define NFLY 10
#define NBAND 2
#define MIX {0, 1, 7, 3, 5, 2, 6, 4}
#define NSMALL 1
constexpr int PW = PWG * NDEC;
constexpr unsigned NTK = 1536;
constexpr u64 DEADLINE = 100000000ull;

constexpr size_t OFF_CNT = 0;
constexpr size_t OFF_TMO = 16384;
constexpr size_t OFF_TKT = 20480;
constexpr size_t OFF_OPOS = 24576;
constexpr size_t OFF_READY = 32768;
constexpr size_t OFF_REC = 131072;
constexpr size_t REC_PAR = (size_t)NR * 3 * 1024 * 8;
constexpr size_t OFF_DBG = 655360;
constexpr size_t OFF_A = 1u << 20;
constexpr size_t OFF_B = OFF_A + (32u << 20);
constexpr int NUNIT = 1024 + 2 * NR * 1024;
constexpr int FIXB = 2560;
constexpr int VCAP = 2624;
constexpr int PAIRB = 2 * FIXB + 2 * VCAP;
constexpr int VMID = 2 * FIXB + VCAP;
constexpr size_t PK_PAR = (size_t)(NUNIT / 2) * PAIRB;
constexpr size_t OFF_PK = OFF_B + (32u << 20);
constexpr size_t WS_NEED = OFF_PK + 2 * PK_PAR;
static_assert(OFF_REC + 2 * REC_PAR <= OFF_DBG, "layout");

struct Params {
    const char* x;
    const char* w;
    char* out;
    char* me;
    char* ws[NR];
    int rank;
    unsigned dstart;
    unsigned dcount[12];
    unsigned order[64];
    unsigned worder[32];
    u64 job[256];
};

#define STAMP(ws, idx) do { } while (0)

__device__ __forceinline__ unsigned rfl(unsigned v) { return __builtin_amdgcn_readfirstlane(v); }
__device__ __forceinline__ unsigned perm(unsigned hi, unsigned lo, unsigned sel) { return __builtin_amdgcn_perm(hi, lo, sel); }
__device__ __forceinline__ unsigned bfi(unsigned m, unsigned a, unsigned b) { return (a & m) | (b & ~m); }
__device__ __forceinline__ int a_swz(int q) { return (0x78 >> (2 * q)) & 3; }
__device__ __forceinline__ unsigned cvt_pk(float lo, float hi) { unsigned r; asm volatile("v_cvt_pk_bf16_f32 %0, %1, %2" : "=v"(r) : "v"(lo), "v"(hi)); return r; }

template <int OFF> __device__ __forceinline__ void st_wt(const char* sbase, unsigned voff, u32x4 v) {
    asm volatile("global_store_dwordx4 %0, %1, %2 offset:%3 sc0 sc1\n\ts_nop 1" :: "v"(voff), "v"(v), "s"(sbase), "i"(OFF) : "memory");
}
#define OUTBITS " sc0 sc1"
template <int OFF> __device__ __forceinline__ void st_out(const char* sbase, unsigned voff, u32x4 v) {
    asm volatile("global_store_dwordx4 %0, %1, %2 offset:%3" OUTBITS "\n\ts_nop 1" :: "v"(voff), "v"(v), "s"(sbase), "i"(OFF) : "memory");
}
__device__ __forceinline__ void wait_vm0() { asm volatile("s_waitcnt vmcnt(0)" ::: "memory"); }

__device__ __forceinline__ bool timed_out(unsigned* tmo, unsigned call, u64 t0, unsigned& spins) {
    if ((++spins & 63u) != 0u) return false;
    if (rfl(__hip_atomic_load(tmo, __ATOMIC_RELAXED, __HIP_MEMORY_SCOPE_AGENT)) == call) return true;
    if (__builtin_amdgcn_s_memrealtime() - t0 > DEADLINE) {
        __hip_atomic_store(tmo, call, __ATOMIC_RELAXED, __HIP_MEMORY_SCOPE_AGENT);
        return true;
    }
    return false;
}

#define LDS_FENCE() asm volatile("" ::: "memory")
template <int CTRL, int ROWMASK> __device__ __forceinline__ unsigned dpp0(unsigned v) {
    return (unsigned)__builtin_amdgcn_update_dpp(0, (int)v, CTRL, ROWMASK, 0xf, false);
}
__device__ __forceinline__ unsigned wave_scan(unsigned v) {
    v += dpp0<0x111, 0xf>(v); v += dpp0<0x112, 0xf>(v); v += dpp0<0x114, 0xf>(v); v += dpp0<0x118, 0xf>(v);
    v += dpp0<0x142, 0xa>(v); v += dpp0<0x143, 0xc>(v);
    return v;
}
__device__ __forceinline__ unsigned lane_rank(u64 m) {
    return __builtin_amdgcn_mbcnt_hi((unsigned)(m >> 32), __builtin_amdgcn_mbcnt_lo((unsigned)m, 0u));
}
__device__ __forceinline__ void build_lut(LAS char* lds, bool dec, int tid) {
    if (tid < 256) {
        u64 e = 0x0c0c0c0c0c0c0c0cull;
        unsigned t = 0;
#pragma unroll
        for (unsigned j = 0; j < 8; ++j)
            if ((tid >> j) & 1) {
                const unsigned slot = (t & 1u) * 4u + (t >> 1);
                const unsigned at = dec ? j : slot, val = dec ? slot : j;
                e = (e & ~(0xffull << (8 * at))) | ((u64)val << (8 * at));
                ++t;
            }
        *(LAS u64*)(lds + LDS_LUT + tid * 8) = e;
    }
}
__device__ __forceinline__ u32x2 lut_at(LAS char* lds, unsigned f) {
    return *(LAS const u32x2*)(lds + __builtin_amdgcn_udot4(f, 0x40201008u, (unsigned)LDS_LUT, false));
}
__device__ __forceinline__ unsigned shuffle8(unsigned lo, unsigned hi, u32x2 sel) {
    return bfi(0x0f0f0f0fu, perm(hi, lo, sel[0]), perm(hi, lo, sel[1]) << 4);
}
__device__ __forceinline__ unsigned half16(unsigned v, int u) { return (u & 1) ? v >> 16 : v & 0xffffu; }
__device__ __forceinline__ unsigned b_start(unsigned n1) { return n1 <= 512u ? ((n1 + 15u) >> 4) << 5 : ((n1 - 249u) >> 3) << 5; }
__device__ __forceinline__ void lds_or(LAS char* p, unsigned v) {
    __hip_atomic_fetch_or((LAS unsigned*)p, v, __ATOMIC_RELAXED, __HIP_MEMORY_SCOPE_WAVEFRONT);
}
__device__ __forceinline__ unsigned bit_addr(LAS char* p) { return (unsigned)(size_t)p * 8u; }
__device__ __forceinline__ void put_bits(unsigned pos, unsigned v) {
    const u64 sh = (u64)v << (pos & 31u);
    LAS char* p = (LAS char*)(size_t)((pos >> 3) & ~3u);
    lds_or(p, (unsigned)sh); lds_or(p + 4, (unsigned)(sh >> 32));
}
__device__ __forceinline__ unsigned get_bits(unsigned pos) {
    LAS const unsigned* p = (LAS const unsigned*)(size_t)((pos >> 3) & ~3u);
    return __builtin_amdgcn_alignbit(p[1], p[0], pos);
}

template <int NU, class Get, class Mid>
__device__ __forceinline__ void pack_store(Get get, char* pk, const unsigned (&g)[NU], LAS char* lds, LAS char* scr, int lane,
                                           unsigned (&info)[NU], Mid mid) {
    constexpr int NS = (NU + 1) / 2;
    const unsigned voff = lane * 16;
#pragma unroll
    for (int u = 0; u < NU; ++u) *(LAS u32x4*)(scr + u * SCR_UNIT + voff) = (u32x4){0u, 0u, 0u, 0u};
    unsigned D[NU][8], pc[NU][3], cnt[NS], x16[NU], top = 0, top4 = 0;
    u32x2 sel[NU][4];
#pragma unroll
    for (int u = 0; u < NU; ++u) {
        u32x4 d[4], S[2];
        unsigned E[8], C[4], c = 0;
        get(u, d);
#pragma unroll
        for (int hf = 0; hf < 2; ++hf)
#pragma unroll
            for (int a = 0; a < 4; ++a) {
                const unsigned p = __builtin_amdgcn_alignbit(d[2 * hf][a], d[2 * hf][a], 31);
                const unsigned r = __builtin_amdgcn_alignbit(d[2 * hf + 1][a], d[2 * hf + 1][a], 31);
                S[hf][a] = perm(r, p, 0x06040200u);
                E[4 * hf + a] = perm(r, p, 0x07050301u);
            }
        char* pair = pk + (size_t)(g[u] >> 1) * PAIRB;
        const char* fix = pair + (g[u] & 1u) * FIXB;
        if (u == 0) {
            unsigned sum = __builtin_amdgcn_sad_u8(E[0], 0u, __builtin_amdgcn_sad_u8(E[2], 0u, __builtin_amdgcn_sad_u8(E[5], 0u, __builtin_amdgcn_sad_u8(E[7], 0u, 0u))));
            sum = __builtin_amdgcn_readlane(wave_scan(sum), 63);
            top = min(((sum + 983u) >> 10) + 2u, 255u);
            top4 = top * 0x01010101u;
        }
        st_wt<0>(fix, voff, S[0]);
#pragma unroll
        for (int a = 0; a < 4; ++a) {
            D[u][a] = top4 - E[a]; D[u][4 + a] = top4 - E[4 + a];
            unsigned n = bfi(0x0f0f0f0fu, D[u][a], D[u][4 + a] << 4);
            asm("" : "+v"(n));
            unsigned o = n | (n >> 1);
            asm("" : "+v"(o));
            const unsigned f = (~o | (o >> 2)) & 0x11111111u;
            unsigned f3 = (f << 1) + f;
            asm("" : "+v"(f3));
            C[a] = n & 0x33333333u & ~f3;
            c += __builtin_popcount(f);
            if (a < 3) pc[u][a] = c;
            sel[u][a] = lut_at(lds, f);
            if (a == 1) st_wt<1024>(fix, voff, S[1]);
        }
        cnt[u >> 1] = (u & 1) ? cnt[u >> 1] | c << 16 : c;
        const u32x2 cc = {C[0] | (C[1] << 2), C[2] | (C[3] << 2)};
        asm volatile("global_store_dwordx2 %0, %1, %2 offset:2048 sc0 sc1" :: "v"(lane * 8), "v"(cc), "s"(fix) : "memory");
        info[u] = top; x16[u] = 0;
        const u64 em = __builtin_amdgcn_ballot_w64(((D[u][0] | D[u][1] | D[u][2] | D[u][3] | D[u][4] | D[u][5] | D[u][6] | D[u][7]) & 0xf0f0f0f0u) != 0u);
        if (__builtin_expect(em != 0ull, 0)) {
            u32x4 X;
#pragma unroll
            for (int a = 0; a < 4; ++a) X[a] = bfi(0xf0f0f0f0u, D[u][a], D[u][4 + a] >> 4);
            const unsigned l = __builtin_ctzll(em);
            const unsigned x0 = __builtin_amdgcn_readlane(X[0], l), x1 = __builtin_amdgcn_readlane(X[1], l);
            const unsigned x2 = __builtin_amdgcn_readlane(X[2], l), x3 = __builtin_amdgcn_readlane(X[3], l);
            const unsigned any = x0 | x1 | x2 | x3, nz = (x0 != 0u) + (x1 != 0u) + (x2 != 0u) + (x3 != 0u);
            const unsigned nib = __builtin_ctz(any) >> 2, val = any >> (4u * nib);
            if ((em & (em - 1ull)) == 0ull && nz == 1u && val < 16u) {
                info[u] |= (l | (x0 ? 0u : x1 ? 1u : x2 ? 2u : 3u) << 6 | nib << 8 | val << 11) << 17;
            } else {
                info[u] |= 1u << 16;
                x16[u] = 1u + __builtin_popcountll(em);
                const unsigned k = 1u + lane_rank(em);
                if (lane == 0) st_wt<0>(pair, (g[u] & 1u) ? VMID : VMID - 16, (u32x4){(unsigned)em, (unsigned)(em >> 32), 0u, 0u});
                if ((em >> lane) & 1ull) st_wt<0>(pair, (g[u] & 1u) ? VMID + 16u * k : VMID - 16 - 16u * k, X);
            }
        }
    }
    unsigned excl[NS], tot[NS];
#pragma unroll
    for (int k = 0; k < NS; ++k) excl[k] = wave_scan(cnt[k]);
#pragma unroll
    for (int k = 0; k < NS; ++k) { tot[k] = __builtin_amdgcn_readlane(excl[k], 63); excl[k] -= cnt[k]; }
#pragma unroll
    for (int u = 0; u < NU; ++u)
        if (__builtin_expect(half16(tot[u >> 1], u) > 1400u, 0) && lane < 24) *(LAS u32x4*)(scr + u * SCR_UNIT + 1024 + voff) = (u32x4){0u, 0u, 0u, 0u};
    LDS_FENCE();
#pragma unroll
    for (int u = 0; u < NU; ++u) {
        unsigned pos = half16(excl[u >> 1], u) * 4u + bit_addr(scr + u * SCR_UNIT);
        asm("" : "+v"(pos));
#pragma unroll
        for (int a = 0; a < 4; ++a) put_bits(a ? pos + pc[u][a - 1] * 4u : pos, shuffle8(D[u][a], D[u][4 + a], sel[u][a]));
    }
    LDS_FENCE();
    mid();
    unsigned w[NU], c2n[NS];
    u32x2 s2[NU];
#pragma unroll
    for (int u = 0; u < NU; ++u) w[u] = *(LAS const unsigned*)(scr + u * SCR_UNIT + lane * 4);
    LDS_FENCE();
#pragma unroll
    for (int u = 0; u < NU; ++u) {
        const unsigned e = ((w[u] >> 1) | (w[u] >> 3)) & 0x11111111u;
        const unsigned c2 = (w[u] & 0x55555555u & ~(e << 2)) | e;
        s2[u] = lut_at(lds, e);
        const unsigned c = __builtin_popcount(e);
        c2n[u >> 1] = (u & 1) ? c2n[u >> 1] | c << 16 : c;
        *(LAS unsigned short*)(scr + u * SCR_UNIT + 128 + lane * 2) = (unsigned short)(c2 | (c2 >> 15));
    }
    unsigned in2[NS];
#pragma unroll
    for (int k = 0; k < NS; ++k) in2[k] = wave_scan(c2n[k]);
    LDS_FENCE();
#pragma unroll
    for (int u = 0; u < NU; ++u)
        put_bits(bit_addr(scr + u * SCR_UNIT + 128) + b_start(half16(tot[u >> 1], u)) + half16(in2[u >> 1] - c2n[u >> 1], u) * 4u,
                 shuffle8(w[u], w[u] >> 4, s2[u]));
    LDS_FENCE();
    u32x4 vv[NU];
#pragma unroll
    for (int u = 0; u < NU; ++u) vv[u] = *(LAS const u32x4*)(scr + u * SCR_UNIT + 128 + voff);
#pragma unroll
    for (int u = 0; u < NU; ++u) {
        const unsigned c16 = (b_start(half16(tot[u >> 1], u)) + half16(__builtin_amdgcn_readlane(in2[u >> 1], 63), u) * 4u + 127u) >> 7;
        const char* vg = pk + (size_t)(g[u] >> 1) * PAIRB + VMID + ((g[u] & 1u) ? 16 * (int)x16[u] : -16 * (int)(x16[u] + c16));
        if ((unsigned)lane < c16) st_wt<0>(vg, voff, vv[u]);
        if (__builtin_expect(c16 > 64u, 0) && 64u + lane < c16) st_wt<1024>(vg, voff, *(LAS const u32x4*)(scr + u * SCR_UNIT + 1152 + voff));
        info[u] |= (x16[u] + c16) << 8;
    }
    LDS_FENCE();
}

template <class Mid>
__device__ __forceinline__ void unpack2(const u32x4 (&p)[2][2], const u32x2 (&cc)[2], const unsigned (&info)[2], unsigned la, LAS char* lds,
                                        LAS char* T, LAS char* M, int lane, u32x4 (&o)[2][4], Mid mid) {
    u32x4 X[2] = {{0u, 0u, 0u, 0u}, {0u, 0u, 0u, 0u}};
    LAS char* A[2] = {M - 16u * la, M};
    const bool anyx = ((info[0] | info[1]) >> 16) != 0u;
    if (__builtin_expect(anyx, 0)) {
#pragma unroll
        for (int u = 0; u < 2; ++u) {
            if (info[u] & 0x10000u) {
                const u32x2 mv = *(LAS const u32x2*)(u ? M : M - 16);
                const u64 em = (u64)rfl(mv[0]) | (u64)rfl(mv[1]) << 32;
                const unsigned k = 1u + lane_rank(em);
                if ((em >> lane) & 1ull) X[u] = *(LAS const u32x4*)(u ? M + 16u * k : M - 16 - 16u * k);
                if (u) A[1] = M + 16u * (1u + __builtin_popcountll(em));
            } else {
                const unsigned xv = (info[u] >> 28) << (((info[u] >> 25) & 7u) * 4u), xa = (info[u] >> 23) & 3u;
                const bool me = (unsigned)lane == ((info[u] >> 17) & 63u);
#pragma unroll
                for (unsigned a = 0; a < 4; ++a) X[u][a] = me && xa == a ? xv : 0u;
            }
        }
    }
    unsigned z[2], rr[2];
#pragma unroll
    for (int u = 0; u < 2; ++u) {
        z[u] = *(LAS const unsigned short*)(A[u] + lane * 2);
        rr[u] = *(LAS const unsigned*)(A[u] + 128 + lane * 4);
    }
    unsigned pc[2][3], cnt = 0;
    u32x2 sel[2][4];
#pragma unroll
    for (int u = 0; u < 2; ++u) {
        unsigned c = 0;
#pragma unroll
        for (int k = 0; k < 2; ++k) {
            const unsigned h = cc[u][k] >> 1;
            const unsigned f0 = ~(cc[u][k] | h) & 0x11111111u, f1 = ~(cc[u][k] | h) & 0x44444444u;
            c += __builtin_popcount(f0);
            pc[u][2 * k] = c;
            c += __builtin_popcount(f1);
            if (k == 0) pc[u][1] = c;
            sel[u][2 * k] = lut_at(lds, f0);
            sel[u][2 * k + 1] = *(LAS const u32x2*)(lds + __builtin_amdgcn_udot4(f1, 0x10080402u, (unsigned)LDS_LUT, false));
        }
        cnt |= c << (16 * u);
    }
    unsigned c2[2], f2[2], c2n = 0;
    u32x2 s2[2];
#pragma unroll
    for (int u = 0; u < 2; ++u) {
        c2[u] = (z[u] | (z[u] << 15)) & 0x55555555u;
        asm("" : "+v"(c2[u]));
        f2[u] = c2[u] & ~(c2[u] >> 2) & 0x11111111u;
        s2[u] = lut_at(lds, f2[u]);
        c2n |= (unsigned)__builtin_popcount(f2[u]) << (16 * u);
        *(LAS unsigned*)(T + u * SCR_T + 256 + lane * 4) = rr[u];
    }
    const unsigned incl = wave_scan(cnt), in2 = wave_scan(c2n), tot = __builtin_amdgcn_readlane(incl, 63);
    unsigned wv[2];
#pragma unroll
    for (int u = 0; u < 2; ++u) wv[u] = get_bits(bit_addr(A[u]) + b_start(half16(tot, u)) + half16(in2 - c2n, u) * 4u);
#pragma unroll
    for (int u = 0; u < 2; ++u) {
        *(LAS unsigned*)(T + u * SCR_T + lane * 4) = (c2[u] ^ f2[u]) | shuffle8(wv[u], wv[u] >> 4, s2[u]);
        if (__builtin_expect(half16(tot, u) > 1024u, 0)) *(LAS u32x2*)(T + u * SCR_T + 512 + lane * 8) = *(LAS const u32x2*)(A[u] + 384 + lane * 8);
    }
    LDS_FENCE();
    unsigned w[2][4];
#pragma unroll
    for (int u = 0; u < 2; ++u) {
        unsigned pos = half16(incl - cnt, u) * 4u + bit_addr(T + u * SCR_T);
        asm("" : "+v"(pos));
#pragma unroll
        for (int a = 0; a < 4; ++a) w[u][a] = get_bits(a ? pos + pc[u][a - 1] * 4u : pos);
    }
    LDS_FENCE();
    mid();
    unsigned e[2][8];
#pragma unroll
    for (int u = 0; u < 2; ++u) {
        const unsigned top4 = (info[u] & 0xffu) * 0x01010101u;
#pragma unroll
        for (int a = 0; a < 4; ++a) {
            const unsigned lo = w[u][a] & 0x0f0f0f0fu, hi = (w[u][a] >> 4) & 0x0f0f0f0fu;
            e[u][a] = top4 - (perm(hi, lo, sel[u][a][0]) | ((cc[u][a >> 1] >> (2 * (a & 1))) & 0x03030303u));
            e[u][4 + a] = top4 - (perm(hi, lo, sel[u][a][1]) | ((cc[u][a >> 1] >> (2 * (a & 1) + 4)) & 0x03030303u));
        }
    }
    if (__builtin_expect(anyx, 0)) {
#pragma unroll
        for (int u = 0; u < 2; ++u)
#pragma unroll
            for (int a = 0; a < 4; ++a) { e[u][a] -= X[u][a] & 0xf0f0f0f0u; e[u][4 + a] -= (X[u][a] << 4) & 0xf0f0f0f0u; }
    }
#pragma unroll
    for (int u = 0; u < 2; ++u)
#pragma unroll
        for (int a = 0; a < 4; ++a) {
            const unsigned q0 = perm(e[u][a], p[u][0][a], 0x05010400u), q1 = perm(e[u][a], p[u][0][a], 0x07030602u);
            const unsigned q2 = perm(e[u][4 + a], p[u][1][a], 0x05010400u), q3 = perm(e[u][4 + a], p[u][1][a], 0x07030602u);
            o[u][0][a] = __builtin_amdgcn_alignbit(q0, q0, 1); o[u][1][a] = __builtin_amdgcn_alignbit(q1, q1, 1);
            o[u][2][a] = __builtin_amdgcn_alignbit(q2, q2, 1); o[u][3][a] = __builtin_amdgcn_alignbit(q3, q3, 1);
        }
}
__device__ __forceinline__ void store_stage(const char* dst, unsigned voff, const u32x4 (&o)[4]) {
    st_wt<0>(dst, voff, o[0]); st_wt<1024>(dst, voff, o[1]); st_wt<2048>(dst, voff, o[2]); st_wt<3072>(dst, voff, o[3]);
}
__device__ __forceinline__ void rows_of(LAS char* buf, int lane, u32x4 (&o)[4]) {
    LAS char* wp = buf + (lane & 15) * 128;
    const unsigned x = (((lane >> 3) & 6) ^ (lane & 7)) * 16;
    LDS_FENCE();
    *(LAS u32x4*)(wp + x) = o[0]; *(LAS u32x4*)(wp + (x ^ 16)) = o[1];
    *(LAS u32x4*)(wp + 2048 + x) = o[2]; *(LAS u32x4*)(wp + 2048 + (x ^ 16)) = o[3];
    LDS_FENCE();
    LAS const char* rp = buf + (lane >> 3) * 128 + (((lane >> 3) ^ lane) & 7) * 16;
#pragma unroll
    for (int c = 0; c < 4; ++c) o[c] = *(LAS const u32x4*)(rp + c * 1024);
    LDS_FENCE();
}
__device__ __forceinline__ void store_out(const char* dst, int lane, const u32x4 (&o)[4]) {
    const unsigned voff = (lane >> 3) * 65536 + (lane & 7) * 16;
#pragma unroll
    for (int c = 0; c < 4; ++c) st_out<0>(dst + c * (8 * 65536), voff, o[c]);
}

__device__ __forceinline__ void pack_piece(const Params& P, char* ws, const char* px, const char* pw, LAS char* lds, int round, unsigned e,
                                           unsigned cntv, int lane, int wid) {
    LAS unsigned* lrec = (LAS unsigned*)(lds + LDS_REC) + 32 + round * 8;
    const int li = lane & 15, lg = lane >> 4;
    unsigned* ready = (unsigned*)(ws + OFF_READY);
    const int h = (e >> 3) & 15, idx = (e & 7) * 8 + wid, id = h * 64 + idx;
    const unsigned kind = e >> 7;
    const bool isx = kind == 0u || kind == 9u;
    const unsigned dest = (kind - 1u) & 7u;
    const bool own = kind == 9u || kind == (unsigned)P.rank + 1u;
    const unsigned rdest = isx ? (unsigned)lane >> 3 : dest;
    char* peer = P.ws[rdest];
    u32x4 d[4];
    u32x2 v[8];
    if (isx) {
        const char* src = px + (size_t)(64 * idx + (lane >> 2)) * 1024 + h * 64 + ((lane & 3) ^ a_swz(lg)) * 16;
#pragma unroll
        for (int c = 0; c < 4; ++c) {
            const u32x4 t = *(const u32x4*)(src + 16 * c * 1024);
            v[2 * c] = (u32x2){t[0], t[1]}; v[2 * c + 1] = (u32x2){t[2], t[3]};
        }
    } else {
        const char* src = pw + (size_t)(32 * h + 8 * lg) * 65536 + (size_t)(dest * 4096 + 64 * idx + 4 * li) * 2;
#pragma unroll
        for (int j = 0; j < 8; ++j) v[j] = *(const u32x2*)(src + (size_t)j * 65536);
    }
    __builtin_amdgcn_sched_barrier(0);
    if (round == 0) {
        build_lut(lds, false, threadIdx.x);
        if (threadIdx.x >= 448) ((LAS unsigned*)(lds + LDS_FLG))[(threadIdx.x - 448) * 32 + 16] = 0u;
        asm volatile("s_waitcnt lgkmcnt(0)" ::: "memory");
        __builtin_amdgcn_s_barrier();
        asm volatile("" ::: "memory");
    }
    if (isx) {
#pragma unroll
        for (int c = 0; c < 4; ++c) d[c] = (u32x4){v[2 * c][0], v[2 * c][1], v[2 * c + 1][0], v[2 * c + 1][1]};
    } else {
#pragma unroll
        for (int hf = 0; hf < 2; ++hf)
#pragma unroll
            for (int a = 0; a < 4; ++a) {
                d[2 * hf][a] = perm(v[2 * a + 1][hf], v[2 * a][hf], 0x05040100u);
                d[2 * hf + 1][a] = perm(v[2 * a + 1][hf], v[2 * a][hf], 0x07060302u);
            }
    }
    asm volatile("" :: "v"(peer));
    const unsigned call = rfl(cntv) + 1u, par = call & 1u;
    if (own) {
        store_stage(ws + (isx ? OFF_A : OFF_B) + (size_t)(h * 256 + 4 * idx) * 1024, lane * 16, d);
        wait_vm0();
        if (lane == 0) __hip_atomic_store(ready + ((isx ? 0 : 64) + idx) * NSLOT + h, call, __ATOMIC_RELAXED, __HIP_MEMORY_SCOPE_SYSTEM);
        return;
    }
    const unsigned g[1] = {(unsigned)(isx ? id : 1024 + (int)dest * 1024 + id)};
    unsigned r[1];
    pack_store<1>([&](int, u32x4 (&o)[4]) { o[0] = d[0]; o[1] = d[1]; o[2] = d[2]; o[3] = d[3]; }, ws + OFF_PK + par * PK_PAR, g, lds, lds + wid * SCR_WAVE, lane, r, [] {});
    if (lane == 0) lrec[wid] = r[0];
    wait_vm0();
    asm volatile("s_waitcnt lgkmcnt(0)" ::: "memory");
    __builtin_amdgcn_s_barrier();
    asm volatile("" ::: "memory");
    if (wid == 0 && (isx || lane < 8) && rdest != (unsigned)P.rank) {
        u64* rp = (u64*)(peer + OFF_REC + par * REC_PAR) + (P.rank * 3 + (isx ? 0 : 1)) * 1024 + h * 64 + (e & 7) * 8 + (lane & 7);
        __hip_atomic_store(rp, ((u64)call << 42) | lrec[lane & 7], __ATOMIC_RELAXED, __HIP_MEMORY_SCOPE_SYSTEM);
    }
}

__device__ __forceinline__ void pack_bulk(const Params& P, char* ws, const char* pw, LAS char* lds, unsigned rank, unsigned call, int lane, int wid) {
    unsigned i = P.dstart + 2u * rank, round = 0;
    const int li = lane & 15, lg = lane >> 4;
    const unsigned par = call & 1u;
    const unsigned lsrc = (unsigned)(8 * lg) * 65536u + (unsigned)(64 * wid + 4 * li) * 2u;
    u32x2 v[2][8];
    unsigned en[2];
    auto load = [&](unsigned i, int u) {
        const unsigned k = (i + u) / 7u, dest = (P.rank + 1u + (i + u) % 7u) & 7u, hp = (P.worder[k >> 2] >> ((k & 3u) * 8u)) & 127u;
        en[u] = dest << 7 | hp;
        const char* src = pw + (size_t)(hp >> 3) * (32 * 65536) + (dest * 4096u + (hp & 7u) * 512u) * 2u;
#pragma unroll
        for (int j = 0; j < 8; ++j) v[u][j] = *(const u32x2*)(src + (size_t)j * 65536 + lsrc);
    };
    load(i, 0);
    load(i, 1);
    __builtin_amdgcn_s_waitcnt(0x0F70);
    for (LAS unsigned* lrec = (LAS unsigned*)(lds + LDS_FLG);; lrec += 32) {
        const unsigned el = (lane & 8) ? en[1] : en[0];
        char* peer = P.ws[el >> 7];
        const unsigned g[2] = {1024u + en[0] * 8u + wid, 1024u + en[1] * 8u + wid};
        u32x4 d[2][4];
#pragma unroll
        for (int u = 0; u < 2; ++u)
#pragma unroll
            for (int hf = 0; hf < 2; ++hf)
#pragma unroll
                for (int a = 0; a < 4; ++a) {
                    d[u][2 * hf][a] = perm(v[u][2 * a + 1][hf], v[u][2 * a][hf], 0x05040100u);
                    d[u][2 * hf + 1][a] = perm(v[u][2 * a + 1][hf], v[u][2 * a][hf], 0x07060302u);
                }
        asm volatile("" : "+v"(d[0][0]), "+v"(d[0][1]), "+v"(d[0][2]), "+v"(d[0][3]), "+v"(d[1][0]), "+v"(d[1][1]), "+v"(d[1][2]), "+v"(d[1][3]));
        i = rank < P.dcount[round + 1u] ? i + 2u * P.dcount[round] : 896u;
        ++round;
        if (i < 896u) load(i, 0);
        __builtin_amdgcn_sched_barrier(0);
        unsigned r[2];
        pack_store<2>([&](int u, u32x4 (&o)[4]) { o[0] = d[u][0]; o[1] = d[u][1]; o[2] = d[u][2]; o[3] = d[u][3]; },
                      ws + OFF_PK + par * PK_PAR, g, lds, lds + wid * SCR_WAVE, lane, r, [&] { if (i < 896u) load(i, 1); });
        if (lane < 2) lrec[lane * 8 + wid] = lane ? r[1] : r[0];
        wait_vm0();
        __builtin_amdgcn_s_waitcnt(0x0F70);
        unsigned arrived = 0;
        if (lane == 0) arrived = __hip_atomic_fetch_add(lrec + 16, 1u, __ATOMIC_RELAXED, __HIP_MEMORY_SCOPE_WORKGROUP);
        LDS_FENCE();
        if (rfl(arrived) == 7u && lane < 16) {
            u64* rp = (u64*)(peer + OFF_REC + par * REC_PAR) + (P.rank * 3 + 1) * 1024 + (el & 127u) * 8u + (lane & 7);
            __hip_atomic_store(rp, ((u64)call << 42) | lrec[lane], __ATOMIC_RELAXED, __HIP_MEMORY_SCOPE_SYSTEM);
        }
        if (i >= 896u) break;
    }
}

constexpr int PSLOT = 15232;
constexpr int PSLOT_M = 2 * FIXB + VCAP;
constexpr int PSLOT_T = PSLOT - 4096;
constexpr int LDS_Q = LDS_REC;
constexpr int LDS_QTAIL = LDS_Q + 256;
constexpr int LDS_FULL = LDS_QTAIL + 16;
constexpr int FLAG_E = (NFLY - 1) / 6 + 1, FLAG_J = (NFLY - 1) % 6;

__device__ __forceinline__ unsigned lds_poll(LAS char* p) {
    return rfl(__hip_atomic_load((LAS unsigned*)p, __ATOMIC_RELAXED, __HIP_MEMORY_SCOPE_WORKGROUP));
}

__device__ __forceinline__ void fetch_link(const Params& P, LAS char* lds, int q, unsigned call, int lane) {
    const unsigned p = (unsigned)(P.rank + 1 + q) & 7u;
    const char* pkb = P.ws[p] + OFF_PK + (call & 1u) * PK_PAR;
    const unsigned ldsbase = (unsigned)(size_t)lds;
    const unsigned v0 = lane * 16, v1 = v0 + 4096, v2 = v0 + (VMID - 512);
    unsigned fa[3] = {0, 0, 0}, fv[3] = {0, 0, 0};
    unsigned ended = 0;
    for (unsigned i = 0;; ++i) {
        LAS char* e = lds + LDS_Q + (i & 15u) * 16u;
        u32x4 ev;
        for (;;) {
            ev = *(volatile LAS u32x4*)e;
            if (rfl(ev[0]) == i + 1u) break;
            if (fa[0] | fa[1] | fa[2]) {
                if (fa[2]) { asm volatile("s_waitcnt vmcnt(12)" ::: "memory"); *(volatile LAS unsigned*)(lds + fa[2]) = fv[2]; fa[2] = 0; }
                else if (fa[1]) { asm volatile("s_waitcnt vmcnt(6)" ::: "memory"); *(volatile LAS unsigned*)(lds + fa[1]) = fv[1]; fa[1] = 0; }
                else { asm volatile("s_waitcnt vmcnt(0)" ::: "memory"); *(volatile LAS unsigned*)(lds + fa[0]) = fv[0]; fa[0] = 0; }
            } else {
                if (ended == NDEC) return;
                __builtin_amdgcn_s_sleep(1);
            }
        }
        const unsigned w1 = rfl(ev[1]);
        if (w1 >> 31) { ++ended; continue; }
        const unsigned owner = w1 >> 24, slot = ldsbase + (owner - 1u) * PSLOT;
        const char* pk = pkb + (size_t)(w1 & 0xffffffu) * PAIRB;
        const u64 vmask = (u64)rfl(ev[2]) | (u64)rfl(ev[3]) << 32;
        const unsigned da = ldsbase + (fa[FLAG_E - 1] ? fa[FLAG_E - 1] : (unsigned)LDS_FULL), dv = fv[FLAG_E - 1];
#define FW(j) "s_waitcnt vmcnt(%10)\n\t.if %11 == " #j "\n\tds_write_b32 %8, %9\n\t.endif\n\t"
        asm volatile("s_mov_b32 m0, %7\n\t"
                     FW(0) "s_mov_b64 exec, %4\n\t"
                     "global_load_lds_dwordx4 %2, %3 sc0 sc1\n\t"
                     "s_mov_b64 exec, -1\n\t"
                     "s_mov_b32 m0, %5\n\t"
                     FW(1) "global_load_lds_dwordx4 %0, %3 offset:2048 sc0 sc1\n\t"
                     FW(2) "global_load_lds_dwordx4 %0, %3 sc0 sc1\n\t"
                     FW(3) "global_load_lds_dwordx4 %0, %3 offset:1024 sc0 sc1\n\t"
                     FW(4) "global_load_lds_dwordx4 %0, %3 offset:3072 sc0 sc1\n\t"
                     "s_mov_b32 m0, %6\n\t"
                     FW(5) "global_load_lds_dwordx4 %1, %3 sc0 sc1"
                     :: "v"(v0), "v"(v1), "v"(v2), "s"(pk), "s"(vmask), "s"(slot), "s"(slot + 4096u), "s"(slot + (PSLOT_M - 512)),
                        "v"(da), "v"(dv), "i"(NFLY - 1), "i"(FLAG_J)
                     : "memory");
#undef FW
        fa[FLAG_E - 1] = 0;
        fa[2] = fa[1]; fv[2] = fv[1]; fa[1] = fa[0]; fv[1] = fv[0];
        fa[0] = LDS_FULL + owner * 4u; fv[0] = i + 1u;
    }
}

struct Desc { unsigned g, dst, rdy, rec; };

__device__ __forceinline__ void decode_link(const Params& P, LAS char* lds, int q, int w, unsigned call, int lane, int wid) {
    char* ws = P.me;
    const unsigned par = call & 1u;
    const unsigned p = (unsigned)(P.rank + 1 + q) & 7u;
    unsigned* tmo = (unsigned*)(ws + OFF_TMO);
    unsigned vorder = P.order[lane];
    asm volatile("" : "+v"(vorder));
    const unsigned rec_base = (unsigned)(OFF_REC + par * REC_PAR) + p * 3072u * 8u;
    auto describe = [&](unsigned n) {
        Desc d;
        if (n < 1024u) {
            const unsigned e = ((unsigned)__builtin_amdgcn_readlane(vorder, n >> 4) >> ((n & 12u) * 2u)) & 255u;
            const unsigned type = e >> 7, h = (e >> 3) & 15u, idx = (e & 7u) * 8u + (n & 3u) * 2u, id = h * 64u + idx, slot = 16u + 7u * h + q;
            d.g = type ? 1024u + P.rank * 1024u + id : id;
            d.dst = (unsigned)(type ? OFF_B : OFF_A) + (slot * 256u + 4u * idx) * 1024u;
            d.rdy = (unsigned)OFF_READY + ((type * 64u + idx) * NSLOT + slot) * 4u;
            d.rec = rec_base + (type * 1024u + id) * 8u;
        } else {
            d.g = 9216u; d.dst = P.rank * 8192u; d.rdy = 0;
            d.rec = rec_base + (2048u + 2u * (n & 511u)) * 8u;
        }
        return d;
    };
    auto accept = [&](u64 v, Desc& d, unsigned& ia, unsigned& ib) {
        if (__builtin_amdgcn_ballot_w64((unsigned)(v >> 42) != (call & 0x3fffffu)) != 0ull) return false;
        ia = __builtin_amdgcn_readlane((unsigned)v, 0); ib = __builtin_amdgcn_readlane((unsigned)v, 1);
        if (!d.rdy) {
            const unsigned id = __builtin_amdgcn_readlane((unsigned)(v >> 32), 0) & 1022u;
            d.g = 9216u + P.rank * 1024u + id;
            d.dst = (id & 15u) * (32u * 65536u) + (p * 4096u + 64u * (id >> 4)) * 2u;
        }
        return true;
    };
    const char* pkb = P.ws[p] + OFF_PK + par * PK_PAR;
    const unsigned voff = lane * 16;
    const unsigned tk_base = (call - 1u) * (NTK - PW) - 2u * PW;
    LAS char* S = lds + (wid - 1) * PSLOT;
    LAS char* M = S + PSLOT_M;
    bool dead = false;
    auto vlen = [](unsigned i) { return min((i >> 8) & 255u, (unsigned)VCAP / 16u); };
    auto post = [&](unsigned w1, unsigned ia, unsigned ib) {
        const unsigned la = vlen(ia), lb = vlen(ib);
        const unsigned mlo = la >= 32u ? ~0u : ~(~0u >> la), mhi = lb >= 32u ? ~0u : ((1u << lb) - 1u) | 1u;
        unsigned i = 0;
        asm volatile("s_waitcnt lgkmcnt(0)" ::: "memory");
        if (lane == 0) {
            i = __hip_atomic_fetch_add((LAS unsigned*)(lds + LDS_QTAIL), 1u, __ATOMIC_RELAXED, __HIP_MEMORY_SCOPE_WORKGROUP);
            *(volatile LAS u32x4*)(lds + LDS_Q + (i & 15u) * 16u) = (u32x4){i + 1u, w1, mlo, mhi};
        }
        return rfl(i) + 1u;
    };
    auto poll = [&](Desc& d, unsigned& ia, unsigned& ib) {
        const u64 t0 = __builtin_amdgcn_s_memrealtime();
        unsigned spins = 0;
        ia = ib = 0;
        while (!dead) {
            const u64 v = __hip_atomic_load((const u64*)(ws + d.rec) + (lane & 1), __ATOMIC_RELAXED, __HIP_MEMORY_SCOPE_SYSTEM);
            if (accept(v, d, ia, ib)) break;
            if (timed_out(tmo, call, t0, spins)) dead = true;
            __builtin_amdgcn_s_sleep(1);
        }
    };
    unsigned n1 = PW + w, ia, ib;
    Desc d0 = describe(w), d1 = describe(n1);
    poll(d0, ia, ib);
    STAMP(ws, 2);
    unsigned want = post(d0.g >> 1 | wid << 24, ia, ib);
    for (;;) {
        const bool more = n1 < NTK;
        const u64 rv = __hip_atomic_load((const u64*)(ws + d1.rec) + (lane & 1), __ATOMIC_RELAXED, __HIP_MEMORY_SCOPE_SYSTEM);
        while (lds_poll(lds + LDS_FULL + wid * 4) != want) __builtin_amdgcn_s_sleep(1);
        LDS_FENCE();
        const unsigned la = vlen(ia), lb = vlen(ib);
        if (la > 32u || lb > 32u) {
            const char* mid = pkb + (size_t)(d0.g >> 1) * PAIRB + VMID;
            for (int c = -3; c <= 3; ++c) {
                const int off = c * 1024 - 512 + (int)voff;
                if (c != 0 && off >= -16 * (int)la && off < 16 * (int)lb) {
                    u32x4 t;
                    asm volatile("global_load_dwordx4 %0, %1, off sc0 sc1\n\ts_waitcnt vmcnt(0)" : "=&v"(t) : "v"(mid + off) : "memory");
                    *(LAS u32x4*)(M + off) = t;
                }
            }
            LDS_FENCE();
        }
        u32x4 o[2][4];
        unsigned ja = 0, jb = 0;
        bool posted = false;
        {
            u32x4 pp[2][2];
            u32x2 cc[2];
#pragma unroll
            for (int u = 0; u < 2; ++u) {
                cc[u] = *(LAS const u32x2*)(S + u * FIXB + 2048 + lane * 8);
                pp[u][0] = *(LAS const u32x4*)(S + u * FIXB + voff);
                pp[u][1] = *(LAS const u32x4*)(S + u * FIXB + 1024 + voff);
            }
            const unsigned inf[2] = {ia, ib};
            unpack2(pp, cc, inf, la, lds, S + PSLOT_T, M, lane, o, [&] {
                asm volatile("" : "+v"(pp[0][0]), "+v"(pp[0][1]), "+v"(pp[1][0]), "+v"(pp[1][1]));
                if (more && accept(rv, d1, ja, jb)) {
                    want = post(d1.g >> 1 | wid << 24, ja, jb);
                    posted = true;
                }
            });
        }
        if (!d0.rdy) {
            if (dead) {
#pragma unroll
                for (int c = 0; c < 8; ++c) o[c >> 2][c & 3] = (u32x4){0x7fc07fc0u, 0x7fc07fc0u, 0x7fc07fc0u, 0x7fc07fc0u};
            }
            rows_of(S + PSLOT_T, lane, o[0]);
            rows_of(S + PSLOT_T, lane, o[1]);
        }
        unsigned tk;
        u64 rv2;
        {
            const unsigned step = d0.rdy ? 1024u : 8u * 65536u, v0 = d0.rdy ? voff : (lane >> 3) * 65536u + (lane & 7) * 16u;
            const char* b0 = (d0.rdy ? ws : P.out) + d0.dst;
            asm volatile("global_load_dwordx2 %1, %2, %5 sc0 sc1\n\t"
                         "s_mov_b64 exec, 1\n\tglobal_atomic_add %0, %3, %4, %5 sc0\n\ts_mov_b64 exec, -1\n\t"
                         "global_store_dwordx4 %8, %12, %6 sc0 sc1\n\tglobal_store_dwordx4 %9, %13, %6 sc0 sc1\n\t"
                         "global_store_dwordx4 %10, %14, %6 sc0 sc1\n\tglobal_store_dwordx4 %11, %15, %6 sc0 sc1\n\t"
                         "global_store_dwordx4 %8, %16, %7 sc0 sc1\n\tglobal_store_dwordx4 %9, %17, %7 sc0 sc1\n\t"
                         "global_store_dwordx4 %10, %18, %7 sc0 sc1\n\tglobal_store_dwordx4 %11, %19, %7 sc0 sc1\n\t"
                         "s_nop 1\n\ts_waitcnt vmcnt(8)"
                         : "=&v"(tk), "=&v"(rv2)
                         : "v"(d1.rec + (lane & 1) * 8u), "v"((unsigned)OFF_TKT + q * 128u), "v"(more ? 1u : 0u), "s"(ws), "s"(b0), "s"(b0 + 4u * step),
                           "v"(v0), "v"(v0 + step), "v"(v0 + 2u * step), "v"(v0 + 3u * step),
                           "v"(o[0][0]), "v"(o[0][1]), "v"(o[0][2]), "v"(o[0][3]), "v"(o[1][0]), "v"(o[1][1]), "v"(o[1][2]), "v"(o[1][3])
                         : "memory");
        }
        if (more && !posted && accept(rv2, d1, ja, jb)) {
            want = post(d1.g >> 1 | wid << 24, ja, jb);
            posted = true;
        }
        if (d0.rdy) {
            wait_vm0();
            if (lane < 2) __hip_atomic_store((unsigned*)(ws + d0.rdy) + lane * NSLOT, call, __ATOMIC_RELAXED, __HIP_MEMORY_SCOPE_SYSTEM);
        }
        const unsigned n2 = rfl(tk) - tk_base;
        if (!more) break;
        if (!posted) {
            poll(d1, ja, jb);
            want = post(d1.g >> 1 | wid << 24, ja, jb);
        }
        ia = ja; ib = jb;
        d0 = d1;
        n1 = n2;
        d1 = describe(n1);
    }
    post(0x80000000u, 0u, 0u);
}

__device__ __forceinline__ void glds4(unsigned m0v, unsigned voff, const char* sb) {
    asm volatile("s_mov_b32 m0, %0\n\ts_nop 0\n\t"
                 "global_load_lds_dwordx4 %1, %2 sc0 sc1\n\t"
                 "global_load_lds_dwordx4 %1, %2 offset:1024 sc0 sc1\n\t"
                 "global_load_lds_dwordx4 %1, %2 offset:2048 sc0 sc1\n\t"
                 "global_load_lds_dwordx4 %1, %2 offset:3072 sc0 sc1" :: "s"(m0v), "v"(voff), "s"(sb) : "memory");
}
__device__ __forceinline__ void glds1(unsigned m0v, unsigned voff, const char* sb) {
    asm volatile("s_mov_b32 m0, %0\n\ts_nop 0\n\tglobal_load_lds_dwordx4 %1, %2 sc0 sc1" :: "s"(m0v), "v"(voff), "s"(sb) : "memory");
}

template <int WR, bool BIG>
__device__ __forceinline__ void k_loop(f32x4 (&acc)[BIG ? 10 : 8][4], char* ws, LAS char* lds, int u0, int pn, unsigned call, int lane, int wid) {
    constexpr int MT = BIG ? 10 : 8;
    constexpr bool has5 = BIG && !WR;
    unsigned* tmo = (unsigned*)(ws + OFF_TMO);
    const int wc = wid & 3;
    const unsigned ldsbase = (unsigned)(size_t)lds;
    const unsigned u1 = WR ? 4 * pn + wc : u0 + wc;
    const char* g1 = ws + (WR ? OFF_B : OFF_A) + (size_t)u1 * 4096;
    const char* g2 = ws + OFF_A + (size_t)(4 * u0 + 16 + wc) * 1024;
    const unsigned l1 = ldsbase + WR * 20480 + wc * 4096, l2 = ldsbase + (16 + wc) * 1024;
    const unsigned voff = lane * 16;
    const unsigned vrow = (unsigned)OFF_READY + (WR * 64 + (lane >= 32 && has5 ? u0 + 4 : u1)) * (NSLOT * 4);
    const unsigned vsnap = vrow + (lane & 31) * 16;
    LAS char* snap = lds + LDS_FLG + wid * 1024;
    *(LAS u32x4*)(snap + voff) = (u32x4){0u, 0u, 0u, 0u};
    asm volatile("s_waitcnt lgkmcnt(0)" ::: "memory");
    unsigned hz = 0;
#define ADVANCE(v) ({ const u64 m_ = __builtin_amdgcn_ballot_w64((v) >= call); \
    (unsigned)__builtin_ctzll(~(u64)((unsigned)m_ & (unsigned)(m_ >> 32))); })
#define ENSURE(t) do { if (__builtin_expect(hz <= (unsigned)(t), 0)) { const u64 t0_ = __builtin_amdgcn_s_memrealtime(); unsigned sp_ = 0; \
        __builtin_amdgcn_s_waitcnt(0xC07F); \
        for (;;) { \
            const unsigned a_ = __hip_atomic_load((const unsigned*)(ws + vrow) + min(hz + (lane & 31u), NSLOT - 1u), __ATOMIC_RELAXED, __HIP_MEMORY_SCOPE_SYSTEM); \
            hz += ADVANCE(a_); \
            if (hz > (unsigned)(t)) break; \
            if (timed_out(tmo, call, t0_, sp_)) { hz = NSLOT; break; } \
            __builtin_amdgcn_s_sleep(2); } } } while (0)
#define STAGE(t, bt) do { const size_t go_ = (size_t)(t) * 262144; const unsigned lo_ = (bt) * LSLOT; \
    glds4(l1 + lo_, voff, g1 + go_); if (has5) glds1(l2 + lo_, voff, g2 + go_); } while (0)
#define WAITV() asm volatile("s_waitcnt vmcnt(%0)" :: "i"(has5 ? 10 : 8) : "memory")
#pragma unroll 1
    for (int t = 0; t < 3; ++t) { ENSURE(t); STAGE(t, t); }
    WAITV();
    __builtin_amdgcn_s_barrier();
    STAMP(ws, 2);
    if (WR == 1) __builtin_amdgcn_s_barrier();
    const unsigned aoff = ((lane & 15) * 4 + ((lane >> 4) ^ a_swz((lane >> 2) & 3))) * 16, boff = 20480 + wc * 4096 + lane * 16;
    for (unsigned s = 0; s < NSLOT; ++s) {
        const unsigned t = s + 3 < NSLOT ? s + 3 : NSLOT - 1;
        ENSURE(t);
        STAGE(t, (s + 3) & 3);

        __builtin_amdgcn_sched_barrier(0);
        bf16x8 A[MT], B[4];
        const unsigned so = (s & 3) * LSLOT;
        LAS const char* pa = lds + aoff + so; LAS const char* pb = lds + boff + so;
#pragma unroll
        for (int n = 0; n < 4; ++n) B[n] = *(LAS const bf16x8*)(pb + n * 1024);
#pragma unroll
        for (int m = 0; m < MT; ++m) A[m] = *(LAS const bf16x8*)(pa + (m < 8 ? 8 * WR + m : 8 + 2 * WR + m) * 1024);
        __builtin_amdgcn_sched_barrier(0);
        if (__builtin_expect((s & FLAG_MASK) == 0 && hz < s + LOOKAHEAD, 0)) {
            const unsigned fv = *(volatile LAS const unsigned*)(snap + (lane >> 5) * 512 + min(hz + (lane & 31u), NSLOT - 1u) * 4u);
            hz += ADVANCE(fv);
            glds1(ldsbase + LDS_FLG + wid * 1024, vsnap, ws);
        }
        WAITV();
        __builtin_amdgcn_s_waitcnt(0xC07F);
        __builtin_amdgcn_sched_barrier(0);
        __builtin_amdgcn_s_barrier();
        __builtin_amdgcn_sched_barrier(0);
        __builtin_amdgcn_s_setprio(1);
#pragma unroll
        for (int m = 0; m < MT; ++m)
#pragma unroll
            for (int n = 0; n < 4; ++n) acc[m][n] = __builtin_amdgcn_mfma_f32_16x16x32_bf16(B[n], A[m], acc[m][n], 0, 0, 0);
        __builtin_amdgcn_s_setprio(0);
        __builtin_amdgcn_sched_barrier(0);
        __builtin_amdgcn_s_barrier();
        __builtin_amdgcn_sched_barrier(0);
    }
    if (WR == 0) __builtin_amdgcn_s_barrier();
#undef ENSURE
#undef ADVANCE
#undef STAGE
#undef WAITV
}

template <bool BIG>
__device__ __forceinline__ void gemm_tile(const Params& P, LAS char* lds, int u0, int pn, unsigned call, int lane, int wid) {
    constexpr int NU = BIG ? 5 : 4;
    char* ws = P.me;
    unsigned* tmo = (unsigned*)(ws + OFF_TMO);
    const int wr = wid >> 2, wc = wid & 3;
    f32x4 acc[2 * NU][4];
#pragma unroll
    for (int m = 0; m < 2 * NU; ++m)
#pragma unroll
        for (int n = 0; n < 4; ++n) acc[m][n] = (f32x4){0.f, 0.f, 0.f, 0.f};
    if (wr == 0) k_loop<0, BIG>(acc, ws, lds, u0, pn, call, lane, wid);
    else k_loop<1, BIG>(acc, ws, lds, u0, pn, call, lane, wid);
    const unsigned tmov = __hip_atomic_load(tmo, __ATOMIC_RELAXED, __HIP_MEMORY_SCOPE_AGENT);
    const unsigned rbl = (lane < 4 ? 2 * u0 + 4 * wr + lane : 2 * u0 + 4 + lane) & 127u;
    char* peer = P.ws[rbl >> 4];
    LAS unsigned* meet = (LAS unsigned*)(lds + LDS_REC + 256) + wc * 4;
    if (BIG && wr == 0 && lane == 0) meet[2] = 0u;
    asm volatile("s_waitcnt lgkmcnt(0)" ::: "memory");
    asm volatile("" :: "v"(peer), "v"(tmov));
    wait_vm0();
    __builtin_amdgcn_s_barrier();
    asm volatile("" ::: "memory");
    STAMP(ws, 3);
    const unsigned par = call & 1u;
    const int cb = 4 * pn + wc;
    if (__builtin_expect(rfl(tmov) == call, 0)) {
#pragma unroll
        for (int m = 0; m < 2 * NU; ++m)
#pragma unroll
            for (int n = 0; n < 4; ++n) acc[m][n] = (f32x4){__builtin_nanf(""), __builtin_nanf(""), __builtin_nanf(""), __builtin_nanf("")};
    }
    unsigned rb[NU], g[NU], nown = 0;
#pragma unroll
    for (int u = 0; u < NU; ++u) {
        rb[u] = (u < 4 ? 2 * u0 + 4 * wr + u : 2 * u0 + 8 + wr) & 127u;
        g[u] = 9216u + (rb[u] >> 4) * 1024u + cb * 16u + (rb[u] & 15u);
        nown += (rb[u] >> 4) == (unsigned)P.rank;
    }
    auto get = [&](int u, u32x4 (&d)[4]) {
#pragma unroll
        for (int c = 0; c < 4; ++c) {
            const int mi = 2 * u + (c >> 1), b = c & 1;
            d[c][0] = cvt_pk(acc[mi][0][2 * b], acc[mi][1][2 * b]);
            d[c][1] = cvt_pk(acc[mi][2][2 * b], acc[mi][3][2 * b]);
            d[c][2] = cvt_pk(acc[mi][0][2 * b + 1], acc[mi][1][2 * b + 1]);
            d[c][3] = cvt_pk(acc[mi][2][2 * b + 1], acc[mi][3][2 * b + 1]);
        }
    };
    if (nown) {
#pragma unroll
        for (int u = 0; u < NU; ++u)
            if ((rb[u] >> 4) == (unsigned)P.rank) {
                u32x4 d[4];
                get(u, d);
                rows_of(lds + wid * SCR_WAVE, lane, d);
                store_out(P.out + (size_t)(rb[u] & 15u) * (32 * 65536) + (size_t)(P.rank * 4096 + 64 * cb) * 2, lane, d);
            }
    }
    if (nown < NU) {
        const bool remote = (rbl >> 4) != (unsigned)P.rank;
        unsigned* opos = (unsigned*)(ws + OFF_OPOS) + (rbl >> 4) * 32;
        u64* ring = (u64*)(peer + OFF_REC + par * REC_PAR) + (P.rank * 3 + 2) * 1024 + (lane & 1);
        const u64 word = (u64)call << 42 | (u64)(cb * 16u + (rbl & 15u)) << 32;
        auto place = [&](bool mine) {
            unsigned pos = 0;
            if (mine && !(lane & 1) && remote) pos = __hip_atomic_fetch_add(opos, 1u, __ATOMIC_RELAXED, __HIP_MEMORY_SCOPE_AGENT);
            asm volatile("" :: "v"(pos));
            wait_vm0();
            return dpp0<0xa0, 0xf>(pos) & 511u;
        };
        char* pk = ws + OFF_PK + par * PK_PAR;
        LAS char* scr = lds + wid * SCR_WAVE;
        unsigned rec, last;
        {
            const unsigned ga[2] = {g[0], g[1]};
            unsigned r[2];
            pack_store<2>(get, pk, ga, lds, scr, lane, r, [] {});
            const unsigned pos = place(lane < 2);
            if (lane < 2 && remote) __hip_atomic_store(ring + 2u * pos, word | (lane ? r[1] : r[0]), __ATOMIC_RELAXED, __HIP_MEMORY_SCOPE_SYSTEM);
        }
        {
            unsigned gb[NU - 2], r[NU - 2];
#pragma unroll
            for (int u = 2; u < NU; ++u) gb[u - 2] = g[u];
            pack_store<NU - 2>([&](int u, u32x4 (&d)[4]) { get(u + 2, d); }, pk, gb, lds, scr, lane, r, [] {});
            rec = (lane & 1) ? r[1] : r[0];
            last = r[NU - 3];
        }
        const bool mine = (lane >> 1) == 1;
        if (BIG && lane == 0) meet[wr] = last;
        const unsigned pos = place(mine || (BIG && wr == 1 && (lane >> 1) == 2));
        if (mine && remote) __hip_atomic_store(ring + 2u * pos, word | rec, __ATOMIC_RELAXED, __HIP_MEMORY_SCOPE_SYSTEM);
        if (BIG) {
            if (wr == 1 && lane == 4) meet[3] = pos;
            unsigned second = 0;
            LDS_FENCE();
            if (lane == 0) second = __hip_atomic_fetch_add(meet + 2, 1u, __ATOMIC_RELAXED, __HIP_MEMORY_SCOPE_WORKGROUP);
            LDS_FENCE();
            if (rfl(second) && (lane >> 1) == 2 && remote)
                __hip_atomic_store(ring + 2u * meet[3], word | meet[lane & 1], __ATOMIC_RELAXED, __HIP_MEMORY_SCOPE_SYSTEM);
        }
    }
}

__global__ void __launch_bounds__(512, 2) fused_gemm_rs(const Params P) {
    __shared__ __attribute__((aligned(16))) char smem[LDS_BYTES];
    const int bid = blockIdx.x;
    const int lane = threadIdx.x & 63, wid = rfl(threadIdx.x >> 6);
    char* ws = P.me; const char* px = P.x; const char* pw = P.w; const u64 job = P.job[bid];
    asm volatile("" :: "s"(ws), "s"(px), "s"(pw), "s"(job), "s"(P.rank));
    STAMP(ws, 0);
    unsigned* cnt = (unsigned*)(ws + OFF_CNT) + bid * 8 + wid;
    const unsigned cntv = __hip_atomic_load(cnt, __ATOMIC_RELAXED, __HIP_MEMORY_SCOPE_AGENT);
    if (bid >= 7 * PWG) {
#pragma unroll 1
        for (int r = 0; r < 3; ++r) {
            const unsigned e = (unsigned)(job >> (11 + 11 * r)) & 0x7ffu;
            if ((e >> 7) == 15u) break;
            pack_piece(P, ws, px, pw, (LAS char*)smem, r, e, cntv, lane, wid);
        }
        if ((unsigned)(job >> 44) < P.dcount[0]) pack_bulk(P, ws, pw, (LAS char*)smem, (unsigned)(job >> 44), rfl(cntv) + 1u, lane, wid);
        asm volatile("s_waitcnt lgkmcnt(0)" ::: "memory");
        __builtin_amdgcn_s_barrier();
    } else {
        build_lut((LAS char*)smem, true, threadIdx.x);
        if (threadIdx.x >= 384) ((LAS unsigned*)(smem + LDS_Q))[threadIdx.x - 384] = 0u;
        asm volatile("s_waitcnt lgkmcnt(0)" ::: "memory");
        __builtin_amdgcn_s_barrier();
        asm volatile("" ::: "memory");
    }
    const unsigned call = rfl(cntv) + 1u;
    if (lane == 0) __hip_atomic_store(cnt, call, __ATOMIC_RELAXED, __HIP_MEMORY_SCOPE_AGENT);
    STAMP(ws, 1);
    if (bid < 7 * PWG) {
        if (wid == 0) fetch_link(P, (LAS char*)smem, bid % 7, call, lane);
        else if (wid <= NDEC) decode_link(P, (LAS char*)smem, bid % 7, (wid - 1) * PWG + bid / 7, call, lane, wid);
    } else if (((unsigned)job & 63u) != 63u) {
        if (job & 64u) gemm_tile<true>(P, (LAS char*)smem, (unsigned)job & 63u, ((unsigned)job >> 7) & 15u, call, lane, wid);
        else gemm_tile<false>(P, (LAS char*)smem, (unsigned)job & 63u, ((unsigned)job >> 7) & 15u, call, lane, wid);
    }
    STAMP(ws, 4);
}

#define LAGS {-1, 0, 19, 29, 41, 52, 63, 74}
#define DCOUNT {104, 104, 78, 52}

void make_plan(Params& P, int rank) {
    const int lag[8] = LAGS;
    unsigned char* order = (unsigned char*)P.order;
    unsigned char* worder = (unsigned char*)P.worder;
    int no = 0, nw = 0;
    for (int pos = -1; pos < 512; ++pos) {
        if (pos >= 0 && pos < 128) order[no++] = (unsigned char)pos;
        for (int pr = 0; pr < 8; ++pr) {
            const int h8 = pos - lag[pr];
            if (h8 >= 0 && h8 < 128 && h8 % 8 == 0) {
                order[no++] = (unsigned char)(128 | h8 | pr);
                worder[nw++] = (unsigned char)(h8 | pr);
            }
        }
    }
    int seq[256], nseq = 0;
    {
        int big[256], nbig = 0;
        const int mix[8] = MIX;
        for (int qt = 0; qt < NBAND; ++qt)
            for (int i = 0; i < 8; ++i) {
                const int pr = mix[i], band = (qt + pr) % NBAND;
                if (pr < NSMALL) continue;
                for (int r = (13 * band + NBAND / 2) / NBAND; r < (13 * (band + 1) + NBAND / 2) / NBAND; ++r)
                    for (int c = 0; c < 2; ++c) big[nbig++] = (5 * r | (r < 12) << 6) | (2 * pr + c) << 7;
            }
        for (int x = 0, k = 0; x < 8; ++x) {
            int room = (256 - 7 * PWG + x) / 8;
            for (int pr = 0; pr < NSMALL; ++pr)
                for (int i = 0; i < 4; ++i, --room) seq[nseq++] = 4 * (2 * x + (i >> 1)) | (2 * pr + (i & 1)) << 7;
            for (; room > 0 && k < nbig; --room) seq[nseq++] = big[k++];
        }
    }
    int tile[256], byp[256], nby = 0;
    static_assert(PWG == 6 && NSMALL == 1, "214 tiles on 214 workgroups");
    for (int x = 0, k = 0; x < 8; ++x)
        for (int b = 248 + x; b >= 7 * PWG; b -= 8) tile[b] = seq[k++];
    for (int pr = 0; pr <= 8; ++pr)
        for (int b = 7 * PWG; b < 256; ++b)
            if ((tile[b] == 63 ? 8 : tile[b] >> 8) == pr) byp[nby++] = b;
    int first[10];
    for (int pr = 0, i = 0; pr <= 9; ++pr) {
        while (i < nby && (tile[byp[i]] == 63 ? 8 : tile[byp[i]] >> 8) < pr) ++i;
        first[pr] = i;
    }
    int nr = 0;
    for (int b = 0; b < 256; ++b) P.job[b] = (u64)(b < 7 * PWG ? 63 : tile[b]) | 0x1ffffffffull << 11 | 511ull << 44;
    for (int r = 0; r < 3; ++r) {
        int list[256], n = 0;
        const int own = (rank + 1) << 7, from = r == 0 ? 0 : first[2 * r];
        if (r < 2) {
            for (int h = 8 * r; h < 8 * r + 8; ++h) {
                for (int q = 0; q < 8; ++q) list[n++] = 9 << 7 | h << 3 | q;
                list[n++] = own | h << 3; list[n++] = own | h << 3 | 1;
            }
            for (int h = 2 * r; h < (r ? 16 : 2); ++h) { list[n++] = own | h << 3 | 2; list[n++] = own | h << 3 | 3; }
        } else {
            for (int h = 0; h < 16; ++h)
                for (int q = 4; q < 8; ++q) list[n++] = own | h << 3 | q;
        }
        for (int i = 48 * r; i < (r < 2 ? 48 * r + 48 : 128); ++i) list[n++] = i;
        while (n < nby - from - (r == 2 ? (nr & 1) != ((nby - from - n) & 1) : 0)) { list[n++] = (((rank + 1 + nr % 7) & 7) + 1) << 7 | worder[nr / 7]; ++nr; }
        for (int i = 0; i < n; ++i) {
            u64& j = P.job[byp[from + i]];
            j = (j & ~(0x7ffull << (11 + 11 * r))) | (u64)list[i] << (11 + 11 * r);
        }
    }
    P.dstart = nr;
    const int dcount[] = DCOUNT;
    for (int r = 0, left = (896 - nr) / 2; r < 12; ++r) {
        const int n = dcount[r < (int)(sizeof dcount / sizeof dcount[0]) ? r : sizeof dcount / sizeof dcount[0] - 1];
        P.dcount[r] = r == 11 ? 0 : n < left ? n : left;
        left -= P.dcount[r];
    }
    for (int i = 0; i < nby; ++i) P.job[byp[i]] = (P.job[byp[i]] & ~(511ull << 44)) | (u64)(nby - 1 - i) << 44;
}

}

extern "C" void kernel_launch(int rank, int nranks, ncclComm_t comm, hipStream_t stream,
                              void* const* d_in, const int* in_sizes, int n_in,
                              void* d_out, int out_size, const int* in_dtypes, int out_dtype,
                              void* const* d_out_all, void* const* d_ws_all, size_t ws_bytes) {
    (void)comm; (void)in_dtypes; (void)out_dtype; (void)d_out_all;
    if (nranks != NR || (unsigned)rank >= (unsigned)NR || n_in < 2 || ws_bytes < WS_NEED || in_sizes[0] != 4096 * 512 ||
        in_sizes[1] != 512 * 32768 || out_size != 512 * 32768) {
        fprintf(stderr, "kernel_launch: unexpected problem shape\n");
        return;
    }
    Params P;
    P.x = (const char*)d_in[0];
    P.w = (const char*)d_in[1];
    P.out = (char*)d_out;
    for (int p = 0; p < NR; ++p) P.ws[p] = (char*)d_ws_all[p];
    P.me = P.ws[rank];
    P.rank = rank;
    make_plan(P, rank);
    hipLaunchKernelGGL(fused_gemm_rs, dim3(256), dim3(512), 0, stream, P);
    hipError_t e = hipGetLastError();
    if (e != hipSuccess) fprintf(stderr, "launch failed: %s\n", hipGetErrorString(e));
}
```

```cpp
#include <hip/hip_runtime.h>
#include <rccl/rccl.h>
#include <cstdio>

#define LAS __attribute__((address_space(3)))
typedef __bf16 bf16x8 __attribute__((ext_vector_type(8)));
typedef __bf16 bf16x4 __attribute__((ext_vector_type(4)));
typedef float f32x4 __attribute__((ext_vector_type(4)));
typedef unsigned u32x4 __attribute__((ext_vector_type(4)));
typedef unsigned u32x2 __attribute__((ext_vector_type(2)));
typedef unsigned long long u64;

namespace {

constexpr int NR = 8;
constexpr int NSLOT = 128;
constexpr int LSLOT = 36 * 1024;
constexpr int LDS_LUT = 4 * LSLOT;
constexpr int LDS_REC = LDS_LUT + 2048;
constexpr int LDS_FLG = LDS_REC + 512;
constexpr int LDS_BYTES = LDS_FLG + 8192;
constexpr int SCR_UNIT = 1408;
constexpr int SCR_WAVE = 5 * SCR_UNIT;
constexpr int SCR_T = 1152;
#define PWG 6
#define NDEC 7
#define LOOKAHEAD 10u
#define NFLY 10
#define NBAND 2
#define MIX {0, 7, 3, 4, 1, 6, 2, 5}
#define NPUSH 6
#define PUSH0 5
#define GSLEEP 2
#define FULL14 0xe0
constexpr int PW = PWG * NDEC;
#define DIAG 1
#define LATE_COL 12u
#define NPRE 46u
constexpr unsigned NLATE = (16u - LATE_COL) * 14u;
static_assert(NPRE <= NLATE, "only jobs that wait for nothing run early");
constexpr unsigned NXQ = DIAG ? 7 : 8;
constexpr unsigned NIN = 64 * NXQ + 512;
constexpr unsigned NTK = NIN + 512;
constexpr u64 DEADLINE = 100000000ull;

constexpr size_t OFF_CNT = 0;
constexpr size_t OFF_TMO = 16384;
constexpr size_t OFF_TKT = 20480;
constexpr size_t OFF_OPOS = 24576;
constexpr size_t OFF_DFL = 28672;
constexpr size_t OFF_READY = 32768;
constexpr size_t OFF_OFL = 98304;
constexpr size_t OFF_REC = 131072;
constexpr size_t REC_PAR = (size_t)NR * 3 * 1024 * 8;
constexpr size_t OFF_DBG = 655360;
constexpr size_t OFF_A = 1u << 20;
constexpr size_t OFF_B = OFF_A + (32u << 20);
constexpr int NUNIT = 1024 + 2 * NR * 1024;
constexpr int FIXB = 2560;
constexpr int VCAP = 2624;
constexpr int PAIRB = 2 * FIXB + 2 * VCAP;
constexpr int VMID = 2 * FIXB + VCAP;
constexpr size_t PK_PAR = (size_t)(NUNIT / 2) * PAIRB;
constexpr size_t OFF_PK = OFF_B + (32u << 20);
constexpr size_t WS_NEED = OFF_PK + 2 * PK_PAR;
static_assert(OFF_REC + 2 * REC_PAR <= OFF_DBG, "layout");

struct Params {
    const char* x;
    const char* w;
    char* out;
    char* me;
    char* ws[NR];
    int rank;
    unsigned dstart;
    unsigned dcount[12];
    unsigned pushk[8];
    unsigned order[64];
    unsigned worder[32];
    u64 job[256];
    unsigned djob[64];
};

#define STAMP(ws, idx) do { } while (0)

__device__ __forceinline__ unsigned rfl(unsigned v) { return __builtin_amdgcn_readfirstlane(v); }
__device__ __forceinline__ unsigned perm(unsigned hi, unsigned lo, unsigned sel) { return __builtin_amdgcn_perm(hi, lo, sel); }
__device__ __forceinline__ unsigned bfi(unsigned m, unsigned a, unsigned b) { return (a & m) | (b & ~m); }
__device__ __forceinline__ int a_swz(int q) { return (0x78 >> (2 * q)) & 3; }
__device__ __forceinline__ unsigned cvt_pk(float lo, float hi) { unsigned r; asm volatile("v_cvt_pk_bf16_f32 %0, %1, %2" : "=v"(r) : "v"(lo), "v"(hi)); return r; }

__device__ __forceinline__ u32x4 add_bf16(u32x4 x, u32x4 y) {
    u32x4 r;
#pragma unroll
    for (int a = 0; a < 4; ++a)
        r[a] = cvt_pk(__uint_as_float(x[a] << 16) + __uint_as_float(y[a] << 16), __uint_as_float(x[a] & 0xffff0000u) + __uint_as_float(y[a] & 0xffff0000u));
    return r;
}

template <int OFF> __device__ __forceinline__ void st_wt(const char* sbase, unsigned voff, u32x4 v) {
    asm volatile("global_store_dwordx4 %0, %1, %2 offset:%3 sc0 sc1\n\ts_nop 1" :: "v"(voff), "v"(v), "s"(sbase), "i"(OFF) : "memory");
}
#define OUTBITS " sc0 sc1"
template <int OFF> __device__ __forceinline__ void st_out(const char* sbase, unsigned voff, u32x4 v) {
    asm volatile("global_store_dwordx4 %0, %1, %2 offset:%3" OUTBITS "\n\ts_nop 1" :: "v"(voff), "v"(v), "s"(sbase), "i"(OFF) : "memory");
}
__device__ __forceinline__ void wait_vm0() { asm volatile("s_waitcnt vmcnt(0)" ::: "memory"); }

__device__ __forceinline__ bool timed_out(unsigned* tmo, unsigned call, u64 t0, unsigned& spins) {
    if ((++spins & 63u) != 0u) return false;
    if (rfl(__hip_atomic_load(tmo, __ATOMIC_RELAXED, __HIP_MEMORY_SCOPE_AGENT)) == call) return true;
    if (__builtin_amdgcn_s_memrealtime() - t0 > DEADLINE) {
        __hip_atomic_store(tmo, call, __ATOMIC_RELAXED, __HIP_MEMORY_SCOPE_AGENT);
        return true;
    }
    return false;
}

#define LDS_FENCE() asm volatile("" ::: "memory")
template <int CTRL, int ROWMASK> __device__ __forceinline__ unsigned dpp0(unsigned v) {
    return (unsigned)__builtin_amdgcn_update_dpp(0, (int)v, CTRL, ROWMASK, 0xf, false);
}
__device__ __forceinline__ unsigned wave_scan(unsigned v) {
    v += dpp0<0x111, 0xf>(v); v += dpp0<0x112, 0xf>(v); v += dpp0<0x114, 0xf>(v); v += dpp0<0x118, 0xf>(v);
    v += dpp0<0x142, 0xa>(v); v += dpp0<0x143, 0xc>(v);
    return v;
}
__device__ __forceinline__ unsigned lane_rank(u64 m) {
    return __builtin_amdgcn_mbcnt_hi((unsigned)(m >> 32), __builtin_amdgcn_mbcnt_lo((unsigned)m, 0u));
}
__device__ __forceinline__ void build_lut(LAS char* lds, bool dec, int tid) {
    if (tid < 256) {
        u64 e = 0x0c0c0c0c0c0c0c0cull;
        unsigned t = 0;
#pragma unroll
        for (unsigned j = 0; j < 8; ++j)
            if ((tid >> j) & 1) {
                const unsigned slot = (t & 1u) * 4u + (t >> 1);
                const unsigned at = dec ? j : slot, val = dec ? slot : j;
                e = (e & ~(0xffull << (8 * at))) | ((u64)val << (8 * at));
                ++t;
            }
        *(LAS u64*)(lds + LDS_LUT + tid * 8) = e;
    }
}
__device__ __forceinline__ u32x2 lut_at(LAS char* lds, unsigned f) {
    return *(LAS const u32x2*)(lds + __builtin_amdgcn_udot4(f, 0x40201008u, (unsigned)LDS_LUT, false));
}
__device__ __forceinline__ unsigned shuffle8(unsigned lo, unsigned hi, u32x2 sel) {
    return bfi(0x0f0f0f0fu, perm(hi, lo, sel[0]), perm(hi, lo, sel[1]) << 4);
}
__device__ __forceinline__ unsigned half16(unsigned v, int u) { return (u & 1) ? v >> 16 : v & 0xffffu; }
__device__ __forceinline__ unsigned b_start(unsigned n1) { return n1 <= 512u ? ((n1 + 15u) >> 4) << 5 : ((n1 - 249u) >> 3) << 5; }
__device__ __forceinline__ void lds_or(LAS char* p, unsigned v) {
    __hip_atomic_fetch_or((LAS unsigned*)p, v, __ATOMIC_RELAXED, __HIP_MEMORY_SCOPE_WAVEFRONT);
}
__device__ __forceinline__ unsigned bit_addr(LAS char* p) { return (unsigned)(size_t)p * 8u; }
__device__ __forceinline__ void put_bits(unsigned pos, unsigned v) {
    const u64 sh = (u64)v << (pos & 31u);
    LAS char* p = (LAS char*)(size_t)((pos >> 3) & ~3u);
    lds_or(p, (unsigned)sh); lds_or(p + 4, (unsigned)(sh >> 32));
}
__device__ __forceinline__ unsigned get_bits(unsigned pos) {
    LAS const unsigned* p = (LAS const unsigned*)(size_t)((pos >> 3) & ~3u);
    return __builtin_amdgcn_alignbit(p[1], p[0], pos);
}

template <int NU, class Get, class Mid>
__device__ __forceinline__ void pack_store(Get get, char* pk, const unsigned (&g)[NU], LAS char* lds, LAS char* scr, int lane,
                                           unsigned (&info)[NU], Mid mid) {
    constexpr int NS = (NU + 1) / 2;
    const unsigned voff = lane * 16;
#pragma unroll
    for (int u = 0; u < NU; ++u) *(LAS u32x4*)(scr + u * SCR_UNIT + voff) = (u32x4){0u, 0u, 0u, 0u};
    unsigned D[NU][8], pc[NU][3], cnt[NS], x16[NU], top = 0, top4 = 0;
    u32x2 sel[NU][4];
#pragma unroll
    for (int u = 0; u < NU; ++u) {
        u32x4 d[4], S[2];
        unsigned E[8], C[4], c = 0;
        get(u, d);
#pragma unroll
        for (int hf = 0; hf < 2; ++hf)
#pragma unroll
            for (int a = 0; a < 4; ++a) {
                const unsigned p = __builtin_amdgcn_alignbit(d[2 * hf][a], d[2 * hf][a], 31);
                const unsigned r = __builtin_amdgcn_alignbit(d[2 * hf + 1][a], d[2 * hf + 1][a], 31);
                S[hf][a] = perm(r, p, 0x06040200u);
                E[4 * hf + a] = perm(r, p, 0x07050301u);
            }
        char* pair = pk + (size_t)(g[u] >> 1) * PAIRB;
        const char* fix = pair + (g[u] & 1u) * FIXB;
        if (u == 0) {
            unsigned sum = __builtin_amdgcn_sad_u8(E[0], 0u, __builtin_amdgcn_sad_u8(E[2], 0u, __builtin_amdgcn_sad_u8(E[5], 0u, __builtin_amdgcn_sad_u8(E[7], 0u, 0u))));
            sum = __builtin_amdgcn_readlane(wave_scan(sum), 63);
            top = min(((sum + 983u) >> 10) + 2u, 255u);
            top4 = top * 0x01010101u;
        }
        st_wt<0>(fix, voff, S[0]);
#pragma unroll
        for (int a = 0; a < 4; ++a) {
            D[u][a] = top4 - E[a]; D[u][4 + a] = top4 - E[4 + a];
            unsigned n = bfi(0x0f0f0f0fu, D[u][a], D[u][4 + a] << 4);
            asm("" : "+v"(n));
            unsigned o = n | (n >> 1);
            asm("" : "+v"(o));
            const unsigned f = (~o | (o >> 2)) & 0x11111111u;
            unsigned f3 = (f << 1) + f;
            asm("" : "+v"(f3));
            C[a] = n & 0x33333333u & ~f3;
            c += __builtin_popcount(f);
            if (a < 3) pc[u][a] = c;
            sel[u][a] = lut_at(lds, f);
            if (a == 1) st_wt<1024>(fix, voff, S[1]);
        }
        cnt[u >> 1] = (u & 1) ? cnt[u >> 1] | c << 16 : c;
        const u32x2 cc = {C[0] | (C[1] << 2), C[2] | (C[3] << 2)};
        asm volatile("global_store_dwordx2 %0, %1, %2 offset:2048 sc0 sc1" :: "v"(lane * 8), "v"(cc), "s"(fix) : "memory");
        info[u] = top; x16[u] = 0;
        const u64 em = __builtin_amdgcn_ballot_w64(((D[u][0] | D[u][1] | D[u][2] | D[u][3] | D[u][4] | D[u][5] | D[u][6] | D[u][7]) & 0xf0f0f0f0u) != 0u);
        if (__builtin_expect(em != 0ull, 0)) {
            u32x4 X;
#pragma unroll
            for (int a = 0; a < 4; ++a) X[a] = bfi(0xf0f0f0f0u, D[u][a], D[u][4 + a] >> 4);
            const unsigned l = __builtin_ctzll(em);
            const unsigned x0 = __builtin_amdgcn_readlane(X[0], l), x1 = __builtin_amdgcn_readlane(X[1], l);
            const unsigned x2 = __builtin_amdgcn_readlane(X[2], l), x3 = __builtin_amdgcn_readlane(X[3], l);
            const unsigned any = x0 | x1 | x2 | x3, nz = (x0 != 0u) + (x1 != 0u) + (x2 != 0u) + (x3 != 0u);
            const unsigned nib = __builtin_ctz(any) >> 2, val = any >> (4u * nib);
            if ((em & (em - 1ull)) == 0ull && nz == 1u && val < 16u) {
                info[u] |= (l | (x0 ? 0u : x1 ? 1u : x2 ? 2u : 3u) << 6 | nib << 8 | val << 11) << 17;
            } else {
                info[u] |= 1u << 16;
                x16[u] = 1u + __builtin_popcountll(em);
                const unsigned k = 1u + lane_rank(em);
                if (lane == 0) st_wt<0>(pair, (g[u] & 1u) ? VMID : VMID - 16, (u32x4){(unsigned)em, (unsigned)(em >> 32), 0u, 0u});
                if ((em >> lane) & 1ull) st_wt<0>(pair, (g[u] & 1u) ? VMID + 16u * k : VMID - 16 - 16u * k, X);
            }
        }
    }
    unsigned excl[NS], tot[NS];
#pragma unroll
    for (int k = 0; k < NS; ++k) excl[k] = wave_scan(cnt[k]);
#pragma unroll
    for (int k = 0; k < NS; ++k) { tot[k] = __builtin_amdgcn_readlane(excl[k], 63); excl[k] -= cnt[k]; }
#pragma unroll
    for (int u = 0; u < NU; ++u)
        if (__builtin_expect(half16(tot[u >> 1], u) > 1400u, 0) && lane < 24) *(LAS u32x4*)(scr + u * SCR_UNIT + 1024 + voff) = (u32x4){0u, 0u, 0u, 0u};
    LDS_FENCE();
#pragma unroll
    for (int u = 0; u < NU; ++u) {
        unsigned pos = half16(excl[u >> 1], u) * 4u + bit_addr(scr + u * SCR_UNIT);
        asm("" : "+v"(pos));
#pragma unroll
        for (int a = 0; a < 4; ++a) put_bits(a ? pos + pc[u][a - 1] * 4u : pos, shuffle8(D[u][a], D[u][4 + a], sel[u][a]));
    }
    LDS_FENCE();
    mid();
    unsigned w[NU], c2n[NS];
    u32x2 s2[NU];
#pragma unroll
    for (int u = 0; u < NU; ++u) w[u] = *(LAS const unsigned*)(scr + u * SCR_UNIT + lane * 4);
    LDS_FENCE();
#pragma unroll
    for (int u = 0; u < NU; ++u) {
        const unsigned e = ((w[u] >> 1) | (w[u] >> 3)) & 0x11111111u;
        const unsigned c2 = (w[u] & 0x55555555u & ~(e << 2)) | e;
        s2[u] = lut_at(lds, e);
        const unsigned c = __builtin_popcount(e);
        c2n[u >> 1] = (u & 1) ? c2n[u >> 1] | c << 16 : c;
        *(LAS unsigned short*)(scr + u * SCR_UNIT + 128 + lane * 2) = (unsigned short)(c2 | (c2 >> 15));
    }
    unsigned in2[NS];
#pragma unroll
    for (int k = 0; k < NS; ++k) in2[k] = wave_scan(c2n[k]);
    LDS_FENCE();
#pragma unroll
    for (int u = 0; u < NU; ++u)
        put_bits(bit_addr(scr + u * SCR_UNIT + 128) + b_start(half16(tot[u >> 1], u)) + half16(in2[u >> 1] - c2n[u >> 1], u) * 4u,
                 shuffle8(w[u], w[u] >> 4, s2[u]));
    LDS_FENCE();
    u32x4 vv[NU];
#pragma unroll
    for (int u = 0; u < NU; ++u) vv[u] = *(LAS const u32x4*)(scr + u * SCR_UNIT + 128 + voff);
#pragma unroll
    for (int u = 0; u < NU; ++u) {
        const unsigned c16 = (b_start(half16(tot[u >> 1], u)) + half16(__builtin_amdgcn_readlane(in2[u >> 1], 63), u) * 4u + 127u) >> 7;
        const char* vg = pk + (size_t)(g[u] >> 1) * PAIRB + VMID + ((g[u] & 1u) ? 16 * (int)x16[u] : -16 * (int)(x16[u] + c16));
        if ((unsigned)lane < c16) st_wt<0>(vg, voff, vv[u]);
        if (__builtin_expect(c16 > 64u, 0) && 64u + lane < c16) st_wt<1024>(vg, voff, *(LAS const u32x4*)(scr + u * SCR_UNIT + 1152 + voff));
        info[u] |= (x16[u] + c16) << 8;
    }
    LDS_FENCE();
}

template <class Mid>
__device__ __forceinline__ void unpack2(const u32x4 (&p)[2][2], const u32x2 (&cc)[2], const unsigned (&info)[2], unsigned la, LAS char* lds,
                                        LAS char* T, LAS char* M, int lane, u32x4 (&o)[2][4], Mid mid) {
    u32x4 X[2] = {{0u, 0u, 0u, 0u}, {0u, 0u, 0u, 0u}};
    LAS char* A[2] = {M - 16u * la, M};
    const bool anyx = ((info[0] | info[1]) >> 16) != 0u;
    if (__builtin_expect(anyx, 0)) {
#pragma unroll
        for (int u = 0; u < 2; ++u) {
            if (info[u] & 0x10000u) {
                const u32x2 mv = *(LAS const u32x2*)(u ? M : M - 16);
                const u64 em = (u64)rfl(mv[0]) | (u64)rfl(mv[1]) << 32;
                const unsigned k = 1u + lane_rank(em);
                if ((em >> lane) & 1ull) X[u] = *(LAS const u32x4*)(u ? M + 16u * k : M - 16 - 16u * k);
                if (u) A[1] = M + 16u * (1u + __builtin_popcountll(em));
            } else {
                const unsigned xv = (info[u] >> 28) << (((info[u] >> 25) & 7u) * 4u), xa = (info[u] >> 23) & 3u;
                const bool me = (unsigned)lane == ((info[u] >> 17) & 63u);
#pragma unroll
                for (unsigned a = 0; a < 4; ++a) X[u][a] = me && xa == a ? xv : 0u;
            }
        }
    }
    unsigned z[2], rr[2];
#pragma unroll
    for (int u = 0; u < 2; ++u) {
        z[u] = *(LAS const unsigned short*)(A[u] + lane * 2);
        rr[u] = *(LAS const unsigned*)(A[u] + 128 + lane * 4);
    }
    unsigned pc[2][3], cnt = 0;
    u32x2 sel[2][4];
#pragma unroll
    for (int u = 0; u < 2; ++u) {
        unsigned c = 0;
#pragma unroll
        for (int k = 0; k < 2; ++k) {
            const unsigned h = cc[u][k] >> 1;
            const unsigned f0 = ~(cc[u][k] | h) & 0x11111111u, f1 = ~(cc[u][k] | h) & 0x44444444u;
            c += __builtin_popcount(f0);
            pc[u][2 * k] = c;
            c += __builtin_popcount(f1);
            if (k == 0) pc[u][1] = c;
            sel[u][2 * k] = lut_at(lds, f0);
            sel[u][2 * k + 1] = *(LAS const u32x2*)(lds + __builtin_amdgcn_udot4(f1, 0x10080402u, (unsigned)LDS_LUT, false));
        }
        cnt |= c << (16 * u);
    }
    unsigned c2[2], f2[2], c2n = 0;
    u32x2 s2[2];
#pragma unroll
    for (int u = 0; u < 2; ++u) {
        c2[u] = (z[u] | (z[u] << 15)) & 0x55555555u;
        asm("" : "+v"(c2[u]));
        f2[u] = c2[u] & ~(c2[u] >> 2) & 0x11111111u;
        s2[u] = lut_at(lds, f2[u]);
        c2n |= (unsigned)__builtin_popcount(f2[u]) << (16 * u);
        *(LAS unsigned*)(T + u * SCR_T + 256 + lane * 4) = rr[u];
    }
    const unsigned incl = wave_scan(cnt), in2 = wave_scan(c2n), tot = __builtin_amdgcn_readlane(incl, 63);
    unsigned wv[2];
#pragma unroll
    for (int u = 0; u < 2; ++u) wv[u] = get_bits(bit_addr(A[u]) + b_start(half16(tot, u)) + half16(in2 - c2n, u) * 4u);
#pragma unroll
    for (int u = 0; u < 2; ++u) {
        *(LAS unsigned*)(T + u * SCR_T + lane * 4) = (c2[u] ^ f2[u]) | shuffle8(wv[u], wv[u] >> 4, s2[u]);
        if (__builtin_expect(half16(tot, u) > 1024u, 0)) *(LAS u32x2*)(T + u * SCR_T + 512 + lane * 8) = *(LAS const u32x2*)(A[u] + 384 + lane * 8);
    }
    LDS_FENCE();
    unsigned w[2][4];
#pragma unroll
    for (int u = 0; u < 2; ++u) {
        unsigned pos = half16(incl - cnt, u) * 4u + bit_addr(T + u * SCR_T);
        asm("" : "+v"(pos));
#pragma unroll
        for (int a = 0; a < 4; ++a) w[u][a] = get_bits(a ? pos + pc[u][a - 1] * 4u : pos);
    }
    LDS_FENCE();
    mid();
    unsigned e[2][8];
#pragma unroll
    for (int u = 0; u < 2; ++u) {
        const unsigned top4 = (info[u] & 0xffu) * 0x01010101u;
#pragma unroll
        for (int a = 0; a < 4; ++a) {
            const unsigned lo = w[u][a] & 0x0f0f0f0fu, hi = (w[u][a] >> 4) & 0x0f0f0f0fu;
            e[u][a] = top4 - (perm(hi, lo, sel[u][a][0]) | ((cc[u][a >> 1] >> (2 * (a & 1))) & 0x03030303u));
            e[u][4 + a] = top4 - (perm(hi, lo, sel[u][a][1]) | ((cc[u][a >> 1] >> (2 * (a & 1) + 4)) & 0x03030303u));
        }
    }
    if (__builtin_expect(anyx, 0)) {
#pragma unroll
        for (int u = 0; u < 2; ++u)
#pragma unroll
            for (int a = 0; a < 4; ++a) { e[u][a] -= X[u][a] & 0xf0f0f0f0u; e[u][4 + a] -= (X[u][a] << 4) & 0xf0f0f0f0u; }
    }
#pragma unroll
    for (int u = 0; u < 2; ++u)
#pragma unroll
        for (int a = 0; a < 4; ++a) {
            const unsigned q0 = perm(e[u][a], p[u][0][a], 0x05010400u), q1 = perm(e[u][a], p[u][0][a], 0x07030602u);
            const unsigned q2 = perm(e[u][4 + a], p[u][1][a], 0x05010400u), q3 = perm(e[u][4 + a], p[u][1][a], 0x07030602u);
            o[u][0][a] = __builtin_amdgcn_alignbit(q0, q0, 1); o[u][1][a] = __builtin_amdgcn_alignbit(q1, q1, 1);
            o[u][2][a] = __builtin_amdgcn_alignbit(q2, q2, 1); o[u][3][a] = __builtin_amdgcn_alignbit(q3, q3, 1);
        }
}
__device__ __forceinline__ void store_stage(const char* dst, unsigned voff, const u32x4 (&o)[4]) {
    st_wt<0>(dst, voff, o[0]); st_wt<1024>(dst, voff, o[1]); st_wt<2048>(dst, voff, o[2]); st_wt<3072>(dst, voff, o[3]);
}
__device__ __forceinline__ void rows_of(LAS char* buf, int lane, u32x4 (&o)[4]) {
    LAS char* wp = buf + (lane & 15) * 128;
    const unsigned x = (((lane >> 3) & 6) ^ (lane & 7)) * 16;
    LDS_FENCE();
    *(LAS u32x4*)(wp + x) = o[0]; *(LAS u32x4*)(wp + (x ^ 16)) = o[1];
    *(LAS u32x4*)(wp + 2048 + x) = o[2]; *(LAS u32x4*)(wp + 2048 + (x ^ 16)) = o[3];
    LDS_FENCE();
    LAS const char* rp = buf + (lane >> 3) * 128 + (((lane >> 3) ^ lane) & 7) * 16;
#pragma unroll
    for (int c = 0; c < 4; ++c) o[c] = *(LAS const u32x4*)(rp + c * 1024);
    LDS_FENCE();
}
__device__ __forceinline__ void store_out(const char* dst, int lane, const u32x4 (&o)[4]) {
    const unsigned voff = (lane >> 3) * 65536 + (lane & 7) * 16;
#pragma unroll
    for (int c = 0; c < 4; ++c) st_out<0>(dst + c * (8 * 65536), voff, o[c]);
}

__device__ __forceinline__ void pack_piece(const Params& P, char* ws, const char* px, const char* pw, LAS char* lds, int round, unsigned e,
                                           unsigned inbox, unsigned cntv, int lane, int wid) {
    LAS unsigned* lrec = (LAS unsigned*)(lds + LDS_REC) + 32 + round * 8;
    const int li = lane & 15, lg = lane >> 4;
    unsigned* ready = (unsigned*)(ws + OFF_READY);
    const int h = (e >> 3) & 15, idx = (e & 7) * 8 + wid, id = h * 64 + idx;
    const unsigned kind = e >> 7;
    const bool isx = kind == 0u || kind == 9u;
    const unsigned dest = (kind - 1u) & 7u;
    const bool own = kind == 9u || kind == (unsigned)P.rank + 1u;
    const unsigned rdest = isx ? (unsigned)lane >> 3 : dest;
    char* peer = P.ws[rdest];
    u32x4 d[4];
    u32x2 v[8];
    if (isx) {
        const char* src = px + (size_t)(64 * idx + (lane >> 2)) * 1024 + h * 64 + ((lane & 3) ^ a_swz(lg)) * 16;
#pragma unroll
        for (int c = 0; c < 4; ++c) {
            const u32x4 t = *(const u32x4*)(src + 16 * c * 1024);
            v[2 * c] = (u32x2){t[0], t[1]}; v[2 * c + 1] = (u32x2){t[2], t[3]};
        }
    } else {
        const char* src = pw + (size_t)(32 * h + 8 * lg) * 65536 + (size_t)(dest * 4096 + 64 * idx + 4 * li) * 2;
#pragma unroll
        for (int j = 0; j < 8; ++j) v[j] = *(const u32x2*)(src + (size_t)j * 65536);
    }
    __builtin_amdgcn_sched_barrier(0);
    if (round == 0) {
        build_lut(lds, false, threadIdx.x);
        if (threadIdx.x >= 448) ((LAS unsigned*)(lds + LDS_FLG))[(threadIdx.x - 448) * 32 + 16] = 0u;
        asm volatile("s_waitcnt lgkmcnt(0)" ::: "memory");
        __builtin_amdgcn_s_barrier();
        asm volatile("" ::: "memory");
    }
    if (isx) {
#pragma unroll
        for (int c = 0; c < 4; ++c) d[c] = (u32x4){v[2 * c][0], v[2 * c][1], v[2 * c + 1][0], v[2 * c + 1][1]};
    } else {
#pragma unroll
        for (int hf = 0; hf < 2; ++hf)
#pragma unroll
            for (int a = 0; a < 4; ++a) {
                d[2 * hf][a] = perm(v[2 * a + 1][hf], v[2 * a][hf], 0x05040100u);
                d[2 * hf + 1][a] = perm(v[2 * a + 1][hf], v[2 * a][hf], 0x07060302u);
            }
    }
    asm volatile("" :: "v"(peer));
    const unsigned call = rfl(cntv) + 1u, par = call & 1u;
    if (own) {
        store_stage(ws + (isx ? OFF_A : OFF_B) + (size_t)(h * 256 + 4 * idx) * 1024, lane * 16, d);
        wait_vm0();
        if (lane == 0) __hip_atomic_store(ready + ((isx ? 0 : 64) + idx) * NSLOT + h, call, __ATOMIC_RELAXED, __HIP_MEMORY_SCOPE_SYSTEM);
        return;
    }
    const unsigned g[1] = {isx ? id : 1024u + dest * 1024u + (inbox ? P.rank * 128u + inbox * 8u - 8u + wid : id)};
    unsigned r[1];
    pack_store<1>([&](int, u32x4 (&o)[4]) { o[0] = d[0]; o[1] = d[1]; o[2] = d[2]; o[3] = d[3]; }, (inbox ? P.ws[dest] : ws) + OFF_PK + par * PK_PAR, g, lds, lds + wid * SCR_WAVE, lane, r, [] {});
    if (lane == 0) lrec[wid] = r[0];
    wait_vm0();
    asm volatile("s_waitcnt lgkmcnt(0)" ::: "memory");
    __builtin_amdgcn_s_barrier();
    asm volatile("" ::: "memory");
    if (wid == 0 && (isx || lane < 8) && rdest != (unsigned)P.rank) {
        u64* rp = (u64*)(peer + OFF_REC + par * REC_PAR) + (P.rank * 3 + (isx ? 0 : 1)) * 1024 + h * 64 + (e & 7) * 8 + (lane & 7);
        __hip_atomic_store(rp, ((u64)call << 42) | lrec[lane & 7], __ATOMIC_RELAXED, __HIP_MEMORY_SCOPE_SYSTEM);
    }
}

__device__ __forceinline__ void pack_bulk(const Params& P, char* ws, const char* pw, LAS char* lds, unsigned rank, unsigned call, int lane, int wid) {
    unsigned i = P.dstart + 2u * rank, round = 0;
    const int li = lane & 15, lg = lane >> 4;
    const unsigned par = call & 1u;
    const unsigned lsrc = (unsigned)(8 * lg) * 65536u + (unsigned)(64 * wid + 4 * li) * 2u;
    u32x2 v[2][8];
    unsigned en[2];
    auto load = [&](unsigned i, int u) {
        const unsigned k = (i + u) / 7u, dest = (P.rank + 1u + (i + u) % 7u) & 7u, hp = (P.worder[k >> 2] >> ((k & 3u) * 8u)) & 127u;
        en[u] = dest << 7 | hp;
        const char* src = pw + (size_t)(hp >> 3) * (32 * 65536) + (dest * 4096u + (hp & 7u) * 512u) * 2u;
#pragma unroll
        for (int j = 0; j < 8; ++j) v[u][j] = *(const u32x2*)(src + (size_t)j * 65536 + lsrc);
    };
    load(i, 0);
    load(i, 1);
    __builtin_amdgcn_s_waitcnt(0x0F70);
    for (LAS unsigned* lrec = (LAS unsigned*)(lds + LDS_FLG);; lrec += 32) {
        const unsigned el = (lane & 8) ? en[1] : en[0];
        char* peer = P.ws[el >> 7];
        const unsigned g[2] = {1024u + en[0] * 8u + wid, 1024u + en[1] * 8u + wid};
        u32x4 d[2][4];
#pragma unroll
        for (int u = 0; u < 2; ++u)
#pragma unroll
            for (int hf = 0; hf < 2; ++hf)
#pragma unroll
                for (int a = 0; a < 4; ++a) {
                    d[u][2 * hf][a] = perm(v[u][2 * a + 1][hf], v[u][2 * a][hf], 0x05040100u);
                    d[u][2 * hf + 1][a] = perm(v[u][2 * a + 1][hf], v[u][2 * a][hf], 0x07060302u);
                }
        asm volatile("" : "+v"(d[0][0]), "+v"(d[0][1]), "+v"(d[0][2]), "+v"(d[0][3]), "+v"(d[1][0]), "+v"(d[1][1]), "+v"(d[1][2]), "+v"(d[1][3]));
        i = rank < P.dcount[round + 1u] ? i + 2u * P.dcount[round] : 896u;
        ++round;
        if (i < 896u) load(i, 0);
        __builtin_amdgcn_sched_barrier(0);
        unsigned r[2];
        pack_store<2>([&](int u, u32x4 (&o)[4]) { o[0] = d[u][0]; o[1] = d[u][1]; o[2] = d[u][2]; o[3] = d[u][3]; },
                      ws + OFF_PK + par * PK_PAR, g, lds, lds + wid * SCR_WAVE, lane, r, [&] { if (i < 896u) load(i, 1); });
        if (lane < 2) lrec[lane * 8 + wid] = lane ? r[1] : r[0];
        wait_vm0();
        __builtin_amdgcn_s_waitcnt(0x0F70);
        unsigned arrived = 0;
        if (lane == 0) arrived = __hip_atomic_fetch_add(lrec + 16, 1u, __ATOMIC_RELAXED, __HIP_MEMORY_SCOPE_WORKGROUP);
        LDS_FENCE();
        if (rfl(arrived) == 7u && lane < 16) {
            u64* rp = (u64*)(peer + OFF_REC + par * REC_PAR) + (P.rank * 3 + 1) * 1024 + (el & 127u) * 8u + (lane & 7);
            __hip_atomic_store(rp, ((u64)call << 42) | lrec[lane], __ATOMIC_RELAXED, __HIP_MEMORY_SCOPE_SYSTEM);
        }
        if (i >= 896u) break;
    }
}

constexpr int PSLOT = 15232;
constexpr int PSLOT_M = 2 * FIXB + VCAP;
constexpr int PSLOT_T = PSLOT - 4096;
constexpr int LDS_Q = LDS_REC;
constexpr int LDS_QTAIL = LDS_Q + 256;
constexpr int LDS_FULL = LDS_QTAIL + 16;
constexpr int FLAG_E = (NFLY - 1) / 6 + 1, FLAG_J = (NFLY - 1) % 6;

__device__ __forceinline__ unsigned lds_poll(LAS char* p) {
    return rfl(__hip_atomic_load((LAS unsigned*)p, __ATOMIC_RELAXED, __HIP_MEMORY_SCOPE_WORKGROUP));
}

__device__ __forceinline__ void fetch_link(const Params& P, LAS char* lds, int q, unsigned call, int lane) {
    const unsigned p = (unsigned)(P.rank + 1 + q) & 7u;
    const char* pkb = P.ws[p] + OFF_PK + (call & 1u) * PK_PAR;
    const char* box = P.me + OFF_PK + (call & 1u) * PK_PAR;
    const unsigned ldsbase = (unsigned)(size_t)lds;
    const unsigned v0 = lane * 16, v1 = v0 + 4096, v2 = v0 + (VMID - 512);
    unsigned fa[3] = {0, 0, 0}, fv[3] = {0, 0, 0};
    if (DIAG && blockIdx.x < 14) {
        const unsigned k = lane + (blockIdx.x / 7u) * 64u;
        __hip_atomic_store((unsigned*)(P.me + OFF_READY) + (8u * p + (k & 7u)) * NSLOT + 16u + 7u * (k >> 3) + q, call, __ATOMIC_RELAXED,
                           __HIP_MEMORY_SCOPE_SYSTEM);
    }
    unsigned ended = 0;
    for (unsigned i = 0;; ++i) {
        LAS char* e = lds + LDS_Q + (i & 15u) * 16u;
        u32x4 ev;
        for (;;) {
            ev = *(volatile LAS u32x4*)e;
            if (rfl(ev[0]) == i + 1u) break;
            if (fa[0] | fa[1] | fa[2]) {
                if (fa[2]) { asm volatile("s_waitcnt vmcnt(12)" ::: "memory"); *(volatile LAS unsigned*)(lds + fa[2]) = fv[2]; fa[2] = 0; }
                else if (fa[1]) { asm volatile("s_waitcnt vmcnt(6)" ::: "memory"); *(volatile LAS unsigned*)(lds + fa[1]) = fv[1]; fa[1] = 0; }
                else { asm volatile("s_waitcnt vmcnt(0)" ::: "memory"); *(volatile LAS unsigned*)(lds + fa[0]) = fv[0]; fa[0] = 0; }
            } else {
                if (ended == NDEC) return;
                __builtin_amdgcn_s_sleep(1);
            }
        }
        const unsigned w1 = rfl(ev[1]);
        if (w1 >> 31) { ++ended; continue; }
        const unsigned owner = w1 >> 24, slot = ldsbase + (owner - 1u) * PSLOT;
        const char* pk = ((w1 >> 23) & 1u ? box : pkb) + (size_t)(w1 & 0x3fffu) * PAIRB;
        const u64 vmask = (u64)rfl(ev[2]) | (u64)rfl(ev[3]) << 32;
        const unsigned da = ldsbase + (fa[FLAG_E - 1] ? fa[FLAG_E - 1] : (unsigned)LDS_FULL), dv = fv[FLAG_E - 1];
#define FW(j) "s_waitcnt vmcnt(%10)\n\t.if %11 == " #j "\n\tds_write_b32 %8, %9\n\t.endif\n\t"
        asm volatile("s_mov_b32 m0, %7\n\t"
                     FW(0) "s_mov_b64 exec, %4\n\t"
                     "global_load_lds_dwordx4 %2, %3 sc0 sc1\n\t"
                     "s_mov_b64 exec, -1\n\t"
                     "s_mov_b32 m0, %5\n\t"
                     FW(1) "global_load_lds_dwordx4 %0, %3 offset:2048 sc0 sc1\n\t"
                     FW(2) "global_load_lds_dwordx4 %0, %3 sc0 sc1\n\t"
                     FW(3) "global_load_lds_dwordx4 %0, %3 offset:1024 sc0 sc1\n\t"
                     FW(4) "global_load_lds_dwordx4 %0, %3 offset:3072 sc0 sc1\n\t"
                     "s_mov_b32 m0, %6\n\t"
                     FW(5) "global_load_lds_dwordx4 %1, %3 sc0 sc1"
                     :: "v"(v0), "v"(v1), "v"(v2), "s"(pk), "s"(vmask), "s"(slot), "s"(slot + 4096u), "s"(slot + (PSLOT_M - 512)),
                        "v"(da), "v"(dv), "i"(NFLY - 1), "i"(FLAG_J)
                     : "memory");
#undef FW
        fa[FLAG_E - 1] = 0;
        fa[2] = fa[1]; fv[2] = fv[1]; fa[1] = fa[0]; fv[1] = fv[0];
        fa[0] = LDS_FULL + owner * 4u; fv[0] = i + 1u;
    }
}

struct Desc { unsigned g, dst, rdy, rec; };

__device__ __forceinline__ void decode_link(const Params& P, LAS char* lds, int q, int w, unsigned call, int lane, int wid) {
    char* ws = P.me;
    const unsigned par = call & 1u;
    const unsigned p = (unsigned)(P.rank + 1 + q) & 7u;
    unsigned* tmo = (unsigned*)(ws + OFF_TMO);
    unsigned vorder = P.order[lane];
    asm volatile("" : "+v"(vorder));
    const unsigned rec_base = (unsigned)(OFF_REC + par * REC_PAR) + p * 3072u * 8u;
    auto describe = [&](unsigned n) {
        Desc d;
        if (n < NIN) {
            unsigned e = ((unsigned)__builtin_amdgcn_readlane(vorder, n >> 4) >> ((n & 12u) * 2u)) & 255u;
            if (DIAG && e < 128u && (e & 7u) >= p) ++e;
            const unsigned type = e >> 7, h = (e >> 3) & 15u, idx = (e & 7u) * 8u + (n & 3u) * 2u, id = h * 64u + idx, slot = 16u + 7u * h + q;
            d.g = type ? 1024u + P.rank * 1024u + id : id;
            if (n < 256u) {
                const unsigned inbox = (P.pushk[n >> 5] >> (n & 28u)) & 15u;
                if (inbox) d.g = (1024u + P.rank * 1024u + p * 128u + inbox * 8u - 8u + (n & 3u) * 2u) | 1u << 24;
            }
            d.dst = (unsigned)(type ? OFF_B : OFF_A) + (slot * 256u + 4u * idx) * 1024u;
            d.rdy = (unsigned)OFF_READY + ((type * 64u + idx) * NSLOT + slot) * 4u;
            d.rec = rec_base + (type * 1024u + id) * 8u;
        } else {
            d.g = 9216u; d.dst = P.rank * 8192u; d.rdy = 0;
            d.rec = rec_base + (2048u + 2u * ((n - NIN) & 511u)) * 8u;
        }
        return d;
    };
    auto accept = [&](u64 v, Desc& d, unsigned& ia, unsigned& ib) {
        if (__builtin_amdgcn_ballot_w64((unsigned)(v >> 42) != (call & 0x3fffffu)) != 0ull) return false;
        ia = __builtin_amdgcn_readlane((unsigned)v, 0); ib = __builtin_amdgcn_readlane((unsigned)v, 1);
        if (!d.rdy) {
            const unsigned id = __builtin_amdgcn_readlane((unsigned)(v >> 32), 0) & 1022u;
            d.g = 9216u + P.rank * 1024u + id;
            d.dst = (id & 15u) * (32u * 65536u) + (p * 4096u + 64u * (id >> 4)) * 2u;
        }
        return true;
    };
    const char* pkb = P.ws[p] + OFF_PK + par * PK_PAR;
    const unsigned voff = lane * 16;
    const unsigned tk_base = (call - 1u) * (NTK - PW) - 2u * PW;
    LAS char* S = lds + (wid - 1) * PSLOT;
    LAS char* M = S + PSLOT_M;
    bool dead = false;
    unsigned dmask = 0;
    const unsigned vrows = (lane >> 3) * 65536u + (lane & 7) * 16u;
    const __amdgpu_buffer_rsrc_t orsrc = __builtin_amdgcn_make_buffer_rsrc(P.out, 0, 512 * 65536, 0x00020000);
    auto vlen = [](unsigned i) { return min((i >> 8) & 255u, (unsigned)VCAP / 16u); };
    auto post = [&](unsigned w1, unsigned ia, unsigned ib) {
        const unsigned la = vlen(ia), lb = vlen(ib);
        const unsigned mlo = la >= 32u ? ~0u : ~(~0u >> la), mhi = lb >= 32u ? ~0u : ((1u << lb) - 1u) | 1u;
        unsigned i = 0;
        asm volatile("s_waitcnt lgkmcnt(0)" ::: "memory");
        if (lane == 0) {
            i = __hip_atomic_fetch_add((LAS unsigned*)(lds + LDS_QTAIL), 1u, __ATOMIC_RELAXED, __HIP_MEMORY_SCOPE_WORKGROUP);
            *(volatile LAS u32x4*)(lds + LDS_Q + (i & 15u) * 16u) = (u32x4){i + 1u, w1, mlo, mhi};
        }
        return rfl(i) + 1u;
    };
    auto poll = [&](Desc& d, unsigned& ia, unsigned& ib) {
        const u64 t0 = __builtin_amdgcn_s_memrealtime();
        unsigned spins = 0;
        ia = ib = 0;
        while (!dead) {
            const u64 v = __hip_atomic_load((const u64*)(ws + d.rec) + (lane & 1), __ATOMIC_RELAXED, __HIP_MEMORY_SCOPE_SYSTEM);
            if (accept(v, d, ia, ib)) break;
            if (timed_out(tmo, call, t0, spins)) dead = true;
            __builtin_amdgcn_s_sleep(1);
        }
    };
    unsigned n1 = PW + w, ia, ib;
    Desc d0 = describe(w), d1 = describe(n1);
    poll(d0, ia, ib);
    STAMP(ws, 2);
    unsigned want = post(d0.g >> 1 | wid << 24, ia, ib);
    for (;;) {
        const bool more = n1 < NTK;
        u32x4 own[8];
        const bool late = DIAG && !d0.rdy && ((d0.dst >> 9) & 15u) >= LATE_COL;
        if (late) {
            const unsigned bit = ((d0.dst >> 8) & 30u) | (d0.dst >> 24);
            if (!((dmask >> bit) & 1u)) {
                const u64 t0 = __builtin_amdgcn_s_memrealtime();
                unsigned spins = 0;
                while (!dead) {
                    const unsigned v = __hip_atomic_load((const unsigned*)(ws + OFF_DFL) + q * 32 + (lane & 31), __ATOMIC_RELAXED, __HIP_MEMORY_SCOPE_SYSTEM);
                    dmask = (unsigned)__builtin_amdgcn_ballot_w64(v >= call);
                    if ((dmask >> bit) & 1u) break;
                    if (timed_out(tmo, call, t0, spins)) dead = true;
                    __builtin_amdgcn_s_sleep(1);
                }
            }
#pragma unroll
            for (int c = 0; c < 8; ++c) own[c] = __builtin_amdgcn_raw_buffer_load_b128(orsrc, vrows, d0.dst + c * (8u * 65536u), 17);
        }
        const u64 rv = __hip_atomic_load((const u64*)(ws + d1.rec) + (lane & 1), __ATOMIC_RELAXED, __HIP_MEMORY_SCOPE_SYSTEM);
        while (lds_poll(lds + LDS_FULL + wid * 4) != want) __builtin_amdgcn_s_sleep(1);
        LDS_FENCE();
        const unsigned la = vlen(ia), lb = vlen(ib);
        if (la > 32u || lb > 32u) {
            const char* mid = (d0.g >> 24 ? ws + OFF_PK + par * PK_PAR : pkb) + (size_t)((d0.g >> 1) & 0x3fffu) * PAIRB + VMID;
            for (int c = -3; c <= 3; ++c) {
                const int off = c * 1024 - 512 + (int)voff;
                if (c != 0 && off >= -16 * (int)la && off < 16 * (int)lb) {
                    u32x4 t;
                    asm volatile("global_load_dwordx4 %0, %1, off sc0 sc1\n\ts_waitcnt vmcnt(0)" : "=&v"(t) : "v"(mid + off) : "memory");
                    *(LAS u32x4*)(M + off) = t;
                }
            }
            LDS_FENCE();
        }
        u32x4 o[2][4];
        unsigned ja = 0, jb = 0;
        bool posted = false;
        {
            u32x4 pp[2][2];
            u32x2 cc[2];
#pragma unroll
            for (int u = 0; u < 2; ++u) {
                cc[u] = *(LAS const u32x2*)(S + u * FIXB + 2048 + lane * 8);
                pp[u][0] = *(LAS const u32x4*)(S + u * FIXB + voff);
                pp[u][1] = *(LAS const u32x4*)(S + u * FIXB + 1024 + voff);
            }
            const unsigned inf[2] = {ia, ib};
            unpack2(pp, cc, inf, la, lds, S + PSLOT_T, M, lane, o, [&] {
                asm volatile("" : "+v"(pp[0][0]), "+v"(pp[0][1]), "+v"(pp[1][0]), "+v"(pp[1][1]));
                if (more && accept(rv, d1, ja, jb)) {
                    want = post(d1.g >> 1 | wid << 24, ja, jb);
                    posted = true;
                }
            });
        }
        if (!d0.rdy) {
            if (dead) {
#pragma unroll
                for (int c = 0; c < 8; ++c) o[c >> 2][c & 3] = (u32x4){0x7fc07fc0u, 0x7fc07fc0u, 0x7fc07fc0u, 0x7fc07fc0u};
            }
            rows_of(S + PSLOT_T, lane, o[0]);
            rows_of(S + PSLOT_T, lane, o[1]);
            if (late) {
#pragma unroll
                for (int c = 0; c < 8; ++c) o[c >> 2][c & 3] = add_bf16(o[c >> 2][c & 3], own[c]);
            }
        }
        unsigned tk;
        u64 rv2;
        {
            const unsigned step = d0.rdy ? 1024u : 8u * 65536u, v0 = d0.rdy ? voff : vrows;
            const char* b0 = (d0.rdy ? ws : P.out) + d0.dst;
            asm volatile("global_load_dwordx2 %1, %2, %5 sc0 sc1\n\t"
                         "s_mov_b64 exec, 1\n\tglobal_atomic_add %0, %3, %4, %5 sc0\n\ts_mov_b64 exec, -1\n\t"
                         "global_store_dwordx4 %8, %12, %6 sc0 sc1\n\tglobal_store_dwordx4 %9, %13, %6 sc0 sc1\n\t"
                         "global_store_dwordx4 %10, %14, %6 sc0 sc1\n\tglobal_store_dwordx4 %11, %15, %6 sc0 sc1\n\t"
                         "global_store_dwordx4 %8, %16, %7 sc0 sc1\n\tglobal_store_dwordx4 %9, %17, %7 sc0 sc1\n\t"
                         "global_store_dwordx4 %10, %18, %7 sc0 sc1\n\tglobal_store_dwordx4 %11, %19, %7 sc0 sc1\n\t"
                         "s_nop 1\n\ts_waitcnt vmcnt(8)"
                         : "=&v"(tk), "=&v"(rv2)
                         : "v"(d1.rec + (lane & 1) * 8u), "v"((unsigned)OFF_TKT + q * 128u), "v"(more ? 1u : 0u), "s"(ws), "s"(b0), "s"(b0 + 4u * step),
                           "v"(v0), "v"(v0 + step), "v"(v0 + 2u * step), "v"(v0 + 3u * step),
                           "v"(o[0][0]), "v"(o[0][1]), "v"(o[0][2]), "v"(o[0][3]), "v"(o[1][0]), "v"(o[1][1]), "v"(o[1][2]), "v"(o[1][3])
                         : "memory");
        }
        if (more && !posted && accept(rv2, d1, ja, jb)) {
            want = post(d1.g >> 1 | wid << 24, ja, jb);
            posted = true;
        }
        if (d0.rdy) {
            wait_vm0();
            if (lane < 2) __hip_atomic_store((unsigned*)(ws + d0.rdy) + lane * NSLOT, call, __ATOMIC_RELAXED, __HIP_MEMORY_SCOPE_SYSTEM);
        } else if (DIAG && !late) {
            wait_vm0();
            if (lane == 0 && !dead)
                __hip_atomic_store((unsigned*)(ws + OFF_OFL) + q * 512 + ((d0.g >> 1) & 511u), call, __ATOMIC_RELAXED, __HIP_MEMORY_SCOPE_SYSTEM);
        }
        const unsigned n2 = rfl(tk) - tk_base;
        if (!more) break;
        if (!posted) {
            poll(d1, ja, jb);
            want = post(d1.g >> 1 | wid << 24, ja, jb);
        }
        ia = ja; ib = jb;
        d0 = d1;
        n1 = n2;
        d1 = describe(n1);
    }
    post(0x80000000u, 0u, 0u);
}

__device__ __forceinline__ void glds4(unsigned m0v, unsigned voff, const char* sb) {
    asm volatile("s_mov_b32 m0, %0\n\ts_nop 0\n\t"
                 "global_load_lds_dwordx4 %1, %2 sc0 sc1\n\t"
                 "global_load_lds_dwordx4 %1, %2 offset:1024 sc0 sc1\n\t"
                 "global_load_lds_dwordx4 %1, %2 offset:2048 sc0 sc1\n\t"
                 "global_load_lds_dwordx4 %1, %2 offset:3072 sc0 sc1" :: "s"(m0v), "v"(voff), "s"(sb) : "memory");
}
__device__ __forceinline__ void glds4w(unsigned m0v, unsigned voff, const char* sb) {
    constexpr int R = 8 * 65536 - 1024;
    asm volatile("s_mov_b32 m0, %0\n\ts_nop 0\n\t"
                 "global_load_lds_dwordx4 %1, %2\n\t"
                 "global_load_lds_dwordx4 %1, %3 offset:1024\n\t"
                 "global_load_lds_dwordx4 %1, %4 offset:2048\n\t"
                 "global_load_lds_dwordx4 %1, %5 offset:3072" :: "s"(m0v), "v"(voff), "s"(sb), "s"(sb + R), "s"(sb + 2 * R), "s"(sb + 3 * R) : "memory");
}
__device__ __forceinline__ bf16x8 read_tr(LAS const char* p) {
    const bf16x4 lo = __builtin_amdgcn_ds_read_tr16_b64_v4bf16((LAS bf16x4*)p), hi = __builtin_amdgcn_ds_read_tr16_b64_v4bf16((LAS bf16x4*)(p + 512));
    return __builtin_shufflevector(lo, hi, 0, 1, 2, 3, 4, 5, 6, 7);
}
__device__ __forceinline__ void glds1(unsigned m0v, unsigned voff, const char* sb) {
    asm volatile("s_mov_b32 m0, %0\n\ts_nop 0\n\tglobal_load_lds_dwordx4 %1, %2 sc0 sc1" :: "s"(m0v), "v"(voff), "s"(sb) : "memory");
}

template <int WR, bool BIG, bool SK5, int MODE>
__device__ __forceinline__ void k_loop(f32x4 (&acc)[BIG ? 10 : 8][4], char* ws, LAS char* lds, int u0, int u5, int pn, const char* wsrc, int rank, unsigned call,
                                       int lane, int wid) {
    constexpr int MT = BIG ? 10 : 8, M0 = MODE == 1 ? 8 : 0;
    constexpr bool OWN = MODE == 2;
    constexpr unsigned ns = OWN ? 16 : NSLOT, dt = MODE == 1 ? 7 : 1;
    const unsigned q4 = ((u0 >> 3) - rank - 1) & 7, q5 = ((u5 >> 3) - rank - 1) & 7;
    const bool cut = DIAG && MODE == 0 && q4 < 7;
    const unsigned steps = MODE ? 16 : cut ? 112 : ns, last = MODE == 1 ? 121 + q4 : cut && q4 == 6 ? ns - 2 : ns - 1;
    unsigned t = MODE == 1 ? 16 + q4 : 0, tskip = cut ? 16 + q4 : 999;
    unsigned skip5 = DIAG && SK5 && q5 != q4 ? 16 + q5 - (q5 > q4) : 999;
    constexpr bool has5 = BIG && !WR, has4 = WR || MODE != 1;
    unsigned* tmo = (unsigned*)(ws + OFF_TMO);
    const int wc = wid & 3;
    const unsigned ldsbase = (unsigned)(size_t)lds;
    const unsigned u1 = WR ? 4 * pn + wc : u0 + wc;
    const char* g1 = OWN && WR ? wsrc + wc * 128 : ws + (WR ? OFF_B : OFF_A) + (size_t)u1 * 4096;
    const unsigned vw = (lane >> 3) * 65536 + ((lane & 7) ^ ((lane >> 4) & 1)) * 16;
    const char* g2 = ws + OFF_A + (size_t)(4 * u5 + wc) * 1024;
    const unsigned l1 = ldsbase + WR * 20480 + wc * 4096, l2 = ldsbase + (16 + wc) * 1024;
    const unsigned voff = lane * 16;
    const unsigned vrow = (unsigned)OFF_READY + (WR * 64 + (lane >= 32 && has5 ? u5 : u1)) * (NSLOT * 4);
    const unsigned vsnap = vrow + (lane & 31) * 16;
    LAS char* snap = lds + LDS_FLG + wid * 1024;
    *(LAS u32x4*)(snap + voff) = (u32x4){0u, 0u, 0u, 0u};
    asm volatile("s_waitcnt lgkmcnt(0)" ::: "memory");
    unsigned hz = OWN && WR ? 4096u : 0u, look = 0;
#define ADVANCE(v) ({ const u64 m_ = __builtin_amdgcn_ballot_w64((v) >= call); \
    (unsigned)__builtin_ctzll(~(u64)((unsigned)m_ & (unsigned)(m_ >> 32))); })
#define ENSURE(t) do { if (__builtin_expect(hz <= (unsigned)(t), 0)) { const u64 t0_ = __builtin_amdgcn_s_memrealtime(); unsigned sp_ = 0; \
        __builtin_amdgcn_s_waitcnt(0xC07F); \
        for (;;) { \
            const unsigned a_ = __hip_atomic_load((const unsigned*)(ws + vrow) + min(hz + (lane & 31u), ns - 1u), __ATOMIC_RELAXED, __HIP_MEMORY_SCOPE_SYSTEM); \
            hz += ADVANCE(a_); \
            if (hz > (unsigned)(t)) break; \
            if (timed_out(tmo, call, t0_, sp_)) { hz = NSLOT; break; } \
            __builtin_amdgcn_s_sleep(GSLEEP); } } } while (0)
#define STAGE(t, lo_) do { const size_t go_ = (size_t)(t) * 262144; \
    if (OWN && WR) glds4w(l1 + lo_, vw, g1 + 8 * go_); else if (has4) glds4(l1 + lo_, voff, g1 + go_); if (has5) glds1(l2 + lo_, voff, g2 + go_); } while (0)
#define WAITV() asm volatile("s_waitcnt vmcnt(%0)" :: "i"(8 * has4 + 2 * has5) : "memory")
#pragma unroll 1
    for (int i = 0; i < 3; ++i, t += dt) { ENSURE(t); STAGE(t, i * LSLOT); }
    WAITV();
    __builtin_amdgcn_s_barrier();
    if (!MODE) STAMP(ws, 2);
    if (WR == 1) __builtin_amdgcn_s_barrier();
    const unsigned aoff = ((lane & 15) * 4 + ((lane >> 4) ^ a_swz((lane >> 2) & 3))) * 16;
    const unsigned bsw = ((lane >> 3) & 1) * 16;
    const unsigned boff = 20480 + wc * 4096 + (OWN ? (lane >> 4) * 1024 + ((lane >> 2) & 3) * 128 + (lane & 3) * 32 : lane * 16);
#define LDB() _Pragma("unroll") for (int n = 0; n < 4; ++n) \
        B[n] = OWN ? read_tr(pb + (((n >> 1) * 16) ^ bsw) + (n & 1) * 8) : *(LAS const bf16x8*)(pb + n * 1024)
#define LDA(m0, m1) _Pragma("unroll") for (int m = m0; m < m1; ++m) A[m] = *(LAS const bf16x8*)(pa + (m < 8 ? 8 * WR + m : 8 + 2 * WR + m) * 1024)
#define MIDDLE() do { WAITV(); __builtin_amdgcn_s_waitcnt(0xC07F); __builtin_amdgcn_sched_barrier(0); __builtin_amdgcn_s_barrier(); \
        __builtin_amdgcn_sched_barrier(0); __builtin_amdgcn_s_setprio(1); } while (0)
#define MMA(m0, m1) _Pragma("unroll") for (int m = m0; m < m1; ++m) _Pragma("unroll") for (int k = 0; k < 4; ++k) { \
        const int n = (m & 1) ? 3 - k : k; \
        acc[m][n] = __builtin_amdgcn_mfma_f32_16x16x32_bf16(B[n], A[m], acc[m][n], 0, 0, 0); __builtin_amdgcn_sched_barrier(0); }
#define TAIL() do { __builtin_amdgcn_s_setprio(0); __builtin_amdgcn_sched_barrier(0); __builtin_amdgcn_s_barrier(); \
        __builtin_amdgcn_sched_barrier(0); } while (0)
    unsigned so = 0, sw = 3 * LSLOT;
    for (unsigned s = 0; s < steps; ++s) {
        ENSURE(t);
        STAGE(t, sw);
        __builtin_amdgcn_sched_barrier(0);
        bf16x8 A[MT], B[4];
        LAS const char* pa = lds + aoff + so; LAS const char* pb = lds + boff + so;
        LDB();
        LDA(M0, MT);
        __builtin_amdgcn_sched_barrier(0);
        sw = so;
        so = so == 3 * LSLOT ? 0u : so + LSLOT;
        t += dt;
        if (MODE == 0) { const bool hit = t == tskip; t += hit ? 1u : 0u; tskip += hit ? 7u : 0u; }
        t = min(t, last);
        const bool do5 = s != skip5;
        if (SK5) skip5 += do5 ? 0u : 6u;
        asm volatile("" : "+s"(t), "+s"(tskip), "+s"(so), "+s"(skip5));
        if (MODE == 0 && __builtin_expect(t >= look, 0)) {
            const unsigned fv = *(volatile LAS const unsigned*)(snap + (lane >> 5) * 512 + min(hz + (lane & 31u), ns - 1u) * 4u);
            hz += ADVANCE(fv);
            glds1(ldsbase + LDS_FLG + wid * 1024, vsnap, ws);
            look = max(t + 4u + LOOKAHEAD, hz) - LOOKAHEAD;
        }
        MIDDLE();
        MMA(M0, 8);
        if (!SK5 || MODE || __builtin_expect(do5, 1)) { MMA(8, MT); }
        TAIL();
    }
#undef TAIL
#undef LDA
#undef LDB
#undef MIDDLE
#undef MMA
#pragma unroll
    for (int m = M0; m < MT; ++m) asm volatile("" : "+v"(acc[m][0]), "+v"(acc[m][1]), "+v"(acc[m][2]), "+v"(acc[m][3]));
    if (WR == 0) __builtin_amdgcn_s_barrier();
#undef ENSURE
#undef ADVANCE
#undef STAGE
#undef WAITV
}

template <bool BIG, bool SK5>
__device__ __forceinline__ void gemm_tile(const Params& P, LAS char* lds, int u0, int u5, int pn, unsigned call, int lane, int wid) {
    constexpr int NU = BIG ? 5 : 4;
    char* ws = P.me;
    unsigned* tmo = (unsigned*)(ws + OFF_TMO);
    const int wr = wid >> 2, wc = wid & 3;
    f32x4 acc[2 * NU][4];
#pragma unroll
    for (int m = 0; m < 2 * NU; ++m)
#pragma unroll
        for (int n = 0; n < 4; ++n) acc[m][n] = (f32x4){0.f, 0.f, 0.f, 0.f};
    if (wr == 0) k_loop<0, BIG, SK5, 0>(acc, ws, lds, u0, u5, pn, nullptr, P.rank, call, lane, wid);
    else k_loop<1, BIG, SK5, 0>(acc, ws, lds, u0, u5, pn, nullptr, P.rank, call, lane, wid);
    const unsigned tmov = __hip_atomic_load(tmo, __ATOMIC_RELAXED, __HIP_MEMORY_SCOPE_AGENT);
    const unsigned rbl = (lane < 4 ? 2 * u0 + 4 * wr + lane : 2 * u5 + (lane & 1)) & 127u;
    char* peer = P.ws[rbl >> 4];
    LAS unsigned* meet = (LAS unsigned*)(lds + LDS_REC + 256) + wc * 4;
    if (SK5 && wr == 0 && lane == 0) meet[2] = 0u;
    asm volatile("s_waitcnt lgkmcnt(0)" ::: "memory");
    asm volatile("" :: "v"(peer), "v"(tmov));
    wait_vm0();
    __builtin_amdgcn_s_barrier();
    asm volatile("" ::: "memory");
    STAMP(ws, 3);
    const unsigned par = call & 1u;
    const int cb = 4 * pn + wc;
    if (__builtin_expect(rfl(tmov) == call, 0)) {
#pragma unroll
        for (int m = 0; m < 2 * NU; ++m)
#pragma unroll
            for (int n = 0; n < 4; ++n) acc[m][n] = (f32x4){__builtin_nanf(""), __builtin_nanf(""), __builtin_nanf(""), __builtin_nanf("")};
    }
    unsigned rb[NU], g[NU];
#pragma unroll
    for (int u = 0; u < NU; ++u) {
        rb[u] = (u < 4 ? 2 * u0 + 4 * wr + u : 2 * u5 + wr) & 127u;
        g[u] = 9216u + (rb[u] >> 4) * 1024u + cb * 16u + (rb[u] & 15u);
    }
    auto get = [&](int u, u32x4 (&d)[4]) {
#pragma unroll
        for (int c = 0; c < 4; ++c) {
            const int mi = 2 * u + (c >> 1), b = c & 1;
            d[c][0] = cvt_pk(acc[mi][0][2 * b], acc[mi][1][2 * b]);
            d[c][1] = cvt_pk(acc[mi][2][2 * b], acc[mi][3][2 * b]);
            d[c][2] = cvt_pk(acc[mi][0][2 * b + 1], acc[mi][1][2 * b + 1]);
            d[c][3] = cvt_pk(acc[mi][2][2 * b + 1], acc[mi][3][2 * b + 1]);
        }
    };
    LAS char* scr = lds + wid * SCR_WAVE;
    auto own_rows = [&](int u) {
        u32x4 d[4];
        get(u, d);
        rows_of(scr, lane, d);
        store_out(P.out + (size_t)(rb[u] & 15u) * (32 * 65536) + (size_t)(P.rank * 4096 + 64 * cb) * 2, lane, d);
    };
    unsigned* opos = (unsigned*)(ws + OFF_OPOS) + (rbl >> 4) * 32;
    u64* ring = (u64*)(peer + OFF_REC + par * REC_PAR) + (P.rank * 3 + 2) * 1024 + (lane & 1);
    const u64 word = (u64)call << 42 | (u64)(cb * 16u + (rbl & 15u)) << 32;
    auto place = [&](bool draw) {
        unsigned pos = 0;
        if (draw) pos = __hip_atomic_fetch_add(opos, 1u, __ATOMIC_RELAXED, __HIP_MEMORY_SCOPE_AGENT);
        asm volatile("" :: "v"(pos));
        wait_vm0();
        return dpp0<0xa0, 0xf>(pos) & 511u;
    };
    char* pk = ws + OFF_PK + par * PK_PAR;
#pragma unroll
    for (int k = 0; k < 4; k += 2) {
        const unsigned ga[2] = {g[k], g[k + 1]};
        unsigned r[2];
        pack_store<2>([&](int u, u32x4 (&d)[4]) { get(u + k, d); }, pk, ga, lds, scr, lane, r, [] {});
        const unsigned pos = place(lane == k);
        if ((lane & ~1) == k) __hip_atomic_store(ring + 2u * pos, word | ((lane & 1) ? r[1] : r[0]), __ATOMIC_RELAXED, __HIP_MEMORY_SCOPE_SYSTEM);
    }
    if (BIG) {
        STAMP(ws, 4);
        if (DIAG && (u5 >> 3) != (u0 >> 3)) {
            asm volatile("s_waitcnt lgkmcnt(0)" ::: "memory");
            __builtin_amdgcn_s_barrier();
            if (wr == 0) k_loop<0, BIG, false, 1>(acc, ws, lds, u0, u5, pn, nullptr, P.rank, call, lane, wid);
            else k_loop<1, BIG, false, 1>(acc, ws, lds, u0, u5, pn, nullptr, P.rank, call, lane, wid);
            if (rfl(__hip_atomic_load(tmo, __ATOMIC_RELAXED, __HIP_MEMORY_SCOPE_AGENT)) == call) acc[2 * NU - 1][0][0] = __builtin_nanf("");
            wait_vm0();
            __builtin_amdgcn_s_barrier();
            asm volatile("" ::: "memory");
        }
        if (!SK5) {
            own_rows(NU - 1);
        } else {
            const unsigned ga[1] = {g[NU - 1]};
            unsigned r[1], second = 0;
            pack_store<1>([&](int, u32x4 (&d)[4]) { get(NU - 1, d); }, pk, ga, lds, scr, lane, r, [] {});
            if (lane == 0) meet[wr] = r[0];
            const unsigned pos = place(wr == 1 && lane == 4);
            if (wr == 1 && lane == 4) meet[3] = pos;
            LDS_FENCE();
            if (lane == 0) second = __hip_atomic_fetch_add(meet + 2, 1u, __ATOMIC_RELAXED, __HIP_MEMORY_SCOPE_WORKGROUP);
            LDS_FENCE();
            if (rfl(second) && (lane >> 1) == 2)
                __hip_atomic_store(ring + 2u * meet[3], word | meet[lane & 1], __ATOMIC_RELAXED, __HIP_MEMORY_SCOPE_SYSTEM);
        }
    }
}

__device__ __forceinline__ void own_part(const Params& P, LAS char* lds, unsigned j, unsigned call, int lane, int wid) {
    char* ws = P.me;
    unsigned* tmo = (unsigned*)(ws + OFF_TMO);
    const int wr = wid >> 2, wc = wid & 3;
    const unsigned half = j & 1u, pn = (j >> 1) / 7u, q = (j >> 1) % 7u, p = (P.rank + 1u + q) & 7u;
    f32x4 acc[8][4];
#pragma unroll
    for (int m = 0; m < 8; ++m)
#pragma unroll
        for (int n = 0; n < 4; ++n) acc[m][n] = (f32x4){0.f, 0.f, 0.f, 0.f};
    const char* wsrc = P.w + p * 8192u + pn * 512u;
    if (wr == 0) k_loop<0, false, false, 2>(acc, ws, lds, 8 * P.rank + 4 * half, 0, pn, wsrc, P.rank, call, lane, wid);
    else k_loop<1, false, false, 2>(acc, ws, lds, 8 * P.rank + 4 * half, 0, pn, wsrc, P.rank, call, lane, wid);
    wait_vm0();
    __builtin_amdgcn_s_barrier();
    asm volatile("" ::: "memory");
    u32x4 d[4][4];
#pragma unroll
    for (int u = 0; u < 4; ++u) {
#pragma unroll
        for (int c = 0; c < 4; ++c) {
            const int mi = 2 * u + (c >> 1), n = 2 * (c & 1);
            d[u][c][0] = cvt_pk(acc[mi][n][0], acc[mi][n][1]);
            d[u][c][1] = cvt_pk(acc[mi][n][2], acc[mi][n][3]);
            d[u][c][2] = cvt_pk(acc[mi][n + 1][0], acc[mi][n + 1][1]);
            d[u][c][3] = cvt_pk(acc[mi][n + 1][2], acc[mi][n + 1][3]);
        }
        rows_of(lds + wid * SCR_WAVE, lane, d[u]);
    }
    const unsigned dst = (8u * half + 4u * wr) * (32u * 65536u) + p * 8192u + (4u * pn + wc) * 128u;
    if (pn < LATE_COL) {
        const unsigned* fl = (const unsigned*)(ws + OFF_OFL) + q * 512u + (4u * pn + wc) * 8u + 4u * half + 2u * wr + (lane & 1);
        const u64 t0 = __builtin_amdgcn_s_memrealtime();
        unsigned spins = 0;
        while (__builtin_amdgcn_ballot_w64(__hip_atomic_load(fl, __ATOMIC_RELAXED, __HIP_MEMORY_SCOPE_SYSTEM) < call) != 0ull) {
            if (timed_out(tmo, call, t0, spins)) break;
            __builtin_amdgcn_s_sleep(8);
        }
        const __amdgpu_buffer_rsrc_t orsrc = __builtin_amdgcn_make_buffer_rsrc(P.out, 0, 512 * 65536, 0x00020000);
        u32x4 y[16];
#pragma unroll
        for (int c = 0; c < 16; ++c) y[c] = __builtin_amdgcn_raw_buffer_load_b128(orsrc, (lane >> 3) * 65536 + (lane & 7) * 16, dst + c * (8u * 65536u), 17);
#pragma unroll
        for (int c = 0; c < 16; ++c) d[c >> 2][c & 3] = add_bf16(d[c >> 2][c & 3], y[c]);
    }
    if (rfl(__hip_atomic_load(tmo, __ATOMIC_RELAXED, __HIP_MEMORY_SCOPE_AGENT)) == call)
        d[0][0] = (u32x4){0x7fc07fc0u, 0x7fc07fc0u, 0x7fc07fc0u, 0x7fc07fc0u};
#pragma unroll
    for (int u = 0; u < 4; ++u) store_out(P.out + dst + u * (32 * 65536), lane, d[u]);
    wait_vm0();
    __builtin_amdgcn_s_barrier();
    asm volatile("" ::: "memory");
    if (threadIdx.x == 0)
        __hip_atomic_store((unsigned*)(ws + OFF_DFL) + q * 32u + pn * 2u + half, call, __ATOMIC_RELAXED, __HIP_MEMORY_SCOPE_SYSTEM);
}

__global__ void __launch_bounds__(512, 2) fused_gemm_rs(const Params P) {
    __shared__ __attribute__((aligned(16))) char smem[LDS_BYTES];
    const int bid = blockIdx.x;
    const int lane = threadIdx.x & 63, wid = rfl(threadIdx.x >> 6);
    char* ws = P.me; const char* px = P.x; const char* pw = P.w; const u64 job = P.job[bid];
    asm volatile("" :: "s"(ws), "s"(px), "s"(pw), "s"(job), "s"(P.rank));
    STAMP(ws, 0);
    unsigned* cnt = (unsigned*)(ws + OFF_CNT) + bid * 8 + wid;
    const unsigned cntv = __hip_atomic_load(cnt, __ATOMIC_RELAXED, __HIP_MEMORY_SCOPE_AGENT);
    if (bid >= 7 * PWG) {
#pragma unroll 1
        for (int r = 0; r < 3; ++r) {
            const unsigned e = (unsigned)(job >> (11 + 11 * r)) & 0x7ffu;
            if ((e >> 7) == 15u) break;
            pack_piece(P, ws, px, pw, (LAS char*)smem, r, e, r ? 0u : (unsigned)(job >> 53) & 15u, cntv, lane, wid);
        }
        const unsigned late = (unsigned)(job >> 44) & 511u;
        if (late < P.dcount[0]) pack_bulk(P, ws, pw, (LAS char*)smem, late, rfl(cntv) + 1u, lane, wid);
        asm volatile("s_waitcnt lgkmcnt(0)" ::: "memory");
        __builtin_amdgcn_s_barrier();
    } else {
        build_lut((LAS char*)smem, true, threadIdx.x);
        if (threadIdx.x >= 384) ((LAS unsigned*)(smem + LDS_Q))[threadIdx.x - 384] = 0u;
        asm volatile("s_waitcnt lgkmcnt(0)" ::: "memory");
        __builtin_amdgcn_s_barrier();
        asm volatile("" ::: "memory");
    }
    const unsigned call = rfl(cntv) + 1u;
    if (lane == 0) __hip_atomic_store(cnt, call, __ATOMIC_RELAXED, __HIP_MEMORY_SCOPE_AGENT);
    STAMP(ws, 1);
    if (bid < 7 * PWG) {
        if (wid == 0) fetch_link(P, (LAS char*)smem, bid % 7, call, lane);
        else if (wid <= NDEC) decode_link(P, (LAS char*)smem, bid % 7, (wid - 1) * PWG + bid / 7, call, lane, wid);
    } else if (((unsigned)job & 63u) != 63u) {
        const int u0 = (unsigned)job & 63u, u5 = (unsigned)(job >> 57) & 63u, pn = ((unsigned)job >> 7) & 15u;
        const unsigned dj = (P.djob[bid >> 2] >> ((bid & 3) * 8)) & 255u;
        if (dj < 224u) own_part(P, (LAS char*)smem, dj, call, lane, wid);
        if (!(job & 64u)) gemm_tile<false, false>(P, (LAS char*)smem, u0, 0, pn, call, lane, wid);
        else if ((u5 >> 3) == P.rank) gemm_tile<true, false>(P, (LAS char*)smem, u0, u5, pn, call, lane, wid);
        else gemm_tile<true, true>(P, (LAS char*)smem, u0, u5, pn, call, lane, wid);
        if (!(job & 64u)) STAMP(ws, 4);
        STAMP(ws, 7);
        for (LAS unsigned* drawn = (LAS unsigned*)(smem + LDS_REC);; drawn = (LAS unsigned*)((size_t)drawn ^ 4)) {
            if (threadIdx.x == 0) *drawn = __hip_atomic_fetch_add((unsigned*)(ws + OFF_TKT + 1024), 1u, __ATOMIC_RELAXED, __HIP_MEMORY_SCOPE_AGENT);
            asm volatile("s_waitcnt lgkmcnt(0)" ::: "memory");
            __builtin_amdgcn_s_barrier();
            const unsigned k = lds_poll((LAS char*)drawn) - (call - 1u) * (224u - NPRE + 256u - 7u * PWG) + NPRE;
            if (k >= 224u) break;
            own_part(P, (LAS char*)smem, k < NLATE ? 223u - k : k - NLATE, call, lane, wid);
        }
        return;
    }
    STAMP(ws, 4);
}

#define LAGS {-1, 4, 19, 30, 42, 58, 72, 86}
#define PRE_TOP 7
#define DCOUNT {110, 110, 84, 56}

void make_plan(Params& P, int rank) {
    const int lag[8] = LAGS;
    unsigned char* order = (unsigned char*)P.order;
    unsigned char* worder = (unsigned char*)P.worder;
    int no = 0, nw = 0;
    for (int i = 0; i < 8; ++i) P.pushk[i] = 0;
    for (int i = 0; i < 64; ++i) P.order[i] = 0;
    for (int pos = -1; pos < 512; ++pos) {
        if (pos >= 0 && pos < 128 && pos % 8 < (int)NXQ) order[no++] = (unsigned char)pos;
        for (int pr = 0; pr < 8; ++pr) {
            const int h8 = pos - lag[pr];
            if (h8 >= 0 && h8 < 128 && h8 % 8 == 0) {
                if (nw >= PUSH0 && nw < PUSH0 + NPUSH) P.pushk[no >> 3] |= (unsigned)(nw - PUSH0 + 1) << (4 * (no & 7));
                order[no++] = (unsigned char)(128 | h8 | pr);
                worder[nw++] = (unsigned char)(h8 | pr);
            }
        }
    }
    int seq[256], nseq = 0;
    const int mix[8] = MIX;
    for (int qt = 0; qt < NBAND; ++qt)
        for (int i = 0; i < 8; ++i) {
            const int pr = mix[i], band = (qt + pr) % NBAND, nt = (FULL14 >> pr) & 1 ? 14 : 13, base = 2 * rank + 2;
            for (int r = (nt * band + NBAND / 2) / NBAND; r < (nt * (band + 1) + NBAND / 2) / NBAND; ++r)
                for (int c = 0; c < 2; ++c) {
                    const int quad = (base + (r + 2 * pr) % 14) & 15, split = (base + (13 + 2 * pr) % 14) & 15;
                    const int k = 12 - r;
                    const int fifth = nt == 14 ? (r % 7 < 4 ? 8 * rank + r / 7 * 4 + r % 7 : -1)
                                               : k == 12 ? -1 : k % 3 == 0 ? 4 * split + k / 3 : 8 * rank + k - 1 - k / 3;
                    seq[nseq++] = (4 * quad | (fifth >= 0) << 6) | (2 * pr + c) << 7 | (fifth >= 0 ? fifth : 0) << 11;
                }
        }
    int tile[256], byp[256], nby = 0;
    static_assert(PWG == 6 && __builtin_popcount(FULL14) == 3 && PUSH0 + NPUSH <= 11, "214 tiles on 214 workgroups; pushed pieces: first round, first 64 entries");
    for (int x = 0, k = 0; x < 8; ++x)
        for (int b = 248 + x; b >= 7 * PWG; b -= 8) tile[b] = seq[k++];
    for (int pr = 0; pr <= 8; ++pr)
        for (int b = 7 * PWG; b < 256; ++b)
            if (((tile[b] >> 8) & 7) == pr) byp[nby++] = b;
    int first[10];
    for (int pr = 0, i = 0; pr <= 9; ++pr) {
        while (i < nby && ((tile[byp[i]] >> 8) & 7) < pr) ++i;
        first[pr] = i;
    }
    int nr = 0;
    for (int b = 0; b < 256; ++b)
        P.job[b] = (b < 7 * PWG ? 63ull : (u64)(tile[b] & 0x7ff) | (u64)(tile[b] >> 11) << 57) | 0x1ffffffffull << 11 | 511ull << 44;
    for (int r = 0; r < 3; ++r) {
        int list[256], n = 0;
        const int own = (rank + 1) << 7, from = r == 0 ? 0 : first[2 * r];
        if (r < 2) {
            for (int h = 8 * r; h < 8 * r + 8; ++h) {
                for (int q = 0; q < 8; ++q) list[n++] = 9 << 7 | h << 3 | q;
                list[n++] = own | h << 3; list[n++] = own | h << 3 | 1;
            }
            for (int h = 2 * r; h < (r ? 16 : 2); ++h) { list[n++] = own | h << 3 | 2; list[n++] = own | h << 3 | 3; }
        } else {
            for (int h = 0; h < 16; ++h)
                for (int q = 4; q < 8; ++q) list[n++] = own | h << 3 | q;
        }
        for (int i = 48 * r; i < (r < 2 ? 48 * r + 48 : 128); ++i)
            if (!DIAG || i % 8 != rank) list[n++] = i;
        while (n < nby - from - (r == 2 ? (nr & 1) != ((nby - from - n) & 1) : 0)) {
            if (nr >= 7 * PUSH0 && nr < 7 * (PUSH0 + NPUSH)) P.job[byp[from + n]] |= (u64)(nr / 7 - PUSH0 + 1) << 53;
            list[n++] = (((rank + 1 + nr % 7) & 7) + 1) << 7 | worder[nr / 7];
            ++nr;
        }
        for (int i = 0; i < n; ++i) {
            u64& j = P.job[byp[from + i]];
            j = (j & ~(0x7ffull << (11 + 11 * r))) | (u64)list[i] << (11 + 11 * r);
        }
    }
    P.dstart = nr;
    const int dcount[] = DCOUNT;
    for (int r = 0, left = (896 - nr) / 2; r < 12; ++r) {
        const int n = dcount[r < (int)(sizeof dcount / sizeof dcount[0]) ? r : sizeof dcount / sizeof dcount[0] - 1];
        P.dcount[r] = r == 11 ? 0 : n < left ? n : left;
        left -= P.dcount[r];
    }
    for (int i = 0; i < nby; ++i) P.job[byp[i]] = (P.job[byp[i]] & ~(511ull << 44)) | (u64)(nby - 1 - i) << 44;
    unsigned char* djob = (unsigned char*)P.djob;
    for (int b = 0; b < 256; ++b) djob[b] = 255;
    unsigned j = 0;
    for (int pr = PRE_TOP; pr >= 0; --pr)
        for (int x = 0; x < 8; ++x)
            for (int i = first[pr]; i < first[pr + 1] && j < NPRE; ++i)
                if (byp[i] % 8 == x && !(tile[byp[i]] & 64)) djob[byp[i]] = (unsigned char)(223 - j++);
}

}

extern "C" void kernel_launch(int rank, int nranks, ncclComm_t comm, hipStream_t stream,
                              void* const* d_in, const int* in_sizes, int n_in,
                              void* d_out, int out_size, const int* in_dtypes, int out_dtype,
                              void* const* d_out_all, void* const* d_ws_all, size_t ws_bytes) {
    (void)comm; (void)in_dtypes; (void)out_dtype; (void)d_out_all;
    if (nranks != NR || (unsigned)rank >= (unsigned)NR || n_in < 2 || ws_bytes < WS_NEED || in_sizes[0] != 4096 * 512 ||
        in_sizes[1] != 512 * 32768 || out_size != 512 * 32768) {
        fprintf(stderr, "kernel_launch: unexpected problem shape\n");
        return;
    }
    Params P;
    P.x = (const char*)d_in[0];
    P.w = (const char*)d_in[1];
    P.out = (char*)d_out;
    for (int p = 0; p < NR; ++p) P.ws[p] = (char*)d_ws_all[p];
    P.me = P.ws[rank];
    P.rank = rank;
    make_plan(P, rank);
    hipLaunchKernelGGL(fused_gemm_rs, dim3(256), dim3(512), 0, stream, P);
    hipError_t e = hipGetLastError();
    if (e != hipSuccess) fprintf(stderr, "launch failed: %s\n", hipGetErrorString(e));
}
```

```cpp
#include <hip/hip_runtime.h>
#include <rccl/rccl.h>
#include <cstdio>

#define LAS __attribute__((address_space(3)))
typedef __bf16 bf16x8 __attribute__((ext_vector_type(8)));
typedef __bf16 bf16x4 __attribute__((ext_vector_type(4)));
typedef float f32x4 __attribute__((ext_vector_type(4)));
typedef unsigned u32x4 __attribute__((ext_vector_type(4)));
typedef unsigned u32x2 __attribute__((ext_vector_type(2)));
typedef unsigned long long u64;

namespace {

constexpr int NR = 8;
constexpr int NSLOT = 128;
constexpr int LSLOT = 36 * 1024;
constexpr int LDS_LUT = 4 * LSLOT;
constexpr int LDS_REC = LDS_LUT + 2048;
constexpr int LDS_FLG = LDS_REC + 512;
constexpr int LDS_BYTES = LDS_FLG + 8192;
constexpr int SCR_UNIT = 1408;
constexpr int SCR_WAVE = 5 * SCR_UNIT;
constexpr int SCR_T = 1152;
#define PWG 6
#define NDEC 7
#define LOOKAHEAD 10u
#define NFLY 10
#define RING_A2 3
#define NBAND 2
#define MIX {0, 7, 3, 4, 1, 6, 2, 5}
#define NPUSH 6
#define PUSH0 5
#define PUSHS 3
#define PUSHP 4
#define GSLEEP 2
#define FULL14 0xc1
constexpr int PW = PWG * NDEC;
#define DIAG 1
#define LATE_COL 14u
#define NPRE 28u
constexpr unsigned NLATE = (16u - LATE_COL) * 14u;
static_assert(NPRE <= NLATE, "only jobs that wait for nothing run early");
constexpr unsigned NXQ = DIAG ? 7 : 8;
constexpr unsigned NIN = 64 * NXQ + 512;
constexpr unsigned NTK = NIN + 512;
constexpr u64 DEADLINE = 100000000ull;

constexpr size_t OFF_CNT = 0;
constexpr size_t OFF_TMO = 16384;
constexpr size_t OFF_TKT = 20480;
constexpr size_t OFF_OPOS = 24576;
constexpr size_t OFF_DFL = 28672;
constexpr size_t OFF_READY = 32768;
constexpr size_t OFF_OFL = 98304;
constexpr size_t OFF_REC = 131072;
constexpr size_t REC_PAR = (size_t)NR * 3 * 1024 * 8;
constexpr size_t OFF_DBG = 655360;
constexpr size_t OFF_A = 1u << 20;
constexpr size_t OFF_B = OFF_A + (32u << 20);
constexpr int NUNIT = 1024 + 2 * NR * 1024;
constexpr int FIXB = 2560;
constexpr int VCAP = 2624;
constexpr int PAIRB = 2 * FIXB + 2 * VCAP;
constexpr int VMID = 2 * FIXB + VCAP;
constexpr size_t PK_PAR = (size_t)(NUNIT / 2) * PAIRB;
constexpr size_t OFF_PK = OFF_B + (32u << 20);
constexpr size_t WS_NEED = OFF_PK + 2 * PK_PAR;
static_assert(OFF_REC + 2 * REC_PAR <= OFF_DBG, "layout");

struct Params {
    const char* x;
    const char* w;
    char* out;
    char* me;
    char* ws[NR];
    int rank;
    unsigned dstart;
    unsigned dcount[12];
    unsigned hsplit[2];
    unsigned pushk[16];
    unsigned order[64];
    unsigned worder[32];
    u64 job[256];
    unsigned djob[64];
};

#define STAMP(ws, idx) do { } while (0)
#define LSTAMP(ws, idx, t) do { } while (0)

__device__ __forceinline__ unsigned rfl(unsigned v) { return __builtin_amdgcn_readfirstlane(v); }
__device__ __forceinline__ unsigned perm(unsigned hi, unsigned lo, unsigned sel) { return __builtin_amdgcn_perm(hi, lo, sel); }
__device__ __forceinline__ unsigned bfi(unsigned m, unsigned a, unsigned b) { return (a & m) | (b & ~m); }
__device__ __forceinline__ int a_swz(int q) { return (0x78 >> (2 * q)) & 3; }
__device__ __forceinline__ unsigned cvt_pk(float lo, float hi) { unsigned r; asm volatile("v_cvt_pk_bf16_f32 %0, %1, %2" : "=v"(r) : "v"(lo), "v"(hi)); return r; }

__device__ __forceinline__ u32x4 add_bf16(u32x4 x, u32x4 y) {
    u32x4 r;
#pragma unroll
    for (int a = 0; a < 4; ++a)
        r[a] = cvt_pk(__uint_as_float(x[a] << 16) + __uint_as_float(y[a] << 16), __uint_as_float(x[a] & 0xffff0000u) + __uint_as_float(y[a] & 0xffff0000u));
    return r;
}

template <int OFF> __device__ __forceinline__ void st_wt(const char* sbase, unsigned voff, u32x4 v) {
    asm volatile("global_store_dwordx4 %0, %1, %2 offset:%3 sc0 sc1\n\ts_nop 1" :: "v"(voff), "v"(v), "s"(sbase), "i"(OFF) : "memory");
}
#define OUTBITS " sc0 sc1"
template <int OFF> __device__ __forceinline__ void st_out(const char* sbase, unsigned voff, u32x4 v) {
    asm volatile("global_store_dwordx4 %0, %1, %2 offset:%3" OUTBITS "\n\ts_nop 1" :: "v"(voff), "v"(v), "s"(sbase), "i"(OFF) : "memory");
}
__device__ __forceinline__ void glds4(unsigned m0v, unsigned voff, const char* sb) {
    asm volatile("s_mov_b32 m0, %0\n\ts_nop 0\n\t"
                 "global_load_lds_dwordx4 %1, %2 sc0 sc1\n\t"
                 "global_load_lds_dwordx4 %1, %2 offset:1024 sc0 sc1\n\t"
                 "global_load_lds_dwordx4 %1, %2 offset:2048 sc0 sc1\n\t"
                 "global_load_lds_dwordx4 %1, %2 offset:3072 sc0 sc1" :: "s"(m0v), "v"(voff), "s"(sb) : "memory");
}
__device__ __forceinline__ void glds1(unsigned m0v, unsigned voff, const char* sb) {
    asm volatile("s_mov_b32 m0, %0\n\ts_nop 0\n\tglobal_load_lds_dwordx4 %1, %2 sc0 sc1" :: "s"(m0v), "v"(voff), "s"(sb) : "memory");
}
__device__ __forceinline__ void wait_vm0() { asm volatile("s_waitcnt vmcnt(0)" ::: "memory"); }

__device__ __forceinline__ bool timed_out(unsigned* tmo, unsigned call, u64 t0, unsigned& spins) {
    if ((++spins & 63u) != 0u) return false;
    if (rfl(__hip_atomic_load(tmo, __ATOMIC_RELAXED, __HIP_MEMORY_SCOPE_AGENT)) == call) return true;
    if (__builtin_amdgcn_s_memrealtime() - t0 > DEADLINE) {
        __hip_atomic_store(tmo, call, __ATOMIC_RELAXED, __HIP_MEMORY_SCOPE_AGENT);
        return true;
    }
    return false;
}

#define LDS_FENCE() asm volatile("" ::: "memory")
template <int CTRL, int ROWMASK> __device__ __forceinline__ unsigned dpp0(unsigned v) {
    return (unsigned)__builtin_amdgcn_update_dpp(0, (int)v, CTRL, ROWMASK, 0xf, false);
}
__device__ __forceinline__ unsigned wave_scan(unsigned v) {
    v += dpp0<0x111, 0xf>(v); v += dpp0<0x112, 0xf>(v); v += dpp0<0x114, 0xf>(v); v += dpp0<0x118, 0xf>(v);
    v += dpp0<0x142, 0xa>(v); v += dpp0<0x143, 0xc>(v);
    return v;
}
__device__ __forceinline__ unsigned lane_rank(u64 m) {
    return __builtin_amdgcn_mbcnt_hi((unsigned)(m >> 32), __builtin_amdgcn_mbcnt_lo((unsigned)m, 0u));
}
__device__ __forceinline__ void build_lut(LAS char* lds, bool dec, int tid) {
    if (tid < 256) {
        u64 e = 0x0c0c0c0c0c0c0c0cull;
        unsigned t = 0;
#pragma unroll
        for (unsigned j = 0; j < 8; ++j)
            if ((tid >> j) & 1) {
                const unsigned slot = (t & 1u) * 4u + (t >> 1);
                const unsigned at = dec ? j : slot, val = dec ? slot : j;
                e = (e & ~(0xffull << (8 * at))) | ((u64)val << (8 * at));
                ++t;
            }
        *(LAS u64*)(lds + LDS_LUT + tid * 8) = e;
    }
}
__device__ __forceinline__ u32x2 lut_at(LAS char* lds, unsigned f) {
    return *(LAS const u32x2*)(lds + __builtin_amdgcn_udot4(f, 0x40201008u, (unsigned)LDS_LUT, false));
}
__device__ __forceinline__ unsigned shuffle8(unsigned lo, unsigned hi, u32x2 sel) {
    return bfi(0x0f0f0f0fu, perm(hi, lo, sel[0]), perm(hi, lo, sel[1]) << 4);
}
__device__ __forceinline__ unsigned half16(unsigned v, int u) { return (u & 1) ? v >> 16 : v & 0xffffu; }
__device__ __forceinline__ unsigned b_start(unsigned n1) { return n1 <= 512u ? ((n1 + 15u) >> 4) << 5 : ((n1 - 249u) >> 3) << 5; }
__device__ __forceinline__ void lds_or(LAS char* p, unsigned v) {
    __hip_atomic_fetch_or((LAS unsigned*)p, v, __ATOMIC_RELAXED, __HIP_MEMORY_SCOPE_WAVEFRONT);
}
__device__ __forceinline__ unsigned bit_addr(LAS char* p) { return (unsigned)(size_t)p * 8u; }
__device__ __forceinline__ void put_bits(unsigned pos, unsigned v) {
    const u64 sh = (u64)v << (pos & 31u);
    LAS char* p = (LAS char*)(size_t)((pos >> 3) & ~3u);
    lds_or(p, (unsigned)sh); lds_or(p + 4, (unsigned)(sh >> 32));
}
__device__ __forceinline__ unsigned get_bits(unsigned pos) {
    LAS const unsigned* p = (LAS const unsigned*)(size_t)((pos >> 3) & ~3u);
    return __builtin_amdgcn_alignbit(p[1], p[0], pos);
}

template <int NU, class Get, class Mid>
__device__ __forceinline__ void pack_store(Get get, char* pk, const unsigned (&g)[NU], LAS char* lds, LAS char* scr, int lane,
                                           unsigned (&info)[NU], Mid mid) {
    constexpr int NS = (NU + 1) / 2;
    const unsigned voff = lane * 16;
#pragma unroll
    for (int u = 0; u < NU; ++u) *(LAS u32x4*)(scr + u * SCR_UNIT + voff) = (u32x4){0u, 0u, 0u, 0u};
    unsigned D[NU][8], pc[NU][3], cnt[NS], x16[NU], top = 0, top4 = 0;
    u32x2 sel[NU][4];
#pragma unroll
    for (int u = 0; u < NU; ++u) {
        u32x4 d[4], S[2];
        unsigned E[8], C[4], c = 0;
        get(u, d);
#pragma unroll
        for (int hf = 0; hf < 2; ++hf)
#pragma unroll
            for (int a = 0; a < 4; ++a) {
                const unsigned p = __builtin_amdgcn_alignbit(d[2 * hf][a], d[2 * hf][a], 31);
                const unsigned r = __builtin_amdgcn_alignbit(d[2 * hf + 1][a], d[2 * hf + 1][a], 31);
                S[hf][a] = perm(r, p, 0x06040200u);
                E[4 * hf + a] = perm(r, p, 0x07050301u);
            }
        char* pair = pk + (size_t)(g[u] >> 1) * PAIRB;
        const char* fix = pair + (g[u] & 1u) * FIXB;
        if (u == 0) {
            unsigned sum = __builtin_amdgcn_sad_u8(E[0], 0u, __builtin_amdgcn_sad_u8(E[2], 0u, __builtin_amdgcn_sad_u8(E[5], 0u, __builtin_amdgcn_sad_u8(E[7], 0u, 0u))));
            sum = __builtin_amdgcn_readlane(wave_scan(sum), 63);
            top = min(((sum + 983u) >> 10) + 2u, 255u);
            top4 = top * 0x01010101u;
        }
        st_wt<0>(fix, voff, S[0]);
#pragma unroll
        for (int a = 0; a < 4; ++a) {
            D[u][a] = top4 - E[a]; D[u][4 + a] = top4 - E[4 + a];
            unsigned n = bfi(0x0f0f0f0fu, D[u][a], D[u][4 + a] << 4);
            asm("" : "+v"(n));
            unsigned o = n | (n >> 1);
            asm("" : "+v"(o));
            const unsigned f = (~o | (o >> 2)) & 0x11111111u;
            unsigned f3 = (f << 1) + f;
            asm("" : "+v"(f3));
            C[a] = n & 0x33333333u & ~f3;
            c += __builtin_popcount(f);
            if (a < 3) pc[u][a] = c;
            sel[u][a] = lut_at(lds, f);
            if (a == 1) st_wt<1024>(fix, voff, S[1]);
        }
        cnt[u >> 1] = (u & 1) ? cnt[u >> 1] | c << 16 : c;
        const u32x2 cc = {C[0] | (C[1] << 2), C[2] | (C[3] << 2)};
        asm volatile("global_store_dwordx2 %0, %1, %2 offset:2048 sc0 sc1" :: "v"(lane * 8), "v"(cc), "s"(fix) : "memory");
        info[u] = top; x16[u] = 0;
        const u64 em = __builtin_amdgcn_ballot_w64(((D[u][0] | D[u][1] | D[u][2] | D[u][3] | D[u][4] | D[u][5] | D[u][6] | D[u][7]) & 0xf0f0f0f0u) != 0u);
        if (__builtin_expect(em != 0ull, 0)) {
            u32x4 X;
#pragma unroll
            for (int a = 0; a < 4; ++a) X[a] = bfi(0xf0f0f0f0u, D[u][a], D[u][4 + a] >> 4);
            const unsigned l = __builtin_ctzll(em);
            const unsigned x0 = __builtin_amdgcn_readlane(X[0], l), x1 = __builtin_amdgcn_readlane(X[1], l);
            const unsigned x2 = __builtin_amdgcn_readlane(X[2], l), x3 = __builtin_amdgcn_readlane(X[3], l);
            const unsigned any = x0 | x1 | x2 | x3, nz = (x0 != 0u) + (x1 != 0u) + (x2 != 0u) + (x3 != 0u);
            const unsigned nib = __builtin_ctz(any) >> 2, val = any >> (4u * nib);
            if ((em & (em - 1ull)) == 0ull && nz == 1u && val < 16u) {
                info[u] |= (l | (x0 ? 0u : x1 ? 1u : x2 ? 2u : 3u) << 6 | nib << 8 | val << 11) << 17;
            } else {
                info[u] |= 1u << 16;
                x16[u] = 1u + __builtin_popcountll(em);
                const unsigned k = 1u + lane_rank(em);
                if (lane == 0) st_wt<0>(pair, (g[u] & 1u) ? VMID : VMID - 16, (u32x4){(unsigned)em, (unsigned)(em >> 32), 0u, 0u});
                if ((em >> lane) & 1ull) st_wt<0>(pair, (g[u] & 1u) ? VMID + 16u * k : VMID - 16 - 16u * k, X);
            }
        }
    }
    unsigned excl[NS], tot[NS];
#pragma unroll
    for (int k = 0; k < NS; ++k) excl[k] = wave_scan(cnt[k]);
#pragma unroll
    for (int k = 0; k < NS; ++k) { tot[k] = __builtin_amdgcn_readlane(excl[k], 63); excl[k] -= cnt[k]; }
#pragma unroll
    for (int u = 0; u < NU; ++u)
        if (__builtin_expect(half16(tot[u >> 1], u) > 1400u, 0) && lane < 24) *(LAS u32x4*)(scr + u * SCR_UNIT + 1024 + voff) = (u32x4){0u, 0u, 0u, 0u};
    LDS_FENCE();
#pragma unroll
    for (int u = 0; u < NU; ++u) {
        unsigned pos = half16(excl[u >> 1], u) * 4u + bit_addr(scr + u * SCR_UNIT);
        asm("" : "+v"(pos));
#pragma unroll
        for (int a = 0; a < 4; ++a) put_bits(a ? pos + pc[u][a - 1] * 4u : pos, shuffle8(D[u][a], D[u][4 + a], sel[u][a]));
    }
    LDS_FENCE();
    mid();
    unsigned w[NU], c2n[NS];
    u32x2 s2[NU];
#pragma unroll
    for (int u = 0; u < NU; ++u) w[u] = *(LAS const unsigned*)(scr + u * SCR_UNIT + lane * 4);
    LDS_FENCE();
#pragma unroll
    for (int u = 0; u < NU; ++u) {
        const unsigned e = ((w[u] >> 1) | (w[u] >> 3)) & 0x11111111u;
        const unsigned c2 = (w[u] & 0x55555555u & ~(e << 2)) | e;
        s2[u] = lut_at(lds, e);
        const unsigned c = __builtin_popcount(e);
        c2n[u >> 1] = (u & 1) ? c2n[u >> 1] | c << 16 : c;
        *(LAS unsigned short*)(scr + u * SCR_UNIT + 128 + lane * 2) = (unsigned short)(c2 | (c2 >> 15));
    }
    unsigned in2[NS];
#pragma unroll
    for (int k = 0; k < NS; ++k) in2[k] = wave_scan(c2n[k]);
    LDS_FENCE();
#pragma unroll
    for (int u = 0; u < NU; ++u)
        put_bits(bit_addr(scr + u * SCR_UNIT + 128) + b_start(half16(tot[u >> 1], u)) + half16(in2[u >> 1] - c2n[u >> 1], u) * 4u,
                 shuffle8(w[u], w[u] >> 4, s2[u]));
    LDS_FENCE();
    u32x4 vv[NU];
#pragma unroll
    for (int u = 0; u < NU; ++u) vv[u] = *(LAS const u32x4*)(scr + u * SCR_UNIT + 128 + voff);
#pragma unroll
    for (int u = 0; u < NU; ++u) {
        const unsigned c16 = (b_start(half16(tot[u >> 1], u)) + half16(__builtin_amdgcn_readlane(in2[u >> 1], 63), u) * 4u + 127u) >> 7;
        const char* vg = pk + (size_t)(g[u] >> 1) * PAIRB + VMID + ((g[u] & 1u) ? 16 * (int)x16[u] : -16 * (int)(x16[u] + c16));
        if ((unsigned)lane < c16) st_wt<0>(vg, voff, vv[u]);
        if (__builtin_expect(c16 > 64u, 0) && 64u + lane < c16) st_wt<1024>(vg, voff, *(LAS const u32x4*)(scr + u * SCR_UNIT + 1152 + voff));
        info[u] |= (x16[u] + c16) << 8;
    }
    LDS_FENCE();
}

template <class Mid>
__device__ __forceinline__ void unpack2(const u32x4 (&p)[2][2], const u32x2 (&cc)[2], const unsigned (&info)[2], unsigned la, LAS char* lds,
                                        LAS char* T, LAS char* M, int lane, u32x4 (&o)[2][4], Mid mid) {
    u32x4 X[2] = {{0u, 0u, 0u, 0u}, {0u, 0u, 0u, 0u}};
    LAS char* A[2] = {M - 16u * la, M};
    const bool anyx = ((info[0] | info[1]) >> 16) != 0u;
    if (__builtin_expect(anyx, 0)) {
#pragma unroll
        for (int u = 0; u < 2; ++u) {
            if (info[u] & 0x10000u) {
                const u32x2 mv = *(LAS const u32x2*)(u ? M : M - 16);
                const u64 em = (u64)rfl(mv[0]) | (u64)rfl(mv[1]) << 32;
                const unsigned k = 1u + lane_rank(em);
                if ((em >> lane) & 1ull) X[u] = *(LAS const u32x4*)(u ? M + 16u * k : M - 16 - 16u * k);
                if (u) A[1] = M + 16u * (1u + __builtin_popcountll(em));
            } else {
                const unsigned xv = (info[u] >> 28) << (((info[u] >> 25) & 7u) * 4u), xa = (info[u] >> 23) & 3u;
                const bool me = (unsigned)lane == ((info[u] >> 17) & 63u);
#pragma unroll
                for (unsigned a = 0; a < 4; ++a) X[u][a] = me && xa == a ? xv : 0u;
            }
        }
    }
    unsigned z[2], rr[2];
#pragma unroll
    for (int u = 0; u < 2; ++u) {
        z[u] = *(LAS const unsigned short*)(A[u] + lane * 2);
        rr[u] = *(LAS const unsigned*)(A[u] + 128 + lane * 4);
    }
    unsigned pc[2][3], cnt = 0;
    u32x2 sel[2][4];
#pragma unroll
    for (int u = 0; u < 2; ++u) {
        unsigned c = 0;
#pragma unroll
        for (int k = 0; k < 2; ++k) {
            const unsigned h = cc[u][k] >> 1;
            const unsigned f0 = ~(cc[u][k] | h) & 0x11111111u, f1 = ~(cc[u][k] | h) & 0x44444444u;
            c += __builtin_popcount(f0);
            pc[u][2 * k] = c;
            c += __builtin_popcount(f1);
            if (k == 0) pc[u][1] = c;
            sel[u][2 * k] = lut_at(lds, f0);
            sel[u][2 * k + 1] = *(LAS const u32x2*)(lds + __builtin_amdgcn_udot4(f1, 0x10080402u, (unsigned)LDS_LUT, false));
        }
        cnt |= c << (16 * u);
    }
    unsigned c2[2], f2[2], c2n = 0;
    u32x2 s2[2];
#pragma unroll
    for (int u = 0; u < 2; ++u) {
        c2[u] = (z[u] | (z[u] << 15)) & 0x55555555u;
        asm("" : "+v"(c2[u]));
        f2[u] = c2[u] & ~(c2[u] >> 2) & 0x11111111u;
        s2[u] = lut_at(lds, f2[u]);
        c2n |= (unsigned)__builtin_popcount(f2[u]) << (16 * u);
        *(LAS unsigned*)(T + u * SCR_T + 256 + lane * 4) = rr[u];
    }
    const unsigned incl = wave_scan(cnt), in2 = wave_scan(c2n), tot = __builtin_amdgcn_readlane(incl, 63);
    unsigned wv[2];
#pragma unroll
    for (int u = 0; u < 2; ++u) wv[u] = get_bits(bit_addr(A[u]) + b_start(half16(tot, u)) + half16(in2 - c2n, u) * 4u);
#pragma unroll
    for (int u = 0; u < 2; ++u) {
        *(LAS unsigned*)(T + u * SCR_T + lane * 4) = (c2[u] ^ f2[u]) | shuffle8(wv[u], wv[u] >> 4, s2[u]);
        if (__builtin_expect(half16(tot, u) > 1024u, 0)) *(LAS u32x2*)(T + u * SCR_T + 512 + lane * 8) = *(LAS const u32x2*)(A[u] + 384 + lane * 8);
    }
    LDS_FENCE();
    unsigned w[2][4];
#pragma unroll
    for (int u = 0; u < 2; ++u) {
        unsigned pos = half16(incl - cnt, u) * 4u + bit_addr(T + u * SCR_T);
        asm("" : "+v"(pos));
#pragma unroll
        for (int a = 0; a < 4; ++a) w[u][a] = get_bits(a ? pos + pc[u][a - 1] * 4u : pos);
    }
    LDS_FENCE();
    mid();
    unsigned e[2][8];
#pragma unroll
    for (int u = 0; u < 2; ++u) {
        const unsigned top4 = (info[u] & 0xffu) * 0x01010101u;
#pragma unroll
        for (int a = 0; a < 4; ++a) {
            const unsigned lo = w[u][a] & 0x0f0f0f0fu, hi = (w[u][a] >> 4) & 0x0f0f0f0fu;
            e[u][a] = top4 - (perm(hi, lo, sel[u][a][0]) | ((cc[u][a >> 1] >> (2 * (a & 1))) & 0x03030303u));
            e[u][4 + a] = top4 - (perm(hi, lo, sel[u][a][1]) | ((cc[u][a >> 1] >> (2 * (a & 1) + 4)) & 0x03030303u));
        }
    }
    if (__builtin_expect(anyx, 0)) {
#pragma unroll
        for (int u = 0; u < 2; ++u)
#pragma unroll
            for (int a = 0; a < 4; ++a) { e[u][a] -= X[u][a] & 0xf0f0f0f0u; e[u][4 + a] -= (X[u][a] << 4) & 0xf0f0f0f0u; }
    }
#pragma unroll
    for (int u = 0; u < 2; ++u)
#pragma unroll
        for (int a = 0; a < 4; ++a) {
            const unsigned q0 = perm(e[u][a], p[u][0][a], 0x05010400u), q1 = perm(e[u][a], p[u][0][a], 0x07030602u);
            const unsigned q2 = perm(e[u][4 + a], p[u][1][a], 0x05010400u), q3 = perm(e[u][4 + a], p[u][1][a], 0x07030602u);
            o[u][0][a] = __builtin_amdgcn_alignbit(q0, q0, 1); o[u][1][a] = __builtin_amdgcn_alignbit(q1, q1, 1);
            o[u][2][a] = __builtin_amdgcn_alignbit(q2, q2, 1); o[u][3][a] = __builtin_amdgcn_alignbit(q3, q3, 1);
        }
}
__device__ __forceinline__ void store_stage(const char* dst, unsigned voff, const u32x4 (&o)[4]) {
    st_wt<0>(dst, voff, o[0]); st_wt<1024>(dst, voff, o[1]); st_wt<2048>(dst, voff, o[2]); st_wt<3072>(dst, voff, o[3]);
}
__device__ __forceinline__ void rows_of(LAS char* buf, int lane, u32x4 (&o)[4]) {
    LAS char* wp = buf + (lane & 15) * 128;
    const unsigned x = (((lane >> 3) & 6) ^ (lane & 7)) * 16;
    LDS_FENCE();
    *(LAS u32x4*)(wp + x) = o[0]; *(LAS u32x4*)(wp + (x ^ 16)) = o[1];
    *(LAS u32x4*)(wp + 2048 + x) = o[2]; *(LAS u32x4*)(wp + 2048 + (x ^ 16)) = o[3];
    LDS_FENCE();
    LAS const char* rp = buf + (lane >> 3) * 128 + (((lane >> 3) ^ lane) & 7) * 16;
#pragma unroll
    for (int c = 0; c < 4; ++c) o[c] = *(LAS const u32x4*)(rp + c * 1024);
    LDS_FENCE();
}
__device__ __forceinline__ void store_out(const char* dst, int lane, const u32x4 (&o)[4]) {
    const unsigned voff = (lane >> 3) * 65536 + (lane & 7) * 16;
#pragma unroll
    for (int c = 0; c < 4; ++c) st_out<0>(dst + c * (8 * 65536), voff, o[c]);
}

__device__ __forceinline__ void pack_piece(const Params& P, char* ws, const char* px, const char* pw, LAS char* lds, int round, unsigned e,
                                           unsigned inbox, unsigned cntv, int lane, int wid) {
    LAS unsigned* lrec = (LAS unsigned*)(lds + LDS_REC) + 32 + round * 8;
    const int li = lane & 15, lg = lane >> 4;
    unsigned* ready = (unsigned*)(ws + OFF_READY);
    const int h = (e >> 3) & 15, idx = (e & 7) * 8 + wid, id = h * 64 + idx;
    const unsigned kind = e >> 7;
    const bool isx = kind == 0u || kind == 9u;
    const unsigned dest = (kind - 1u) & 7u;
    const bool own = kind == 9u || kind == (unsigned)P.rank + 1u;
    const unsigned rdest = isx ? (unsigned)lane >> 3 : dest;
    char* peer = P.ws[rdest];
    u32x4 d[4];
    u32x2 v[8];
    if (isx) {
        const char* src = px + (size_t)(64 * idx + (lane >> 2)) * 1024 + h * 64 + ((lane & 3) ^ a_swz(lg)) * 16;
#pragma unroll
        for (int c = 0; c < 4; ++c) {
            const u32x4 t = *(const u32x4*)(src + 16 * c * 1024);
            v[2 * c] = (u32x2){t[0], t[1]}; v[2 * c + 1] = (u32x2){t[2], t[3]};
        }
    } else {
        const char* src = pw + (size_t)(32 * h + 8 * lg) * 65536 + (size_t)(dest * 4096 + 64 * idx + 4 * li) * 2;
#pragma unroll
        for (int j = 0; j < 8; ++j) v[j] = *(const u32x2*)(src + (size_t)j * 65536);
    }
    __builtin_amdgcn_sched_barrier(0);
    if (round == 0) {
        build_lut(lds, false, threadIdx.x);
        if (threadIdx.x >= 448) ((LAS unsigned*)(lds + LDS_FLG))[(threadIdx.x - 448) * 32 + 16] = 0u;
        asm volatile("s_waitcnt lgkmcnt(0)" ::: "memory");
        __builtin_amdgcn_s_barrier();
        asm volatile("" ::: "memory");
    }
    if (isx) {
#pragma unroll
        for (int c = 0; c < 4; ++c) d[c] = (u32x4){v[2 * c][0], v[2 * c][1], v[2 * c + 1][0], v[2 * c + 1][1]};
    } else {
#pragma unroll
        for (int hf = 0; hf < 2; ++hf)
#pragma unroll
            for (int a = 0; a < 4; ++a) {
                d[2 * hf][a] = perm(v[2 * a + 1][hf], v[2 * a][hf], 0x05040100u);
                d[2 * hf + 1][a] = perm(v[2 * a + 1][hf], v[2 * a][hf], 0x07060302u);
            }
    }
    asm volatile("" :: "v"(peer));
    const unsigned call = rfl(cntv) + 1u, par = call & 1u;
    if (own) {
        store_stage(ws + (isx ? OFF_A : OFF_B) + (size_t)(h * 256 + 4 * idx) * 1024, lane * 16, d);
        wait_vm0();
        if (lane == 0) __hip_atomic_store(ready + ((isx ? 0 : 64) + idx) * NSLOT + h, call, __ATOMIC_RELAXED, __HIP_MEMORY_SCOPE_SYSTEM);
        return;
    }
    const unsigned g[1] = {isx ? id : 1024u + dest * 1024u + (inbox ? P.rank * 128u + inbox * 8u - 8u + wid : id)};
    unsigned r[1];
    pack_store<1>([&](int, u32x4 (&o)[4]) { o[0] = d[0]; o[1] = d[1]; o[2] = d[2]; o[3] = d[3]; }, (inbox ? P.ws[dest] : ws) + OFF_PK + par * PK_PAR, g, lds, lds + wid * SCR_WAVE, lane, r, [] {});
    if (lane == 0) lrec[wid] = r[0];
    wait_vm0();
    asm volatile("s_waitcnt lgkmcnt(0)" ::: "memory");
    __builtin_amdgcn_s_barrier();
    asm volatile("" ::: "memory");
    if (wid == 0 && (isx || lane < 8) && rdest != (unsigned)P.rank) {
        u64* rp = (u64*)(peer + OFF_REC + par * REC_PAR) + (P.rank * 3 + (isx ? 0 : 1)) * 1024 + h * 64 + (e & 7) * 8 + (lane & 7);
        __hip_atomic_store(rp, ((u64)call << 42) | lrec[lane & 7], __ATOMIC_RELAXED, __HIP_MEMORY_SCOPE_SYSTEM);
    }
}

__device__ __forceinline__ void pack_bulk(const Params& P, char* ws, const char* pw, LAS char* lds, unsigned rank, unsigned call, int lane, int wid) {
    unsigned i = P.dstart + 2u * rank, round = 0;
    const int li = lane & 15, lg = lane >> 4;
    const unsigned par = call & 1u;
    const unsigned lsrc = (unsigned)(8 * lg) * 65536u + (unsigned)(64 * wid + 4 * li) * 2u;
    u32x2 v[2][8];
    unsigned en[2];
    auto load = [&](unsigned i, int u) {
        const unsigned k = (i + u) / 7u, dest = (P.rank + 1u + (i + u) % 7u) & 7u, hp = (P.worder[k >> 2] >> ((k & 3u) * 8u)) & 127u;
        en[u] = dest << 7 | hp;
        const char* src = pw + (size_t)(hp >> 3) * (32 * 65536) + (dest * 4096u + (hp & 7u) * 512u) * 2u;
#pragma unroll
        for (int j = 0; j < 8; ++j) v[u][j] = *(const u32x2*)(src + (size_t)j * 65536 + lsrc);
    };
    load(i, 0);
    load(i, 1);
    __builtin_amdgcn_s_waitcnt(0x0F70);
    for (LAS unsigned* lrec = (LAS unsigned*)(lds + LDS_FLG);; lrec += 32) {
        const unsigned el = (lane & 8) ? en[1] : en[0];
        char* peer = P.ws[el >> 7];
        const unsigned g[2] = {1024u + en[0] * 8u + wid, 1024u + en[1] * 8u + wid};
        u32x4 d[2][4];
#pragma unroll
        for (int u = 0; u < 2; ++u)
#pragma unroll
            for (int hf = 0; hf < 2; ++hf)
#pragma unroll
                for (int a = 0; a < 4; ++a) {
                    d[u][2 * hf][a] = perm(v[u][2 * a + 1][hf], v[u][2 * a][hf], 0x05040100u);
                    d[u][2 * hf + 1][a] = perm(v[u][2 * a + 1][hf], v[u][2 * a][hf], 0x07060302u);
                }
        asm volatile("" : "+v"(d[0][0]), "+v"(d[0][1]), "+v"(d[0][2]), "+v"(d[0][3]), "+v"(d[1][0]), "+v"(d[1][1]), "+v"(d[1][2]), "+v"(d[1][3]));
        i = rank < P.dcount[round + 1u] ? i + 2u * P.dcount[round] : 896u;
        ++round;
        if (i < 896u) load(i, 0);
        __builtin_amdgcn_sched_barrier(0);
        unsigned r[2];
        pack_store<2>([&](int u, u32x4 (&o)[4]) { o[0] = d[u][0]; o[1] = d[u][1]; o[2] = d[u][2]; o[3] = d[u][3]; },
                      ws + OFF_PK + par * PK_PAR, g, lds, lds + wid * SCR_WAVE, lane, r, [&] { if (i < 896u) load(i, 1); });
        if (lane < 2) lrec[lane * 8 + wid] = lane ? r[1] : r[0];
        wait_vm0();
        __builtin_amdgcn_s_waitcnt(0x0F70);
        unsigned arrived = 0;
        if (lane == 0) arrived = __hip_atomic_fetch_add(lrec + 16, 1u, __ATOMIC_RELAXED, __HIP_MEMORY_SCOPE_WORKGROUP);
        LDS_FENCE();
        if (rfl(arrived) == 7u && lane < 16) {
            u64* rp = (u64*)(peer + OFF_REC + par * REC_PAR) + (P.rank * 3 + 1) * 1024 + (el & 127u) * 8u + (lane & 7);
            __hip_atomic_store(rp, ((u64)call << 42) | lrec[lane], __ATOMIC_RELAXED, __HIP_MEMORY_SCOPE_SYSTEM);
        }
        if (i >= 896u) break;
    }
}

constexpr int PSLOT = 15232;
constexpr int PSLOT_M = 2 * FIXB + VCAP;
constexpr int PSLOT_T = PSLOT - 4096;
constexpr int LDS_Q = LDS_REC;
constexpr int LDS_QTAIL = LDS_Q + 256;
constexpr int LDS_FULL = LDS_QTAIL + 16;

__device__ __forceinline__ unsigned lds_poll(LAS char* p) {
    return rfl(__hip_atomic_load((LAS unsigned*)p, __ATOMIC_RELAXED, __HIP_MEMORY_SCOPE_WORKGROUP));
}

template <int N>
__device__ __forceinline__ void fetch_pair(u32x4& next, unsigned naddr, unsigned v0, unsigned v1, unsigned v2, const char* pk, u64 vmask,
                                           unsigned slot, unsigned da, unsigned dv) {
    static_assert(N >= 7 && N <= 12, "window");
#define FW(j) "s_waitcnt vmcnt(%12)\n\t.if %13 == " #j "\n\tds_write_b32 %10, %11\n\t.endif\n\t"
    asm volatile("s_mov_b32 m0, %7\n\t"
                 FW(0) "global_load_lds_dwordx4 %2, %5 offset:2048 sc0 sc1\n\t"
                 FW(1) "global_load_lds_dwordx4 %2, %5 sc0 sc1\n\t"
                 FW(2) "global_load_lds_dwordx4 %2, %5 offset:1024 sc0 sc1\n\t"
                 FW(3) "global_load_lds_dwordx4 %2, %5 offset:3072 sc0 sc1\n\t"
                 "s_mov_b32 m0, %9\n\t"
                 FW(4) "s_mov_b64 exec, %6\n\t"
                 "global_load_lds_dwordx4 %4, %5 sc0 sc1\n\t"
                 "s_mov_b64 exec, -1\n\t"
                 "s_mov_b32 m0, %8\n\t"
                 "ds_read_b128 %0, %1\n\t"
                 FW(5) "global_load_lds_dwordx4 %3, %5 sc0 sc1\n\t"
                 "s_waitcnt lgkmcnt(0)"
                 : "=&v"(next)
                 : "v"(naddr), "v"(v0), "v"(v1), "v"(v2), "s"(pk), "s"(vmask), "s"(slot), "s"(slot + 4096u), "s"(slot + (PSLOT_M - 512)),
                   "v"(da), "v"(dv), "i"(N - 1), "i"((N - 1) % 6)
                 : "memory");
#undef FW
}

__device__ __forceinline__ void fetch_link(const Params& P, LAS char* lds, int q, unsigned call, int lane) {
    const unsigned p = (unsigned)(P.rank + 1 + q) & 7u;
    const char* pkb = P.ws[p] + OFF_PK + (call & 1u) * PK_PAR;
    const unsigned ldsbase = (unsigned)(size_t)lds;
    const unsigned v0 = lane * 16, v1 = v0 + 4096, v2 = v0 + (VMID - 512);
    unsigned fa[2] = {0, 0}, fv[2] = {0, 0};
    if (DIAG && blockIdx.x < 14) {
        const unsigned k = lane + (blockIdx.x / 7u) * 64u;
        __hip_atomic_store((unsigned*)(P.me + OFF_READY) + (8u * p + (k & 7u)) * NSLOT + 16u + 7u * (k >> 3) + q, call, __ATOMIC_RELAXED,
                           __HIP_MEMORY_SCOPE_SYSTEM);
    }
    __builtin_amdgcn_s_setprio(3);
    unsigned ended = 0;
    u32x4 ev = {0u, 0u, 0u, 0u};
    for (unsigned i = 0;; ++i) {
        while (__builtin_expect(rfl(ev[0]) != i + 1u, 0)) {
            ev = *(volatile LAS u32x4*)(lds + LDS_Q + (i & 15u) * 16u);
            if (rfl(ev[0]) == i + 1u) break;
            if (fa[1]) {
                asm volatile("s_waitcnt vmcnt(6)" ::: "memory"); *(volatile LAS unsigned*)(lds + fa[1]) = fv[1]; fa[1] = 0;
            } else if (fa[0]) {
                asm volatile("s_waitcnt vmcnt(0)" ::: "memory"); *(volatile LAS unsigned*)(lds + fa[0]) = fv[0]; fa[0] = 0;
            } else {
                if (ended == NDEC) return;
                __builtin_amdgcn_s_sleep(1);
            }
        }
        const unsigned w1 = rfl(ev[1]);
        if (__builtin_expect(w1 >> 31, 0)) { ++ended; continue; }
        const unsigned owner = w1 >> 24, slot = ldsbase + (owner - 1u) * PSLOT;
        const char* pk = pkb + (size_t)(w1 & 0x3fffu) * PAIRB;
        const u64 vmask = (u64)rfl(ev[2]) | (u64)rfl(ev[3]) << 32;
        const unsigned da = ldsbase + (fa[1] ? fa[1] : (unsigned)LDS_FULL), dv = fv[1];
        fetch_pair<NFLY>(ev, ldsbase + LDS_Q + ((i + 1u) & 15u) * 16u, v0, v1, v2, pk, vmask, slot, da, dv);
        fa[1] = fa[0]; fv[1] = fv[0];
        fa[0] = LDS_FULL + owner * 4u; fv[0] = i + 1u;
    }
}

struct Desc { unsigned g, dst, rdy, rec; };

__device__ __forceinline__ void decode_link(const Params& P, LAS char* lds, int q, int w, unsigned call, int lane, int wid) {
    char* ws = P.me;
    const unsigned par = call & 1u;
    const unsigned p = (unsigned)(P.rank + 1 + q) & 7u;
    unsigned* tmo = (unsigned*)(ws + OFF_TMO);
    unsigned vorder = P.order[lane];
    asm volatile("" : "+v"(vorder));
    const unsigned rec_base = (unsigned)(OFF_REC + par * REC_PAR) + p * 3072u * 8u;
    auto describe = [&](unsigned n) {
        Desc d;
        if (n < NIN) {
            unsigned e = ((unsigned)__builtin_amdgcn_readlane(vorder, n >> 4) >> ((n & 12u) * 2u)) & 255u;
            if (DIAG && e < 128u && (e & 7u) >= p) ++e;
            const unsigned type = e >> 7, h = (e >> 3) & 15u, idx = (e & 7u) * 8u + (n & 3u) * 2u, id = h * 64u + idx, slot = 16u + 7u * h + q;
            d.g = type ? 1024u + P.rank * 1024u + id : id;
            if (n < 512u) {
                const unsigned inbox = (P.pushk[n >> 5] >> (n & 28u)) & 15u;
                if (inbox) d.g = (1024u + P.rank * 1024u + p * 128u + inbox * 8u - 8u + (n & 3u) * 2u) | 1u << 24;
            }
            d.dst = (unsigned)(type ? OFF_B : OFF_A) + (slot * 256u + 4u * idx) * 1024u;
            d.rdy = (unsigned)OFF_READY + ((type * 64u + idx) * NSLOT + slot) * 4u;
            d.rec = rec_base + (type * 1024u + id) * 8u;
        } else {
            d.g = 9216u; d.dst = P.rank * 8192u; d.rdy = 0;
            d.rec = rec_base + (2048u + 2u * ((n - NIN) & 511u)) * 8u;
        }
        return d;
    };
    auto accept = [&](u64 v, Desc& d, unsigned& ia, unsigned& ib) {
        if (__builtin_amdgcn_ballot_w64((unsigned)(v >> 42) != (call & 0x3fffffu)) != 0ull) return false;
        ia = __builtin_amdgcn_readlane((unsigned)v, 0); ib = __builtin_amdgcn_readlane((unsigned)v, 1);
        if (!d.rdy) {
            const unsigned id = __builtin_amdgcn_readlane((unsigned)(v >> 32), 0) & 1022u;
            d.g = 9216u + P.rank * 1024u + id;
            d.dst = (id & 15u) * (32u * 65536u) + (p * 4096u + 64u * (id >> 4)) * 2u;
        }
        return true;
    };
    const char* pkb = P.ws[p] + OFF_PK + par * PK_PAR;
    const unsigned voff = lane * 16;
    const unsigned tk_base = (call - 1u) * (NTK - PW) - 2u * PW;
    LAS char* S = lds + (wid - 1) * PSLOT;
    LAS char* M = S + PSLOT_M;
    bool dead = false;
    unsigned dmask = 0;
    const unsigned vrows = (lane >> 3) * 65536u + (lane & 7) * 16u;
    const __amdgpu_buffer_rsrc_t orsrc = __builtin_amdgcn_make_buffer_rsrc(P.out, 0, 512 * 65536, 0x00020000);
    auto vlen = [](unsigned i) { return min((i >> 8) & 255u, (unsigned)VCAP / 16u); };
    auto post = [&](unsigned w1, unsigned ia, unsigned ib) {
        const unsigned la = vlen(ia), lb = vlen(ib);
        const unsigned mlo = la >= 32u ? ~0u : ~(~0u >> la), mhi = lb >= 32u ? ~0u : ((1u << lb) - 1u) | 1u;
        unsigned i = 0;
        asm volatile("s_waitcnt lgkmcnt(0)" ::: "memory");
        if (__builtin_expect((w1 >> 23) & 1u, 0)) {
            const char* pk = ws + OFF_PK + par * PK_PAR + (size_t)(w1 & 0x3fffu) * PAIRB;
            glds4((unsigned)(size_t)S, voff, pk);
            glds1((unsigned)(size_t)S + 4096u, voff, pk + 4096);
            glds1((unsigned)(size_t)M - 512u, voff, pk + (VMID - 512));
            return 0u;
        }
        if (lane == 0) {
            i = __hip_atomic_fetch_add((LAS unsigned*)(lds + LDS_QTAIL), 1u, __ATOMIC_RELAXED, __HIP_MEMORY_SCOPE_WORKGROUP);
            *(volatile LAS u32x4*)(lds + LDS_Q + (i & 15u) * 16u) = (u32x4){i + 1u, w1, mlo, mhi};
        }
        return rfl(i) + 1u;
    };
    auto poll = [&](Desc& d, unsigned& ia, unsigned& ib) {
        const u64 t0 = __builtin_amdgcn_s_memrealtime();
        unsigned spins = 0;
        ia = ib = 0;
        while (!dead) {
            const u64 v = __hip_atomic_load((const u64*)(ws + d.rec) + (lane & 1), __ATOMIC_RELAXED, __HIP_MEMORY_SCOPE_SYSTEM);
            if (accept(v, d, ia, ib)) break;
            if (timed_out(tmo, call, t0, spins)) dead = true;
            __builtin_amdgcn_s_sleep(1);
        }
    };
    unsigned n1 = PW + w, ia, ib;
    Desc d0 = describe(w), d1 = describe(n1);
    poll(d0, ia, ib);
    STAMP(ws, 2);
    unsigned want = post(d0.g >> 1 | wid << 24, ia, ib);
    for (;;) {
        const bool more = n1 < NTK;
        u32x4 own[8];
        const bool late = DIAG && !d0.rdy && ((d0.dst >> 9) & 15u) >= LATE_COL;
        if (late) {
            const unsigned bit = ((d0.dst >> 8) & 30u) | (d0.dst >> 24);
            if (!((dmask >> bit) & 1u)) {
                const u64 t0 = __builtin_amdgcn_s_memrealtime();
                unsigned spins = 0;
                while (!dead) {
                    const unsigned v = __hip_atomic_load((const unsigned*)(ws + OFF_DFL) + q * 32 + (lane & 31), __ATOMIC_RELAXED, __HIP_MEMORY_SCOPE_SYSTEM);
                    dmask = (unsigned)__builtin_amdgcn_ballot_w64(v >= call);
                    if ((dmask >> bit) & 1u) break;
                    if (timed_out(tmo, call, t0, spins)) dead = true;
                    __builtin_amdgcn_s_sleep(1);
                }
            }
            asm volatile("" ::: "memory");
#pragma unroll
            for (int c = 0; c < 8; ++c) own[c] = __builtin_amdgcn_raw_buffer_load_b128(orsrc, vrows, d0.dst + c * (8u * 65536u), 17);
        }
        unsigned dv = 0;
        if (n1 + 2u * PW >= NIN + 32u * LATE_COL && dmask == 0u)
            dv = __hip_atomic_load((const unsigned*)(ws + OFF_DFL) + q * 32 + (lane & 31), __ATOMIC_RELAXED, __HIP_MEMORY_SCOPE_SYSTEM);
        const u64 rv = __hip_atomic_load((const u64*)(ws + d1.rec) + (lane & 1), __ATOMIC_RELAXED, __HIP_MEMORY_SCOPE_SYSTEM);
        if (want == 0u) wait_vm0();
        else while (lds_poll(lds + LDS_FULL + wid * 4) != want) __builtin_amdgcn_s_sleep(1);
        LDS_FENCE();
        const unsigned la = vlen(ia), lb = vlen(ib);
        if (la > 32u || lb > 32u) {
            const char* mid = (d0.g >> 24 ? ws + OFF_PK + par * PK_PAR : pkb) + (size_t)((d0.g >> 1) & 0x3fffu) * PAIRB + VMID;
            for (int c = -3; c <= 3; ++c) {
                const int off = c * 1024 - 512 + (int)voff;
                if (c != 0 && off >= -16 * (int)la && off < 16 * (int)lb) {
                    u32x4 t;
                    asm volatile("global_load_dwordx4 %0, %1, off sc0 sc1\n\ts_waitcnt vmcnt(0)" : "=&v"(t) : "v"(mid + off) : "memory");
                    *(LAS u32x4*)(M + off) = t;
                }
            }
            LDS_FENCE();
        }
        u32x4 o[2][4];
        unsigned ja = 0, jb = 0;
        bool posted = false;
        {
            u32x4 pp[2][2];
            u32x2 cc[2];
#pragma unroll
            for (int u = 0; u < 2; ++u) {
                cc[u] = *(LAS const u32x2*)(S + u * FIXB + 2048 + lane * 8);
                pp[u][0] = *(LAS const u32x4*)(S + u * FIXB + voff);
                pp[u][1] = *(LAS const u32x4*)(S + u * FIXB + 1024 + voff);
            }
            const unsigned inf[2] = {ia, ib};
            unpack2(pp, cc, inf, la, lds, S + PSLOT_T, M, lane, o, [&] {
                asm volatile("" : "+v"(pp[0][0]), "+v"(pp[0][1]), "+v"(pp[1][0]), "+v"(pp[1][1]));
                if (more && accept(rv, d1, ja, jb)) {
                    want = post(d1.g >> 1 | wid << 24, ja, jb);
                    posted = true;
                }
            });
        }
        dmask |= (unsigned)__builtin_amdgcn_ballot_w64(dv >= call);
        if (!d0.rdy) {
            if (dead) {
#pragma unroll
                for (int c = 0; c < 8; ++c) o[c >> 2][c & 3] = (u32x4){0x7fc07fc0u, 0x7fc07fc0u, 0x7fc07fc0u, 0x7fc07fc0u};
            }
            rows_of(S + PSLOT_T, lane, o[0]);
            rows_of(S + PSLOT_T, lane, o[1]);
            if (late) {
#pragma unroll
                for (int c = 0; c < 8; ++c) o[c >> 2][c & 3] = add_bf16(o[c >> 2][c & 3], own[c]);
            }
        }
        unsigned tk;
        u64 rv2;
        {
            const unsigned step = d0.rdy ? 1024u : 8u * 65536u, v0 = d0.rdy ? voff : vrows;
            const char* b0 = (d0.rdy ? ws : P.out) + d0.dst;
            asm volatile("global_load_dwordx2 %1, %2, %5 sc0 sc1\n\t"
                         "s_mov_b64 exec, 1\n\tglobal_atomic_add %0, %3, %4, %5 sc0\n\ts_mov_b64 exec, -1\n\t"
                         "global_store_dwordx4 %8, %12, %6 sc0 sc1\n\tglobal_store_dwordx4 %9, %13, %6 sc0 sc1\n\t"
                         "global_store_dwordx4 %10, %14, %6 sc0 sc1\n\tglobal_store_dwordx4 %11, %15, %6 sc0 sc1\n\t"
                         "global_store_dwordx4 %8, %16, %7 sc0 sc1\n\tglobal_store_dwordx4 %9, %17, %7 sc0 sc1\n\t"
                         "global_store_dwordx4 %10, %18, %7 sc0 sc1\n\tglobal_store_dwordx4 %11, %19, %7 sc0 sc1\n\t"
                         "s_nop 1\n\ts_waitcnt vmcnt(8)"
                         : "=&v"(tk), "=&v"(rv2)
                         : "v"(d1.rec + (lane & 1) * 8u), "v"((unsigned)OFF_TKT + q * 128u), "v"(more ? 1u : 0u), "s"(ws), "s"(b0), "s"(b0 + 4u * step),
                           "v"(v0), "v"(v0 + step), "v"(v0 + 2u * step), "v"(v0 + 3u * step),
                           "v"(o[0][0]), "v"(o[0][1]), "v"(o[0][2]), "v"(o[0][3]), "v"(o[1][0]), "v"(o[1][1]), "v"(o[1][2]), "v"(o[1][3])
                         : "memory");
        }
        if (more && !posted && accept(rv2, d1, ja, jb)) {
            want = post(d1.g >> 1 | wid << 24, ja, jb);
            posted = true;
        }
        if (d0.rdy) {
            wait_vm0();
            if (lane < 2) __hip_atomic_store((unsigned*)(ws + d0.rdy) + lane * NSLOT, call, __ATOMIC_RELAXED, __HIP_MEMORY_SCOPE_SYSTEM);
        } else if (DIAG && !late) {
            wait_vm0();
            if (lane == 0 && !dead)
                __hip_atomic_store((unsigned*)(ws + OFF_OFL) + q * 512 + ((d0.g >> 1) & 511u), call, __ATOMIC_RELAXED, __HIP_MEMORY_SCOPE_SYSTEM);
        }
        const unsigned n2 = rfl(tk) - tk_base;
        if (!more) break;
        if (!posted) {
            poll(d1, ja, jb);
            want = post(d1.g >> 1 | wid << 24, ja, jb);
        }
        ia = ja; ib = jb;
        d0 = d1;
        n1 = n2;
        d1 = describe(n1);
    }
    post(0x80000000u, 0u, 0u);
}

__device__ __forceinline__ void glds4w(unsigned m0v, unsigned voff, const char* sb) {
    constexpr int R = 8 * 65536 - 1024;
    asm volatile("s_mov_b32 m0, %0\n\ts_nop 0\n\t"
                 "global_load_lds_dwordx4 %1, %2\n\t"
                 "global_load_lds_dwordx4 %1, %3 offset:1024\n\t"
                 "global_load_lds_dwordx4 %1, %4 offset:2048\n\t"
                 "global_load_lds_dwordx4 %1, %5 offset:3072" :: "s"(m0v), "v"(voff), "s"(sb), "s"(sb + R), "s"(sb + 2 * R), "s"(sb + 3 * R) : "memory");
}
__device__ __forceinline__ bf16x8 read_tr(LAS const char* p) {
    const bf16x4 lo = __builtin_amdgcn_ds_read_tr16_b64_v4bf16((LAS bf16x4*)p), hi = __builtin_amdgcn_ds_read_tr16_b64_v4bf16((LAS bf16x4*)(p + 512));
    return __builtin_shufflevector(lo, hi, 0, 1, 2, 3, 4, 5, 6, 7);
}

template <int WR, bool BIG, bool SK5, int MODE>
__device__ __forceinline__ void k_loop(f32x4 (&acc)[BIG ? 10 : 8][4], char* ws, LAS char* lds, int u0, int u5, int pn, unsigned H, const char* wsrc, int rank,
                                       unsigned call, int lane, int wid) {
    constexpr int MT = BIG ? 10 : 8, M0 = MODE == 1 ? 8 : 0;
    constexpr bool OWN = MODE == 2;
    constexpr unsigned ns = OWN ? 16 : NSLOT, dt = MODE == 1 ? 7 : 1;
    const unsigned q4 = ((u0 >> 3) - rank - 1) & 7, q5 = ((u5 >> 3) - rank - 1) & 7;
    const bool cut = DIAG && MODE == 0 && q4 < 7;
    const unsigned steps = MODE == 2 ? 16 : MODE == 1 ? 16 - H : 112 + H, last = MODE == 1 ? 121 + q4 : cut && q4 == 6 && H < 16 ? ns - 2 : ns - 1;
    unsigned t = MODE == 1 ? 16 + q4 + 7 * H : 0, tskip = cut ? 16 + q4 + 7 * H : 999;
    unsigned skip4 = 16 + q4, skip5 = 16 + q5;
    constexpr bool has5 = BIG && !WR, has4 = WR || MODE != 1;
    constexpr int RA = MODE == 0 ? 4 : MODE == 1 ? 7 : RING_A2, RB = MODE == 0 ? 4 : MODE == 1 ? 7 : 9 - RING_A2, RW = WR ? RB : RA;
    constexpr int RS = MODE == 0 ? LSLOT : MODE == 1 ? 20480 : 16384;
    constexpr int A0 = MODE == 1 ? -16384 : 0, B0 = MODE == 0 ? 20480 : MODE == 1 ? 4096 : RA * RS;
    unsigned* tmo = (unsigned*)(ws + OFF_TMO);
    const int wc = wid & 3;
    const unsigned ldsbase = (unsigned)(size_t)lds;
    const unsigned u1 = WR ? 4 * pn + wc : u0 + wc;
    const char* g1 = OWN && WR ? wsrc + wc * 128 : ws + (WR ? OFF_B : OFF_A) + (size_t)u1 * 4096;
    const unsigned vw = (lane >> 3) * 65536 + ((lane & 7) ^ ((lane >> 4) & 1)) * 16;
    const char* g2 = ws + OFF_A + (size_t)(4 * u5 + wc) * 1024;
    const unsigned l1 = ldsbase + WR * B0 + wc * 4096, l2 = ldsbase + A0 + (16 + wc) * 1024;
    const unsigned voff = lane * 16;
    const unsigned vrow = (unsigned)OFF_READY + (WR * 64 + (lane >= 32 && has5 ? u5 : u1)) * (NSLOT * 4);
    const unsigned vsnap = vrow + (lane & 31) * 16;
    LAS char* snap = lds + LDS_FLG + wid * 1024;
    *(LAS u32x4*)(snap + voff) = (u32x4){0u, 0u, 0u, 0u};
    asm volatile("s_waitcnt lgkmcnt(0)" ::: "memory");
    unsigned hz = OWN && WR ? 4096u : 0u, look = 0;
#define ADVANCE(v) ({ const u64 m_ = __builtin_amdgcn_ballot_w64((v) >= call); \
    (unsigned)__builtin_ctzll(~(u64)((unsigned)m_ & (unsigned)(m_ >> 32))); })
#define ENSURE(t) do { if (__builtin_expect(hz <= (unsigned)(t), 0)) { const u64 t0_ = __builtin_amdgcn_s_memrealtime(); unsigned sp_ = 0; \
        __builtin_amdgcn_s_waitcnt(0xC07F); \
        for (;;) { \
            const unsigned a_ = __hip_atomic_load((const unsigned*)(ws + vrow) + min(hz + (lane & 31u), ns - 1u), __ATOMIC_RELAXED, __HIP_MEMORY_SCOPE_SYSTEM); \
            hz += ADVANCE(a_); \
            if (hz > (unsigned)(t)) break; \
            if (timed_out(tmo, call, t0_, sp_)) { hz = NSLOT; break; } \
            __builtin_amdgcn_s_sleep(GSLEEP); } } } while (0)
#define STAGE(t, lo_) do { const size_t go_ = (size_t)(t) * 262144; \
    if (OWN && WR) glds4w(l1 + lo_, vw, g1 + 8 * go_); else if (has4) glds4(l1 + lo_, voff, g1 + go_); if (has5) glds1(l2 + lo_, voff, g2 + go_); } while (0)
#define WAITV() asm volatile("s_waitcnt vmcnt(%0)" :: "i"((4 * has4 + has5) * (RW - 2)) : "memory")
#pragma unroll 1
    for (int i = 0; i < RW - 1; ++i, t = min(t + dt, last)) { ENSURE(t); STAGE(t, i * RS); }
    WAITV();
    __builtin_amdgcn_s_barrier();
    if (!MODE) STAMP(ws, 2);
    if (WR == 1) __builtin_amdgcn_s_barrier();
    const unsigned aoff = ((lane & 15) * 4 + ((lane >> 4) ^ a_swz((lane >> 2) & 3))) * 16;
    const unsigned bsw = ((lane >> 3) & 1) * 16;
    const unsigned boff = B0 + wc * 4096 + (OWN ? (lane >> 4) * 1024 + ((lane >> 2) & 3) * 128 + (lane & 3) * 32 : lane * 16);
#define LDB() _Pragma("unroll") for (int n = 0; n < 4; ++n) \
        B[n] = OWN ? read_tr(pb + (((n >> 1) * 16) ^ bsw) + (n & 1) * 8) : *(LAS const bf16x8*)(pb + n * 1024)
#define LDA(m0, m1) _Pragma("unroll") for (int m = m0; m < m1; ++m) A[m] = *(LAS const bf16x8*)(pa + A0 + (m < 8 ? 8 * WR + m : 8 + 2 * WR + m) * 1024)
#define MIDDLE() do { WAITV(); __builtin_amdgcn_s_waitcnt(0xC07F); __builtin_amdgcn_sched_barrier(0); __builtin_amdgcn_s_barrier(); \
        __builtin_amdgcn_sched_barrier(0); __builtin_amdgcn_s_setprio(1); } while (0)
#define MMA(m0, m1) _Pragma("unroll") for (int m = m0; m < m1; ++m) _Pragma("unroll") for (int k = 0; k < 4; ++k) { \
        const int n = (m & 1) ? 3 - k : k; \
        acc[m][n] = __builtin_amdgcn_mfma_f32_16x16x32_bf16(B[n], A[m], acc[m][n], 0, 0, 0); __builtin_amdgcn_sched_barrier(0); }
#define TAIL() do { __builtin_amdgcn_s_setprio(0); __builtin_amdgcn_sched_barrier(0); __builtin_amdgcn_s_barrier(); \
        __builtin_amdgcn_sched_barrier(0); } while (0)
    unsigned so = 0, sb = 0, sw = (RW - 1) * RS;
#define TRACE()
#define HEAD() TRACE(); ENSURE(t); STAGE(t, sw); __builtin_amdgcn_sched_barrier(0); \
        bf16x8 A[MT], B[4]; LAS const char* pa = lds + aoff + so; LAS const char* pb = lds + boff + (RA == RB ? so : sb)
#define NEXT(d5) sw = RW == RA ? so : sb; so = so == (RA - 1) * RS ? 0u : so + RS; \
        if (RA != RB) sb = sb == (RB - 1) * RS ? 0u : sb + RS; \
        t += dt; \
        if (MODE == 0) { const bool hit = t == tskip; t += hit ? 1u : 0u; tskip += hit ? 7u : 0u; } \
        t = min(t, last); \
        const bool do5 = s != skip5; \
        if (SK5) skip5 += do5 ? 0u : d5; \
        asm volatile("" : "+s"(t), "+s"(tskip), "+s"(so), "+s"(skip5))
#define LOOK() if (MODE == 0 && __builtin_expect(t >= look, 0)) { \
        const unsigned fv = *(volatile LAS const unsigned*)(snap + (lane >> 5) * 512 + min(hz + (lane & 31u), ns - 1u) * 4u); \
        hz += ADVANCE(fv); \
        glds1(ldsbase + LDS_FLG + wid * 1024, vsnap, ws); \
        look = max(t + 4u + LOOKAHEAD, hz) - LOOKAHEAD; }
    unsigned s = 0;
    if (BIG && MODE == 0) {
        do {
            HEAD();
            LDB();
            LDA(8, MT);
            NEXT(7u);
            unsigned cmp4 = skip4;
            if (__builtin_expect(s == skip4, 0)) {
                skip4 += 7u;
                MIDDLE();
            }
            asm volatile("" : "+s"(cmp4));
            if (__builtin_expect(s != cmp4, 1)) {
                LDA(0, 8);
                __builtin_amdgcn_sched_barrier(0);
                LOOK();
                MIDDLE();
                MMA(0, 8);
            }
            if (!SK5 || __builtin_expect(do5, 1)) { MMA(8, MT); }
            TAIL();
        } while (++s < 7u * H + (H ? 16u : 1u));
        skip5 = q5 != q4 ? 16u + 7u * H + q5 - (q5 > q4) : 999u;
    }
    for (; s < steps; ++s) {
        HEAD();
        LDB();
        LDA(M0, MT);
        __builtin_amdgcn_sched_barrier(0);
        NEXT(6u);
        LOOK();
        MIDDLE();
        MMA(M0, 8);
        if (!SK5 || __builtin_expect(do5, 1)) { MMA(8, MT); }
        TAIL();
    }
#undef TRACE
#undef HEAD
#undef NEXT
#undef LOOK
#undef TAIL
#undef LDA
#undef LDB
#undef MIDDLE
#undef MMA
#pragma unroll
    for (int m = M0; m < MT; ++m) asm volatile("" : "+v"(acc[m][0]), "+v"(acc[m][1]), "+v"(acc[m][2]), "+v"(acc[m][3]));
    if (WR == 0) __builtin_amdgcn_s_barrier();
#undef ENSURE
#undef ADVANCE
#undef STAGE
#undef WAITV
}

template <bool BIG, bool SK5>
__device__ __forceinline__ void gemm_tile(const Params& P, LAS char* lds, int u0, int u5, int pn, unsigned call, int lane, int wid) {
    constexpr int NU = BIG ? 5 : 4;
    char* ws = P.me;
    unsigned* tmo = (unsigned*)(ws + OFF_TMO);
    const int wr = wid >> 2, wc = wid & 3;
    const unsigned H = BIG && (u5 >> 3) != (u0 >> 3) ? (P.hsplit[SK5] >> (4 * (pn >> 1))) & 15u : 0u;
    f32x4 acc[2 * NU][4];
#pragma unroll
    for (int m = 0; m < 2 * NU; ++m)
#pragma unroll
        for (int n = 0; n < 4; ++n) acc[m][n] = (f32x4){0.f, 0.f, 0.f, 0.f};
    if (wr == 0) k_loop<0, BIG, SK5, 0>(acc, ws, lds, u0, u5, pn, H, nullptr, P.rank, call, lane, wid);
    else k_loop<1, BIG, SK5, 0>(acc, ws, lds, u0, u5, pn, H, nullptr, P.rank, call, lane, wid);
    const unsigned tmov = __hip_atomic_load(tmo, __ATOMIC_RELAXED, __HIP_MEMORY_SCOPE_AGENT);
    const unsigned rbl = (lane < 4 ? 2 * u0 + 4 * wr + lane : 2 * u5 + (lane & 1)) & 127u;
    char* peer = P.ws[rbl >> 4];
    LAS unsigned* meet = (LAS unsigned*)(lds + LDS_REC + 256) + wc * 4;
    if (SK5 && wr == 0 && lane == 0) meet[2] = 0u;
    asm volatile("s_waitcnt lgkmcnt(0)" ::: "memory");
    asm volatile("" :: "v"(peer), "v"(tmov));
    wait_vm0();
    __builtin_amdgcn_s_barrier();
    asm volatile("" ::: "memory");
    STAMP(ws, 3);
    const unsigned par = call & 1u;
    const int cb = 4 * pn + wc;
    if (__builtin_expect(rfl(tmov) == call, 0)) {
#pragma unroll
        for (int m = 0; m < 2 * NU; ++m)
#pragma unroll
            for (int n = 0; n < 4; ++n) acc[m][n] = (f32x4){__builtin_nanf(""), __builtin_nanf(""), __builtin_nanf(""), __builtin_nanf("")};
    }
    unsigned rb[NU], g[NU];
#pragma unroll
    for (int u = 0; u < NU; ++u) {
        rb[u] = (u < 4 ? 2 * u0 + 4 * wr + u : 2 * u5 + wr) & 127u;
        g[u] = 9216u + (rb[u] >> 4) * 1024u + cb * 16u + (rb[u] & 15u);
    }
    auto get = [&](int u, u32x4 (&d)[4]) {
#pragma unroll
        for (int c = 0; c < 4; ++c) {
            const int mi = 2 * u + (c >> 1), b = c & 1;
            d[c][0] = cvt_pk(acc[mi][0][2 * b], acc[mi][1][2 * b]);
            d[c][1] = cvt_pk(acc[mi][2][2 * b], acc[mi][3][2 * b]);
            d[c][2] = cvt_pk(acc[mi][0][2 * b + 1], acc[mi][1][2 * b + 1]);
            d[c][3] = cvt_pk(acc[mi][2][2 * b + 1], acc[mi][3][2 * b + 1]);
        }
    };
    LAS char* scr = lds + wid * SCR_WAVE;
    auto own_rows = [&](int u) {
        u32x4 d[4];
        get(u, d);
        rows_of(scr, lane, d);
        store_out(P.out + (size_t)(rb[u] & 15u) * (32 * 65536) + (size_t)(P.rank * 4096 + 64 * cb) * 2, lane, d);
    };
    const bool late = pn >= LATE_COL;
    unsigned* opos = (unsigned*)(ws + OFF_OPOS) + (rbl >> 4) * 32 + (late ? 256 : 0);
    const unsigned pos0 = late ? 32u * LATE_COL - (call - 1u) * (512u - 32u * LATE_COL) : (1u - call) * (32u * LATE_COL);
    u64* ring = (u64*)(peer + OFF_REC + par * REC_PAR) + (P.rank * 3 + 2) * 1024 + (lane & 1);
    const u64 word = (u64)call << 42 | (u64)(cb * 16u + (rbl & 15u)) << 32;
    auto place = [&](bool draw) {
        unsigned pos = 0;
        if (draw) pos = __hip_atomic_fetch_add(opos, 1u, __ATOMIC_RELAXED, __HIP_MEMORY_SCOPE_AGENT) + pos0;
        asm volatile("" :: "v"(pos));
        wait_vm0();
        return dpp0<0xa0, 0xf>(pos) & 511u;
    };
    char* pk = ws + OFF_PK + par * PK_PAR;
#pragma unroll
    for (int k = 0; k < 4; k += 2) {
        const unsigned ga[2] = {g[k], g[k + 1]};
        unsigned r[2];
        pack_store<2>([&](int u, u32x4 (&d)[4]) { get(u + k, d); }, pk, ga, lds, scr, lane, r, [] {});
        const unsigned pos = place(lane == k);
        if ((lane & ~1) == k) __hip_atomic_store(ring + 2u * pos, word | ((lane & 1) ? r[1] : r[0]), __ATOMIC_RELAXED, __HIP_MEMORY_SCOPE_SYSTEM);
    }
    if (BIG) {
        STAMP(ws, 4);
        if (DIAG && (u5 >> 3) != (u0 >> 3)) {
            asm volatile("s_waitcnt lgkmcnt(0)" ::: "memory");
            __builtin_amdgcn_s_barrier();
            if (wr == 0) k_loop<0, BIG, false, 1>(acc, ws, lds, u0, u5, pn, H, nullptr, P.rank, call, lane, wid);
            else k_loop<1, BIG, false, 1>(acc, ws, lds, u0, u5, pn, H, nullptr, P.rank, call, lane, wid);
            if (rfl(__hip_atomic_load(tmo, __ATOMIC_RELAXED, __HIP_MEMORY_SCOPE_AGENT)) == call) acc[2 * NU - 1][0][0] = __builtin_nanf("");
            wait_vm0();
            __builtin_amdgcn_s_barrier();
            asm volatile("" ::: "memory");
        }
        if (!SK5) {
            own_rows(NU - 1);
        } else {
            const unsigned ga[1] = {g[NU - 1]};
            unsigned r[1], second = 0;
            pack_store<1>([&](int, u32x4 (&d)[4]) { get(NU - 1, d); }, pk, ga, lds, scr, lane, r, [] {});
            if (lane == 0) meet[wr] = r[0];
            const unsigned pos = place(wr == 1 && lane == 4);
            if (wr == 1 && lane == 4) meet[3] = pos;
            LDS_FENCE();
            if (lane == 0) second = __hip_atomic_fetch_add(meet + 2, 1u, __ATOMIC_RELAXED, __HIP_MEMORY_SCOPE_WORKGROUP);
            LDS_FENCE();
            if (rfl(second) && (lane >> 1) == 2)
                __hip_atomic_store(ring + 2u * meet[3], word | meet[lane & 1], __ATOMIC_RELAXED, __HIP_MEMORY_SCOPE_SYSTEM);
        }
    }
}

__device__ __forceinline__ void own_part(const Params& P, LAS char* lds, unsigned j, unsigned call, int lane, int wid) {
    char* ws = P.me;
    unsigned* tmo = (unsigned*)(ws + OFF_TMO);
    const int wr = wid >> 2, wc = wid & 3;
    const unsigned half = j & 1u, pn = (j >> 1) / 7u, q = (j >> 1) % 7u, p = (P.rank + 1u + q) & 7u;
    f32x4 acc[8][4];
#pragma unroll
    for (int m = 0; m < 8; ++m)
#pragma unroll
        for (int n = 0; n < 4; ++n) acc[m][n] = (f32x4){0.f, 0.f, 0.f, 0.f};
    const char* wsrc = P.w + p * 8192u + pn * 512u;
    if (wr == 0) k_loop<0, false, false, 2>(acc, ws, lds, 8 * P.rank + 4 * half, 0, pn, 0, wsrc, P.rank, call, lane, wid);
    else k_loop<1, false, false, 2>(acc, ws, lds, 8 * P.rank + 4 * half, 0, pn, 0, wsrc, P.rank, call, lane, wid);
    wait_vm0();
    __builtin_amdgcn_s_barrier();
    asm volatile("" ::: "memory");
    u32x4 d[4][4];
#pragma unroll
    for (int u = 0; u < 4; ++u) {
#pragma unroll
        for (int c = 0; c < 4; ++c) {
            const int mi = 2 * u + (c >> 1), n = 2 * (c & 1);
            d[u][c][0] = cvt_pk(acc[mi][n][0], acc[mi][n][1]);
            d[u][c][1] = cvt_pk(acc[mi][n][2], acc[mi][n][3]);
            d[u][c][2] = cvt_pk(acc[mi][n + 1][0], acc[mi][n + 1][1]);
            d[u][c][3] = cvt_pk(acc[mi][n + 1][2], acc[mi][n + 1][3]);
        }
        rows_of(lds + wid * SCR_WAVE, lane, d[u]);
    }
    const unsigned dst = (8u * half + 4u * wr) * (32u * 65536u) + p * 8192u + (4u * pn + wc) * 128u;
    if (pn < LATE_COL) {
        const unsigned* fl = (const unsigned*)(ws + OFF_OFL) + q * 512u + (4u * pn + wc) * 8u + 4u * half + 2u * wr + (lane & 1);
        const u64 t0 = __builtin_amdgcn_s_memrealtime();
        unsigned spins = 0;
        while (__builtin_amdgcn_ballot_w64(__hip_atomic_load(fl, __ATOMIC_RELAXED, __HIP_MEMORY_SCOPE_SYSTEM) < call) != 0ull) {
            if (timed_out(tmo, call, t0, spins)) break;
            __builtin_amdgcn_s_sleep(8);
        }
        asm volatile("" ::: "memory");
        const __amdgpu_buffer_rsrc_t orsrc = __builtin_amdgcn_make_buffer_rsrc(P.out, 0, 512 * 65536, 0x00020000);
        u32x4 y[16];
#pragma unroll
        for (int c = 0; c < 16; ++c) y[c] = __builtin_amdgcn_raw_buffer_load_b128(orsrc, (lane >> 3) * 65536 + (lane & 7) * 16, dst + c * (8u * 65536u), 17);
#pragma unroll
        for (int c = 0; c < 16; ++c) d[c >> 2][c & 3] = add_bf16(d[c >> 2][c & 3], y[c]);
    }
    if (rfl(__hip_atomic_load(tmo, __ATOMIC_RELAXED, __HIP_MEMORY_SCOPE_AGENT)) == call)
        d[0][0] = (u32x4){0x7fc07fc0u, 0x7fc07fc0u, 0x7fc07fc0u, 0x7fc07fc0u};
#pragma unroll
    for (int u = 0; u < 4; ++u) store_out(P.out + dst + u * (32 * 65536), lane, d[u]);
    wait_vm0();
    __builtin_amdgcn_s_barrier();
    asm volatile("" ::: "memory");
    if (threadIdx.x == 0)
        __hip_atomic_store((unsigned*)(ws + OFF_DFL) + q * 32u + pn * 2u + half, call, __ATOMIC_RELAXED, __HIP_MEMORY_SCOPE_SYSTEM);
}

__global__ void __launch_bounds__(512, 2) fused_gemm_rs(const Params P) {
    __shared__ __attribute__((aligned(16))) char smem[LDS_BYTES];
    const int bid = blockIdx.x;
    const int lane = threadIdx.x & 63, wid = rfl(threadIdx.x >> 6);
    char* ws = P.me; const char* px = P.x; const char* pw = P.w; const u64 job = P.job[bid];
    asm volatile("" :: "s"(ws), "s"(px), "s"(pw), "s"(job), "s"(P.rank));
    STAMP(ws, 0);
    unsigned* cnt = (unsigned*)(ws + OFF_CNT) + bid * 8 + wid;
    const unsigned cntv = __hip_atomic_load(cnt, __ATOMIC_RELAXED, __HIP_MEMORY_SCOPE_AGENT);
    if (bid >= 7 * PWG) {
#pragma unroll 1
        for (int r = 0; r < 3; ++r) {
            const unsigned e = (unsigned)(job >> (11 + 11 * r)) & 0x7ffu;
            if ((e >> 7) == 15u) break;
            pack_piece(P, ws, px, pw, (LAS char*)smem, r, e, r ? 0u : (unsigned)(job >> 53) & 15u, cntv, lane, wid);
        }
        const unsigned late = (unsigned)(job >> 44) & 511u;
        if (late < P.dcount[0]) pack_bulk(P, ws, pw, (LAS char*)smem, late, rfl(cntv) + 1u, lane, wid);
        asm volatile("s_waitcnt lgkmcnt(0)" ::: "memory");
        __builtin_amdgcn_s_barrier();
    } else {
        build_lut((LAS char*)smem, true, threadIdx.x);
        if (threadIdx.x >= 384) ((LAS unsigned*)(smem + LDS_Q))[threadIdx.x - 384] = 0u;
        asm volatile("s_waitcnt lgkmcnt(0)" ::: "memory");
        __builtin_amdgcn_s_barrier();
        asm volatile("" ::: "memory");
    }
    const unsigned call = rfl(cntv) + 1u;
    if (lane == 0) __hip_atomic_store(cnt, call, __ATOMIC_RELAXED, __HIP_MEMORY_SCOPE_AGENT);
    STAMP(ws, 1);
    if (bid < 7 * PWG) {
        if (wid == 0) fetch_link(P, (LAS char*)smem, bid % 7, call, lane);
        else if (wid <= NDEC) decode_link(P, (LAS char*)smem, bid % 7, (wid - 1) * PWG + bid / 7, call, lane, wid);
    } else if (((unsigned)job & 63u) != 63u) {
        const int u0 = (unsigned)job & 63u, u5 = (unsigned)(job >> 57) & 63u, pn = ((unsigned)job >> 7) & 15u;
        const unsigned dj = (P.djob[bid >> 2] >> ((bid & 3) * 8)) & 255u;
        if (dj < 224u) own_part(P, (LAS char*)smem, dj, call, lane, wid);
        if (!(job & 64u)) gemm_tile<false, false>(P, (LAS char*)smem, u0, 0, pn, call, lane, wid);
        else if ((u5 >> 3) == P.rank) gemm_tile<true, false>(P, (LAS char*)smem, u0, u5, pn, call, lane, wid);
        else gemm_tile<true, true>(P, (LAS char*)smem, u0, u5, pn, call, lane, wid);
        if (!(job & 64u)) STAMP(ws, 4);
        STAMP(ws, 7);
        LSTAMP(ws, 1, __builtin_amdgcn_s_memrealtime());
        if (!(pn >= 12 && (job & 64u))) for (LAS unsigned* drawn = (LAS unsigned*)(smem + LDS_REC);; drawn = (LAS unsigned*)((size_t)drawn ^ 4)) {
            if (threadIdx.x == 0) *drawn = __hip_atomic_fetch_add((unsigned*)(ws + OFF_TKT + 1024), 1u, __ATOMIC_RELAXED, __HIP_MEMORY_SCOPE_AGENT);
            asm volatile("s_waitcnt lgkmcnt(0)" ::: "memory");
            __builtin_amdgcn_s_barrier();
            const unsigned k = lds_poll((LAS char*)drawn) - (call - 1u) * (224u - NPRE + 256u - 7u * PWG - 32u) + NPRE;
            if (k >= 224u) break;
            own_part(P, (LAS char*)smem, k < NLATE ? 223u - k : k - NLATE, call, lane, wid);
        }
        LSTAMP(ws, 0, lt0);
        LSTAMP(ws, 2, __builtin_amdgcn_s_memrealtime());
        return;
    }
    wait_vm0();
    LSTAMP(ws, 0, lt0);
    LSTAMP(ws, 2, __builtin_amdgcn_s_memrealtime());
    STAMP(ws, 4);
}

#define LAGS {-1, 4, 18, 29, 42, 56, 72, 86}
#define HS_OWN 0x46800000
#define HS_SK5 0x468a8640
#define PRE_TOP 7
#define DCOUNT {108, 108, 82, 56}

void make_plan(Params& P, int rank) {
    const int lag[8] = LAGS;
    unsigned char* order = (unsigned char*)P.order;
    unsigned char* worder = (unsigned char*)P.worder;
    int no = 0, nw = 0;
    P.hsplit[0] = HS_OWN; P.hsplit[1] = HS_SK5;
    for (int i = 0; i < 16; ++i) P.pushk[i] = 0;
    for (int i = 0; i < 64; ++i) P.order[i] = 0;
    for (int pos = -1; pos < 512; ++pos) {
        if (pos >= 0 && pos < 128 && pos % 8 < (int)NXQ) order[no++] = (unsigned char)pos;
        for (int pr = 0; pr < 8; ++pr) {
            const int h8 = pos - lag[pr];
            if (h8 >= 0 && h8 < 128 && h8 % 8 == 0) {
                if (nw >= PUSH0 && (nw - PUSH0) % PUSHS == 0 && (nw - PUSH0) / PUSHS < NPUSH)
                    P.pushk[no >> 3] |= (unsigned)((nw - PUSH0) / PUSHS + 1) << (4 * (no & 7));
                order[no++] = (unsigned char)(128 | h8 | pr);
                worder[nw++] = (unsigned char)(h8 | pr);
            }
        }
    }
    for (int k = 0; k < NPUSH; ++k) {
        const unsigned char v = worder[PUSH0 + k * PUSHS];
        for (int i = PUSH0 + k * PUSHS; i > PUSHP + k; --i) worder[i] = worder[i - 1];
        worder[PUSHP + k] = v;
    }
    int seq[256], nseq = 0;
    const int mix[8] = MIX;
    for (int qt = 0; qt < NBAND; ++qt)
        for (int i = 0; i < 8; ++i) {
            const int pr = mix[i], band = (qt + pr) % NBAND, nt = (FULL14 >> pr) & 1 ? 14 : 13, base = 2 * rank + 2;
            for (int r = (nt * band + NBAND / 2) / NBAND; r < (nt * (band + 1) + NBAND / 2) / NBAND; ++r)
                for (int c = 0; c < 2; ++c) {
                    const int quad = (base + (r + 2 * pr) % 14) & 15, split = (base + (13 + 2 * pr) % 14) & 15;
                    const int k = 12 - r;
                    const int fifth = nt == 14 ? (r % 2 == 0 ? 8 * rank + r / 2 : r / 2 == 6 * c ? 8 * rank + 7 : -1)
                                               : k == 12 ? -1 : k % 3 == 0 ? 4 * split + k / 3 : 8 * rank + k - 1 - k / 3;
                    seq[nseq++] = (4 * quad | (fifth >= 0) << 6) | (2 * pr + c) << 7 | (fifth >= 0 ? fifth : 0) << 11;
                }
        }
    int tile[256], byp[256], nby = 0;
    static_assert(PWG == 6 && __builtin_popcount(FULL14) == 3 && (FULL14 & 0xc0) == 0xc0 && PUSHP + NPUSH <= 12 && PUSHP <= PUSH0, "214 tiles on 214 workgroups; pushed pieces: first round, first 64 entries");
    for (int x = 0, k = 0; x < 8; ++x)
        for (int b = 248 + x; b >= 7 * PWG; b -= 8) tile[b] = seq[k++];
    for (int pr = 0; pr <= 8; ++pr)
        for (int b = 7 * PWG; b < 256; ++b)
            if (((tile[b] >> 8) & 7) == pr) byp[nby++] = b;
    int first[10];
    for (int pr = 0, i = 0; pr <= 9; ++pr) {
        while (i < nby && ((tile[byp[i]] >> 8) & 7) < pr) ++i;
        first[pr] = i;
    }
    int nr = 0;
    for (int b = 0; b < 256; ++b)
        P.job[b] = (b < 7 * PWG ? 63ull : (u64)(tile[b] & 0x7ff) | (u64)(tile[b] >> 11) << 57) | 0x1ffffffffull << 11 | 511ull << 44;
    for (int r = 0; r < 3; ++r) {
        int list[256], n = 0;
        const int own = (rank + 1) << 7, from = r == 0 ? 0 : first[2 * r];
        if (r < 2) {
            for (int h = 8 * r; h < 8 * r + 8; ++h) {
                for (int q = 0; q < 8; ++q) list[n++] = 9 << 7 | h << 3 | q;
                list[n++] = own | h << 3; list[n++] = own | h << 3 | 1;
            }
            for (int h = 2 * r; h < (r ? 16 : 2); ++h) { list[n++] = own | h << 3 | 2; list[n++] = own | h << 3 | 3; }
        } else {
            for (int h = 0; h < 16; ++h)
                for (int q = 4; q < 8; ++q) list[n++] = own | h << 3 | q;
        }
        for (int i = 48 * r; i < (r < 2 ? 48 * r + 48 : 128); ++i)
            if (!DIAG || i % 8 != rank) list[n++] = i;
        while (n < nby - from - (r == 2 ? (nr & 1) != ((nby - from - n) & 1) : 0)) {
            if (nr >= 7 * PUSHP && nr < 7 * (PUSHP + NPUSH)) P.job[byp[from + n]] |= (u64)(nr / 7 - PUSHP + 1) << 53;
            list[n++] = (((rank + 1 + nr % 7) & 7) + 1) << 7 | worder[nr / 7];
            ++nr;
        }
        for (int i = 0; i < n; ++i) {
            u64& j = P.job[byp[from + i]];
            j = (j & ~(0x7ffull << (11 + 11 * r))) | (u64)list[i] << (11 + 11 * r);
        }
    }
    P.dstart = nr;
    const int dcount[] = DCOUNT;
    for (int r = 0, left = (896 - nr) / 2; r < 12; ++r) {
        const int n = dcount[r < (int)(sizeof dcount / sizeof dcount[0]) ? r : sizeof dcount / sizeof dcount[0] - 1];
        P.dcount[r] = r == 11 ? 0 : n < left ? n : left;
        left -= P.dcount[r];
    }
    for (int i = 0; i < nby; ++i) P.job[byp[i]] = (P.job[byp[i]] & ~(511ull << 44)) | (u64)(nby - 1 - i) << 44;
    unsigned char* djob = (unsigned char*)P.djob;
    for (int b = 0; b < 256; ++b) djob[b] = 255;
    unsigned j = 0;
    for (int pr = PRE_TOP; pr >= 0; --pr)
        for (int x = 0; x < 8; ++x)
            for (int i = first[pr]; i < first[pr + 1] && j < NPRE; ++i)
                if (byp[i] % 8 == x && !(tile[byp[i]] & 64)) djob[byp[i]] = (unsigned char)(223 - j++);
}

}

extern "C" void kernel_launch(int rank, int nranks, ncclComm_t comm, hipStream_t stream,
                              void* const* d_in, const int* in_sizes, int n_in,
                              void* d_out, int out_size, const int* in_dtypes, int out_dtype,
                              void* const* d_out_all, void* const* d_ws_all, size_t ws_bytes) {
    (void)comm; (void)in_dtypes; (void)out_dtype; (void)d_out_all;
    if (nranks != NR || (unsigned)rank >= (unsigned)NR || n_in < 2 || ws_bytes < WS_NEED || in_sizes[0] != 4096 * 512 ||
        in_sizes[1] != 512 * 32768 || out_size != 512 * 32768) {
        fprintf(stderr, "kernel_launch: unexpected problem shape\n");
        return;
    }
    Params P;
    P.x = (const char*)d_in[0];
    P.w = (const char*)d_in[1];
    P.out = (char*)d_out;
    for (int p = 0; p < NR; ++p) P.ws[p] = (char*)d_ws_all[p];
    P.me = P.ws[rank];
    P.rank = rank;
    make_plan(P, rank);
    hipLaunchKernelGGL(fused_gemm_rs, dim3(256), dim3(512), 0, stream, P);
    hipError_t e = hipGetLastError();
    if (e != hipSuccess) fprintf(stderr, "launch failed: %s\n", hipGetErrorString(e));
}
```

```cpp
#include <hip/hip_runtime.h>
#include <rccl/rccl.h>
#include <cstdio>

#define LAS __attribute__((address_space(3)))
typedef __bf16 bf16x8 __attribute__((ext_vector_type(8)));
typedef __bf16 bf16x4 __attribute__((ext_vector_type(4)));
typedef float f32x4 __attribute__((ext_vector_type(4)));
typedef unsigned u32x4 __attribute__((ext_vector_type(4)));
typedef unsigned u32x2 __attribute__((ext_vector_type(2)));
typedef unsigned long long u64;

namespace {

constexpr int NR = 8;
constexpr int NSLOT = 128;
constexpr int LSLOT = 36 * 1024;
constexpr int LDS_LUT = 4 * LSLOT;
constexpr int LDS_REC = LDS_LUT + 2048;
constexpr int LDS_FLG = LDS_REC + 512;
constexpr int LDS_BYTES = LDS_FLG + 8192;
constexpr int SCR_UNIT = 1408;
constexpr int SCR_WAVE = 5 * SCR_UNIT;
constexpr int SCR_T = 1152;
#define PWG 6
#define NDEC 7
#define LOOKAHEAD 10u
#define NFLY 10
#define RING_A2 3
#define NBAND 2
#define MIX {0, 7, 3, 4, 1, 6, 2, 5}
#define NPUSH 6
#define PUSH0 5
#define PUSHS 3
#define PUSHP 4
#define FULL14 0xc1
constexpr int PW = PWG * NDEC;
#define LATE_COL 14u
#define NPRE 28u
constexpr unsigned NLATE = (16u - LATE_COL) * 14u;
static_assert(NPRE <= NLATE, "only jobs that wait for nothing run early");
constexpr unsigned NXQ = 7;
constexpr unsigned NIN = 64 * NXQ + 512;
constexpr unsigned NTK = NIN + 512;
constexpr u64 DEADLINE = 100000000ull;

constexpr size_t OFF_CNT = 0;
constexpr size_t OFF_TMO = 16384;
constexpr size_t OFF_TKT = 20480;
constexpr size_t OFF_OPOS = 24576;
constexpr size_t OFF_DFL = 28672;
constexpr size_t OFF_READY = 32768;
constexpr size_t OFF_OFL = 98304;
constexpr size_t OFF_REC = 131072;
constexpr size_t REC_PAR = (size_t)NR * 3 * 1024 * 8;
constexpr size_t OFF_DBG = 655360;
constexpr size_t OFF_A = 1u << 20;
constexpr size_t OFF_B = OFF_A + (32u << 20);
constexpr int NUNIT = 1024 + 2 * NR * 1024;
constexpr int FIXB = 2560;
constexpr int VCAP = 2624;
constexpr int PAIRB = 2 * FIXB + 2 * VCAP;
constexpr int VMID = 2 * FIXB + VCAP;
constexpr size_t PK_PAR = (size_t)(NUNIT / 2) * PAIRB;
constexpr size_t OFF_PK = OFF_B + (32u << 20);
constexpr size_t WS_NEED = OFF_PK + 2 * PK_PAR;
static_assert(OFF_REC + 2 * REC_PAR <= OFF_DBG, "layout");

struct Params {
    const char* x;
    const char* w;
    char* out;
    char* me;
    char* ws[NR];
    int rank;
    unsigned dstart;
    unsigned dcount[12];
    unsigned hsplit[2];
    unsigned pushk[16];
    unsigned order[64];
    unsigned worder[32];
    u64 job[256];
    unsigned djob[64];
};

#define STAMP(ws, idx) do { } while (0)
#define LSTAMP(ws, idx, t) do { } while (0)

__device__ __forceinline__ unsigned rfl(unsigned v) { return __builtin_amdgcn_readfirstlane(v); }
__device__ __forceinline__ unsigned perm(unsigned hi, unsigned lo, unsigned sel) { return __builtin_amdgcn_perm(hi, lo, sel); }
__device__ __forceinline__ unsigned bfi(unsigned m, unsigned a, unsigned b) { return (a & m) | (b & ~m); }
__device__ __forceinline__ int a_swz(int q) { return (0x78 >> (2 * q)) & 3; }
__device__ __forceinline__ unsigned cvt_pk(float lo, float hi) { unsigned r; asm volatile("v_cvt_pk_bf16_f32 %0, %1, %2" : "=v"(r) : "v"(lo), "v"(hi)); return r; }

__device__ __forceinline__ u32x4 add_bf16(u32x4 x, u32x4 y) {
    u32x4 r;
#pragma unroll
    for (int a = 0; a < 4; ++a)
        r[a] = cvt_pk(__uint_as_float(x[a] << 16) + __uint_as_float(y[a] << 16), __uint_as_float(x[a] & 0xffff0000u) + __uint_as_float(y[a] & 0xffff0000u));
    return r;
}

template <int OFF> __device__ __forceinline__ void st_wt(const char* sbase, unsigned voff, u32x4 v) {
    asm volatile("global_store_dwordx4 %0, %1, %2 offset:%3 sc0 sc1\n\ts_nop 1" :: "v"(voff), "v"(v), "s"(sbase), "i"(OFF) : "memory");
}
__device__ __forceinline__ void glds4(unsigned m0v, unsigned voff, const char* sb) {
    asm volatile("s_mov_b32 m0, %0\n\ts_nop 0\n\t"
                 "global_load_lds_dwordx4 %1, %2 sc0 sc1\n\t"
                 "global_load_lds_dwordx4 %1, %2 offset:1024 sc0 sc1\n\t"
                 "global_load_lds_dwordx4 %1, %2 offset:2048 sc0 sc1\n\t"
                 "global_load_lds_dwordx4 %1, %2 offset:3072 sc0 sc1" :: "s"(m0v), "v"(voff), "s"(sb) : "memory");
}
__device__ __forceinline__ void glds1(unsigned m0v, unsigned voff, const char* sb) {
    asm volatile("s_mov_b32 m0, %0\n\ts_nop 0\n\tglobal_load_lds_dwordx4 %1, %2 sc0 sc1" :: "s"(m0v), "v"(voff), "s"(sb) : "memory");
}
__device__ __forceinline__ void wait_vm0() { asm volatile("s_waitcnt vmcnt(0)" ::: "memory"); }

__device__ __forceinline__ bool timed_out(unsigned* tmo, unsigned call, u64 t0, unsigned& spins) {
    if ((++spins & 63u) != 0u) return false;
    if (rfl(__hip_atomic_load(tmo, __ATOMIC_RELAXED, __HIP_MEMORY_SCOPE_AGENT)) == call) return true;
    if (__builtin_amdgcn_s_memrealtime() - t0 > DEADLINE) {
        __hip_atomic_store(tmo, call, __ATOMIC_RELAXED, __HIP_MEMORY_SCOPE_AGENT);
        return true;
    }
    return false;
}

#define LDS_FENCE() asm volatile("" ::: "memory")
template <int CTRL, int ROWMASK> __device__ __forceinline__ unsigned dpp0(unsigned v) {
    return (unsigned)__builtin_amdgcn_update_dpp(0, (int)v, CTRL, ROWMASK, 0xf, false);
}
__device__ __forceinline__ unsigned wave_scan(unsigned v) {
    v += dpp0<0x111, 0xf>(v); v += dpp0<0x112, 0xf>(v); v += dpp0<0x114, 0xf>(v); v += dpp0<0x118, 0xf>(v);
    v += dpp0<0x142, 0xa>(v); v += dpp0<0x143, 0xc>(v);
    return v;
}
__device__ __forceinline__ unsigned lane_rank(u64 m) {
    return __builtin_amdgcn_mbcnt_hi((unsigned)(m >> 32), __builtin_amdgcn_mbcnt_lo((unsigned)m, 0u));
}
__device__ __forceinline__ void build_lut(LAS char* lds, bool dec, int tid) {
    if (tid < 256) {
        u64 e = 0x0c0c0c0c0c0c0c0cull;
        unsigned t = 0;
#pragma unroll
        for (unsigned j = 0; j < 8; ++j)
            if ((tid >> j) & 1) {
                const unsigned slot = (t & 1u) * 4u + (t >> 1);
                const unsigned at = dec ? j : slot, val = dec ? slot : j;
                e = (e & ~(0xffull << (8 * at))) | ((u64)val << (8 * at));
                ++t;
            }
        *(LAS u64*)(lds + LDS_LUT + tid * 8) = e;
    }
}
__device__ __forceinline__ u32x2 lut_at(LAS char* lds, unsigned f) {
    return *(LAS const u32x2*)(lds + __builtin_amdgcn_udot4(f, 0x40201008u, (unsigned)LDS_LUT, false));
}
__device__ __forceinline__ unsigned shuffle8(unsigned lo, unsigned hi, u32x2 sel) {
    return bfi(0x0f0f0f0fu, perm(hi, lo, sel[0]), perm(hi, lo, sel[1]) << 4);
}
__device__ __forceinline__ unsigned half16(unsigned v, int u) { return (u & 1) ? v >> 16 : v & 0xffffu; }
__device__ __forceinline__ unsigned b_start(unsigned n1) { return n1 <= 512u ? ((n1 + 15u) >> 4) << 5 : ((n1 - 249u) >> 3) << 5; }
__device__ __forceinline__ void lds_or(LAS char* p, unsigned v) {
    __hip_atomic_fetch_or((LAS unsigned*)p, v, __ATOMIC_RELAXED, __HIP_MEMORY_SCOPE_WAVEFRONT);
}
__device__ __forceinline__ unsigned bit_addr(LAS char* p) { return (unsigned)(size_t)p * 8u; }
__device__ __forceinline__ void put_bits(unsigned pos, unsigned v) {
    const u64 sh = (u64)v << (pos & 31u);
    LAS char* p = (LAS char*)(size_t)((pos >> 3) & ~3u);
    lds_or(p, (unsigned)sh); lds_or(p + 4, (unsigned)(sh >> 32));
}
__device__ __forceinline__ unsigned get_bits(unsigned pos) {
    LAS const unsigned* p = (LAS const unsigned*)(size_t)((pos >> 3) & ~3u);
    return __builtin_amdgcn_alignbit(p[1], p[0], pos);
}

template <int NU, class Get, class Mid>
__device__ __forceinline__ void pack_store(Get get, char* pk, const unsigned (&g)[NU], LAS char* lds, LAS char* scr, int lane,
                                           unsigned (&info)[NU], Mid mid) {
    constexpr int NS = (NU + 1) / 2;
    const unsigned voff = lane * 16;
#pragma unroll
    for (int u = 0; u < NU; ++u) *(LAS u32x4*)(scr + u * SCR_UNIT + voff) = (u32x4){0u, 0u, 0u, 0u};
    unsigned D[NU][8], pc[NU][3], cnt[NS], x16[NU], top = 0, top4 = 0;
    u32x2 sel[NU][4];
#pragma unroll
    for (int u = 0; u < NU; ++u) {
        u32x4 d[4], S[2];
        unsigned E[8], C[4], c = 0;
        get(u, d);
#pragma unroll
        for (int hf = 0; hf < 2; ++hf)
#pragma unroll
            for (int a = 0; a < 4; ++a) {
                const unsigned p = __builtin_amdgcn_alignbit(d[2 * hf][a], d[2 * hf][a], 31);
                const unsigned r = __builtin_amdgcn_alignbit(d[2 * hf + 1][a], d[2 * hf + 1][a], 31);
                S[hf][a] = perm(r, p, 0x06040200u);
                E[4 * hf + a] = perm(r, p, 0x07050301u);
            }
        char* pair = pk + (size_t)(g[u] >> 1) * PAIRB;
        const char* fix = pair + (g[u] & 1u) * FIXB;
        if (u == 0) {
            unsigned sum = __builtin_amdgcn_sad_u8(E[0], 0u, __builtin_amdgcn_sad_u8(E[2], 0u, __builtin_amdgcn_sad_u8(E[5], 0u, __builtin_amdgcn_sad_u8(E[7], 0u, 0u))));
            sum = __builtin_amdgcn_readlane(wave_scan(sum), 63);
            top = min(((sum + 983u) >> 10) + 2u, 255u);
            top4 = top * 0x01010101u;
        }
        st_wt<0>(fix, voff, S[0]);
#pragma unroll
        for (int a = 0; a < 4; ++a) {
            D[u][a] = top4 - E[a]; D[u][4 + a] = top4 - E[4 + a];
            unsigned n = bfi(0x0f0f0f0fu, D[u][a], D[u][4 + a] << 4);
            asm("" : "+v"(n));
            unsigned o = n | (n >> 1);
            asm("" : "+v"(o));
            const unsigned f = (~o | (o >> 2)) & 0x11111111u;
            unsigned f3 = (f << 1) + f;
            asm("" : "+v"(f3));
            C[a] = n & 0x33333333u & ~f3;
            c += __builtin_popcount(f);
            if (a < 3) pc[u][a] = c;
            sel[u][a] = lut_at(lds, f);
            if (a == 1) st_wt<1024>(fix, voff, S[1]);
        }
        cnt[u >> 1] = (u & 1) ? cnt[u >> 1] | c << 16 : c;
        const u32x2 cc = {C[0] | (C[1] << 2), C[2] | (C[3] << 2)};
        asm volatile("global_store_dwordx2 %0, %1, %2 offset:2048 sc0 sc1" :: "v"(lane * 8), "v"(cc), "s"(fix) : "memory");
        info[u] = top; x16[u] = 0;
        const u64 em = __builtin_amdgcn_ballot_w64(((D[u][0] | D[u][1] | D[u][2] | D[u][3] | D[u][4] | D[u][5] | D[u][6] | D[u][7]) & 0xf0f0f0f0u) != 0u);
        if (__builtin_expect(em != 0ull, 0)) {
            u32x4 X;
#pragma unroll
            for (int a = 0; a < 4; ++a) X[a] = bfi(0xf0f0f0f0u, D[u][a], D[u][4 + a] >> 4);
            const unsigned l = __builtin_ctzll(em);
            const unsigned x0 = __builtin_amdgcn_readlane(X[0], l), x1 = __builtin_amdgcn_readlane(X[1], l);
            const unsigned x2 = __builtin_amdgcn_readlane(X[2], l), x3 = __builtin_amdgcn_readlane(X[3], l);
            const unsigned any = x0 | x1 | x2 | x3, nz = (x0 != 0u) + (x1 != 0u) + (x2 != 0u) + (x3 != 0u);
            const unsigned nib = __builtin_ctz(any) >> 2, val = any >> (4u * nib);
            if ((em & (em - 1ull)) == 0ull && nz == 1u && val < 16u) {
                info[u] |= (l | (x0 ? 0u : x1 ? 1u : x2 ? 2u : 3u) << 6 | nib << 8 | val << 11) << 17;
            } else {
                info[u] |= 1u << 16;
                x16[u] = 1u + __builtin_popcountll(em);
                const unsigned k = 1u + lane_rank(em);
                if (lane == 0) st_wt<0>(pair, (g[u] & 1u) ? VMID : VMID - 16, (u32x4){(unsigned)em, (unsigned)(em >> 32), 0u, 0u});
                if ((em >> lane) & 1ull) st_wt<0>(pair, (g[u] & 1u) ? VMID + 16u * k : VMID - 16 - 16u * k, X);
            }
        }
    }
    unsigned excl[NS], tot[NS];
#pragma unroll
    for (int k = 0; k < NS; ++k) excl[k] = wave_scan(cnt[k]);
#pragma unroll
    for (int k = 0; k < NS; ++k) { tot[k] = __builtin_amdgcn_readlane(excl[k], 63); excl[k] -= cnt[k]; }
#pragma unroll
    for (int u = 0; u < NU; ++u)
        if (__builtin_expect(half16(tot[u >> 1], u) > 1400u, 0) && lane < 24) *(LAS u32x4*)(scr + u * SCR_UNIT + 1024 + voff) = (u32x4){0u, 0u, 0u, 0u};
    LDS_FENCE();
#pragma unroll
    for (int u = 0; u < NU; ++u) {
        unsigned pos = half16(excl[u >> 1], u) * 4u + bit_addr(scr + u * SCR_UNIT);
        asm("" : "+v"(pos));
#pragma unroll
        for (int a = 0; a < 4; ++a) put_bits(a ? pos + pc[u][a - 1] * 4u : pos, shuffle8(D[u][a], D[u][4 + a], sel[u][a]));
    }
    LDS_FENCE();
    mid();
    unsigned w[NU], c2n[NS];
    u32x2 s2[NU];
#pragma unroll
    for (int u = 0; u < NU; ++u) w[u] = *(LAS const unsigned*)(scr + u * SCR_UNIT + lane * 4);
    LDS_FENCE();
#pragma unroll
    for (int u = 0; u < NU; ++u) {
        const unsigned e = ((w[u] >> 1) | (w[u] >> 3)) & 0x11111111u;
        const unsigned c2 = (w[u] & 0x55555555u & ~(e << 2)) | e;
        s2[u] = lut_at(lds, e);
        const unsigned c = __builtin_popcount(e);
        c2n[u >> 1] = (u & 1) ? c2n[u >> 1] | c << 16 : c;
        *(LAS unsigned short*)(scr + u * SCR_UNIT + 128 + lane * 2) = (unsigned short)(c2 | (c2 >> 15));
    }
    unsigned in2[NS];
#pragma unroll
    for (int k = 0; k < NS; ++k) in2[k] = wave_scan(c2n[k]);
    LDS_FENCE();
#pragma unroll
    for (int u = 0; u < NU; ++u)
        put_bits(bit_addr(scr + u * SCR_UNIT + 128) + b_start(half16(tot[u >> 1], u)) + half16(in2[u >> 1] - c2n[u >> 1], u) * 4u,
                 shuffle8(w[u], w[u] >> 4, s2[u]));
    LDS_FENCE();
    u32x4 vv[NU];
#pragma unroll
    for (int u = 0; u < NU; ++u) vv[u] = *(LAS const u32x4*)(scr + u * SCR_UNIT + 128 + voff);
#pragma unroll
    for (int u = 0; u < NU; ++u) {
        const unsigned c16 = (b_start(half16(tot[u >> 1], u)) + half16(__builtin_amdgcn_readlane(in2[u >> 1], 63), u) * 4u + 127u) >> 7;
        const char* vg = pk + (size_t)(g[u] >> 1) * PAIRB + VMID + ((g[u] & 1u) ? 16 * (int)x16[u] : -16 * (int)(x16[u] + c16));
        if ((unsigned)lane < c16) st_wt<0>(vg, voff, vv[u]);
        if (__builtin_expect(c16 > 64u, 0) && 64u + lane < c16) st_wt<1024>(vg, voff, *(LAS const u32x4*)(scr + u * SCR_UNIT + 1152 + voff));
        info[u] |= (x16[u] + c16) << 8;
    }
    LDS_FENCE();
}

template <class Mid>
__device__ __forceinline__ void unpack2(const u32x4 (&p)[2][2], const u32x2 (&cc)[2], const unsigned (&info)[2], unsigned la, LAS char* lds,
                                        LAS char* T, LAS char* M, int lane, u32x4 (&o)[2][4], Mid mid) {
    u32x4 X[2] = {{0u, 0u, 0u, 0u}, {0u, 0u, 0u, 0u}};
    LAS char* A[2] = {M - 16u * la, M};
    const bool anyx = ((info[0] | info[1]) >> 16) != 0u;
    if (__builtin_expect(anyx, 0)) {
#pragma unroll
        for (int u = 0; u < 2; ++u) {
            if (info[u] & 0x10000u) {
                const u32x2 mv = *(LAS const u32x2*)(u ? M : M - 16);
                const u64 em = (u64)rfl(mv[0]) | (u64)rfl(mv[1]) << 32;
                const unsigned k = 1u + lane_rank(em);
                if ((em >> lane) & 1ull) X[u] = *(LAS const u32x4*)(u ? M + 16u * k : M - 16 - 16u * k);
                if (u) A[1] = M + 16u * (1u + __builtin_popcountll(em));
            } else {
                const unsigned xv = (info[u] >> 28) << (((info[u] >> 25) & 7u) * 4u), xa = (info[u] >> 23) & 3u;
                const bool me = (unsigned)lane == ((info[u] >> 17) & 63u);
#pragma unroll
                for (unsigned a = 0; a < 4; ++a) X[u][a] = me && xa == a ? xv : 0u;
            }
        }
    }
    unsigned z[2], rr[2];
#pragma unroll
    for (int u = 0; u < 2; ++u) {
        z[u] = *(LAS const unsigned short*)(A[u] + lane * 2);
        rr[u] = *(LAS const unsigned*)(A[u] + 128 + lane * 4);
    }
    unsigned pc[2][3], cnt = 0;
    u32x2 sel[2][4];
#pragma unroll
    for (int u = 0; u < 2; ++u) {
        unsigned c = 0;
#pragma unroll
        for (int k = 0; k < 2; ++k) {
            const unsigned h = cc[u][k] >> 1;
            const unsigned f0 = ~(cc[u][k] | h) & 0x11111111u, f1 = ~(cc[u][k] | h) & 0x44444444u;
            c += __builtin_popcount(f0);
            pc[u][2 * k] = c;
            c += __builtin_popcount(f1);
            if (k == 0) pc[u][1] = c;
            sel[u][2 * k] = lut_at(lds, f0);
            sel[u][2 * k + 1] = *(LAS const u32x2*)(lds + __builtin_amdgcn_udot4(f1, 0x10080402u, (unsigned)LDS_LUT, false));
        }
        cnt |= c << (16 * u);
    }
    unsigned c2[2], f2[2], c2n = 0;
    u32x2 s2[2];
#pragma unroll
    for (int u = 0; u < 2; ++u) {
        c2[u] = (z[u] | (z[u] << 15)) & 0x55555555u;
        asm("" : "+v"(c2[u]));
        f2[u] = c2[u] & ~(c2[u] >> 2) & 0x11111111u;
        s2[u] = lut_at(lds, f2[u]);
        c2n |= (unsigned)__builtin_popcount(f2[u]) << (16 * u);
        *(LAS unsigned*)(T + u * SCR_T + 256 + lane * 4) = rr[u];
    }
    const unsigned incl = wave_scan(cnt), in2 = wave_scan(c2n), tot = __builtin_amdgcn_readlane(incl, 63);
    unsigned wv[2];
#pragma unroll
    for (int u = 0; u < 2; ++u) wv[u] = get_bits(bit_addr(A[u]) + b_start(half16(tot, u)) + half16(in2 - c2n, u) * 4u);
#pragma unroll
    for (int u = 0; u < 2; ++u) {
        *(LAS unsigned*)(T + u * SCR_T + lane * 4) = (c2[u] ^ f2[u]) | shuffle8(wv[u], wv[u] >> 4, s2[u]);
        if (__builtin_expect(half16(tot, u) > 1024u, 0)) *(LAS u32x2*)(T + u * SCR_T + 512 + lane * 8) = *(LAS const u32x2*)(A[u] + 384 + lane * 8);
    }
    LDS_FENCE();
    unsigned w[2][4];
#pragma unroll
    for (int u = 0; u < 2; ++u) {
        unsigned pos = half16(incl - cnt, u) * 4u + bit_addr(T + u * SCR_T);
        asm("" : "+v"(pos));
#pragma unroll
        for (int a = 0; a < 4; ++a) w[u][a] = get_bits(a ? pos + pc[u][a - 1] * 4u : pos);
    }
    LDS_FENCE();
    mid();
    unsigned e[2][8];
#pragma unroll
    for (int u = 0; u < 2; ++u) {
        const unsigned top4 = (info[u] & 0xffu) * 0x01010101u;
#pragma unroll
        for (int a = 0; a < 4; ++a) {
            const unsigned lo = w[u][a] & 0x0f0f0f0fu, hi = (w[u][a] >> 4) & 0x0f0f0f0fu;
            e[u][a] = top4 - (perm(hi, lo, sel[u][a][0]) | ((cc[u][a >> 1] >> (2 * (a & 1))) & 0x03030303u));
            e[u][4 + a] = top4 - (perm(hi, lo, sel[u][a][1]) | ((cc[u][a >> 1] >> (2 * (a & 1) + 4)) & 0x03030303u));
        }
    }
    if (__builtin_expect(anyx, 0)) {
#pragma unroll
        for (int u = 0; u < 2; ++u)
#pragma unroll
            for (int a = 0; a < 4; ++a) { e[u][a] -= X[u][a] & 0xf0f0f0f0u; e[u][4 + a] -= (X[u][a] << 4) & 0xf0f0f0f0u; }
    }
#pragma unroll
    for (int u = 0; u < 2; ++u)
#pragma unroll
        for (int a = 0; a < 4; ++a) {
            const unsigned q0 = perm(e[u][a], p[u][0][a], 0x05010400u), q1 = perm(e[u][a], p[u][0][a], 0x07030602u);
            const unsigned q2 = perm(e[u][4 + a], p[u][1][a], 0x05010400u), q3 = perm(e[u][4 + a], p[u][1][a], 0x07030602u);
            o[u][0][a] = __builtin_amdgcn_alignbit(q0, q0, 1); o[u][1][a] = __builtin_amdgcn_alignbit(q1, q1, 1);
            o[u][2][a] = __builtin_amdgcn_alignbit(q2, q2, 1); o[u][3][a] = __builtin_amdgcn_alignbit(q3, q3, 1);
        }
}
__device__ __forceinline__ void store_stage(const char* dst, unsigned voff, const u32x4 (&o)[4]) {
    st_wt<0>(dst, voff, o[0]); st_wt<1024>(dst, voff, o[1]); st_wt<2048>(dst, voff, o[2]); st_wt<3072>(dst, voff, o[3]);
}
__device__ __forceinline__ void rows_of(LAS char* buf, int lane, u32x4 (&o)[4]) {
    LAS char* wp = buf + (lane & 15) * 128;
    const unsigned x = (((lane >> 3) & 6) ^ (lane & 7)) * 16;
    LDS_FENCE();
    *(LAS u32x4*)(wp + x) = o[0]; *(LAS u32x4*)(wp + (x ^ 16)) = o[1];
    *(LAS u32x4*)(wp + 2048 + x) = o[2]; *(LAS u32x4*)(wp + 2048 + (x ^ 16)) = o[3];
    LDS_FENCE();
    LAS const char* rp = buf + (lane >> 3) * 128 + (((lane >> 3) ^ lane) & 7) * 16;
#pragma unroll
    for (int c = 0; c < 4; ++c) o[c] = *(LAS const u32x4*)(rp + c * 1024);
    LDS_FENCE();
}
__device__ __forceinline__ void store_out(const char* dst, int lane, const u32x4 (&o)[4]) {
    const unsigned voff = (lane >> 3) * 65536 + (lane & 7) * 16;
#pragma unroll
    for (int c = 0; c < 4; ++c) st_wt<0>(dst + c * (8 * 65536), voff, o[c]);
}

__device__ __forceinline__ void pack_piece(const Params& P, char* ws, const char* px, const char* pw, LAS char* lds, int round, unsigned e,
                                           unsigned inbox, unsigned cntv, int lane, int wid) {
    LAS unsigned* lrec = (LAS unsigned*)(lds + LDS_REC) + 32 + round * 8;
    const int li = lane & 15, lg = lane >> 4;
    unsigned* ready = (unsigned*)(ws + OFF_READY);
    const int h = (e >> 3) & 15, idx = (e & 7) * 8 + wid, id = h * 64 + idx;
    const unsigned kind = e >> 7;
    const bool isx = kind == 0u || kind == 9u;
    const unsigned dest = (kind - 1u) & 7u;
    const bool own = kind == 9u || kind == (unsigned)P.rank + 1u;
    const unsigned rdest = isx ? (unsigned)lane >> 3 : dest;
    char* peer = P.ws[rdest];
    u32x4 d[4];
    u32x2 v[8];
    if (isx) {
        const char* src = px + (size_t)(64 * idx + (lane >> 2)) * 1024 + h * 64 + ((lane & 3) ^ a_swz(lg)) * 16;
#pragma unroll
        for (int c = 0; c < 4; ++c) {
            const u32x4 t = *(const u32x4*)(src + 16 * c * 1024);
            v[2 * c] = (u32x2){t[0], t[1]}; v[2 * c + 1] = (u32x2){t[2], t[3]};
        }
    } else {
        const char* src = pw + (size_t)(32 * h + 8 * lg) * 65536 + (size_t)(dest * 4096 + 64 * idx + 4 * li) * 2;
#pragma unroll
        for (int j = 0; j < 8; ++j) v[j] = *(const u32x2*)(src + (size_t)j * 65536);
    }
    __builtin_amdgcn_sched_barrier(0);
    if (round == 0) {
        build_lut(lds, false, threadIdx.x);
        if (threadIdx.x >= 448) ((LAS unsigned*)(lds + LDS_FLG))[(threadIdx.x - 448) * 32 + 16] = 0u;
        asm volatile("s_waitcnt lgkmcnt(0)" ::: "memory");
        __builtin_amdgcn_s_barrier();
        asm volatile("" ::: "memory");
    }
    if (isx) {
#pragma unroll
        for (int c = 0; c < 4; ++c) d[c] = (u32x4){v[2 * c][0], v[2 * c][1], v[2 * c + 1][0], v[2 * c + 1][1]};
    } else {
#pragma unroll
        for (int hf = 0; hf < 2; ++hf)
#pragma unroll
            for (int a = 0; a < 4; ++a) {
                d[2 * hf][a] = perm(v[2 * a + 1][hf], v[2 * a][hf], 0x05040100u);
                d[2 * hf + 1][a] = perm(v[2 * a + 1][hf], v[2 * a][hf], 0x07060302u);
            }
    }
    asm volatile("" :: "v"(peer));
    const unsigned call = rfl(cntv) + 1u, par = call & 1u;
    if (own) {
        store_stage(ws + (isx ? OFF_A : OFF_B) + (size_t)(h * 256 + 4 * idx) * 1024, lane * 16, d);
        wait_vm0();
        if (lane == 0) __hip_atomic_store(ready + ((isx ? 0 : 64) + idx) * NSLOT + h, call, __ATOMIC_RELAXED, __HIP_MEMORY_SCOPE_SYSTEM);
        return;
    }
    const unsigned g[1] = {isx ? id : 1024u + dest * 1024u + (inbox ? P.rank * 128u + inbox * 8u - 8u + wid : id)};
    unsigned r[1];
    pack_store<1>([&](int, u32x4 (&o)[4]) { o[0] = d[0]; o[1] = d[1]; o[2] = d[2]; o[3] = d[3]; }, (inbox ? P.ws[dest] : ws) + OFF_PK + par * PK_PAR, g, lds, lds + wid * SCR_WAVE, lane, r, [] {});
    if (lane == 0) lrec[wid] = r[0];
    wait_vm0();
    asm volatile("s_waitcnt lgkmcnt(0)" ::: "memory");
    __builtin_amdgcn_s_barrier();
    asm volatile("" ::: "memory");
    if (wid == 0 && (isx || lane < 8) && rdest != (unsigned)P.rank) {
        u64* rp = (u64*)(peer + OFF_REC + par * REC_PAR) + (P.rank * 3 + (isx ? 0 : 1)) * 1024 + h * 64 + (e & 7) * 8 + (lane & 7);
        __hip_atomic_store(rp, ((u64)call << 42) | lrec[lane & 7], __ATOMIC_RELAXED, __HIP_MEMORY_SCOPE_SYSTEM);
    }
}

__device__ __forceinline__ void pack_bulk(const Params& P, char* ws, const char* pw, LAS char* lds, unsigned rank, unsigned call, int lane, int wid) {
    unsigned i = P.dstart + 2u * rank, round = 0;
    const int li = lane & 15, lg = lane >> 4;
    const unsigned par = call & 1u;
    const unsigned lsrc = (unsigned)(8 * lg) * 65536u + (unsigned)(64 * wid + 4 * li) * 2u;
    u32x2 v[2][8];
    unsigned en[2];
    auto load = [&](unsigned i, int u) {
        const unsigned k = (i + u) / 7u, dest = (P.rank + 1u + (i + u) % 7u) & 7u, hp = (P.worder[k >> 2] >> ((k & 3u) * 8u)) & 127u;
        en[u] = dest << 7 | hp;
        const char* src = pw + (size_t)(hp >> 3) * (32 * 65536) + (dest * 4096u + (hp & 7u) * 512u) * 2u;
#pragma unroll
        for (int j = 0; j < 8; ++j) v[u][j] = *(const u32x2*)(src + (size_t)j * 65536 + lsrc);
    };
    load(i, 0);
    load(i, 1);
    __builtin_amdgcn_s_waitcnt(0x0F70);
    for (LAS unsigned* lrec = (LAS unsigned*)(lds + LDS_FLG);; lrec += 32) {
        const unsigned el = (lane & 8) ? en[1] : en[0];
        char* peer = P.ws[el >> 7];
        const unsigned g[2] = {1024u + en[0] * 8u + wid, 1024u + en[1] * 8u + wid};
        u32x4 d[2][4];
#pragma unroll
        for (int u = 0; u < 2; ++u)
#pragma unroll
            for (int hf = 0; hf < 2; ++hf)
#pragma unroll
                for (int a = 0; a < 4; ++a) {
                    d[u][2 * hf][a] = perm(v[u][2 * a + 1][hf], v[u][2 * a][hf], 0x05040100u);
                    d[u][2 * hf + 1][a] = perm(v[u][2 * a + 1][hf], v[u][2 * a][hf], 0x07060302u);
                }
        asm volatile("" : "+v"(d[0][0]), "+v"(d[0][1]), "+v"(d[0][2]), "+v"(d[0][3]), "+v"(d[1][0]), "+v"(d[1][1]), "+v"(d[1][2]), "+v"(d[1][3]));
        i = rank < P.dcount[round + 1u] ? i + 2u * P.dcount[round] : 896u;
        ++round;
        if (i < 896u) load(i, 0);
        __builtin_amdgcn_sched_barrier(0);
        unsigned r[2];
        pack_store<2>([&](int u, u32x4 (&o)[4]) { o[0] = d[u][0]; o[1] = d[u][1]; o[2] = d[u][2]; o[3] = d[u][3]; },
                      ws + OFF_PK + par * PK_PAR, g, lds, lds + wid * SCR_WAVE, lane, r, [&] { if (i < 896u) load(i, 1); });
        if (lane < 2) lrec[lane * 8 + wid] = lane ? r[1] : r[0];
        wait_vm0();
        __builtin_amdgcn_s_waitcnt(0x0F70);
        unsigned arrived = 0;
        if (lane == 0) arrived = __hip_atomic_fetch_add(lrec + 16, 1u, __ATOMIC_RELAXED, __HIP_MEMORY_SCOPE_WORKGROUP);
        LDS_FENCE();
        if (rfl(arrived) == 7u && lane < 16) {
            u64* rp = (u64*)(peer + OFF_REC + par * REC_PAR) + (P.rank * 3 + 1) * 1024 + (el & 127u) * 8u + (lane & 7);
            __hip_atomic_store(rp, ((u64)call << 42) | lrec[lane], __ATOMIC_RELAXED, __HIP_MEMORY_SCOPE_SYSTEM);
        }
        if (i >= 896u) break;
    }
}

constexpr int PSLOT = 15232;
constexpr int PSLOT_M = 2 * FIXB + VCAP;
constexpr int PSLOT_T = PSLOT - 4096;
constexpr int LDS_Q = LDS_REC;
constexpr int LDS_QTAIL = LDS_Q + 256;
constexpr int LDS_FULL = LDS_QTAIL + 16;

__device__ __forceinline__ unsigned lds_poll(LAS char* p) {
    return rfl(__hip_atomic_load((LAS unsigned*)p, __ATOMIC_RELAXED, __HIP_MEMORY_SCOPE_WORKGROUP));
}

template <int N>
__device__ __forceinline__ void fetch_pair(u32x4& next, unsigned naddr, unsigned v0, unsigned v1, unsigned v2, const char* pk, u64 vmask,
                                           unsigned slot, unsigned da, unsigned dv) {
    static_assert(N >= 7 && N <= 12, "window");
#define FW(j) "s_waitcnt vmcnt(%12)\n\t.if %13 == " #j "\n\tds_write_b32 %10, %11\n\t.endif\n\t"
    asm volatile("s_mov_b32 m0, %7\n\t"
                 FW(0) "global_load_lds_dwordx4 %2, %5 offset:2048 sc0 sc1\n\t"
                 FW(1) "global_load_lds_dwordx4 %2, %5 sc0 sc1\n\t"
                 FW(2) "global_load_lds_dwordx4 %2, %5 offset:1024 sc0 sc1\n\t"
                 FW(3) "global_load_lds_dwordx4 %2, %5 offset:3072 sc0 sc1\n\t"
                 "s_mov_b32 m0, %9\n\t"
                 FW(4) "s_mov_b64 exec, %6\n\t"
                 "global_load_lds_dwordx4 %4, %5 sc0 sc1\n\t"
                 "s_mov_b64 exec, -1\n\t"
                 "s_mov_b32 m0, %8\n\t"
                 "ds_read_b128 %0, %1\n\t"
                 FW(5) "global_load_lds_dwordx4 %3, %5 sc0 sc1\n\t"
                 "s_waitcnt lgkmcnt(0)"
                 : "=&v"(next)
                 : "v"(naddr), "v"(v0), "v"(v1), "v"(v2), "s"(pk), "s"(vmask), "s"(slot), "s"(slot + 4096u), "s"(slot + (PSLOT_M - 512)),
                   "v"(da), "v"(dv), "i"(N - 1), "i"((N - 1) % 6)
                 : "memory");
#undef FW
}

__device__ __forceinline__ void fetch_link(const Params& P, LAS char* lds, int q, unsigned call, int lane) {
    const unsigned p = (unsigned)(P.rank + 1 + q) & 7u;
    const char* pkb = P.ws[p] + OFF_PK + (call & 1u) * PK_PAR;
    const unsigned ldsbase = (unsigned)(size_t)lds;
    const unsigned v0 = lane * 16, v1 = v0 + 4096, v2 = v0 + (VMID - 512);
    unsigned fa[2] = {0, 0}, fv[2] = {0, 0};
    if (blockIdx.x < 14) {
        const unsigned k = lane + (blockIdx.x / 7u) * 64u;
        __hip_atomic_store((unsigned*)(P.me + OFF_READY) + (8u * p + (k & 7u)) * NSLOT + 16u + 7u * (k >> 3) + q, call, __ATOMIC_RELAXED,
                           __HIP_MEMORY_SCOPE_SYSTEM);
    }
    __builtin_amdgcn_s_setprio(3);
    unsigned ended = 0;
    u32x4 ev = {0u, 0u, 0u, 0u};
    for (unsigned i = 0;; ++i) {
        while (__builtin_expect(rfl(ev[0]) != i + 1u, 0)) {
            ev = *(volatile LAS u32x4*)(lds + LDS_Q + (i & 15u) * 16u);
            if (rfl(ev[0]) == i + 1u) break;
            if (fa[1]) {
                asm volatile("s_waitcnt vmcnt(6)" ::: "memory"); *(volatile LAS unsigned*)(lds + fa[1]) = fv[1]; fa[1] = 0;
            } else if (fa[0]) {
                asm volatile("s_waitcnt vmcnt(0)" ::: "memory"); *(volatile LAS unsigned*)(lds + fa[0]) = fv[0]; fa[0] = 0;
            } else {
                if (ended == NDEC) return;
                __builtin_amdgcn_s_sleep(1);
            }
        }
        const unsigned w1 = rfl(ev[1]);
        if (__builtin_expect(w1 >> 31, 0)) { ++ended; continue; }
        const unsigned owner = w1 >> 24, slot = ldsbase + (owner - 1u) * PSLOT;
        const char* pk = pkb + (size_t)(w1 & 0x3fffu) * PAIRB;
        const u64 vmask = (u64)rfl(ev[2]) | (u64)rfl(ev[3]) << 32;
        const unsigned da = ldsbase + (fa[1] ? fa[1] : (unsigned)LDS_FULL), dv = fv[1];
        fetch_pair<NFLY>(ev, ldsbase + LDS_Q + ((i + 1u) & 15u) * 16u, v0, v1, v2, pk, vmask, slot, da, dv);
        fa[1] = fa[0]; fv[1] = fv[0];
        fa[0] = LDS_FULL + owner * 4u; fv[0] = i + 1u;
    }
}

struct Desc { unsigned g, dst, rdy, rec; };

__device__ __forceinline__ void decode_link(const Params& P, LAS char* lds, int q, int w, unsigned call, int lane, int wid) {
    char* ws = P.me;
    const unsigned par = call & 1u;
    const unsigned p = (unsigned)(P.rank + 1 + q) & 7u;
    unsigned* tmo = (unsigned*)(ws + OFF_TMO);
    unsigned vorder = P.order[lane];
    asm volatile("" : "+v"(vorder));
    const unsigned rec_base = (unsigned)(OFF_REC + par * REC_PAR) + p * 3072u * 8u;
    auto describe = [&](unsigned n) {
        Desc d;
        if (n < NIN) {
            unsigned e = ((unsigned)__builtin_amdgcn_readlane(vorder, n >> 4) >> ((n & 12u) * 2u)) & 255u;
            if (e < 128u) e = (e & ~7u) | ((P.rank + (e & 7u) + ((e & 7u) > (unsigned)q)) & 7u);
            const unsigned type = e >> 7, h = (e >> 3) & 15u, idx = (e & 7u) * 8u + (n & 3u) * 2u, id = h * 64u + idx, slot = 16u + 7u * h + q;
            d.g = type ? 1024u + P.rank * 1024u + id : id;
            if (n < 512u) {
                const unsigned inbox = (P.pushk[n >> 5] >> (n & 28u)) & 15u;
                if (inbox) d.g = (1024u + P.rank * 1024u + p * 128u + inbox * 8u - 8u + (n & 3u) * 2u) | 1u << 24;
            }
            d.dst = (unsigned)(type ? OFF_B : OFF_A) + (slot * 256u + 4u * idx) * 1024u;
            d.rdy = (unsigned)OFF_READY + ((type * 64u + idx) * NSLOT + slot) * 4u;
            d.rec = rec_base + (type * 1024u + id) * 8u;
        } else {
            d.g = 9216u; d.dst = P.rank * 8192u; d.rdy = 0;
            d.rec = rec_base + (2048u + 2u * ((n - NIN) & 511u)) * 8u;
        }
        return d;
    };
    auto accept = [&](u64 v, Desc& d, unsigned& ia, unsigned& ib) {
        if (__builtin_amdgcn_ballot_w64((unsigned)(v >> 42) != (call & 0x3fffffu)) != 0ull) return false;
        ia = __builtin_amdgcn_readlane((unsigned)v, 0); ib = __builtin_amdgcn_readlane((unsigned)v, 1);
        if (!d.rdy) {
            const unsigned id = __builtin_amdgcn_readlane((unsigned)(v >> 32), 0) & 1022u;
            d.g = 9216u + P.rank * 1024u + id;
            d.dst = (id & 15u) * (32u * 65536u) + (p * 4096u + 64u * (id >> 4)) * 2u;
        }
        return true;
    };
    const char* pkb = P.ws[p] + OFF_PK + par * PK_PAR;
    const unsigned voff = lane * 16;
    const unsigned tk_base = (call - 1u) * (NTK - PW) - 2u * PW;
    LAS char* S = lds + (wid - 1) * PSLOT;
    LAS char* M = S + PSLOT_M;
    bool dead = false;
    unsigned dmask = 0;
    const unsigned vrows = (lane >> 3) * 65536u + (lane & 7) * 16u;
    const __amdgpu_buffer_rsrc_t orsrc = __builtin_amdgcn_make_buffer_rsrc(P.out, 0, 512 * 65536, 0x00020000);
    auto vlen = [](unsigned i) { return min((i >> 8) & 255u, (unsigned)VCAP / 16u); };
    auto post = [&](unsigned w1, unsigned ia, unsigned ib) {
        const unsigned la = vlen(ia), lb = vlen(ib);
        const unsigned mlo = la >= 32u ? ~0u : ~(~0u >> la), mhi = lb >= 32u ? ~0u : ((1u << lb) - 1u) | 1u;
        unsigned i = 0;
        asm volatile("s_waitcnt lgkmcnt(0)" ::: "memory");
        if (__builtin_expect((w1 >> 23) & 1u, 0)) {
            const char* pk = ws + OFF_PK + par * PK_PAR + (size_t)(w1 & 0x3fffu) * PAIRB;
            glds4((unsigned)(size_t)S, voff, pk);
            glds1((unsigned)(size_t)S + 4096u, voff, pk + 4096);
            glds1((unsigned)(size_t)M - 512u, voff, pk + (VMID - 512));
            return 0u;
        }
        if (lane == 0) {
            i = __hip_atomic_fetch_add((LAS unsigned*)(lds + LDS_QTAIL), 1u, __ATOMIC_RELAXED, __HIP_MEMORY_SCOPE_WORKGROUP);
            *(volatile LAS u32x4*)(lds + LDS_Q + (i & 15u) * 16u) = (u32x4){i + 1u, w1, mlo, mhi};
        }
        return rfl(i) + 1u;
    };
    auto poll = [&](Desc& d, unsigned& ia, unsigned& ib) {
        const u64 t0 = __builtin_amdgcn_s_memrealtime();
        unsigned spins = 0;
        ia = ib = 0;
        while (!dead) {
            const u64 v = __hip_atomic_load((const u64*)(ws + d.rec) + (lane & 1), __ATOMIC_RELAXED, __HIP_MEMORY_SCOPE_SYSTEM);
            if (accept(v, d, ia, ib)) break;
            if (timed_out(tmo, call, t0, spins)) dead = true;
            __builtin_amdgcn_s_sleep(1);
        }
    };
    unsigned n1 = PW + w, ia, ib;
    Desc d0 = describe(w), d1 = describe(n1);
    poll(d0, ia, ib);
    STAMP(ws, 2);
    unsigned want = post(d0.g >> 1 | wid << 24, ia, ib);
    for (;;) {
        const bool more = n1 < NTK;
        u32x4 own[8];
        const bool late = !d0.rdy && ((d0.dst >> 9) & 15u) >= LATE_COL;
        if (late) {
            const unsigned bit = ((d0.dst >> 8) & 30u) | (d0.dst >> 24);
            if (!((dmask >> bit) & 1u)) {
                const u64 t0 = __builtin_amdgcn_s_memrealtime();
                unsigned spins = 0;
                while (!dead) {
                    const unsigned v = __hip_atomic_load((const unsigned*)(ws + OFF_DFL) + q * 32 + (lane & 31), __ATOMIC_RELAXED, __HIP_MEMORY_SCOPE_SYSTEM);
                    dmask = (unsigned)__builtin_amdgcn_ballot_w64(v >= call);
                    if ((dmask >> bit) & 1u) break;
                    if (timed_out(tmo, call, t0, spins)) dead = true;
                    __builtin_amdgcn_s_sleep(1);
                }
            }
            asm volatile("" ::: "memory");
#pragma unroll
            for (int c = 0; c < 8; ++c) own[c] = __builtin_amdgcn_raw_buffer_load_b128(orsrc, vrows, d0.dst + c * (8u * 65536u), 17);
        }
        unsigned dv = 0;
        if (n1 + 2u * PW >= NIN + 32u * LATE_COL && dmask == 0u)
            dv = __hip_atomic_load((const unsigned*)(ws + OFF_DFL) + q * 32 + (lane & 31), __ATOMIC_RELAXED, __HIP_MEMORY_SCOPE_SYSTEM);
        const u64 rv = __hip_atomic_load((const u64*)(ws + d1.rec) + (lane & 1), __ATOMIC_RELAXED, __HIP_MEMORY_SCOPE_SYSTEM);
        if (want == 0u) wait_vm0();
        else while (lds_poll(lds + LDS_FULL + wid * 4) != want) __builtin_amdgcn_s_sleep(1);
        LDS_FENCE();
        const unsigned la = vlen(ia), lb = vlen(ib);
        if (la > 32u || lb > 32u) {
            const char* mid = (d0.g >> 24 ? ws + OFF_PK + par * PK_PAR : pkb) + (size_t)((d0.g >> 1) & 0x3fffu) * PAIRB + VMID;
            for (int c = -3; c <= 3; ++c) {
                const int off = c * 1024 - 512 + (int)voff;
                if (c != 0 && off >= -16 * (int)la && off < 16 * (int)lb) {
                    u32x4 t;
                    asm volatile("global_load_dwordx4 %0, %1, off sc0 sc1\n\ts_waitcnt vmcnt(0)" : "=&v"(t) : "v"(mid + off) : "memory");
                    *(LAS u32x4*)(M + off) = t;
                }
            }
            LDS_FENCE();
        }
        u32x4 o[2][4];
        unsigned ja = 0, jb = 0;
        bool posted = false;
        {
            u32x4 pp[2][2];
            u32x2 cc[2];
#pragma unroll
            for (int u = 0; u < 2; ++u) {
                cc[u] = *(LAS const u32x2*)(S + u * FIXB + 2048 + lane * 8);
                pp[u][0] = *(LAS const u32x4*)(S + u * FIXB + voff);
                pp[u][1] = *(LAS const u32x4*)(S + u * FIXB + 1024 + voff);
            }
            const unsigned inf[2] = {ia, ib};
            unpack2(pp, cc, inf, la, lds, S + PSLOT_T, M, lane, o, [&] {
                asm volatile("" : "+v"(pp[0][0]), "+v"(pp[0][1]), "+v"(pp[1][0]), "+v"(pp[1][1]));
                if (more && accept(rv, d1, ja, jb)) {
                    want = post(d1.g >> 1 | wid << 24, ja, jb);
                    posted = true;
                }
            });
        }
        dmask |= (unsigned)__builtin_amdgcn_ballot_w64(dv >= call);
        if (!d0.rdy) {
            if (dead) {
#pragma unroll
                for (int c = 0; c < 8; ++c) o[c >> 2][c & 3] = (u32x4){0x7fc07fc0u, 0x7fc07fc0u, 0x7fc07fc0u, 0x7fc07fc0u};
            }
            rows_of(S + PSLOT_T, lane, o[0]);
            rows_of(S + PSLOT_T, lane, o[1]);
            if (late) {
#pragma unroll
                for (int c = 0; c < 8; ++c) o[c >> 2][c & 3] = add_bf16(o[c >> 2][c & 3], own[c]);
            }
        }
        unsigned tk;
        u64 rv2;
        {
            const unsigned step = d0.rdy ? 1024u : 8u * 65536u, v0 = d0.rdy ? voff : vrows;
            const char* b0 = (d0.rdy ? ws : P.out) + d0.dst;
            asm volatile("global_load_dwordx2 %1, %2, %5 sc0 sc1\n\t"
                         "s_mov_b64 exec, 1\n\tglobal_atomic_add %0, %3, %4, %5 sc0\n\ts_mov_b64 exec, -1\n\t"
                         "global_store_dwordx4 %8, %12, %6 sc0 sc1\n\tglobal_store_dwordx4 %9, %13, %6 sc0 sc1\n\t"
                         "global_store_dwordx4 %10, %14, %6 sc0 sc1\n\tglobal_store_dwordx4 %11, %15, %6 sc0 sc1\n\t"
                         "global_store_dwordx4 %8, %16, %7 sc0 sc1\n\tglobal_store_dwordx4 %9, %17, %7 sc0 sc1\n\t"
                         "global_store_dwordx4 %10, %18, %7 sc0 sc1\n\tglobal_store_dwordx4 %11, %19, %7 sc0 sc1\n\t"
                         "s_nop 1\n\ts_waitcnt vmcnt(8)"
                         : "=&v"(tk), "=&v"(rv2)
                         : "v"(d1.rec + (lane & 1) * 8u), "v"((unsigned)OFF_TKT + q * 128u), "v"(more ? 1u : 0u), "s"(ws), "s"(b0), "s"(b0 + 4u * step),
                           "v"(v0), "v"(v0 + step), "v"(v0 + 2u * step), "v"(v0 + 3u * step),
                           "v"(o[0][0]), "v"(o[0][1]), "v"(o[0][2]), "v"(o[0][3]), "v"(o[1][0]), "v"(o[1][1]), "v"(o[1][2]), "v"(o[1][3])
                         : "memory");
        }
        if (more && !posted && accept(rv2, d1, ja, jb)) {
            want = post(d1.g >> 1 | wid << 24, ja, jb);
            posted = true;
        }
        if (d0.rdy) {
            wait_vm0();
            if (lane < 2) __hip_atomic_store((unsigned*)(ws + d0.rdy) + lane * NSLOT, call, __ATOMIC_RELAXED, __HIP_MEMORY_SCOPE_SYSTEM);
        } else if (!late) {
            wait_vm0();
            if (lane == 0 && !dead)
                __hip_atomic_store((unsigned*)(ws + OFF_OFL) + q * 512 + ((d0.g >> 1) & 511u), call, __ATOMIC_RELAXED, __HIP_MEMORY_SCOPE_SYSTEM);
        }
        const unsigned n2 = rfl(tk) - tk_base;
        if (!more) break;
        if (!posted) {
            poll(d1, ja, jb);
            want = post(d1.g >> 1 | wid << 24, ja, jb);
        }
        ia = ja; ib = jb;
        d0 = d1;
        n1 = n2;
        d1 = describe(n1);
    }
    post(0x80000000u, 0u, 0u);
}

__device__ __forceinline__ void glds4w(unsigned m0v, unsigned voff, const char* sb) {
    constexpr int R = 8 * 65536 - 1024;
    asm volatile("s_mov_b32 m0, %0\n\ts_nop 0\n\t"
                 "global_load_lds_dwordx4 %1, %2\n\t"
                 "global_load_lds_dwordx4 %1, %3 offset:1024\n\t"
                 "global_load_lds_dwordx4 %1, %4 offset:2048\n\t"
                 "global_load_lds_dwordx4 %1, %5 offset:3072" :: "s"(m0v), "v"(voff), "s"(sb), "s"(sb + R), "s"(sb + 2 * R), "s"(sb + 3 * R) : "memory");
}
__device__ __forceinline__ bf16x8 read_tr(LAS const char* p) {
    const bf16x4 lo = __builtin_amdgcn_ds_read_tr16_b64_v4bf16((LAS bf16x4*)p), hi = __builtin_amdgcn_ds_read_tr16_b64_v4bf16((LAS bf16x4*)(p + 512));
    return __builtin_shufflevector(lo, hi, 0, 1, 2, 3, 4, 5, 6, 7);
}

template <int WR, bool BIG, bool SK5, int MODE>
__device__ __forceinline__ void k_loop(f32x4 (&acc)[BIG ? 10 : 8][4], char* ws, LAS char* lds, int u0, int u5, int pn, unsigned H, const char* wsrc, int rank,
                                       unsigned call, int lane, int wid) {
    constexpr int MT = BIG ? 10 : 8, M0 = MODE == 1 ? 8 : 0;
    constexpr bool OWN = MODE == 2;
    constexpr unsigned ns = OWN ? 16 : NSLOT, dt = MODE == 1 ? 7 : 1;
    const unsigned q4 = ((u0 >> 3) - rank - 1) & 7, q5 = ((u5 >> 3) - rank - 1) & 7;
    const bool cut = MODE == 0 && q4 < 7;
    const unsigned steps = MODE == 2 ? 16 : MODE == 1 ? 16 - H : 112 + H, last = MODE == 1 ? 121 + q4 : cut && q4 == 6 && H < 16 ? ns - 2 : ns - 1;
    unsigned t = MODE == 1 ? 16 + q4 + 7 * H : 0, tskip = cut ? 16 + q4 + 7 * H : 999;
    unsigned skip4 = 16 + q4, skip5 = 16 + q5;
    constexpr bool has5 = BIG && !WR, has4 = WR || MODE != 1;
    constexpr int RA = MODE == 0 ? 4 : MODE == 1 ? 7 : RING_A2, RB = MODE == 0 ? 4 : MODE == 1 ? 7 : 9 - RING_A2, RW = WR ? RB : RA;
    constexpr int RS = MODE == 0 ? LSLOT : MODE == 1 ? 20480 : 16384;
    constexpr int A0 = MODE == 1 ? -16384 : 0, B0 = MODE == 0 ? 20480 : MODE == 1 ? 4096 : RA * RS;
    unsigned* tmo = (unsigned*)(ws + OFF_TMO);
    const int wc = wid & 3;
    const unsigned ldsbase = (unsigned)(size_t)lds;
    const unsigned u1 = WR ? 4 * pn + wc : u0 + wc;
    const char* g1 = OWN && WR ? wsrc + wc * 128 : ws + (WR ? OFF_B : OFF_A) + (size_t)u1 * 4096;
    const unsigned vw = (lane >> 3) * 65536 + ((lane & 7) ^ ((lane >> 4) & 1)) * 16;
    const char* g2 = ws + OFF_A + (size_t)(4 * u5 + wc) * 1024;
    const unsigned l1 = ldsbase + WR * B0 + wc * 4096, l2 = ldsbase + A0 + (16 + wc) * 1024;
    const unsigned voff = lane * 16;
    const unsigned vrow = (unsigned)OFF_READY + (WR * 64 + (lane >= 32 && has5 ? u5 : u1)) * (NSLOT * 4);
    const unsigned vsnap = vrow + (lane & 31) * 16;
    LAS char* snap = lds + LDS_FLG + wid * 1024;
    *(LAS u32x4*)(snap + voff) = (u32x4){0u, 0u, 0u, 0u};
    asm volatile("s_waitcnt lgkmcnt(0)" ::: "memory");
    unsigned hz = OWN && WR ? 4096u : 0u, look = 0;
#define ADVANCE(v) ({ const u64 m_ = __builtin_amdgcn_ballot_w64((v) >= call); \
    (unsigned)__builtin_ctzll(~(u64)((unsigned)m_ & (unsigned)(m_ >> 32))); })
#define ENSURE(t) do { if (__builtin_expect(hz <= (unsigned)(t), 0)) { const u64 t0_ = __builtin_amdgcn_s_memrealtime(); unsigned sp_ = 0; \
        __builtin_amdgcn_s_waitcnt(0xC07F); \
        for (;;) { \
            const unsigned a_ = __hip_atomic_load((const unsigned*)(ws + vrow) + min(hz + (lane & 31u), ns - 1u), __ATOMIC_RELAXED, __HIP_MEMORY_SCOPE_SYSTEM); \
            hz += ADVANCE(a_); \
            if (hz > (unsigned)(t)) break; \
            if (timed_out(tmo, call, t0_, sp_)) { hz = NSLOT; break; } \
            __builtin_amdgcn_s_sleep(2); } } } while (0)
#define STAGE(t, lo_) do { const size_t go_ = (size_t)(t) * 262144; \
    if (OWN && WR) glds4w(l1 + lo_, vw, g1 + 8 * go_); else if (has4) glds4(l1 + lo_, voff, g1 + go_); if (has5) glds1(l2 + lo_, voff, g2 + go_); } while (0)
#define WAITV() asm volatile("s_waitcnt vmcnt(%0)" :: "i"((4 * has4 + has5) * (RW - 2)) : "memory")
#pragma unroll 1
    for (int i = 0; i < RW - 1; ++i, t = min(t + dt, last)) { ENSURE(t); STAGE(t, i * RS); }
    WAITV();
    __builtin_amdgcn_s_barrier();
    if (!MODE) STAMP(ws, 2);
    if (WR == 1) __builtin_amdgcn_s_barrier();
    const unsigned aoff = ((lane & 15) * 4 + ((lane >> 4) ^ a_swz((lane >> 2) & 3))) * 16;
    const unsigned bsw = ((lane >> 3) & 1) * 16;
    const unsigned boff = B0 + wc * 4096 + (OWN ? (lane >> 4) * 1024 + ((lane >> 2) & 3) * 128 + (lane & 3) * 32 : lane * 16);
#define LDB() _Pragma("unroll") for (int n = 0; n < 4; ++n) \
        B[n] = OWN ? read_tr(pb + (((n >> 1) * 16) ^ bsw) + (n & 1) * 8) : *(LAS const bf16x8*)(pb + n * 1024)
#define LDA(m0, m1) _Pragma("unroll") for (int m = m0; m < m1; ++m) A[m] = *(LAS const bf16x8*)(pa + A0 + (m < 8 ? 8 * WR + m : 8 + 2 * WR + m) * 1024)
#define MIDDLE() do { WAITV(); __builtin_amdgcn_s_waitcnt(0xC07F); __builtin_amdgcn_sched_barrier(0); __builtin_amdgcn_s_barrier(); \
        __builtin_amdgcn_sched_barrier(0); __builtin_amdgcn_s_setprio(1); } while (0)
#define MMA(m0, m1) _Pragma("unroll") for (int m = m0; m < m1; ++m) _Pragma("unroll") for (int k = 0; k < 4; ++k) { \
        const int n = (m & 1) ? 3 - k : k; \
        acc[m][n] = __builtin_amdgcn_mfma_f32_16x16x32_bf16(B[n], A[m], acc[m][n], 0, 0, 0); __builtin_amdgcn_sched_barrier(0); }
#define TAIL() do { __builtin_amdgcn_s_setprio(0); __builtin_amdgcn_sched_barrier(0); __builtin_amdgcn_s_barrier(); \
        __builtin_amdgcn_sched_barrier(0); } while (0)
    unsigned so = 0, sb = 0, sw = (RW - 1) * RS;
#define TRACE()
#define HEAD() TRACE(); ENSURE(t); STAGE(t, sw); __builtin_amdgcn_sched_barrier(0); \
        bf16x8 A[MT], B[4]; LAS const char* pa = lds + aoff + so; LAS const char* pb = lds + boff + (RA == RB ? so : sb)
#define NEXT(d5) sw = RW == RA ? so : sb; so = so == (RA - 1) * RS ? 0u : so + RS; \
        if (RA != RB) sb = sb == (RB - 1) * RS ? 0u : sb + RS; \
        t += dt; \
        if (MODE == 0) { const bool hit = t == tskip; t += hit ? 1u : 0u; tskip += hit ? 7u : 0u; } \
        t = min(t, last); \
        const bool do5 = s != skip5; \
        if (SK5) skip5 += do5 ? 0u : d5; \
        asm volatile("" : "+s"(t), "+s"(tskip), "+s"(so), "+s"(skip5))
#define LOOK() if (MODE == 0 && __builtin_expect(t >= look, 0)) { \
        const unsigned fv = *(volatile LAS const unsigned*)(snap + (lane >> 5) * 512 + min(hz + (lane & 31u), ns - 1u) * 4u); \
        hz += ADVANCE(fv); \
        glds1(ldsbase + LDS_FLG + wid * 1024, vsnap, ws); \
        look = max(t + 4u + LOOKAHEAD, hz) - LOOKAHEAD; }
    unsigned s = 0;
    if (BIG && MODE == 0) {
        do {
            HEAD();
            LDB();
            LDA(8, MT);
            NEXT(7u);
            unsigned cmp4 = skip4;
            if (__builtin_expect(s == skip4, 0)) {
                skip4 += 7u;
                MIDDLE();
            }
            asm volatile("" : "+s"(cmp4));
            if (__builtin_expect(s != cmp4, 1)) {
                LDA(0, 8);
                __builtin_amdgcn_sched_barrier(0);
                LOOK();
                MIDDLE();
                MMA(0, 8);
            }
            if (!SK5 || __builtin_expect(do5, 1)) { MMA(8, MT); }
            TAIL();
        } while (++s < 7u * H + (H ? 16u : 1u));
        skip5 = q5 != q4 ? 16u + 7u * H + q5 - (q5 > q4) : 999u;
    }
    for (; s < steps; ++s) {
        HEAD();
        LDB();
        LDA(M0, MT);
        __builtin_amdgcn_sched_barrier(0);
        NEXT(6u);
        LOOK();
        MIDDLE();
        MMA(M0, 8);
        if (!SK5 || __builtin_expect(do5, 1)) { MMA(8, MT); }
        TAIL();
    }
#undef TRACE
#undef HEAD
#undef NEXT
#undef LOOK
#undef TAIL
#undef LDA
#undef LDB
#undef MIDDLE
#undef MMA
#pragma unroll
    for (int m = M0; m < MT; ++m) asm volatile("" : "+v"(acc[m][0]), "+v"(acc[m][1]), "+v"(acc[m][2]), "+v"(acc[m][3]));
    if (WR == 0) __builtin_amdgcn_s_barrier();
#undef ENSURE
#undef ADVANCE
#undef STAGE
#undef WAITV
}

template <bool BIG, bool SK5>
__device__ __forceinline__ void gemm_tile(const Params& P, LAS char* lds, int u0, int u5, int pn, unsigned call, int lane, int wid) {
    constexpr int NU = BIG ? 5 : 4;
    char* ws = P.me;
    unsigned* tmo = (unsigned*)(ws + OFF_TMO);
    const int wr = wid >> 2, wc = wid & 3;
    const unsigned H = BIG && (u5 >> 3) != (u0 >> 3) ? (P.hsplit[SK5] >> (4 * (pn >> 1))) & 15u : 0u;
    f32x4 acc[2 * NU][4];
#pragma unroll
    for (int m = 0; m < 2 * NU; ++m)
#pragma unroll
        for (int n = 0; n < 4; ++n) acc[m][n] = (f32x4){0.f, 0.f, 0.f, 0.f};
    if (wr == 0) k_loop<0, BIG, SK5, 0>(acc, ws, lds, u0, u5, pn, H, nullptr, P.rank, call, lane, wid);
    else k_loop<1, BIG, SK5, 0>(acc, ws, lds, u0, u5, pn, H, nullptr, P.rank, call, lane, wid);
    const unsigned tmov = __hip_atomic_load(tmo, __ATOMIC_RELAXED, __HIP_MEMORY_SCOPE_AGENT);
    LAS unsigned* meet = (LAS unsigned*)(lds + LDS_REC + 128);
    if (wid == 0 && lane < 5) meet[lane * 16 + 8] = 0u;
    asm volatile("s_waitcnt lgkmcnt(0)" ::: "memory");
    asm volatile("" :: "v"(tmov));
    wait_vm0();
    __builtin_amdgcn_s_barrier();
    asm volatile("" ::: "memory");
    STAMP(ws, 3);
    const unsigned par = call & 1u;
    const int cb = 4 * pn + wc;
    if (__builtin_expect(rfl(tmov) == call, 0)) {
#pragma unroll
        for (int m = 0; m < 2 * NU; ++m)
#pragma unroll
            for (int n = 0; n < 4; ++n) acc[m][n] = (f32x4){__builtin_nanf(""), __builtin_nanf(""), __builtin_nanf(""), __builtin_nanf("")};
    }
    unsigned rb[NU], g[NU];
#pragma unroll
    for (int u = 0; u < NU; ++u) {
        rb[u] = (u < 4 ? 2 * u0 + 4 * wr + u : 2 * u5 + wr) & 127u;
        g[u] = 9216u + (rb[u] >> 4) * 1024u + cb * 16u + (rb[u] & 15u);
    }
    auto get = [&](int u, u32x4 (&d)[4]) {
#pragma unroll
        for (int c = 0; c < 4; ++c) {
            const int mi = 2 * u + (c >> 1), b = c & 1;
            d[c][0] = cvt_pk(acc[mi][0][2 * b], acc[mi][1][2 * b]);
            d[c][1] = cvt_pk(acc[mi][2][2 * b], acc[mi][3][2 * b]);
            d[c][2] = cvt_pk(acc[mi][0][2 * b + 1], acc[mi][1][2 * b + 1]);
            d[c][3] = cvt_pk(acc[mi][2][2 * b + 1], acc[mi][3][2 * b + 1]);
        }
    };
    LAS char* scr = lds + wid * SCR_WAVE;
    auto own_rows = [&](int u) {
        u32x4 d[4];
        get(u, d);
        rows_of(scr, lane, d);
        store_out(P.out + (size_t)(rb[u] & 15u) * (32 * 65536) + (size_t)(P.rank * 4096 + 64 * cb) * 2, lane, d);
    };
    const bool late = pn >= LATE_COL;
    const unsigned pos0 = late ? 32u * LATE_COL - (call - 1u) * (512u - 32u * LATE_COL) : (1u - call) * (32u * LATE_COL);
    auto publish = [&](LAS unsigned* m, unsigned rb0, bool draw, unsigned n) {
        const unsigned dest = rb0 >> 4;
        char* peer = P.ws[dest];
        if (draw && lane == 0)
            m[9] = __hip_atomic_fetch_add((unsigned*)(ws + OFF_OPOS) + dest * 32 + (late ? 256 : 0), 4u, __ATOMIC_RELAXED, __HIP_MEMORY_SCOPE_AGENT) + pos0;
        wait_vm0();
        unsigned before = 0;
        if (lane == 0) before = __hip_atomic_fetch_add(m + 8, 1u, __ATOMIC_RELAXED, __HIP_MEMORY_SCOPE_WORKGROUP);
        LDS_FENCE();
        if (rfl(before) == n - 1u && lane < 8) {
            u64* ring = (u64*)(peer + OFF_REC + par * REC_PAR) + (P.rank * 3 + 2) * 1024 + 2u * (m[9] & 508u) + lane;
            __hip_atomic_store(ring, (u64)call << 42 | (u64)((4u * pn + (lane >> 1)) * 16u + (rb0 & 15u)) << 32 | m[lane], __ATOMIC_RELAXED,
                               __HIP_MEMORY_SCOPE_SYSTEM);
        }
    };
    char* pk = ws + OFF_PK + par * PK_PAR;
#pragma unroll
    for (int k = 0; k < 4; k += 2) {
        const unsigned ga[2] = {g[k], g[k + 1]};
        unsigned r[2];
        pack_store<2>([&](int u, u32x4 (&d)[4]) { get(u + k, d); }, pk, ga, lds, scr, lane, r, [] {});
        LAS unsigned* m = meet + (k + wr) * 16;
        if (lane < 2) m[2 * wc + lane] = lane ? r[1] : r[0];
        publish(m, rb[k], wc == 0, 4u);
    }
    if (BIG) {
        STAMP(ws, 4);
        if ((u5 >> 3) != (u0 >> 3)) {
            asm volatile("s_waitcnt lgkmcnt(0)" ::: "memory");
            __builtin_amdgcn_s_barrier();
            if (wr == 0) k_loop<0, BIG, false, 1>(acc, ws, lds, u0, u5, pn, H, nullptr, P.rank, call, lane, wid);
            else k_loop<1, BIG, false, 1>(acc, ws, lds, u0, u5, pn, H, nullptr, P.rank, call, lane, wid);
            if (rfl(__hip_atomic_load(tmo, __ATOMIC_RELAXED, __HIP_MEMORY_SCOPE_AGENT)) == call) acc[2 * NU - 1][0][0] = __builtin_nanf("");
            wait_vm0();
            __builtin_amdgcn_s_barrier();
            asm volatile("" ::: "memory");
        }
        if (!SK5) {
            own_rows(NU - 1);
        } else {
            const unsigned ga[1] = {g[NU - 1]};
            unsigned r[1];
            pack_store<1>([&](int, u32x4 (&d)[4]) { get(NU - 1, d); }, pk, ga, lds, scr, lane, r, [] {});
            if (lane == 0) meet[64 + 2 * wc + wr] = r[0];
            publish(meet + 64, rb[NU - 1] & ~1u, wid == 4, 8u);
        }
    }
}

__device__ __forceinline__ void own_part(const Params& P, LAS char* lds, unsigned j, unsigned call, int lane, int wid) {
    char* ws = P.me;
    unsigned* tmo = (unsigned*)(ws + OFF_TMO);
    const int wr = wid >> 2, wc = wid & 3;
    const unsigned half = j & 1u, pn = (j >> 1) / 7u, q = (j >> 1) % 7u, p = (P.rank + 1u + q) & 7u;
    f32x4 acc[8][4];
#pragma unroll
    for (int m = 0; m < 8; ++m)
#pragma unroll
        for (int n = 0; n < 4; ++n) acc[m][n] = (f32x4){0.f, 0.f, 0.f, 0.f};
    const char* wsrc = P.w + p * 8192u + pn * 512u;
    if (wr == 0) k_loop<0, false, false, 2>(acc, ws, lds, 8 * P.rank + 4 * half, 0, pn, 0, wsrc, P.rank, call, lane, wid);
    else k_loop<1, false, false, 2>(acc, ws, lds, 8 * P.rank + 4 * half, 0, pn, 0, wsrc, P.rank, call, lane, wid);
    wait_vm0();
    __builtin_amdgcn_s_barrier();
    asm volatile("" ::: "memory");
    u32x4 d[4][4];
#pragma unroll
    for (int u = 0; u < 4; ++u) {
#pragma unroll
        for (int c = 0; c < 4; ++c) {
            const int mi = 2 * u + (c >> 1), n = 2 * (c & 1);
            d[u][c][0] = cvt_pk(acc[mi][n][0], acc[mi][n][1]);
            d[u][c][1] = cvt_pk(acc[mi][n][2], acc[mi][n][3]);
            d[u][c][2] = cvt_pk(acc[mi][n + 1][0], acc[mi][n + 1][1]);
            d[u][c][3] = cvt_pk(acc[mi][n + 1][2], acc[mi][n + 1][3]);
        }
        rows_of(lds + wid * SCR_WAVE, lane, d[u]);
    }
    const unsigned dst = (8u * half + 4u * wr) * (32u * 65536u) + p * 8192u + (4u * pn + wc) * 128u;
    if (pn < LATE_COL) {
        const unsigned* fl = (const unsigned*)(ws + OFF_OFL) + q * 512u + (4u * pn + wc) * 8u + 4u * half + 2u * wr + (lane & 1);
        const u64 t0 = __builtin_amdgcn_s_memrealtime();
        unsigned spins = 0;
        while (__builtin_amdgcn_ballot_w64(__hip_atomic_load(fl, __ATOMIC_RELAXED, __HIP_MEMORY_SCOPE_SYSTEM) < call) != 0ull) {
            if (timed_out(tmo, call, t0, spins)) break;
            __builtin_amdgcn_s_sleep(8);
        }
        asm volatile("" ::: "memory");
        const __amdgpu_buffer_rsrc_t orsrc = __builtin_amdgcn_make_buffer_rsrc(P.out, 0, 512 * 65536, 0x00020000);
        u32x4 y[16];
#pragma unroll
        for (int c = 0; c < 16; ++c) y[c] = __builtin_amdgcn_raw_buffer_load_b128(orsrc, (lane >> 3) * 65536 + (lane & 7) * 16, dst + c * (8u * 65536u), 17);
#pragma unroll
        for (int c = 0; c < 16; ++c) d[c >> 2][c & 3] = add_bf16(d[c >> 2][c & 3], y[c]);
    }
    if (rfl(__hip_atomic_load(tmo, __ATOMIC_RELAXED, __HIP_MEMORY_SCOPE_AGENT)) == call)
        d[0][0] = (u32x4){0x7fc07fc0u, 0x7fc07fc0u, 0x7fc07fc0u, 0x7fc07fc0u};
#pragma unroll
    for (int u = 0; u < 4; ++u) store_out(P.out + dst + u * (32 * 65536), lane, d[u]);
    wait_vm0();
    __builtin_amdgcn_s_barrier();
    asm volatile("" ::: "memory");
    if (threadIdx.x == 0)
        __hip_atomic_store((unsigned*)(ws + OFF_DFL) + q * 32u + pn * 2u + half, call, __ATOMIC_RELAXED, __HIP_MEMORY_SCOPE_SYSTEM);
}

__global__ void __launch_bounds__(512, 2) fused_gemm_rs(const Params P) {
    __shared__ __attribute__((aligned(16))) char smem[LDS_BYTES];
    const int bid = blockIdx.x;
    const int lane = threadIdx.x & 63, wid = rfl(threadIdx.x >> 6);
    char* ws = P.me; const char* px = P.x; const char* pw = P.w; const u64 job = P.job[bid];
    asm volatile("" :: "s"(ws), "s"(px), "s"(pw), "s"(job), "s"(P.rank));
    STAMP(ws, 0);
    unsigned* cnt = (unsigned*)(ws + OFF_CNT) + bid * 8 + wid;
    const unsigned cntv = __hip_atomic_load(cnt, __ATOMIC_RELAXED, __HIP_MEMORY_SCOPE_AGENT);
    if (bid >= 7 * PWG) {
#pragma unroll 1
        for (int r = 0; r < 3; ++r) {
            const unsigned e = (unsigned)(job >> (11 + 11 * r)) & 0x7ffu;
            if ((e >> 7) == 15u) break;
            pack_piece(P, ws, px, pw, (LAS char*)smem, r, e, r ? 0u : (unsigned)(job >> 53) & 15u, cntv, lane, wid);
        }
        const unsigned late = (unsigned)(job >> 44) & 511u;
        if (late < P.dcount[0]) pack_bulk(P, ws, pw, (LAS char*)smem, late, rfl(cntv) + 1u, lane, wid);
        asm volatile("s_waitcnt lgkmcnt(0)" ::: "memory");
        __builtin_amdgcn_s_barrier();
    } else {
        build_lut((LAS char*)smem, true, threadIdx.x);
        if (threadIdx.x >= 384) ((LAS unsigned*)(smem + LDS_Q))[threadIdx.x - 384] = 0u;
        asm volatile("s_waitcnt lgkmcnt(0)" ::: "memory");
        __builtin_amdgcn_s_barrier();
        asm volatile("" ::: "memory");
    }
    const unsigned call = rfl(cntv) + 1u;
    if (lane == 0) __hip_atomic_store(cnt, call, __ATOMIC_RELAXED, __HIP_MEMORY_SCOPE_AGENT);
    STAMP(ws, 1);
    if (bid < 7 * PWG) {
        if (wid == 0) fetch_link(P, (LAS char*)smem, bid % 7, call, lane);
        else if (wid <= NDEC) decode_link(P, (LAS char*)smem, bid % 7, (wid - 1) * PWG + bid / 7, call, lane, wid);
    } else if (((unsigned)job & 63u) != 63u) {
        const int u0 = (unsigned)job & 63u, u5 = (unsigned)(job >> 57) & 63u, pn = ((unsigned)job >> 7) & 15u;
        const unsigned dj = (P.djob[bid >> 2] >> ((bid & 3) * 8)) & 255u;
        if (dj < 224u) own_part(P, (LAS char*)smem, dj, call, lane, wid);
        if (!(job & 64u)) gemm_tile<false, false>(P, (LAS char*)smem, u0, 0, pn, call, lane, wid);
        else if ((u5 >> 3) == P.rank) gemm_tile<true, false>(P, (LAS char*)smem, u0, u5, pn, call, lane, wid);
        else gemm_tile<true, true>(P, (LAS char*)smem, u0, u5, pn, call, lane, wid);
        if (!(job & 64u)) STAMP(ws, 4);
        STAMP(ws, 7);
        LSTAMP(ws, 1, __builtin_amdgcn_s_memrealtime());
        if (!(pn >= 12 && (job & 64u))) for (LAS unsigned* drawn = (LAS unsigned*)(smem + LDS_REC);; drawn = (LAS unsigned*)((size_t)drawn ^ 4)) {
            if (threadIdx.x == 0) *drawn = __hip_atomic_fetch_add((unsigned*)(ws + OFF_TKT + 1024), 1u, __ATOMIC_RELAXED, __HIP_MEMORY_SCOPE_AGENT);
            asm volatile("s_waitcnt lgkmcnt(0)" ::: "memory");
            __builtin_amdgcn_s_barrier();
            const unsigned k = lds_poll((LAS char*)drawn) - (call - 1u) * (224u - NPRE + 256u - 7u * PWG - 32u) + NPRE;
            if (k >= 224u) break;
            own_part(P, (LAS char*)smem, k < NLATE ? 223u - k : k - NLATE, call, lane, wid);
        }
        LSTAMP(ws, 0, lt0);
        LSTAMP(ws, 2, __builtin_amdgcn_s_memrealtime());
        return;
    }
    wait_vm0();
    LSTAMP(ws, 0, lt0);
    LSTAMP(ws, 2, __builtin_amdgcn_s_memrealtime());
    STAMP(ws, 4);
}

#define LAGS {-1, 4, 18, 29, 42, 56, 72, 86}
#define HS_OWN 0x46800000
#define HS_SK5 0x468a8640
#define PRE_TOP 7
#define DCOUNT {108, 108, 82, 56}

void make_plan(Params& P, int rank) {
    const int lag[8] = LAGS;
    unsigned char* order = (unsigned char*)P.order;
    unsigned char* worder = (unsigned char*)P.worder;
    int no = 0, nw = 0;
    P.hsplit[0] = HS_OWN; P.hsplit[1] = HS_SK5;
    for (int i = 0; i < 16; ++i) P.pushk[i] = 0;
    for (int i = 0; i < 64; ++i) P.order[i] = 0;
    for (int pos = -1; pos < 512; ++pos) {
        if (pos >= 0 && pos < 128 && pos % 8 < (int)NXQ) order[no++] = (unsigned char)pos;
        for (int pr = 0; pr < 8; ++pr) {
            const int h8 = pos - lag[pr];
            if (h8 >= 0 && h8 < 128 && h8 % 8 == 0) {
                if (nw >= PUSH0 && (nw - PUSH0) % PUSHS == 0 && (nw - PUSH0) / PUSHS < NPUSH)
                    P.pushk[no >> 3] |= (unsigned)((nw - PUSH0) / PUSHS + 1) << (4 * (no & 7));
                order[no++] = (unsigned char)(128 | h8 | pr);
                worder[nw++] = (unsigned char)(h8 | pr);
            }
        }
    }
    for (int k = 0; k < NPUSH; ++k) {
        const unsigned char v = worder[PUSH0 + k * PUSHS];
        for (int i = PUSH0 + k * PUSHS; i > PUSHP + k; --i) worder[i] = worder[i - 1];
        worder[PUSHP + k] = v;
    }
    int seq[256], nseq = 0;
    const int mix[8] = MIX;
    for (int qt = 0; qt < NBAND; ++qt)
        for (int i = 0; i < 8; ++i) {
            const int pr = mix[i], band = (qt + pr) % NBAND, nt = (FULL14 >> pr) & 1 ? 14 : 13, base = 2 * rank + 2;
            for (int r = (nt * band + NBAND / 2) / NBAND; r < (nt * (band + 1) + NBAND / 2) / NBAND; ++r)
                for (int c = 0; c < 2; ++c) {
                    const int quad = (base + (r + 2 * pr) % 14) & 15, split = (base + (13 + 2 * pr) % 14) & 15;
                    const int k = 12 - r;
                    const int fifth = nt == 14 ? (r % 2 == 0 ? 8 * rank + r / 2 : r / 2 == 6 * c ? 8 * rank + 7 : -1)
                                               : k == 12 ? -1 : k % 3 == 0 ? 4 * split + k / 3 : 8 * rank + k - 1 - k / 3;
                    seq[nseq++] = (4 * quad | (fifth >= 0) << 6) | (2 * pr + c) << 7 | (fifth >= 0 ? fifth : 0) << 11;
                }
        }
    int tile[256], byp[256], nby = 0;
    static_assert(PWG == 6 && __builtin_popcount(FULL14) == 3 && (FULL14 & 0xc0) == 0xc0 && PUSHP + NPUSH <= 12 && PUSHP <= PUSH0, "214 tiles on 214 workgroups; pushed pieces: first round, first 64 entries");
    for (int x = 0, k = 0; x < 8; ++x)
        for (int b = 248 + x; b >= 7 * PWG; b -= 8) tile[b] = seq[k++];
    for (int pr = 0; pr <= 8; ++pr)
        for (int b = 7 * PWG; b < 256; ++b)
            if (((tile[b] >> 8) & 7) == pr) byp[nby++] = b;
    int first[10];
    for (int pr = 0, i = 0; pr <= 9; ++pr) {
        while (i < nby && ((tile[byp[i]] >> 8) & 7) < pr) ++i;
        first[pr] = i;
    }
    int nr = 0;
    for (int b = 0; b < 256; ++b)
        P.job[b] = (b < 7 * PWG ? 63ull : (u64)(tile[b] & 0x7ff) | (u64)(tile[b] >> 11) << 57) | 0x1ffffffffull << 11 | 511ull << 44;
    for (int r = 0; r < 3; ++r) {
        int list[256], n = 0;
        const int own = (rank + 1) << 7, from = r == 0 ? 0 : first[2 * r];
        if (r < 2) {
            for (int h = 8 * r; h < 8 * r + 8; ++h) {
                for (int q = 0; q < 8; ++q) list[n++] = 9 << 7 | h << 3 | q;
                list[n++] = own | h << 3; list[n++] = own | h << 3 | 1;
            }
            for (int h = 2 * r; h < (r ? 16 : 2); ++h) { list[n++] = own | h << 3 | 2; list[n++] = own | h << 3 | 3; }
        } else {
            for (int h = 0; h < 16; ++h)
                for (int q = 4; q < 8; ++q) list[n++] = own | h << 3 | q;
        }
        for (int i = 48 * r; i < (r < 2 ? 48 * r + 48 : 128); ++i)
            if (i % 8 != rank) list[n++] = i;
        while (n < nby - from - (r == 2 ? (nr & 1) != ((nby - from - n) & 1) : 0)) {
            if (nr >= 7 * PUSHP && nr < 7 * (PUSHP + NPUSH)) P.job[byp[from + n]] |= (u64)(nr / 7 - PUSHP + 1) << 53;
            list[n++] = (((rank + 1 + nr % 7) & 7) + 1) << 7 | worder[nr / 7];
            ++nr;
        }
        for (int i = 0; i < n; ++i) {
            u64& j = P.job[byp[from + i]];
            j = (j & ~(0x7ffull << (11 + 11 * r))) | (u64)list[i] << (11 + 11 * r);
        }
    }
    P.dstart = nr;
    const int dcount[] = DCOUNT;
    for (int r = 0, left = (896 - nr) / 2; r < 12; ++r) {
        const int n = dcount[r < (int)(sizeof dcount / sizeof dcount[0]) ? r : sizeof dcount / sizeof dcount[0] - 1];
        P.dcount[r] = r == 11 ? 0 : n < left ? n : left;
        left -= P.dcount[r];
    }
    for (int i = 0; i < nby; ++i) P.job[byp[i]] = (P.job[byp[i]] & ~(511ull << 44)) | (u64)(nby - 1 - i) << 44;
    unsigned char* djob = (unsigned char*)P.djob;
    for (int b = 0; b < 256; ++b) djob[b] = 255;
    unsigned j = 0;
    for (int pr = PRE_TOP; pr >= 0; --pr)
        for (int x = 0; x < 8; ++x)
            for (int i = first[pr]; i < first[pr + 1] && j < NPRE; ++i)
                if (byp[i] % 8 == x && !(tile[byp[i]] & 64)) djob[byp[i]] = (unsigned char)(223 - j++);
}

}

extern "C" void kernel_launch(int rank, int nranks, ncclComm_t comm, hipStream_t stream,
                              void* const* d_in, const int* in_sizes, int n_in,
                              void* d_out, int out_size, const int* in_dtypes, int out_dtype,
                              void* const* d_out_all, void* const* d_ws_all, size_t ws_bytes) {
    (void)comm; (void)in_dtypes; (void)out_dtype; (void)d_out_all;
    if (nranks != NR || (unsigned)rank >= (unsigned)NR || n_in < 2 || ws_bytes < WS_NEED || in_sizes[0] != 4096 * 512 ||
        in_sizes[1] != 512 * 32768 || out_size != 512 * 32768) {
        fprintf(stderr, "kernel_launch: unexpected problem shape\n");
        return;
    }
    Params P;
    P.x = (const char*)d_in[0];
    P.w = (const char*)d_in[1];
    P.out = (char*)d_out;
    for (int p = 0; p < NR; ++p) P.ws[p] = (char*)d_ws_all[p];
    P.me = P.ws[rank];
    P.rank = rank;
    make_plan(P, rank);
    hipLaunchKernelGGL(fused_gemm_rs, dim3(256), dim3(512), 0, stream, P);
    hipError_t e = hipGetLastError();
    if (e != hipSuccess) fprintf(stderr, "launch failed: %s\n", hipGetErrorString(e));
}
```

```cpp
#include <hip/hip_runtime.h>
#include <rccl/rccl.h>
#include <cstdio>

#define LAS __attribute__((address_space(3)))
typedef __bf16 bf16x8 __attribute__((ext_vector_type(8)));
typedef __bf16 bf16x4 __attribute__((ext_vector_type(4)));
typedef float f32x4 __attribute__((ext_vector_type(4)));
typedef unsigned u32x4 __attribute__((ext_vector_type(4)));
typedef unsigned u32x2 __attribute__((ext_vector_type(2)));
typedef unsigned long long u64;

namespace {

constexpr int NR = 8;
constexpr int NSLOT = 128;
constexpr int LSLOT = 36 * 1024;
constexpr int LDS_LUT = 4 * LSLOT;
constexpr int LDS_REC = LDS_LUT + 2048;
constexpr int LDS_FLG = LDS_REC + 512;
constexpr int LDS_LUT2 = LDS_FLG + 8192;
constexpr int LDS_BYTES = LDS_LUT2 + 2048;
constexpr int SCR_UNIT = 1408;
constexpr int SCR_WAVE = 5 * SCR_UNIT;
constexpr int SCR_T = 1152;
#define PWG 6
#define NDEC 7
#define LOOKAHEAD 10u
#define NFLY 10
#define RING_A2 3
#define NBAND 2
#define MIX {0, 7, 3, 4, 1, 6, 2, 5}
#define NPUSH 6
#define PUSH0 4
#define PUSHS 1
#define PUSHP 4
#define FULL14 0xc1
constexpr int PW = PWG * NDEC;
#define LATE_COL 14u
#define NPRE 28u
constexpr unsigned NLATE = (16u - LATE_COL) * 14u;
static_assert(NPRE <= NLATE, "only jobs that wait for nothing run early");
constexpr unsigned NXQ = 7;
constexpr unsigned NIN = 64 * NXQ + 512 - 4 * NPUSH;
constexpr unsigned NTK = NIN + 512;
constexpr u64 DEADLINE = 100000000ull;

constexpr size_t OFF_CNT = 0;
constexpr size_t OFF_TMO = 16384;
constexpr size_t OFF_TKT = 20480;
constexpr size_t OFF_OPOS = 24576;
constexpr size_t OFF_DFL = 28672;
constexpr size_t OFF_READY = 32768;
constexpr size_t OFF_OFL = 98304;
constexpr size_t OFF_REC = 131072;
constexpr size_t REC_PAR = (size_t)NR * 3 * 1024 * 8;
constexpr size_t OFF_A = 1u << 20;
constexpr size_t OFF_B = OFF_A + (32u << 20);
constexpr int NUNIT = 1024 + 2 * NR * 1024;
constexpr int FIXB = 2560;
constexpr int VCAP = 2624;
constexpr int PAIRB = 2 * FIXB + 2 * VCAP;
constexpr int VMID = 2 * FIXB + VCAP;
constexpr size_t PK_PAR = (size_t)(NUNIT / 2) * PAIRB;
constexpr size_t OFF_PK = OFF_B + (32u << 20);
constexpr size_t WS_NEED = OFF_PK + 2 * PK_PAR;
static_assert(OFF_REC + 2 * REC_PAR <= OFF_A, "layout");

struct Params {
    const char* x;
    const char* w;
    char* out;
    char* me;
    char* ws[NR];
    int rank;
    unsigned dstart;
    unsigned dcount[12];
    unsigned hsplit[2];
    unsigned unbox[64];
    unsigned order[64];
    unsigned worder[32];
    u64 job[256];
    unsigned djob[64];
};

__device__ __forceinline__ unsigned rfl(unsigned v) { return __builtin_amdgcn_readfirstlane(v); }
__device__ __forceinline__ unsigned perm(unsigned hi, unsigned lo, unsigned sel) { return __builtin_amdgcn_perm(hi, lo, sel); }
__device__ __forceinline__ unsigned bfi(unsigned m, unsigned a, unsigned b) { return (a & m) | (b & ~m); }
__device__ __forceinline__ int a_swz(int q) { return (0x78 >> (2 * q)) & 3; }
__device__ __forceinline__ unsigned cvt_pk(float lo, float hi) { unsigned r; asm volatile("v_cvt_pk_bf16_f32 %0, %1, %2" : "=v"(r) : "v"(lo), "v"(hi)); return r; }

__device__ __forceinline__ u32x4 add_bf16(u32x4 x, u32x4 y) {
    u32x4 r;
#pragma unroll
    for (int a = 0; a < 4; ++a)
        r[a] = cvt_pk(__uint_as_float(x[a] << 16) + __uint_as_float(y[a] << 16), __uint_as_float(x[a] & 0xffff0000u) + __uint_as_float(y[a] & 0xffff0000u));
    return r;
}

template <int OFF> __device__ __forceinline__ void st_wt(const char* sbase, unsigned voff, u32x4 v) {
    asm volatile("global_store_dwordx4 %0, %1, %2 offset:%3 sc0 sc1\n\ts_nop 1" :: "v"(voff), "v"(v), "s"(sbase), "i"(OFF) : "memory");
}
__device__ __forceinline__ void glds4(unsigned m0v, unsigned voff, const char* sb) {
    asm volatile("s_mov_b32 m0, %0\n\ts_nop 0\n\t"
                 "global_load_lds_dwordx4 %1, %2 sc0 sc1\n\t"
                 "global_load_lds_dwordx4 %1, %2 offset:1024 sc0 sc1\n\t"
                 "global_load_lds_dwordx4 %1, %2 offset:2048 sc0 sc1\n\t"
                 "global_load_lds_dwordx4 %1, %2 offset:3072 sc0 sc1" :: "s"(m0v), "v"(voff), "s"(sb) : "memory");
}
__device__ __forceinline__ void glds1(unsigned m0v, unsigned voff, const char* sb) {
    asm volatile("s_mov_b32 m0, %0\n\ts_nop 0\n\tglobal_load_lds_dwordx4 %1, %2 sc0 sc1" :: "s"(m0v), "v"(voff), "s"(sb) : "memory");
}
__device__ __forceinline__ void wait_vm0() { asm volatile("s_waitcnt vmcnt(0)" ::: "memory"); }

__device__ __forceinline__ bool timed_out(unsigned* tmo, unsigned call, u64 t0, unsigned& spins) {
    if ((++spins & 63u) != 0u) return false;
    if (rfl(__hip_atomic_load(tmo, __ATOMIC_RELAXED, __HIP_MEMORY_SCOPE_AGENT)) == call) return true;
    if (__builtin_amdgcn_s_memrealtime() - t0 > DEADLINE) {
        __hip_atomic_store(tmo, call, __ATOMIC_RELAXED, __HIP_MEMORY_SCOPE_AGENT);
        return true;
    }
    return false;
}

#define LDS_FENCE() asm volatile("" ::: "memory")
template <int CTRL, int ROWMASK> __device__ __forceinline__ unsigned dpp0(unsigned v) {
    return (unsigned)__builtin_amdgcn_update_dpp(0, (int)v, CTRL, ROWMASK, 0xf, false);
}
__device__ __forceinline__ unsigned wave_scan(unsigned v) {
    v += dpp0<0x111, 0xf>(v); v += dpp0<0x112, 0xf>(v); v += dpp0<0x114, 0xf>(v); v += dpp0<0x118, 0xf>(v);
    v += dpp0<0x142, 0xa>(v); v += dpp0<0x143, 0xc>(v);
    return v;
}
__device__ __forceinline__ unsigned lane_rank(u64 m) {
    return __builtin_amdgcn_mbcnt_hi((unsigned)(m >> 32), __builtin_amdgcn_mbcnt_lo((unsigned)m, 0u));
}
__device__ __forceinline__ void build_lut(LAS char* lds, bool dec, int tid) {
    if (tid < 256) {
        u64 e = 0x0c0c0c0c0c0c0c0cull;
        unsigned t = 0;
#pragma unroll
        for (unsigned j = 0; j < 8; ++j)
            if ((tid >> j) & 1) {
                const unsigned slot = (t & 1u) * 4u + (t >> 1);
                const unsigned at = dec ? j : slot, val = dec ? slot : j;
                e = (e & ~(0xffull << (8 * at))) | ((u64)val << (8 * at));
                ++t;
            }
        *(LAS u64*)(lds + LDS_LUT + tid * 8) = e;
    }
}
__device__ __forceinline__ u32x2 lut_at(LAS char* lds, unsigned f) {
    return *(LAS const u32x2*)(lds + __builtin_amdgcn_udot4(f, 0x40201008u, (unsigned)LDS_LUT, false));
}
__device__ __forceinline__ unsigned shuffle8(unsigned lo, unsigned hi, u32x2 sel) {
    return bfi(0x0f0f0f0fu, perm(hi, lo, sel[0]), perm(hi, lo, sel[1]) << 4);
}
__device__ __forceinline__ unsigned half16(unsigned v, int u) { return (u & 1) ? v >> 16 : v & 0xffffu; }
__device__ __forceinline__ unsigned b_start(unsigned n1) { return n1 <= 512u ? ((n1 + 15u) >> 4) << 5 : ((n1 - 249u) >> 3) << 5; }
__device__ __forceinline__ void lds_or(LAS char* p, unsigned v) {
    __hip_atomic_fetch_or((LAS unsigned*)p, v, __ATOMIC_RELAXED, __HIP_MEMORY_SCOPE_WAVEFRONT);
}
__device__ __forceinline__ unsigned bit_addr(LAS char* p) { return (unsigned)(size_t)p * 8u; }
__device__ __forceinline__ void put_bits(unsigned pos, unsigned v) {
    const u64 sh = (u64)v << (pos & 31u);
    LAS char* p = (LAS char*)(size_t)((pos >> 3) & ~3u);
    lds_or(p, (unsigned)sh); lds_or(p + 4, (unsigned)(sh >> 32));
}
__device__ __forceinline__ unsigned get_bits(unsigned pos) {
    LAS const unsigned* p = (LAS const unsigned*)(size_t)((pos >> 3) & ~3u);
    return __builtin_amdgcn_alignbit(p[1], p[0], pos);
}

template <int NU, class Get, class Mid>
__device__ __forceinline__ void pack_store(Get get, char* pk, const unsigned (&g)[NU], LAS char* lds, LAS char* scr, int lane,
                                           unsigned (&info)[NU], Mid mid) {
    constexpr int NS = (NU + 1) / 2;
    const unsigned voff = lane * 16;
#pragma unroll
    for (int u = 0; u < NU; ++u) *(LAS u32x4*)(scr + u * SCR_UNIT + voff) = (u32x4){0u, 0u, 0u, 0u};
    unsigned D[NU][8], pc[NU][3], cnt[NS], x16[NU], top = 0, top4 = 0;
    u32x2 sel[NU][4];
#pragma unroll
    for (int u = 0; u < NU; ++u) {
        u32x4 d[4], S[2];
        unsigned E[8], C[4], c = 0;
        get(u, d);
#pragma unroll
        for (int hf = 0; hf < 2; ++hf)
#pragma unroll
            for (int a = 0; a < 4; ++a) {
                const unsigned p = __builtin_amdgcn_alignbit(d[2 * hf][a], d[2 * hf][a], 31);
                const unsigned r = __builtin_amdgcn_alignbit(d[2 * hf + 1][a], d[2 * hf + 1][a], 31);
                S[hf][a] = perm(r, p, 0x06040200u);
                E[4 * hf + a] = perm(r, p, 0x07050301u);
            }
        char* pair = pk + (size_t)(g[u] >> 1) * PAIRB;
        const char* fix = pair + (g[u] & 1u) * FIXB;
        if (u == 0) {
            unsigned sum = __builtin_amdgcn_sad_u8(E[0], 0u, __builtin_amdgcn_sad_u8(E[2], 0u, __builtin_amdgcn_sad_u8(E[5], 0u, __builtin_amdgcn_sad_u8(E[7], 0u, 0u))));
            sum = __builtin_amdgcn_readlane(wave_scan(sum), 63);
            top = min(((sum + 983u) >> 10) + 2u, 255u);
            top4 = top * 0x01010101u;
        }
        st_wt<0>(fix, voff, S[0]);
#pragma unroll
        for (int a = 0; a < 4; ++a) {
            D[u][a] = top4 - E[a]; D[u][4 + a] = top4 - E[4 + a];
            unsigned n = bfi(0x0f0f0f0fu, D[u][a], D[u][4 + a] << 4);
            asm("" : "+v"(n));
            unsigned o = n | (n >> 1);
            asm("" : "+v"(o));
            const unsigned f = (~o | (o >> 2)) & 0x11111111u;
            unsigned f3 = (f << 1) + f;
            asm("" : "+v"(f3));
            C[a] = n & 0x33333333u & ~f3;
            c += __builtin_popcount(f);
            if (a < 3) pc[u][a] = c;
            sel[u][a] = lut_at(lds, f);
            if (a == 1) st_wt<1024>(fix, voff, S[1]);
        }
        cnt[u >> 1] = (u & 1) ? cnt[u >> 1] | c << 16 : c;
        const u32x2 cc = {C[0] | (C[1] << 2), C[2] | (C[3] << 2)};
        asm volatile("global_store_dwordx2 %0, %1, %2 offset:2048 sc0 sc1" :: "v"(lane * 8), "v"(cc), "s"(fix) : "memory");
        info[u] = top; x16[u] = 0;
        const u64 em = __builtin_amdgcn_ballot_w64(((D[u][0] | D[u][1] | D[u][2] | D[u][3] | D[u][4] | D[u][5] | D[u][6] | D[u][7]) & 0xf0f0f0f0u) != 0u);
        if (__builtin_expect(em != 0ull, 0)) {
            u32x4 X;
#pragma unroll
            for (int a = 0; a < 4; ++a) X[a] = bfi(0xf0f0f0f0u, D[u][a], D[u][4 + a] >> 4);
            const unsigned l = __builtin_ctzll(em);
            const unsigned x0 = __builtin_amdgcn_readlane(X[0], l), x1 = __builtin_amdgcn_readlane(X[1], l);
            const unsigned x2 = __builtin_amdgcn_readlane(X[2], l), x3 = __builtin_amdgcn_readlane(X[3], l);
            const unsigned any = x0 | x1 | x2 | x3, nz = (x0 != 0u) + (x1 != 0u) + (x2 != 0u) + (x3 != 0u);
            const unsigned nib = __builtin_ctz(any) >> 2, val = any >> (4u * nib);
            if ((em & (em - 1ull)) == 0ull && nz == 1u && val < 16u) {
                info[u] |= (l | (x0 ? 0u : x1 ? 1u : x2 ? 2u : 3u) << 6 | nib << 8 | val << 11) << 17;
            } else {
                info[u] |= 1u << 16;
                x16[u] = 1u + __builtin_popcountll(em);
                const unsigned k = 1u + lane_rank(em);
                if (lane == 0) st_wt<0>(pair, (g[u] & 1u) ? VMID : VMID - 16, (u32x4){(unsigned)em, (unsigned)(em >> 32), 0u, 0u});
                if ((em >> lane) & 1ull) st_wt<0>(pair, (g[u] & 1u) ? VMID + 16u * k : VMID - 16 - 16u * k, X);
            }
        }
    }
    unsigned excl[NS], tot[NS];
#pragma unroll
    for (int k = 0; k < NS; ++k) excl[k] = wave_scan(cnt[k]);
#pragma unroll
    for (int k = 0; k < NS; ++k) { tot[k] = __builtin_amdgcn_readlane(excl[k], 63); excl[k] -= cnt[k]; }
#pragma unroll
    for (int u = 0; u < NU; ++u)
        if (__builtin_expect(half16(tot[u >> 1], u) > 1400u, 0) && lane < 24) *(LAS u32x4*)(scr + u * SCR_UNIT + 1024 + voff) = (u32x4){0u, 0u, 0u, 0u};
    LDS_FENCE();
#pragma unroll
    for (int u = 0; u < NU; ++u) {
        unsigned pos = half16(excl[u >> 1], u) * 4u + bit_addr(scr + u * SCR_UNIT);
        asm("" : "+v"(pos));
#pragma unroll
        for (int a = 0; a < 4; ++a) put_bits(a ? pos + pc[u][a - 1] * 4u : pos, shuffle8(D[u][a], D[u][4 + a], sel[u][a]));
    }
    LDS_FENCE();
    mid();
    unsigned w[NU], c2n[NS];
    u32x2 s2[NU];
#pragma unroll
    for (int u = 0; u < NU; ++u) w[u] = *(LAS const unsigned*)(scr + u * SCR_UNIT + lane * 4);
    LDS_FENCE();
#pragma unroll
    for (int u = 0; u < NU; ++u) {
        const unsigned e = ((w[u] >> 1) | (w[u] >> 3)) & 0x11111111u;
        const unsigned c2 = (w[u] & 0x55555555u & ~(e << 2)) | e;
        s2[u] = lut_at(lds, e);
        const unsigned c = __builtin_popcount(e);
        c2n[u >> 1] = (u & 1) ? c2n[u >> 1] | c << 16 : c;
        *(LAS unsigned short*)(scr + u * SCR_UNIT + 128 + lane * 2) = (unsigned short)(c2 | (c2 >> 15));
    }
    unsigned in2[NS];
#pragma unroll
    for (int k = 0; k < NS; ++k) in2[k] = wave_scan(c2n[k]);
    LDS_FENCE();
#pragma unroll
    for (int u = 0; u < NU; ++u)
        put_bits(bit_addr(scr + u * SCR_UNIT + 128) + b_start(half16(tot[u >> 1], u)) + half16(in2[u >> 1] - c2n[u >> 1], u) * 4u,
                 shuffle8(w[u], w[u] >> 4, s2[u]));
    LDS_FENCE();
    u32x4 vv[NU];
#pragma unroll
    for (int u = 0; u < NU; ++u) vv[u] = *(LAS const u32x4*)(scr + u * SCR_UNIT + 128 + voff);
#pragma unroll
    for (int u = 0; u < NU; ++u) {
        const unsigned c16 = (b_start(half16(tot[u >> 1], u)) + half16(__builtin_amdgcn_readlane(in2[u >> 1], 63), u) * 4u + 127u) >> 7;
        const char* vg = pk + (size_t)(g[u] >> 1) * PAIRB + VMID + ((g[u] & 1u) ? 16 * (int)x16[u] : -16 * (int)(x16[u] + c16));
        if ((unsigned)lane < c16) st_wt<0>(vg, voff, vv[u]);
        if (__builtin_expect(c16 > 64u, 0) && 64u + lane < c16) st_wt<1024>(vg, voff, *(LAS const u32x4*)(scr + u * SCR_UNIT + 1152 + voff));
        info[u] |= (x16[u] + c16) << 8;
    }
    LDS_FENCE();
}

template <class Mid>
__device__ __forceinline__ void unpack2(const u32x4 (&p)[2][2], const u32x2 (&cc)[2], const unsigned (&info)[2], unsigned la, LAS char* lds,
                                        LAS char* T, LAS char* M, int lane, u32x4 (&o)[2][4], Mid mid) {
    u32x4 X[2] = {{0u, 0u, 0u, 0u}, {0u, 0u, 0u, 0u}};
    LAS char* A[2] = {M - 16u * la, M};
    const bool anyx = ((info[0] | info[1]) >> 16) != 0u;
    if (__builtin_expect(anyx, 0)) {
#pragma unroll
        for (int u = 0; u < 2; ++u) {
            if (info[u] & 0x10000u) {
                const u32x2 mv = *(LAS const u32x2*)(u ? M : M - 16);
                const u64 em = (u64)rfl(mv[0]) | (u64)rfl(mv[1]) << 32;
                const unsigned k = 1u + lane_rank(em);
                if ((em >> lane) & 1ull) X[u] = *(LAS const u32x4*)(u ? M + 16u * k : M - 16 - 16u * k);
                if (u) A[1] = M + 16u * (1u + __builtin_popcountll(em));
            } else {
                const unsigned xv = (info[u] >> 28) << (((info[u] >> 25) & 7u) * 4u), xa = (info[u] >> 23) & 3u;
                const bool me = (unsigned)lane == ((info[u] >> 17) & 63u);
#pragma unroll
                for (unsigned a = 0; a < 4; ++a) X[u][a] = me && xa == a ? xv : 0u;
            }
        }
    }
    unsigned z[2], rr[2];
#pragma unroll
    for (int u = 0; u < 2; ++u) {
        z[u] = *(LAS const unsigned short*)(A[u] + lane * 2);
        rr[u] = *(LAS const unsigned*)(A[u] + 128 + lane * 4);
    }
    unsigned pc[2][3], cnt = 0;
    u32x2 sel[2][4];
#pragma unroll
    for (int u = 0; u < 2; ++u) {
        unsigned c = 0;
#pragma unroll
        for (int k = 0; k < 2; ++k) {
            const unsigned h = cc[u][k] >> 1;
            const unsigned f0 = ~(cc[u][k] | h) & 0x11111111u, f1 = ~(cc[u][k] | h) & 0x44444444u;
            c += __builtin_popcount(f0);
            pc[u][2 * k] = c;
            c += __builtin_popcount(f1);
            if (k == 0) pc[u][1] = c;
            sel[u][2 * k] = lut_at(lds, f0);
            sel[u][2 * k + 1] = *(LAS const u32x2*)(lds + __builtin_amdgcn_udot4(f1, 0x10080402u, (unsigned)LDS_LUT, false));
        }
        cnt |= c << (16 * u);
    }
    unsigned c2[2], f2[2], c2n = 0;
    u32x2 s2[2];
#pragma unroll
    for (int u = 0; u < 2; ++u) {
        c2[u] = (z[u] | (z[u] << 15)) & 0x55555555u;
        asm("" : "+v"(c2[u]));
        f2[u] = c2[u] & ~(c2[u] >> 2) & 0x11111111u;
        s2[u] = lut_at(lds, f2[u]);
        c2n |= (unsigned)__builtin_popcount(f2[u]) << (16 * u);
        *(LAS unsigned*)(T + u * SCR_T + 256 + lane * 4) = rr[u];
    }
    const unsigned incl = wave_scan(cnt), in2 = wave_scan(c2n), tot = __builtin_amdgcn_readlane(incl, 63);
    unsigned wv[2];
#pragma unroll
    for (int u = 0; u < 2; ++u) wv[u] = get_bits(bit_addr(A[u]) + b_start(half16(tot, u)) + half16(in2 - c2n, u) * 4u);
#pragma unroll
    for (int u = 0; u < 2; ++u) {
        *(LAS unsigned*)(T + u * SCR_T + lane * 4) = (c2[u] ^ f2[u]) | shuffle8(wv[u], wv[u] >> 4, s2[u]);
        if (__builtin_expect(half16(tot, u) > 1024u, 0)) *(LAS u32x2*)(T + u * SCR_T + 512 + lane * 8) = *(LAS const u32x2*)(A[u] + 384 + lane * 8);
    }
    LDS_FENCE();
    unsigned w[2][4];
#pragma unroll
    for (int u = 0; u < 2; ++u) {
        unsigned pos = half16(incl - cnt, u) * 4u + bit_addr(T + u * SCR_T);
        asm("" : "+v"(pos));
#pragma unroll
        for (int a = 0; a < 4; ++a) w[u][a] = get_bits(a ? pos + pc[u][a - 1] * 4u : pos);
    }
    LDS_FENCE();
    mid();
    unsigned e[2][8];
#pragma unroll
    for (int u = 0; u < 2; ++u) {
        const unsigned top4 = (info[u] & 0xffu) * 0x01010101u;
#pragma unroll
        for (int a = 0; a < 4; ++a) {
            const unsigned lo = w[u][a] & 0x0f0f0f0fu, hi = (w[u][a] >> 4) & 0x0f0f0f0fu;
            e[u][a] = top4 - (perm(hi, lo, sel[u][a][0]) | ((cc[u][a >> 1] >> (2 * (a & 1))) & 0x03030303u));
            e[u][4 + a] = top4 - (perm(hi, lo, sel[u][a][1]) | ((cc[u][a >> 1] >> (2 * (a & 1) + 4)) & 0x03030303u));
        }
    }
    if (__builtin_expect(anyx, 0)) {
#pragma unroll
        for (int u = 0; u < 2; ++u)
#pragma unroll
            for (int a = 0; a < 4; ++a) { e[u][a] -= X[u][a] & 0xf0f0f0f0u; e[u][4 + a] -= (X[u][a] << 4) & 0xf0f0f0f0u; }
    }
#pragma unroll
    for (int u = 0; u < 2; ++u)
#pragma unroll
        for (int a = 0; a < 4; ++a) {
            const unsigned q0 = perm(e[u][a], p[u][0][a], 0x05010400u), q1 = perm(e[u][a], p[u][0][a], 0x07030602u);
            const unsigned q2 = perm(e[u][4 + a], p[u][1][a], 0x05010400u), q3 = perm(e[u][4 + a], p[u][1][a], 0x07030602u);
            o[u][0][a] = __builtin_amdgcn_alignbit(q0, q0, 1); o[u][1][a] = __builtin_amdgcn_alignbit(q1, q1, 1);
            o[u][2][a] = __builtin_amdgcn_alignbit(q2, q2, 1); o[u][3][a] = __builtin_amdgcn_alignbit(q3, q3, 1);
        }
}
__device__ __forceinline__ void store_stage(const char* dst, unsigned voff, const u32x4 (&o)[4]) {
    st_wt<0>(dst, voff, o[0]); st_wt<1024>(dst, voff, o[1]); st_wt<2048>(dst, voff, o[2]); st_wt<3072>(dst, voff, o[3]);
}
__device__ __forceinline__ void rows_of(LAS char* buf, int lane, u32x4 (&o)[4]) {
    LAS char* wp = buf + (lane & 15) * 128;
    const unsigned x = (((lane >> 3) & 6) ^ (lane & 7)) * 16;
    LDS_FENCE();
    *(LAS u32x4*)(wp + x) = o[0]; *(LAS u32x4*)(wp + (x ^ 16)) = o[1];
    *(LAS u32x4*)(wp + 2048 + x) = o[2]; *(LAS u32x4*)(wp + 2048 + (x ^ 16)) = o[3];
    LDS_FENCE();
    LAS const char* rp = buf + (lane >> 3) * 128 + (((lane >> 3) ^ lane) & 7) * 16;
#pragma unroll
    for (int c = 0; c < 4; ++c) o[c] = *(LAS const u32x4*)(rp + c * 1024);
    LDS_FENCE();
}
__device__ __forceinline__ void store_out(const char* dst, int lane, const u32x4 (&o)[4]) {
    const unsigned voff = (lane >> 3) * 65536 + (lane & 7) * 16;
#pragma unroll
    for (int c = 0; c < 4; ++c) st_wt<0>(dst + c * (8 * 65536), voff, o[c]);
}

__device__ __forceinline__ void pack_piece(const Params& P, char* ws, const char* px, const char* pw, LAS char* lds, int round, unsigned e,
                                           unsigned inbox, unsigned cntv, int lane, int wid) {
    LAS unsigned* lrec = (LAS unsigned*)(lds + LDS_REC) + 32 + round * 8;
    const int li = lane & 15, lg = lane >> 4;
    unsigned* ready = (unsigned*)(ws + OFF_READY);
    const int h = (e >> 3) & 15, idx = (e & 7) * 8 + wid, id = h * 64 + idx;
    const unsigned kind = e >> 7;
    const bool isx = kind == 0u || kind == 9u;
    const unsigned dest = (kind - 1u) & 7u;
    const bool own = kind == 9u || kind == (unsigned)P.rank + 1u;
    const unsigned rdest = isx ? (unsigned)lane >> 3 : dest;
    char* peer = P.ws[rdest];
    u32x4 d[4];
    u32x2 v[8];
    if (isx) {
        const char* src = px + (size_t)(64 * idx + (lane >> 2)) * 1024 + h * 64 + ((lane & 3) ^ a_swz(lg)) * 16;
#pragma unroll
        for (int c = 0; c < 4; ++c) {
            const u32x4 t = *(const u32x4*)(src + 16 * c * 1024);
            v[2 * c] = (u32x2){t[0], t[1]}; v[2 * c + 1] = (u32x2){t[2], t[3]};
        }
    } else {
        const char* src = pw + (size_t)(32 * h + 8 * lg) * 65536 + (size_t)(dest * 4096 + 64 * idx + 4 * li) * 2;
#pragma unroll
        for (int j = 0; j < 8; ++j) v[j] = *(const u32x2*)(src + (size_t)j * 65536);
    }
    __builtin_amdgcn_sched_barrier(0);
    if (round == 0) {
        build_lut(lds, false, threadIdx.x);
        if (threadIdx.x >= 448) ((LAS unsigned*)(lds + LDS_FLG))[(threadIdx.x - 448) * 32 + 16] = 0u;
        asm volatile("s_waitcnt lgkmcnt(0)" ::: "memory");
        __builtin_amdgcn_s_barrier();
        asm volatile("" ::: "memory");
    }
    if (isx) {
#pragma unroll
        for (int c = 0; c < 4; ++c) d[c] = (u32x4){v[2 * c][0], v[2 * c][1], v[2 * c + 1][0], v[2 * c + 1][1]};
    } else {
#pragma unroll
        for (int hf = 0; hf < 2; ++hf)
#pragma unroll
            for (int a = 0; a < 4; ++a) {
                d[2 * hf][a] = perm(v[2 * a + 1][hf], v[2 * a][hf], 0x05040100u);
                d[2 * hf + 1][a] = perm(v[2 * a + 1][hf], v[2 * a][hf], 0x07060302u);
            }
    }
    asm volatile("" :: "v"(peer));
    const unsigned call = rfl(cntv) + 1u, par = call & 1u;
    if (own) {
        store_stage(ws + (isx ? OFF_A : OFF_B) + (size_t)(h * 256 + 4 * idx) * 1024, lane * 16, d);
        wait_vm0();
        if (lane == 0) __hip_atomic_store(ready + ((isx ? 0 : 64) + idx) * NSLOT + h, call, __ATOMIC_RELAXED, __HIP_MEMORY_SCOPE_SYSTEM);
        return;
    }
    const unsigned g[1] = {isx ? id : 1024u + dest * 1024u + (inbox ? P.rank * 128u + inbox * 8u - 8u + wid : id)};
    unsigned r[1];
    pack_store<1>([&](int, u32x4 (&o)[4]) { o[0] = d[0]; o[1] = d[1]; o[2] = d[2]; o[3] = d[3]; }, (inbox ? P.ws[dest] : ws) + OFF_PK + par * PK_PAR, g, lds, lds + wid * SCR_WAVE, lane, r, [] {});
    if (lane == 0) lrec[wid] = r[0];
    wait_vm0();
    asm volatile("s_waitcnt lgkmcnt(0)" ::: "memory");
    __builtin_amdgcn_s_barrier();
    asm volatile("" ::: "memory");
    if (wid == 0 && (isx || lane < 8) && rdest != (unsigned)P.rank) {
        u64* rp = (u64*)(peer + OFF_REC + par * REC_PAR) + (P.rank * 3 + (isx ? 0 : 1)) * 1024 + h * 64 + (e & 7) * 8 + (lane & 7);
        __hip_atomic_store(rp, ((u64)call << 42) | lrec[lane & 7], __ATOMIC_RELAXED, __HIP_MEMORY_SCOPE_SYSTEM);
    }
}

__device__ __forceinline__ void pack_bulk(const Params& P, char* ws, const char* pw, LAS char* lds, unsigned rank, unsigned call, int lane, int wid) {
    unsigned i = P.dstart + 2u * rank, round = 0;
    const int li = lane & 15, lg = lane >> 4;
    const unsigned par = call & 1u;
    const unsigned lsrc = (unsigned)(8 * lg) * 65536u + (unsigned)(64 * wid + 4 * li) * 2u;
    u32x2 v[2][8];
    unsigned en[2];
    auto load = [&](unsigned i, int u) {
        const unsigned k = (i + u) / 7u, dest = (P.rank + 1u + (i + u) % 7u) & 7u, hp = (P.worder[k >> 2] >> ((k & 3u) * 8u)) & 127u;
        en[u] = dest << 7 | hp;
        const char* src = pw + (size_t)(hp >> 3) * (32 * 65536) + (dest * 4096u + (hp & 7u) * 512u) * 2u;
#pragma unroll
        for (int j = 0; j < 8; ++j) v[u][j] = *(const u32x2*)(src + (size_t)j * 65536 + lsrc);
    };
    load(i, 0);
    load(i, 1);
    __builtin_amdgcn_s_waitcnt(0x0F70);
    for (LAS unsigned* lrec = (LAS unsigned*)(lds + LDS_FLG);; lrec += 32) {
        const unsigned el = (lane & 8) ? en[1] : en[0];
        char* peer = P.ws[el >> 7];
        const unsigned g[2] = {1024u + en[0] * 8u + wid, 1024u + en[1] * 8u + wid};
        u32x4 d[2][4];
#pragma unroll
        for (int u = 0; u < 2; ++u)
#pragma unroll
            for (int hf = 0; hf < 2; ++hf)
#pragma unroll
                for (int a = 0; a < 4; ++a) {
                    d[u][2 * hf][a] = perm(v[u][2 * a + 1][hf], v[u][2 * a][hf], 0x05040100u);
                    d[u][2 * hf + 1][a] = perm(v[u][2 * a + 1][hf], v[u][2 * a][hf], 0x07060302u);
                }
        asm volatile("" : "+v"(d[0][0]), "+v"(d[0][1]), "+v"(d[0][2]), "+v"(d[0][3]), "+v"(d[1][0]), "+v"(d[1][1]), "+v"(d[1][2]), "+v"(d[1][3]));
        i = rank < P.dcount[round + 1u] ? i + 2u * P.dcount[round] : 896u;
        ++round;
        if (i < 896u) load(i, 0);
        __builtin_amdgcn_sched_barrier(0);
        unsigned r[2];
        pack_store<2>([&](int u, u32x4 (&o)[4]) { o[0] = d[u][0]; o[1] = d[u][1]; o[2] = d[u][2]; o[3] = d[u][3]; },
                      ws + OFF_PK + par * PK_PAR, g, lds, lds + wid * SCR_WAVE, lane, r, [&] { if (i < 896u) load(i, 1); });
        if (lane < 2) lrec[lane * 8 + wid] = lane ? r[1] : r[0];
        wait_vm0();
        __builtin_amdgcn_s_waitcnt(0x0F70);
        unsigned arrived = 0;
        if (lane == 0) arrived = __hip_atomic_fetch_add(lrec + 16, 1u, __ATOMIC_RELAXED, __HIP_MEMORY_SCOPE_WORKGROUP);
        LDS_FENCE();
        if (rfl(arrived) == 7u && lane < 16) {
            u64* rp = (u64*)(peer + OFF_REC + par * REC_PAR) + (P.rank * 3 + 1) * 1024 + (el & 127u) * 8u + (lane & 7);
            __hip_atomic_store(rp, ((u64)call << 42) | lrec[lane], __ATOMIC_RELAXED, __HIP_MEMORY_SCOPE_SYSTEM);
        }
        if (i >= 896u) break;
    }
}

constexpr int PSLOT = 15232;
constexpr int PSLOT_M = 2 * FIXB + VCAP;
constexpr int PSLOT_T = PSLOT - 4096;
constexpr int LDS_Q = LDS_REC;
constexpr int LDS_QTAIL = LDS_Q + 256;
constexpr int LDS_FULL = LDS_QTAIL + 16;

__device__ __forceinline__ unsigned lds_poll(LAS char* p) {
    return rfl(__hip_atomic_load((LAS unsigned*)p, __ATOMIC_RELAXED, __HIP_MEMORY_SCOPE_WORKGROUP));
}

template <int N>
__device__ __forceinline__ void fetch_pair(u32x4& next, unsigned naddr, unsigned v0, unsigned v1, unsigned v2, const char* pk, u64 vmask,
                                           unsigned slot, unsigned da, unsigned dv) {
    static_assert(N >= 7 && N <= 12, "window");
#define FW(j) "s_waitcnt vmcnt(%12)\n\t.if %13 == " #j "\n\tds_write_b32 %10, %11\n\t.endif\n\t"
    asm volatile("s_mov_b32 m0, %7\n\t"
                 FW(0) "global_load_lds_dwordx4 %2, %5 offset:2048 sc0 sc1\n\t"
                 FW(1) "global_load_lds_dwordx4 %2, %5 sc0 sc1\n\t"
                 FW(2) "global_load_lds_dwordx4 %2, %5 offset:1024 sc0 sc1\n\t"
                 FW(3) "global_load_lds_dwordx4 %2, %5 offset:3072 sc0 sc1\n\t"
                 "s_mov_b32 m0, %9\n\t"
                 FW(4) "s_mov_b64 exec, %6\n\t"
                 "global_load_lds_dwordx4 %4, %5 sc0 sc1\n\t"
                 "s_mov_b64 exec, -1\n\t"
                 "s_mov_b32 m0, %8\n\t"
                 "ds_read_b128 %0, %1\n\t"
                 FW(5) "global_load_lds_dwordx4 %3, %5 sc0 sc1\n\t"
                 "s_waitcnt lgkmcnt(0)"
                 : "=&v"(next)
                 : "v"(naddr), "v"(v0), "v"(v1), "v"(v2), "s"(pk), "s"(vmask), "s"(slot), "s"(slot + 4096u), "s"(slot + (PSLOT_M - 512)),
                   "v"(da), "v"(dv), "i"(N - 1), "i"((N - 1) % 6)
                 : "memory");
#undef FW
}

__device__ __forceinline__ void fetch_link(const Params& P, LAS char* lds, int q, unsigned call, int lane) {
    const unsigned p = (unsigned)(P.rank + 1 + q) & 7u;
    const char* pkb = P.ws[p] + OFF_PK + (call & 1u) * PK_PAR;
    const unsigned ldsbase = (unsigned)(size_t)lds;
    const unsigned v0 = lane * 16, v1 = v0 + 4096, v2 = v0 + (VMID - 512);
    unsigned fa[2] = {0, 0}, fv[2] = {0, 0};
    if (blockIdx.x < 14) {
        const unsigned k = lane + (blockIdx.x / 7u) * 64u;
        __hip_atomic_store((unsigned*)(P.me + OFF_READY) + (8u * p + (k & 7u)) * NSLOT + 16u + 7u * (k >> 3) + q, call, __ATOMIC_RELAXED,
                           __HIP_MEMORY_SCOPE_SYSTEM);
    }
    __builtin_amdgcn_s_setprio(3);
    unsigned ended = 0;
    u32x4 ev = {0u, 0u, 0u, 0u};
    for (unsigned i = 0;; ++i) {
        while (__builtin_expect(rfl(ev[0]) != i + 1u, 0)) {
            ev = *(volatile LAS u32x4*)(lds + LDS_Q + (i & 15u) * 16u);
            if (rfl(ev[0]) == i + 1u) break;
            if (fa[1]) {
                asm volatile("s_waitcnt vmcnt(6)" ::: "memory"); *(volatile LAS unsigned*)(lds + fa[1]) = fv[1]; fa[1] = 0;
            } else if (fa[0]) {
                asm volatile("s_waitcnt vmcnt(0)" ::: "memory"); *(volatile LAS unsigned*)(lds + fa[0]) = fv[0]; fa[0] = 0;
            } else {
                if (ended == NDEC) return;
                __builtin_amdgcn_s_sleep(1);
            }
        }
        const unsigned w1 = rfl(ev[1]);
        if (__builtin_expect(w1 >> 31, 0)) { ++ended; continue; }
        const unsigned owner = w1 >> 24, slot = ldsbase + (owner - 1u) * PSLOT;
        const char* pk = pkb + (size_t)(w1 & 0x3fffu) * PAIRB;
        const u64 vmask = (u64)rfl(ev[2]) | (u64)rfl(ev[3]) << 32;
        const unsigned da = ldsbase + (fa[1] ? fa[1] : (unsigned)LDS_FULL), dv = fv[1];
        fetch_pair<NFLY>(ev, ldsbase + LDS_Q + ((i + 1u) & 15u) * 16u, v0, v1, v2, pk, vmask, slot, da, dv);
        fa[1] = fa[0]; fv[1] = fv[0];
        fa[0] = LDS_FULL + owner * 4u; fv[0] = i + 1u;
    }
}

struct Desc { unsigned g, dst, rdy, rec; };

__device__ __forceinline__ void decode_link(const Params& P, LAS char* lds, int q, int w, unsigned call, int lane, int wid) {
    char* ws = P.me;
    const unsigned par = call & 1u;
    const unsigned p = (unsigned)(P.rank + 1 + q) & 7u;
    unsigned* tmo = (unsigned*)(ws + OFF_TMO);
    unsigned vorder = P.order[lane];
    asm volatile("" : "+v"(vorder));
    const unsigned rec_base = (unsigned)(OFF_REC + par * REC_PAR) + p * 3072u * 8u;
    auto describe = [&](unsigned n) {
        Desc d;
        if (n < NIN) {
            unsigned e = ((unsigned)__builtin_amdgcn_readlane(vorder, n >> 4) >> ((n & 12u) * 2u)) & 255u;
            if (e < 128u) e = (e & ~7u) | ((P.rank + (e & 7u) + ((e & 7u) > (unsigned)q)) & 7u);
            const unsigned type = e >> 7, h = (e >> 3) & 15u, idx = (e & 7u) * 8u + (n & 3u) * 2u, id = h * 64u + idx, slot = 16u + 7u * h + q;
            d.g = type ? 1024u + P.rank * 1024u + id : id;
            d.dst = (unsigned)(type ? OFF_B : OFF_A) + (slot * 256u + 4u * idx) * 1024u;
            d.rdy = (unsigned)OFF_READY + ((type * 64u + idx) * NSLOT + slot) * 4u;
            d.rec = rec_base + (type * 1024u + id) * 8u;
        } else {
            d.g = 9216u; d.dst = P.rank * 8192u; d.rdy = 0;
            d.rec = rec_base + (2048u + 2u * ((n - NIN) & 511u)) * 8u;
        }
        return d;
    };
    auto accept = [&](u64 v, Desc& d, unsigned& ia, unsigned& ib) {
        if (__builtin_amdgcn_ballot_w64((unsigned)(v >> 42) != (call & 0x3fffffu)) != 0ull) return false;
        ia = __builtin_amdgcn_readlane((unsigned)v, 0); ib = __builtin_amdgcn_readlane((unsigned)v, 1);
        if (!d.rdy) {
            const unsigned id = __builtin_amdgcn_readlane((unsigned)(v >> 32), 0) & 1022u;
            d.g = 9216u + P.rank * 1024u + id;
            d.dst = (id & 15u) * (32u * 65536u) + (p * 4096u + 64u * (id >> 4)) * 2u;
        }
        return true;
    };
    const char* pkb = P.ws[p] + OFF_PK + par * PK_PAR;
    const unsigned voff = lane * 16;
    const unsigned tk_base = (call - 1u) * (NTK - PW) - 2u * PW;
    LAS char* S = lds + (wid - 1) * PSLOT;
    LAS char* M = S + PSLOT_M;
    bool dead = false;
    unsigned dmask = 0;
    const unsigned vrows = (lane >> 3) * 65536u + (lane & 7) * 16u;
    const __amdgpu_buffer_rsrc_t orsrc = __builtin_amdgcn_make_buffer_rsrc(P.out, 0, 512 * 65536, 0x00020000);
    auto vlen = [](unsigned i) { return min((i >> 8) & 255u, (unsigned)VCAP / 16u); };
    auto post = [&](unsigned w1, unsigned ia, unsigned ib) {
        const unsigned la = vlen(ia), lb = vlen(ib);
        const unsigned mlo = la >= 32u ? ~0u : ~(~0u >> la), mhi = lb >= 32u ? ~0u : ((1u << lb) - 1u) | 1u;
        unsigned i = 0;
        asm volatile("s_waitcnt lgkmcnt(0)" ::: "memory");
        if (lane == 0) {
            i = __hip_atomic_fetch_add((LAS unsigned*)(lds + LDS_QTAIL), 1u, __ATOMIC_RELAXED, __HIP_MEMORY_SCOPE_WORKGROUP);
            *(volatile LAS u32x4*)(lds + LDS_Q + (i & 15u) * 16u) = (u32x4){i + 1u, w1, mlo, mhi};
        }
        return rfl(i) + 1u;
    };
    auto poll = [&](Desc& d, unsigned& ia, unsigned& ib) {
        const u64 t0 = __builtin_amdgcn_s_memrealtime();
        unsigned spins = 0;
        ia = ib = 0;
        while (!dead) {
            const u64 v = __hip_atomic_load((const u64*)(ws + d.rec) + (lane & 1), __ATOMIC_RELAXED, __HIP_MEMORY_SCOPE_SYSTEM);
            if (accept(v, d, ia, ib)) break;
            if (timed_out(tmo, call, t0, spins)) dead = true;
            __builtin_amdgcn_s_sleep(1);
        }
    };
    unsigned n1 = PW + w, ia, ib;
    Desc d0 = describe(w), d1 = describe(n1);
    poll(d0, ia, ib);
    unsigned want = post(d0.g >> 1 | wid << 24, ia, ib);
    for (;;) {
        const bool more = n1 < NTK;
        u32x4 own[8];
        const bool late = !d0.rdy && ((d0.dst >> 9) & 15u) >= LATE_COL;
        if (late) {
            const unsigned bit = ((d0.dst >> 8) & 30u) | (d0.dst >> 24);
            if (!((dmask >> bit) & 1u)) {
                const u64 t0 = __builtin_amdgcn_s_memrealtime();
                unsigned spins = 0;
                while (!dead) {
                    const unsigned v = __hip_atomic_load((const unsigned*)(ws + OFF_DFL) + q * 32 + (lane & 31), __ATOMIC_RELAXED, __HIP_MEMORY_SCOPE_SYSTEM);
                    dmask = (unsigned)__builtin_amdgcn_ballot_w64(v >= call);
                    if ((dmask >> bit) & 1u) break;
                    if (timed_out(tmo, call, t0, spins)) dead = true;
                    __builtin_amdgcn_s_sleep(1);
                }
            }
            asm volatile("" ::: "memory");
#pragma unroll
            for (int c = 0; c < 8; ++c) own[c] = __builtin_amdgcn_raw_buffer_load_b128(orsrc, vrows, d0.dst + c * (8u * 65536u), 17);
        }
        unsigned dv = 0;
        if (n1 + 2u * PW >= NIN + 32u * LATE_COL && dmask == 0u)
            dv = __hip_atomic_load((const unsigned*)(ws + OFF_DFL) + q * 32 + (lane & 31), __ATOMIC_RELAXED, __HIP_MEMORY_SCOPE_SYSTEM);
        const u64 rv = __hip_atomic_load((const u64*)(ws + d1.rec) + (lane & 1), __ATOMIC_RELAXED, __HIP_MEMORY_SCOPE_SYSTEM);
        while (lds_poll(lds + LDS_FULL + wid * 4) != want) __builtin_amdgcn_s_sleep(1);
        LDS_FENCE();
        const unsigned la = vlen(ia), lb = vlen(ib);
        if (la > 32u || lb > 32u) {
            const char* mid = pkb + (size_t)(d0.g >> 1) * PAIRB + VMID;
            for (int c = -3; c <= 3; ++c) {
                const int off = c * 1024 - 512 + (int)voff;
                if (c != 0 && off >= -16 * (int)la && off < 16 * (int)lb) {
                    u32x4 t;
                    asm volatile("global_load_dwordx4 %0, %1, off sc0 sc1\n\ts_waitcnt vmcnt(0)" : "=&v"(t) : "v"(mid + off) : "memory");
                    *(LAS u32x4*)(M + off) = t;
                }
            }
            LDS_FENCE();
        }
        u32x4 o[2][4];
        unsigned ja = 0, jb = 0;
        bool posted = false;
        {
            u32x4 pp[2][2];
            u32x2 cc[2];
#pragma unroll
            for (int u = 0; u < 2; ++u) {
                cc[u] = *(LAS const u32x2*)(S + u * FIXB + 2048 + lane * 8);
                pp[u][0] = *(LAS const u32x4*)(S + u * FIXB + voff);
                pp[u][1] = *(LAS const u32x4*)(S + u * FIXB + 1024 + voff);
            }
            const unsigned inf[2] = {ia, ib};
            unpack2(pp, cc, inf, la, lds, S + PSLOT_T, M, lane, o, [&] {
                asm volatile("" : "+v"(pp[0][0]), "+v"(pp[0][1]), "+v"(pp[1][0]), "+v"(pp[1][1]));
                if (more && accept(rv, d1, ja, jb)) {
                    want = post(d1.g >> 1 | wid << 24, ja, jb);
                    posted = true;
                }
            });
        }
        dmask |= (unsigned)__builtin_amdgcn_ballot_w64(dv >= call);
        if (!d0.rdy) {
            if (dead) {
#pragma unroll
                for (int c = 0; c < 8; ++c) o[c >> 2][c & 3] = (u32x4){0x7fc07fc0u, 0x7fc07fc0u, 0x7fc07fc0u, 0x7fc07fc0u};
            }
            rows_of(S + PSLOT_T, lane, o[0]);
            rows_of(S + PSLOT_T, lane, o[1]);
            if (late) {
#pragma unroll
                for (int c = 0; c < 8; ++c) o[c >> 2][c & 3] = add_bf16(o[c >> 2][c & 3], own[c]);
            }
        }
        unsigned tk;
        u64 rv2;
        {
            const unsigned step = d0.rdy ? 1024u : 8u * 65536u, v0 = d0.rdy ? voff : vrows;
            const char* b0 = (d0.rdy ? ws : P.out) + d0.dst;
            asm volatile("global_load_dwordx2 %1, %2, %5 sc0 sc1\n\t"
                         "s_mov_b64 exec, 1\n\tglobal_atomic_add %0, %3, %4, %5 sc0\n\ts_mov_b64 exec, -1\n\t"
                         "global_store_dwordx4 %8, %12, %6 sc0 sc1\n\tglobal_store_dwordx4 %9, %13, %6 sc0 sc1\n\t"
                         "global_store_dwordx4 %10, %14, %6 sc0 sc1\n\tglobal_store_dwordx4 %11, %15, %6 sc0 sc1\n\t"
                         "global_store_dwordx4 %8, %16, %7 sc0 sc1\n\tglobal_store_dwordx4 %9, %17, %7 sc0 sc1\n\t"
                         "global_store_dwordx4 %10, %18, %7 sc0 sc1\n\tglobal_store_dwordx4 %11, %19, %7 sc0 sc1\n\t"
                         "s_nop 1\n\ts_waitcnt vmcnt(8)"
                         : "=&v"(tk), "=&v"(rv2)
                         : "v"(d1.rec + (lane & 1) * 8u), "v"((unsigned)OFF_TKT + q * 128u), "v"(more ? 1u : 0u), "s"(ws), "s"(b0), "s"(b0 + 4u * step),
                           "v"(v0), "v"(v0 + step), "v"(v0 + 2u * step), "v"(v0 + 3u * step),
                           "v"(o[0][0]), "v"(o[0][1]), "v"(o[0][2]), "v"(o[0][3]), "v"(o[1][0]), "v"(o[1][1]), "v"(o[1][2]), "v"(o[1][3])
                         : "memory");
        }
        if (more && !posted && accept(rv2, d1, ja, jb)) {
            want = post(d1.g >> 1 | wid << 24, ja, jb);
            posted = true;
        }
        if (d0.rdy) {
            wait_vm0();
            if (lane < 2) __hip_atomic_store((unsigned*)(ws + d0.rdy) + lane * NSLOT, call, __ATOMIC_RELAXED, __HIP_MEMORY_SCOPE_SYSTEM);
        } else if (!late) {
            wait_vm0();
            if (lane == 0 && !dead)
                __hip_atomic_store((unsigned*)(ws + OFF_OFL) + q * 512 + ((d0.g >> 1) & 511u), call, __ATOMIC_RELAXED, __HIP_MEMORY_SCOPE_SYSTEM);
        }
        const unsigned n2 = rfl(tk) - tk_base;
        if (!more) break;
        if (!posted) {
            poll(d1, ja, jb);
            want = post(d1.g >> 1 | wid << 24, ja, jb);
        }
        ia = ja; ib = jb;
        d0 = d1;
        n1 = n2;
        d1 = describe(n1);
    }
    post(0x80000000u, 0u, 0u);
}

__device__ __forceinline__ void unbox(const Params& P, LAS char* lds, unsigned first, unsigned call, int lane, int wid) {
    char* ws = P.me;
    build_lut(lds + (LDS_LUT2 - LDS_LUT), true, threadIdx.x);
    asm volatile("s_waitcnt lgkmcnt(0)" ::: "memory");
    __builtin_amdgcn_s_barrier();
    asm volatile("" ::: "memory");
    const unsigned piece = first + (wid >> 2);
    if (piece >= 7u * NPUSH) return;
    const unsigned k = piece / 7u, q = piece % 7u, p = (P.rank + 1u + q) & 7u, par = call & 1u;
    const unsigned hp = (P.worder[(PUSHP + k) >> 2] >> (((PUSHP + k) & 3u) * 8u)) & 127u, idx = (hp & 7u) * 8u + (wid & 3) * 2u, slot = 16u + 7u * (hp >> 3) + q;
    const u64* rec = (const u64*)(ws + OFF_REC + par * REC_PAR) + (p * 3u + 1u) * 1024u + (hp >> 3) * 64u + idx + (lane & 1);
    unsigned inf[2] = {0u, 0u}, spins = 0;
    const u64 t0 = __builtin_amdgcn_s_memrealtime();
    for (;;) {
        const u64 v = __hip_atomic_load(rec, __ATOMIC_RELAXED, __HIP_MEMORY_SCOPE_SYSTEM);
        if (__builtin_amdgcn_ballot_w64((unsigned)(v >> 42) != (call & 0x3fffffu)) == 0ull) {
            inf[0] = __builtin_amdgcn_readlane((unsigned)v, 0); inf[1] = __builtin_amdgcn_readlane((unsigned)v, 1);
            break;
        }
        if (timed_out((unsigned*)(ws + OFF_TMO), call, t0, spins)) break;
        __builtin_amdgcn_s_sleep(2);
    }
    LAS char* S = lds + wid * PSLOT;
    const char* pk = ws + OFF_PK + par * PK_PAR + (size_t)(512u + P.rank * 512u + p * 64u + k * 4u + (wid & 3)) * PAIRB;
    const unsigned voff = lane * 16;
    glds4((unsigned)(size_t)S, voff, pk);
    glds4((unsigned)(size_t)S + 4096u, voff, pk + 4096);
    glds1((unsigned)(size_t)S + 8192u, voff, pk + 8192);
    glds1((unsigned)(size_t)S + 9216u, voff, pk + 9216);
    wait_vm0();
    u32x4 pp[2][2], o[2][4];
    u32x2 cc[2];
#pragma unroll
    for (int u = 0; u < 2; ++u) {
        cc[u] = *(LAS const u32x2*)(S + u * FIXB + 2048 + lane * 8);
        pp[u][0] = *(LAS const u32x4*)(S + u * FIXB + voff);
        pp[u][1] = *(LAS const u32x4*)(S + u * FIXB + 1024 + voff);
    }
    unpack2(pp, cc, inf, min((inf[0] >> 8) & 255u, (unsigned)VCAP / 16u), lds + (LDS_LUT2 - LDS_LUT), S + PSLOT_T, S + PSLOT_M, lane, o, [] {});
    store_stage(ws + OFF_B + (size_t)(slot * 256u + 4u * idx) * 1024, voff, o[0]);
    store_stage(ws + OFF_B + (size_t)(slot * 256u + 4u * idx) * 1024 + 4096, voff, o[1]);
    wait_vm0();
    if (lane < 2) __hip_atomic_store((unsigned*)(ws + OFF_READY) + (64u + idx + lane) * NSLOT + slot, call, __ATOMIC_RELAXED, __HIP_MEMORY_SCOPE_SYSTEM);
}

__device__ __forceinline__ void glds4w(unsigned m0v, unsigned voff, const char* sb) {
    constexpr int R = 8 * 65536 - 1024;
    asm volatile("s_mov_b32 m0, %0\n\ts_nop 0\n\t"
                 "global_load_lds_dwordx4 %1, %2\n\t"
                 "global_load_lds_dwordx4 %1, %3 offset:1024\n\t"
                 "global_load_lds_dwordx4 %1, %4 offset:2048\n\t"
                 "global_load_lds_dwordx4 %1, %5 offset:3072" :: "s"(m0v), "v"(voff), "s"(sb), "s"(sb + R), "s"(sb + 2 * R), "s"(sb + 3 * R) : "memory");
}
__device__ __forceinline__ bf16x8 read_tr(LAS const char* p) {
    const bf16x4 lo = __builtin_amdgcn_ds_read_tr16_b64_v4bf16((LAS bf16x4*)p), hi = __builtin_amdgcn_ds_read_tr16_b64_v4bf16((LAS bf16x4*)(p + 512));
    return __builtin_shufflevector(lo, hi, 0, 1, 2, 3, 4, 5, 6, 7);
}

template <int WR, bool BIG, bool SK5, int MODE>
__device__ __forceinline__ void k_loop(f32x4 (&acc)[BIG ? 10 : 8][4], char* ws, LAS char* lds, int u0, int u5, int pn, unsigned H, const char* wsrc, int rank,
                                       unsigned call, int lane, int wid) {
    constexpr int MT = BIG ? 10 : 8, M0 = MODE == 1 ? 8 : 0;
    constexpr bool OWN = MODE == 2;
    constexpr unsigned ns = OWN ? 16 : NSLOT, dt = MODE == 1 ? 7 : 1;
    const unsigned q4 = ((u0 >> 3) - rank - 1) & 7, q5 = ((u5 >> 3) - rank - 1) & 7;
    const bool cut = MODE == 0 && q4 < 7;
    const unsigned steps = MODE == 2 ? 16 : MODE == 1 ? 16 - H : 112 + H, last = MODE == 1 ? 121 + q4 : cut && q4 == 6 && H < 16 ? ns - 2 : ns - 1;
    unsigned t = MODE == 1 ? 16 + q4 + 7 * H : 0, tskip = cut ? 16 + q4 + 7 * H : 999;
    unsigned skip4 = 16 + q4, skip5 = 16 + q5;
    constexpr bool has5 = BIG && !WR, has4 = WR || MODE != 1;
    constexpr int RA = MODE == 0 ? 4 : MODE == 1 ? 7 : RING_A2, RB = MODE == 0 ? 4 : MODE == 1 ? 7 : 9 - RING_A2, RW = WR ? RB : RA;
    constexpr int RS = MODE == 0 ? LSLOT : MODE == 1 ? 20480 : 16384;
    constexpr int A0 = MODE == 1 ? -16384 : 0, B0 = MODE == 0 ? 20480 : MODE == 1 ? 4096 : RA * RS;
    unsigned* tmo = (unsigned*)(ws + OFF_TMO);
    const int wc = wid & 3;
    const unsigned ldsbase = (unsigned)(size_t)lds;
    const unsigned u1 = WR ? 4 * pn + wc : u0 + wc;
    const char* g1 = OWN && WR ? wsrc + wc * 128 : ws + (WR ? OFF_B : OFF_A) + (size_t)u1 * 4096;
    const unsigned vw = (lane >> 3) * 65536 + ((lane & 7) ^ ((lane >> 4) & 1)) * 16;
    const char* g2 = ws + OFF_A + (size_t)(4 * u5 + wc) * 1024;
    const unsigned l1 = ldsbase + WR * B0 + wc * 4096, l2 = ldsbase + A0 + (16 + wc) * 1024;
    const unsigned voff = lane * 16;
    const unsigned vrow = (unsigned)OFF_READY + (WR * 64 + (lane >= 32 && has5 ? u5 : u1)) * (NSLOT * 4);
    const unsigned vsnap = vrow + (lane & 31) * 16;
    LAS char* snap = lds + LDS_FLG + wid * 1024;
    *(LAS u32x4*)(snap + voff) = (u32x4){0u, 0u, 0u, 0u};
    asm volatile("s_waitcnt lgkmcnt(0)" ::: "memory");
    unsigned hz = OWN && WR ? 4096u : 0u, look = 0;
#define ADVANCE(v) ({ const u64 m_ = __builtin_amdgcn_ballot_w64((v) >= call); \
    (unsigned)__builtin_ctzll(~(u64)((unsigned)m_ & (unsigned)(m_ >> 32))); })
#define ENSURE(t) do { if (__builtin_expect(hz <= (unsigned)(t), 0)) { const u64 t0_ = __builtin_amdgcn_s_memrealtime(); unsigned sp_ = 0; \
        __builtin_amdgcn_s_waitcnt(0xC07F); \
        for (;;) { \
            const unsigned a_ = __hip_atomic_load((const unsigned*)(ws + vrow) + min(hz + (lane & 31u), ns - 1u), __ATOMIC_RELAXED, __HIP_MEMORY_SCOPE_SYSTEM); \
            hz += ADVANCE(a_); \
            if (hz > (unsigned)(t)) break; \
            if (timed_out(tmo, call, t0_, sp_)) { hz = NSLOT; break; } \
            __builtin_amdgcn_s_sleep(2); } } } while (0)
#define STAGE(t, lo_) do { const size_t go_ = (size_t)(t) * 262144; \
    if (OWN && WR) glds4w(l1 + lo_, vw, g1 + 8 * go_); else if (has4) glds4(l1 + lo_, voff, g1 + go_); if (has5) glds1(l2 + lo_, voff, g2 + go_); } while (0)
#define WAITV() asm volatile("s_waitcnt vmcnt(%0)" :: "i"((4 * has4 + has5) * (RW - 2)) : "memory")
#pragma unroll 1
    for (int i = 0; i < RW - 1; ++i, t = min(t + dt, last)) { ENSURE(t); STAGE(t, i * RS); }
    WAITV();
    __builtin_amdgcn_s_barrier();
    if (WR == 1) __builtin_amdgcn_s_barrier();
    const unsigned aoff = ((lane & 15) * 4 + ((lane >> 4) ^ a_swz((lane >> 2) & 3))) * 16;
    const unsigned bsw = ((lane >> 3) & 1) * 16;
    const unsigned boff = B0 + wc * 4096 + (OWN ? (lane >> 4) * 1024 + ((lane >> 2) & 3) * 128 + (lane & 3) * 32 : lane * 16);
#define LDB() _Pragma("unroll") for (int n = 0; n < 4; ++n) \
        B[n] = OWN ? read_tr(pb + (((n >> 1) * 16) ^ bsw) + (n & 1) * 8) : *(LAS const bf16x8*)(pb + n * 1024)
#define LDA(m0, m1) _Pragma("unroll") for (int m = m0; m < m1; ++m) A[m] = *(LAS const bf16x8*)(pa + A0 + (m < 8 ? 8 * WR + m : 8 + 2 * WR + m) * 1024)
#define MIDDLE() do { WAITV(); __builtin_amdgcn_s_waitcnt(0xC07F); __builtin_amdgcn_sched_barrier(0); __builtin_amdgcn_s_barrier(); \
        __builtin_amdgcn_sched_barrier(0); __builtin_amdgcn_s_setprio(1); } while (0)
#define MMA(m0, m1) _Pragma("unroll") for (int m = m0; m < m1; ++m) _Pragma("unroll") for (int k = 0; k < 4; ++k) { \
        const int n = (m & 1) ? 3 - k : k; \
        acc[m][n] = __builtin_amdgcn_mfma_f32_16x16x32_bf16(B[n], A[m], acc[m][n], 0, 0, 0); __builtin_amdgcn_sched_barrier(0); }
#define TAIL() do { __builtin_amdgcn_s_setprio(0); __builtin_amdgcn_sched_barrier(0); __builtin_amdgcn_s_barrier(); \
        __builtin_amdgcn_sched_barrier(0); } while (0)
    unsigned so = 0, sb = 0, sw = (RW - 1) * RS;
#define HEAD() ENSURE(t); STAGE(t, sw); __builtin_amdgcn_sched_barrier(0); \
        bf16x8 A[MT], B[4]; LAS const char* pa = lds + aoff + so; LAS const char* pb = lds + boff + (RA == RB ? so : sb)
#define NEXT(d5) sw = RW == RA ? so : sb; so = so == (RA - 1) * RS ? 0u : so + RS; \
        if (RA != RB) sb = sb == (RB - 1) * RS ? 0u : sb + RS; \
        t += dt; \
        if (MODE == 0) { const bool hit = t == tskip; t += hit ? 1u : 0u; tskip += hit ? 7u : 0u; } \
        t = min(t, last); \
        const bool do5 = s != skip5; \
        if (SK5) skip5 += do5 ? 0u : d5; \
        asm volatile("" : "+s"(t), "+s"(tskip), "+s"(so), "+s"(skip5))
#define LOOK() if (MODE == 0 && __builtin_expect(t >= look, 0)) { \
        const unsigned fv = *(volatile LAS const unsigned*)(snap + (lane >> 5) * 512 + min(hz + (lane & 31u), ns - 1u) * 4u); \
        hz += ADVANCE(fv); \
        glds1(ldsbase + LDS_FLG + wid * 1024, vsnap, ws); \
        look = max(t + 4u + LOOKAHEAD, hz) - LOOKAHEAD; }
    unsigned s = 0;
    if (BIG && MODE == 0) {
        do {
            HEAD();
            LDB();
            LDA(8, MT);
            NEXT(7u);
            unsigned cmp4 = skip4;
            if (__builtin_expect(s == skip4, 0)) {
                skip4 += 7u;
                MIDDLE();
            }
            asm volatile("" : "+s"(cmp4));
            if (__builtin_expect(s != cmp4, 1)) {
                LDA(0, 8);
                __builtin_amdgcn_sched_barrier(0);
                LOOK();
                MIDDLE();
                MMA(0, 8);
            }
            if (!SK5 || __builtin_expect(do5, 1)) { MMA(8, MT); }
            TAIL();
        } while (++s < 7u * H + (H ? 16u : 1u));
        skip5 = q5 != q4 ? 16u + 7u * H + q5 - (q5 > q4) : 999u;
    }
    for (; s < steps; ++s) {
        HEAD();
        LDB();
        LDA(M0, MT);
        __builtin_amdgcn_sched_barrier(0);
        NEXT(6u);
        LOOK();
        MIDDLE();
        MMA(M0, 8);
        if (!SK5 || __builtin_expect(do5, 1)) { MMA(8, MT); }
        TAIL();
    }
#undef HEAD
#undef NEXT
#undef LOOK
#undef TAIL
#undef LDA
#undef LDB
#undef MIDDLE
#undef MMA
#pragma unroll
    for (int m = M0; m < MT; ++m) asm volatile("" : "+v"(acc[m][0]), "+v"(acc[m][1]), "+v"(acc[m][2]), "+v"(acc[m][3]));
    if (WR == 0) __builtin_amdgcn_s_barrier();
#undef ENSURE
#undef ADVANCE
#undef STAGE
#undef WAITV
}

template <bool BIG, bool SK5>
__device__ __forceinline__ void gemm_tile(const Params& P, LAS char* lds, int u0, int u5, int pn, unsigned call, int lane, int wid) {
    constexpr int NU = BIG ? 5 : 4;
    char* ws = P.me;
    unsigned* tmo = (unsigned*)(ws + OFF_TMO);
    const int wr = wid >> 2, wc = wid & 3;
    const unsigned H = BIG && (u5 >> 3) != (u0 >> 3) ? (P.hsplit[SK5] >> (4 * (pn >> 1))) & 15u : 0u;
    f32x4 acc[2 * NU][4];
#pragma unroll
    for (int m = 0; m < 2 * NU; ++m)
#pragma unroll
        for (int n = 0; n < 4; ++n) acc[m][n] = (f32x4){0.f, 0.f, 0.f, 0.f};
    if (wr == 0) k_loop<0, BIG, SK5, 0>(acc, ws, lds, u0, u5, pn, H, nullptr, P.rank, call, lane, wid);
    else k_loop<1, BIG, SK5, 0>(acc, ws, lds, u0, u5, pn, H, nullptr, P.rank, call, lane, wid);
    const unsigned tmov = __hip_atomic_load(tmo, __ATOMIC_RELAXED, __HIP_MEMORY_SCOPE_AGENT);
    LAS unsigned* meet = (LAS unsigned*)(lds + LDS_REC + 128);
    if (wid == 0 && lane < 5) meet[lane * 16 + 8] = 0u;
    asm volatile("s_waitcnt lgkmcnt(0)" ::: "memory");
    asm volatile("" :: "v"(tmov));
    wait_vm0();
    __builtin_amdgcn_s_barrier();
    asm volatile("" ::: "memory");
    const unsigned par = call & 1u;
    const int cb = 4 * pn + wc;
    if (__builtin_expect(rfl(tmov) == call, 0)) {
#pragma unroll
        for (int m = 0; m < 2 * NU; ++m)
#pragma unroll
            for (int n = 0; n < 4; ++n) acc[m][n] = (f32x4){__builtin_nanf(""), __builtin_nanf(""), __builtin_nanf(""), __builtin_nanf("")};
    }
    unsigned rb[NU], g[NU];
#pragma unroll
    for (int u = 0; u < NU; ++u) {
        rb[u] = (u < 4 ? 2 * u0 + 4 * wr + u : 2 * u5 + wr) & 127u;
        g[u] = 9216u + (rb[u] >> 4) * 1024u + cb * 16u + (rb[u] & 15u);
    }
    auto get = [&](int u, u32x4 (&d)[4]) {
#pragma unroll
        for (int c = 0; c < 4; ++c) {
            const int mi = 2 * u + (c >> 1), b = c & 1;
            d[c][0] = cvt_pk(acc[mi][0][2 * b], acc[mi][1][2 * b]);
            d[c][1] = cvt_pk(acc[mi][2][2 * b], acc[mi][3][2 * b]);
            d[c][2] = cvt_pk(acc[mi][0][2 * b + 1], acc[mi][1][2 * b + 1]);
            d[c][3] = cvt_pk(acc[mi][2][2 * b + 1], acc[mi][3][2 * b + 1]);
        }
    };
    LAS char* scr = lds + wid * SCR_WAVE;
    auto own_rows = [&](int u) {
        u32x4 d[4];
        get(u, d);
        rows_of(scr, lane, d);
        store_out(P.out + (size_t)(rb[u] & 15u) * (32 * 65536) + (size_t)(P.rank * 4096 + 64 * cb) * 2, lane, d);
    };
    const bool late = pn >= LATE_COL;
    const unsigned pos0 = late ? 32u * LATE_COL - (call - 1u) * (512u - 32u * LATE_COL) : (1u - call) * (32u * LATE_COL);
    auto publish = [&](LAS unsigned* m, unsigned rb0, bool draw, unsigned n) {
        const unsigned dest = rb0 >> 4;
        char* peer = P.ws[dest];
        if (draw && lane == 0)
            m[9] = __hip_atomic_fetch_add((unsigned*)(ws + OFF_OPOS) + dest * 32 + (late ? 256 : 0), 4u, __ATOMIC_RELAXED, __HIP_MEMORY_SCOPE_AGENT) + pos0;
        wait_vm0();
        unsigned before = 0;
        if (lane == 0) before = __hip_atomic_fetch_add(m + 8, 1u, __ATOMIC_RELAXED, __HIP_MEMORY_SCOPE_WORKGROUP);
        LDS_FENCE();
        if (rfl(before) == n - 1u && lane < 8) {
            u64* ring = (u64*)(peer + OFF_REC + par * REC_PAR) + (P.rank * 3 + 2) * 1024 + 2u * (m[9] & 508u) + lane;
            __hip_atomic_store(ring, (u64)call << 42 | (u64)((4u * pn + (lane >> 1)) * 16u + (rb0 & 15u)) << 32 | m[lane], __ATOMIC_RELAXED,
                               __HIP_MEMORY_SCOPE_SYSTEM);
        }
    };
    char* pk = ws + OFF_PK + par * PK_PAR;
#pragma unroll
    for (int k = 0; k < 4; k += 2) {
        const unsigned ga[2] = {g[k], g[k + 1]};
        unsigned r[2];
        pack_store<2>([&](int u, u32x4 (&d)[4]) { get(u + k, d); }, pk, ga, lds, scr, lane, r, [] {});
        LAS unsigned* m = meet + (k + wr) * 16;
        if (lane < 2) m[2 * wc + lane] = lane ? r[1] : r[0];
        publish(m, rb[k], wc == 0, 4u);
    }
    if (BIG) {
        if ((u5 >> 3) != (u0 >> 3)) {
            asm volatile("s_waitcnt lgkmcnt(0)" ::: "memory");
            __builtin_amdgcn_s_barrier();
            if (wr == 0) k_loop<0, BIG, false, 1>(acc, ws, lds, u0, u5, pn, H, nullptr, P.rank, call, lane, wid);
            else k_loop<1, BIG, false, 1>(acc, ws, lds, u0, u5, pn, H, nullptr, P.rank, call, lane, wid);
            if (rfl(__hip_atomic_load(tmo, __ATOMIC_RELAXED, __HIP_MEMORY_SCOPE_AGENT)) == call) acc[2 * NU - 1][0][0] = __builtin_nanf("");
            wait_vm0();
            __builtin_amdgcn_s_barrier();
            asm volatile("" ::: "memory");
        }
        if (!SK5) {
            own_rows(NU - 1);
        } else {
            const unsigned ga[1] = {g[NU - 1]};
            unsigned r[1];
            pack_store<1>([&](int, u32x4 (&d)[4]) { get(NU - 1, d); }, pk, ga, lds, scr, lane, r, [] {});
            if (lane == 0) meet[64 + 2 * wc + wr] = r[0];
            publish(meet + 64, rb[NU - 1] & ~1u, wid == 4, 8u);
        }
    }
}

__device__ __forceinline__ void own_part(const Params& P, LAS char* lds, unsigned j, unsigned call, int lane, int wid) {
    char* ws = P.me;
    unsigned* tmo = (unsigned*)(ws + OFF_TMO);
    const int wr = wid >> 2, wc = wid & 3;
    const unsigned half = j & 1u, pn = (j >> 1) / 7u, q = (j >> 1) % 7u, p = (P.rank + 1u + q) & 7u;
    f32x4 acc[8][4];
#pragma unroll
    for (int m = 0; m < 8; ++m)
#pragma unroll
        for (int n = 0; n < 4; ++n) acc[m][n] = (f32x4){0.f, 0.f, 0.f, 0.f};
    const char* wsrc = P.w + p * 8192u + pn * 512u;
    if (wr == 0) k_loop<0, false, false, 2>(acc, ws, lds, 8 * P.rank + 4 * half, 0, pn, 0, wsrc, P.rank, call, lane, wid);
    else k_loop<1, false, false, 2>(acc, ws, lds, 8 * P.rank + 4 * half, 0, pn, 0, wsrc, P.rank, call, lane, wid);
    wait_vm0();
    __builtin_amdgcn_s_barrier();
    asm volatile("" ::: "memory");
    u32x4 d[4][4];
#pragma unroll
    for (int u = 0; u < 4; ++u) {
#pragma unroll
        for (int c = 0; c < 4; ++c) {
            const int mi = 2 * u + (c >> 1), n = 2 * (c & 1);
            d[u][c][0] = cvt_pk(acc[mi][n][0], acc[mi][n][1]);
            d[u][c][1] = cvt_pk(acc[mi][n][2], acc[mi][n][3]);
            d[u][c][2] = cvt_pk(acc[mi][n + 1][0], acc[mi][n + 1][1]);
            d[u][c][3] = cvt_pk(acc[mi][n + 1][2], acc[mi][n + 1][3]);
        }
        rows_of(lds + wid * SCR_WAVE, lane, d[u]);
    }
    const unsigned dst = (8u * half + 4u * wr) * (32u * 65536u) + p * 8192u + (4u * pn + wc) * 128u;
    if (pn < LATE_COL) {
        const unsigned* fl = (const unsigned*)(ws + OFF_OFL) + q * 512u + (4u * pn + wc) * 8u + 4u * half + 2u * wr + (lane & 1);
        const u64 t0 = __builtin_amdgcn_s_memrealtime();
        unsigned spins = 0;
        while (__builtin_amdgcn_ballot_w64(__hip_atomic_load(fl, __ATOMIC_RELAXED, __HIP_MEMORY_SCOPE_SYSTEM) < call) != 0ull) {
            if (timed_out(tmo, call, t0, spins)) break;
            __builtin_amdgcn_s_sleep(8);
        }
        asm volatile("" ::: "memory");
        const __amdgpu_buffer_rsrc_t orsrc = __builtin_amdgcn_make_buffer_rsrc(P.out, 0, 512 * 65536, 0x00020000);
        u32x4 y[16];
#pragma unroll
        for (int c = 0; c < 16; ++c) y[c] = __builtin_amdgcn_raw_buffer_load_b128(orsrc, (lane >> 3) * 65536 + (lane & 7) * 16, dst + c * (8u * 65536u), 17);
#pragma unroll
        for (int c = 0; c < 16; ++c) d[c >> 2][c & 3] = add_bf16(d[c >> 2][c & 3], y[c]);
    }
    if (rfl(__hip_atomic_load(tmo, __ATOMIC_RELAXED, __HIP_MEMORY_SCOPE_AGENT)) == call)
        d[0][0] = (u32x4){0x7fc07fc0u, 0x7fc07fc0u, 0x7fc07fc0u, 0x7fc07fc0u};
#pragma unroll
    for (int u = 0; u < 4; ++u) store_out(P.out + dst + u * (32 * 65536), lane, d[u]);
    wait_vm0();
    __builtin_amdgcn_s_barrier();
    asm volatile("" ::: "memory");
    if (threadIdx.x == 0)
        __hip_atomic_store((unsigned*)(ws + OFF_DFL) + q * 32u + pn * 2u + half, call, __ATOMIC_RELAXED, __HIP_MEMORY_SCOPE_SYSTEM);
}

__global__ void __launch_bounds__(512, 2) fused_gemm_rs(const Params P) {
    __shared__ __attribute__((aligned(16))) char smem[LDS_BYTES];
    const int bid = blockIdx.x;
    const int lane = threadIdx.x & 63, wid = rfl(threadIdx.x >> 6);
    char* ws = P.me; const char* px = P.x; const char* pw = P.w; const u64 job = P.job[bid];
    asm volatile("" :: "s"(ws), "s"(px), "s"(pw), "s"(job), "s"(P.rank));
    unsigned* cnt = (unsigned*)(ws + OFF_CNT) + bid * 8 + wid;
    const unsigned cntv = __hip_atomic_load(cnt, __ATOMIC_RELAXED, __HIP_MEMORY_SCOPE_AGENT);
    if (bid >= 7 * PWG) {
#pragma unroll 1
        for (int r = 0; r < 3; ++r) {
            const unsigned e = (unsigned)(job >> (11 + 11 * r)) & 0x7ffu;
            if ((e >> 7) == 15u) break;
            pack_piece(P, ws, px, pw, (LAS char*)smem, r, e, r ? 0u : (unsigned)(job >> 53) & 15u, cntv, lane, wid);
        }
        const unsigned late = (unsigned)(job >> 44) & 511u;
        if (late < P.dcount[0]) pack_bulk(P, ws, pw, (LAS char*)smem, late, rfl(cntv) + 1u, lane, wid);
        const unsigned ub = (P.unbox[bid >> 2] >> ((bid & 3) * 8)) & 255u;
        if (ub) unbox(P, (LAS char*)smem, ub - 1u, rfl(cntv) + 1u, lane, wid);
        asm volatile("s_waitcnt lgkmcnt(0)" ::: "memory");
        __builtin_amdgcn_s_barrier();
    } else {
        build_lut((LAS char*)smem, true, threadIdx.x);
        if (threadIdx.x >= 384) ((LAS unsigned*)(smem + LDS_Q))[threadIdx.x - 384] = 0u;
        asm volatile("s_waitcnt lgkmcnt(0)" ::: "memory");
        __builtin_amdgcn_s_barrier();
        asm volatile("" ::: "memory");
    }
    const unsigned call = rfl(cntv) + 1u;
    if (lane == 0) __hip_atomic_store(cnt, call, __ATOMIC_RELAXED, __HIP_MEMORY_SCOPE_AGENT);
    if (bid < 7 * PWG) {
        if (wid == 0) fetch_link(P, (LAS char*)smem, bid % 7, call, lane);
        else if (wid <= NDEC) decode_link(P, (LAS char*)smem, bid % 7, (wid - 1) * PWG + bid / 7, call, lane, wid);
    } else if (((unsigned)job & 63u) != 63u) {
        const int u0 = (unsigned)job & 63u, u5 = (unsigned)(job >> 57) & 63u, pn = ((unsigned)job >> 7) & 15u;
        const unsigned dj = (P.djob[bid >> 2] >> ((bid & 3) * 8)) & 255u;
        if (dj < 224u) own_part(P, (LAS char*)smem, dj, call, lane, wid);
        if (!(job & 64u)) gemm_tile<false, false>(P, (LAS char*)smem, u0, 0, pn, call, lane, wid);
        else if ((u5 >> 3) == P.rank) gemm_tile<true, false>(P, (LAS char*)smem, u0, u5, pn, call, lane, wid);
        else gemm_tile<true, true>(P, (LAS char*)smem, u0, u5, pn, call, lane, wid);
        if (!(pn >= 12 && (job & 64u))) for (LAS unsigned* drawn = (LAS unsigned*)(smem + LDS_REC);; drawn = (LAS unsigned*)((size_t)drawn ^ 4)) {
            if (threadIdx.x == 0) *drawn = __hip_atomic_fetch_add((unsigned*)(ws + OFF_TKT + 1024), 1u, __ATOMIC_RELAXED, __HIP_MEMORY_SCOPE_AGENT);
            asm volatile("s_waitcnt lgkmcnt(0)" ::: "memory");
            __builtin_amdgcn_s_barrier();
            const unsigned k = lds_poll((LAS char*)drawn) - (call - 1u) * (224u - NPRE + 256u - 7u * PWG - 32u) + NPRE;
            if (k >= 224u) break;
            own_part(P, (LAS char*)smem, k < NLATE ? 223u - k : k - NLATE, call, lane, wid);
        }
    }
}

#define LAGS {-1, 4, 18, 29, 42, 56, 72, 86}
#define HS_OWN 0x46800000
#define HS_SK5 0x468a8640
#define UNB 3
#define PRE_TOP 7
#define DCOUNT {108, 108, 82, 56}

void make_plan(Params& P, int rank) {
    const int lag[8] = LAGS;
    unsigned char* order = (unsigned char*)P.order;
    unsigned char* worder = (unsigned char*)P.worder;
    int no = 0, nw = 0;
    P.hsplit[0] = HS_OWN; P.hsplit[1] = HS_SK5;
    for (int i = 0; i < 64; ++i) P.order[i] = 0;
    for (int pos = -1; pos < 512; ++pos) {
        if (pos >= 0 && pos < 128 && pos % 8 < (int)NXQ) order[no++] = (unsigned char)pos;
        for (int pr = 0; pr < 8; ++pr) {
            const int h8 = pos - lag[pr];
            if (h8 >= 0 && h8 < 128 && h8 % 8 == 0) {
                if (!(nw >= PUSH0 && (nw - PUSH0) % PUSHS == 0 && (nw - PUSH0) / PUSHS < NPUSH)) order[no++] = (unsigned char)(128 | h8 | pr);
                worder[nw++] = (unsigned char)(h8 | pr);
            }
        }
    }
    for (int k = 0; k < NPUSH; ++k) {
        const unsigned char v = worder[PUSH0 + k * PUSHS];
        for (int i = PUSH0 + k * PUSHS; i > PUSHP + k; --i) worder[i] = worder[i - 1];
        worder[PUSHP + k] = v;
    }
    int seq[256], nseq = 0;
    const int mix[8] = MIX;
    for (int qt = 0; qt < NBAND; ++qt)
        for (int i = 0; i < 8; ++i) {
            const int pr = mix[i], band = (qt + pr) % NBAND, nt = (FULL14 >> pr) & 1 ? 14 : 13, base = 2 * rank + 2;
            for (int r = (nt * band + NBAND / 2) / NBAND; r < (nt * (band + 1) + NBAND / 2) / NBAND; ++r)
                for (int c = 0; c < 2; ++c) {
                    const int quad = (base + (r + 2 * pr) % 14) & 15, split = (base + (13 + 2 * pr) % 14) & 15;
                    const int k = 12 - r;
                    const int fifth = nt == 14 ? (r % 2 == 0 ? 8 * rank + r / 2 : r / 2 == 6 * c ? 8 * rank + 7 : -1)
                                               : k == 12 ? -1 : k % 3 == 0 ? 4 * split + k / 3 : 8 * rank + k - 1 - k / 3;
                    seq[nseq++] = (4 * quad | (fifth >= 0) << 6) | (2 * pr + c) << 7 | (fifth >= 0 ? fifth : 0) << 11;
                }
        }
    int tile[256], byp[256], nby = 0;
    static_assert(PWG == 6 && __builtin_popcount(FULL14) == 3 && (FULL14 & 0xc0) == 0xc0 && PUSHP + NPUSH <= 12 && PUSHP <= PUSH0, "214 tiles on 214 workgroups; pushed pieces: first round, first 64 entries");
    for (int x = 0, k = 0; x < 8; ++x)
        for (int b = 248 + x; b >= 7 * PWG; b -= 8) tile[b] = seq[k++];
    for (int pr = 0; pr <= 8; ++pr)
        for (int b = 7 * PWG; b < 256; ++b)
            if (((tile[b] >> 8) & 7) == pr) byp[nby++] = b;
    int first[10];
    for (int pr = 0, i = 0; pr <= 9; ++pr) {
        while (i < nby && ((tile[byp[i]] >> 8) & 7) < pr) ++i;
        first[pr] = i;
    }
    int nr = 0;
    for (int b = 0; b < 256; ++b)
        P.job[b] = (b < 7 * PWG ? 63ull : (u64)(tile[b] & 0x7ff) | (u64)(tile[b] >> 11) << 57) | 0x1ffffffffull << 11 | 511ull << 44;
    for (int r = 0; r < 3; ++r) {
        int list[256], n = 0;
        const int own = (rank + 1) << 7, from = r == 0 ? 0 : first[2 * r];
        if (r < 2) {
            for (int h = 8 * r; h < 8 * r + 8; ++h) {
                for (int q = 0; q < 8; ++q) list[n++] = 9 << 7 | h << 3 | q;
                list[n++] = own | h << 3; list[n++] = own | h << 3 | 1;
            }
            for (int h = 2 * r; h < (r ? 16 : 2); ++h) { list[n++] = own | h << 3 | 2; list[n++] = own | h << 3 | 3; }
        } else {
            for (int h = 0; h < 16; ++h)
                for (int q = 4; q < 8; ++q) list[n++] = own | h << 3 | q;
        }
        for (int i = 48 * r; i < (r < 2 ? 48 * r + 48 : 128); ++i)
            if (i % 8 != rank) list[n++] = i;
        while (n < nby - from - (r == 2 ? (nr & 1) != ((nby - from - n) & 1) : 0)) {
            if (nr >= 7 * PUSHP && nr < 7 * (PUSHP + NPUSH)) P.job[byp[from + n]] |= (u64)(nr / 7 - PUSHP + 1) << 53;
            list[n++] = (((rank + 1 + nr % 7) & 7) + 1) << 7 | worder[nr / 7];
            ++nr;
        }
        for (int i = 0; i < n; ++i) {
            u64& j = P.job[byp[from + i]];
            j = (j & ~(0x7ffull << (11 + 11 * r))) | (u64)list[i] << (11 + 11 * r);
        }
    }
    P.dstart = nr;
    const int dcount[] = DCOUNT;
    for (int r = 0, left = (896 - nr) / 2; r < 12; ++r) {
        const int n = dcount[r < (int)(sizeof dcount / sizeof dcount[0]) ? r : sizeof dcount / sizeof dcount[0] - 1];
        P.dcount[r] = r == 11 ? 0 : n < left ? n : left;
        left -= P.dcount[r];
    }
    for (int i = 0; i < nby; ++i) P.job[byp[i]] = (P.job[byp[i]] & ~(511ull << 44)) | (u64)(nby - 1 - i) << 44;
    unsigned char* unbox = (unsigned char*)P.unbox;
    for (int b = 0; b < 256; ++b) unbox[b] = 0;
    for (int i = 0; 2 * i < 7 * NPUSH; ++i) unbox[byp[first[UNB] + i]] = (unsigned char)(1 + 2 * i);
    unsigned char* djob = (unsigned char*)P.djob;
    for (int b = 0; b < 256; ++b) djob[b] = 255;
    unsigned j = 0;
    for (int pr = PRE_TOP; pr >= 0; --pr)
        for (int x = 0; x < 8; ++x)
            for (int i = first[pr]; i < first[pr + 1] && j < NPRE; ++i)
                if (byp[i] % 8 == x && !(tile[byp[i]] & 64)) djob[byp[i]] = (unsigned char)(223 - j++);
}

}

extern "C" void kernel_launch(int rank, int nranks, ncclComm_t comm, hipStream_t stream,
                              void* const* d_in, const int* in_sizes, int n_in,
                              void* d_out, int out_size, const int* in_dtypes, int out_dtype,
                              void* const* d_out_all, void* const* d_ws_all, size_t ws_bytes) {
    (void)comm; (void)in_dtypes; (void)out_dtype; (void)d_out_all;
    if (nranks != NR || (unsigned)rank >= (unsigned)NR || n_in < 2 || ws_bytes < WS_NEED || in_sizes[0] != 4096 * 512 ||
        in_sizes[1] != 512 * 32768 || out_size != 512 * 32768) {
        fprintf(stderr, "kernel_launch: unexpected problem shape\n");
        return;
    }
    Params P;
    P.x = (const char*)d_in[0];
    P.w = (const char*)d_in[1];
    P.out = (char*)d_out;
    for (int p = 0; p < NR; ++p) P.ws[p] = (char*)d_ws_all[p];
    P.me = P.ws[rank];
    P.rank = rank;
    make_plan(P, rank);
    hipLaunchKernelGGL(fused_gemm_rs, dim3(256), dim3(512), 0, stream, P);
    hipError_t e = hipGetLastError();
    if (e != hipSuccess) fprintf(stderr, "launch failed: %s\n", hipGetErrorString(e));
}
```

```cpp
#include <hip/hip_runtime.h>
#include <rccl/rccl.h>
#include <cstdio>

#define LAS __attribute__((address_space(3)))
typedef __bf16 bf16x8 __attribute__((ext_vector_type(8)));
typedef __bf16 bf16x4 __attribute__((ext_vector_type(4)));
typedef float f32x4 __attribute__((ext_vector_type(4)));
typedef unsigned u32x4 __attribute__((ext_vector_type(4)));
typedef unsigned u32x2 __attribute__((ext_vector_type(2)));
typedef unsigned long long u64;

namespace {

constexpr int NR = 8;
constexpr int NSLOT = 128;
constexpr int LSLOT = 36 * 1024;
constexpr int LDS_LUT = 4 * LSLOT;
constexpr int LDS_REC = LDS_LUT + 2048;
constexpr int LDS_FLG = LDS_REC + 512;
constexpr int LDS_LUT2 = LDS_FLG + 8192;
constexpr int LDS_BYTES = LDS_LUT2 + 2048;
constexpr int SCR_UNIT = 1408;
constexpr int SCR_WAVE = 5 * SCR_UNIT;
constexpr int SCR_T = 1152;
#define PWG 6
#define NDEC 7
#define LOOKAHEAD 10u
#define NFLY 10
#define XH0 4u
#define RING_A2 3
#define NBAND 2
#define MIX {1, 7, 2, 6, 0, 5, 3, 4}
#define NPUSH 6
#define PUSH0 4
#define PUSHS 1
#define PUSHP 4
constexpr int PW = PWG * NDEC;
#define LATE_COL 14u
#define NPRE 0u
constexpr unsigned NLATE = (16u - LATE_COL) * 14u;
static_assert(NPRE <= NLATE, "only jobs that wait for nothing run early");
constexpr unsigned NXQ = 7;
constexpr unsigned NIN = 64 * NXQ + 512 - 4 * NPUSH;
constexpr unsigned NTK = NIN + 512;
constexpr u64 DEADLINE = 100000000ull;

constexpr size_t OFF_CNT = 0;
constexpr size_t OFF_TMO = 16384;
constexpr size_t OFF_TKT = 20480;
constexpr size_t OFF_OPOS = 24576;
constexpr size_t OFF_DFL = 28672;
constexpr size_t OFF_READY = 32768;
constexpr size_t OFF_OFL = 98304;
constexpr size_t OFF_REC = 131072;
constexpr size_t REC_PAR = (size_t)NR * 3 * 1024 * 8;
constexpr size_t OFF_A = 1u << 20;
constexpr size_t OFF_B = OFF_A + (32u << 20);
constexpr int NUNIT = 1024 + 2 * NR * 1024;
constexpr int FIXB = 2560;
constexpr int VCAP = 2624;
constexpr int PAIRB = 2 * FIXB + 2 * VCAP;
constexpr int VMID = 2 * FIXB + VCAP;
constexpr size_t PK_PAR = (size_t)(NUNIT / 2) * PAIRB;
constexpr size_t OFF_PK = OFF_B + (32u << 20);
constexpr size_t WS_NEED = OFF_PK + 2 * PK_PAR;
static_assert(OFF_REC + 2 * REC_PAR <= OFF_A, "layout");

struct Params {
    const char* x;
    const char* w;
    char* out;
    char* me;
    char* ws[NR];
    int rank;
    unsigned dstart;
    unsigned dcount[12];
    unsigned drawers, pad;
    unsigned order[NIN / 2];
    unsigned worder[32];
    unsigned djob[54];
    u64 job[214];
};
static_assert(sizeof(Params) <= 4096 && sizeof(Params) == 96 + 16 + 48 + 2 * NIN + 128 + 216 + 1712, "kernel arguments, no padding");

__device__ __forceinline__ unsigned rfl(unsigned v) { return __builtin_amdgcn_readfirstlane(v); }
__device__ __forceinline__ unsigned perm(unsigned hi, unsigned lo, unsigned sel) { return __builtin_amdgcn_perm(hi, lo, sel); }
__device__ __forceinline__ unsigned bfi(unsigned m, unsigned a, unsigned b) { return (a & m) | (b & ~m); }
__device__ __forceinline__ int a_swz(int q) { return (0x78 >> (2 * q)) & 3; }
__device__ __forceinline__ unsigned cvt_pk(float lo, float hi) { unsigned r; asm volatile("v_cvt_pk_bf16_f32 %0, %1, %2" : "=v"(r) : "v"(lo), "v"(hi)); return r; }

__device__ __forceinline__ u32x4 add_bf16(u32x4 x, u32x4 y) {
    u32x4 r;
#pragma unroll
    for (int a = 0; a < 4; ++a)
        r[a] = cvt_pk(__uint_as_float(x[a] << 16) + __uint_as_float(y[a] << 16), __uint_as_float(x[a] & 0xffff0000u) + __uint_as_float(y[a] & 0xffff0000u));
    return r;
}

template <int OFF> __device__ __forceinline__ void st_wt(const char* sbase, unsigned voff, u32x4 v) {
    asm volatile("global_store_dwordx4 %0, %1, %2 offset:%3 sc0 sc1\n\ts_nop 1" :: "v"(voff), "v"(v), "s"(sbase), "i"(OFF) : "memory");
}
__device__ __forceinline__ void glds4(unsigned m0v, unsigned voff, const char* sb) {
    asm volatile("s_mov_b32 m0, %0\n\ts_nop 0\n\t"
                 "global_load_lds_dwordx4 %1, %2 sc0 sc1\n\t"
                 "global_load_lds_dwordx4 %1, %2 offset:1024 sc0 sc1\n\t"
                 "global_load_lds_dwordx4 %1, %2 offset:2048 sc0 sc1\n\t"
                 "global_load_lds_dwordx4 %1, %2 offset:3072 sc0 sc1" :: "s"(m0v), "v"(voff), "s"(sb) : "memory");
}
__device__ __forceinline__ void glds1(unsigned m0v, unsigned voff, const char* sb) {
    asm volatile("s_mov_b32 m0, %0\n\ts_nop 0\n\tglobal_load_lds_dwordx4 %1, %2 sc0 sc1" :: "s"(m0v), "v"(voff), "s"(sb) : "memory");
}
__device__ __forceinline__ void wait_vm0() { asm volatile("s_waitcnt vmcnt(0)" ::: "memory"); }

__device__ __forceinline__ bool timed_out(unsigned* tmo, unsigned call, u64 t0, unsigned& spins) {
    if ((++spins & 63u) != 0u) return false;
    if (rfl(__hip_atomic_load(tmo, __ATOMIC_RELAXED, __HIP_MEMORY_SCOPE_AGENT)) == call) return true;
    if (__builtin_amdgcn_s_memrealtime() - t0 > DEADLINE) {
        __hip_atomic_store(tmo, call, __ATOMIC_RELAXED, __HIP_MEMORY_SCOPE_AGENT);
        return true;
    }
    return false;
}

#define LDS_FENCE() asm volatile("" ::: "memory")
template <int CTRL, int ROWMASK> __device__ __forceinline__ unsigned dpp0(unsigned v) {
    return (unsigned)__builtin_amdgcn_update_dpp(0, (int)v, CTRL, ROWMASK, 0xf, false);
}
__device__ __forceinline__ unsigned wave_scan(unsigned v) {
    v += dpp0<0x111, 0xf>(v); v += dpp0<0x112, 0xf>(v); v += dpp0<0x114, 0xf>(v); v += dpp0<0x118, 0xf>(v);
    v += dpp0<0x142, 0xa>(v); v += dpp0<0x143, 0xc>(v);
    return v;
}
__device__ __forceinline__ unsigned lane_rank(u64 m) {
    return __builtin_amdgcn_mbcnt_hi((unsigned)(m >> 32), __builtin_amdgcn_mbcnt_lo((unsigned)m, 0u));
}
__device__ __forceinline__ void build_lut(LAS char* lds, bool dec, int tid) {
    if (tid < 256) {
        u64 e = 0x0c0c0c0c0c0c0c0cull;
        unsigned t = 0;
#pragma unroll
        for (unsigned j = 0; j < 8; ++j)
            if ((tid >> j) & 1) {
                const unsigned slot = (t & 1u) * 4u + (t >> 1);
                const unsigned at = dec ? j : slot, val = dec ? slot : j;
                e = (e & ~(0xffull << (8 * at))) | ((u64)val << (8 * at));
                ++t;
            }
        *(LAS u64*)(lds + LDS_LUT + tid * 8) = e;
    }
}
__device__ __forceinline__ u32x2 lut_at(LAS char* lds, unsigned f) {
    return *(LAS const u32x2*)(lds + __builtin_amdgcn_udot4(f, 0x40201008u, (unsigned)LDS_LUT, false));
}
__device__ __forceinline__ unsigned shuffle8(unsigned lo, unsigned hi, u32x2 sel) {
    return bfi(0x0f0f0f0fu, perm(hi, lo, sel[0]), perm(hi, lo, sel[1]) << 4);
}
__device__ __forceinline__ unsigned half16(unsigned v, int u) { return (u & 1) ? v >> 16 : v & 0xffffu; }
__device__ __forceinline__ unsigned b_start(unsigned n1) { return n1 <= 512u ? ((n1 + 15u) >> 4) << 5 : ((n1 - 249u) >> 3) << 5; }
__device__ __forceinline__ void lds_or(LAS char* p, unsigned v) {
    __hip_atomic_fetch_or((LAS unsigned*)p, v, __ATOMIC_RELAXED, __HIP_MEMORY_SCOPE_WAVEFRONT);
}
__device__ __forceinline__ unsigned bit_addr(LAS char* p) { return (unsigned)(size_t)p * 8u; }
__device__ __forceinline__ void put_bits(unsigned pos, unsigned v) {
    const u64 sh = (u64)v << (pos & 31u);
    LAS char* p = (LAS char*)(size_t)((pos >> 3) & ~3u);
    lds_or(p, (unsigned)sh); lds_or(p + 4, (unsigned)(sh >> 32));
}
__device__ __forceinline__ unsigned get_bits(unsigned pos) {
    LAS const unsigned* p = (LAS const unsigned*)(size_t)((pos >> 3) & ~3u);
    return __builtin_amdgcn_alignbit(p[1], p[0], pos);
}

template <int NU, class Get, class Mid>
__device__ __forceinline__ void pack_store(Get get, char* pk, const unsigned (&g)[NU], LAS char* lds, LAS char* scr, int lane,
                                           unsigned (&info)[NU], Mid mid) {
    constexpr int NS = (NU + 1) / 2;
    const unsigned voff = lane * 16;
#pragma unroll
    for (int u = 0; u < NU; ++u) *(LAS u32x4*)(scr + u * SCR_UNIT + voff) = (u32x4){0u, 0u, 0u, 0u};
    unsigned D[NU][8], pc[NU][3], cnt[NS], x16[NU], top = 0, top4 = 0;
    u32x2 sel[NU][4];
#pragma unroll
    for (int u = 0; u < NU; ++u) {
        u32x4 d[4], S[2];
        unsigned E[8], C[4], c = 0;
        get(u, d);
#pragma unroll
        for (int hf = 0; hf < 2; ++hf)
#pragma unroll
            for (int a = 0; a < 4; ++a) {
                const unsigned p = __builtin_amdgcn_alignbit(d[2 * hf][a], d[2 * hf][a], 31);
                const unsigned r = __builtin_amdgcn_alignbit(d[2 * hf + 1][a], d[2 * hf + 1][a], 31);
                S[hf][a] = perm(r, p, 0x06040200u);
                E[4 * hf + a] = perm(r, p, 0x07050301u);
            }
        char* pair = pk + (size_t)(g[u] >> 1) * PAIRB;
        const char* fix = pair + (g[u] & 1u) * FIXB;
        if (u == 0) {
            unsigned sum = __builtin_amdgcn_sad_u8(E[0], 0u, __builtin_amdgcn_sad_u8(E[2], 0u, __builtin_amdgcn_sad_u8(E[5], 0u, __builtin_amdgcn_sad_u8(E[7], 0u, 0u))));
            sum = __builtin_amdgcn_readlane(wave_scan(sum), 63);
            top = min(((sum + 983u) >> 10) + 2u, 255u);
            top4 = top * 0x01010101u;
        }
        st_wt<0>(fix, voff, S[0]);
#pragma unroll
        for (int a = 0; a < 4; ++a) {
            D[u][a] = top4 - E[a]; D[u][4 + a] = top4 - E[4 + a];
            unsigned n = bfi(0x0f0f0f0fu, D[u][a], D[u][4 + a] << 4);
            asm("" : "+v"(n));
            unsigned o = n | (n >> 1);
            asm("" : "+v"(o));
            const unsigned f = (~o | (o >> 2)) & 0x11111111u;
            unsigned f3 = (f << 1) + f;
            asm("" : "+v"(f3));
            C[a] = n & 0x33333333u & ~f3;
            c += __builtin_popcount(f);
            if (a < 3) pc[u][a] = c;
            sel[u][a] = lut_at(lds, f);
            if (a == 1) st_wt<1024>(fix, voff, S[1]);
        }
        cnt[u >> 1] = (u & 1) ? cnt[u >> 1] | c << 16 : c;
        const u32x2 cc = {C[0] | (C[1] << 2), C[2] | (C[3] << 2)};
        asm volatile("global_store_dwordx2 %0, %1, %2 offset:2048 sc0 sc1" :: "v"(lane * 8), "v"(cc), "s"(fix) : "memory");
        info[u] = top; x16[u] = 0;
        const u64 em = __builtin_amdgcn_ballot_w64(((D[u][0] | D[u][1] | D[u][2] | D[u][3] | D[u][4] | D[u][5] | D[u][6] | D[u][7]) & 0xf0f0f0f0u) != 0u);
        if (__builtin_expect(em != 0ull, 0)) {
            u32x4 X;
#pragma unroll
            for (int a = 0; a < 4; ++a) X[a] = bfi(0xf0f0f0f0u, D[u][a], D[u][4 + a] >> 4);
            const unsigned l = __builtin_ctzll(em);
            const unsigned x0 = __builtin_amdgcn_readlane(X[0], l), x1 = __builtin_amdgcn_readlane(X[1], l);
            const unsigned x2 = __builtin_amdgcn_readlane(X[2], l), x3 = __builtin_amdgcn_readlane(X[3], l);
            const unsigned any = x0 | x1 | x2 | x3, nz = (x0 != 0u) + (x1 != 0u) + (x2 != 0u) + (x3 != 0u);
            const unsigned nib = __builtin_ctz(any) >> 2, val = any >> (4u * nib);
            if ((em & (em - 1ull)) == 0ull && nz == 1u && val < 16u) {
                info[u] |= (l | (x0 ? 0u : x1 ? 1u : x2 ? 2u : 3u) << 6 | nib << 8 | val << 11) << 17;
            } else {
                info[u] |= 1u << 16;
                x16[u] = 1u + __builtin_popcountll(em);
                const unsigned k = 1u + lane_rank(em);
                if (lane == 0) st_wt<0>(pair, (g[u] & 1u) ? VMID : VMID - 16, (u32x4){(unsigned)em, (unsigned)(em >> 32), 0u, 0u});
                if ((em >> lane) & 1ull) st_wt<0>(pair, (g[u] & 1u) ? VMID + 16u * k : VMID - 16 - 16u * k, X);
            }
        }
    }
    unsigned excl[NS], tot[NS];
#pragma unroll
    for (int k = 0; k < NS; ++k) excl[k] = wave_scan(cnt[k]);
#pragma unroll
    for (int k = 0; k < NS; ++k) { tot[k] = __builtin_amdgcn_readlane(excl[k], 63); excl[k] -= cnt[k]; }
#pragma unroll
    for (int u = 0; u < NU; ++u)
        if (__builtin_expect(half16(tot[u >> 1], u) > 1400u, 0) && lane < 24) *(LAS u32x4*)(scr + u * SCR_UNIT + 1024 + voff) = (u32x4){0u, 0u, 0u, 0u};
    LDS_FENCE();
#pragma unroll
    for (int u = 0; u < NU; ++u) {
        unsigned pos = half16(excl[u >> 1], u) * 4u + bit_addr(scr + u * SCR_UNIT);
        asm("" : "+v"(pos));
#pragma unroll
        for (int a = 0; a < 4; ++a) put_bits(a ? pos + pc[u][a - 1] * 4u : pos, shuffle8(D[u][a], D[u][4 + a], sel[u][a]));
    }
    LDS_FENCE();
    mid();
    unsigned w[NU], c2n[NS];
    u32x2 s2[NU];
#pragma unroll
    for (int u = 0; u < NU; ++u) w[u] = *(LAS const unsigned*)(scr + u * SCR_UNIT + lane * 4);
    LDS_FENCE();
#pragma unroll
    for (int u = 0; u < NU; ++u) {
        const unsigned e = ((w[u] >> 1) | (w[u] >> 3)) & 0x11111111u;
        const unsigned c2 = (w[u] & 0x55555555u & ~(e << 2)) | e;
        s2[u] = lut_at(lds, e);
        const unsigned c = __builtin_popcount(e);
        c2n[u >> 1] = (u & 1) ? c2n[u >> 1] | c << 16 : c;
        *(LAS unsigned short*)(scr + u * SCR_UNIT + 128 + lane * 2) = (unsigned short)(c2 | (c2 >> 15));
    }
    unsigned in2[NS];
#pragma unroll
    for (int k = 0; k < NS; ++k) in2[k] = wave_scan(c2n[k]);
    LDS_FENCE();
#pragma unroll
    for (int u = 0; u < NU; ++u)
        put_bits(bit_addr(scr + u * SCR_UNIT + 128) + b_start(half16(tot[u >> 1], u)) + half16(in2[u >> 1] - c2n[u >> 1], u) * 4u,
                 shuffle8(w[u], w[u] >> 4, s2[u]));
    LDS_FENCE();
    u32x4 vv[NU];
#pragma unroll
    for (int u = 0; u < NU; ++u) vv[u] = *(LAS const u32x4*)(scr + u * SCR_UNIT + 128 + voff);
#pragma unroll
    for (int u = 0; u < NU; ++u) {
        const unsigned c16 = (b_start(half16(tot[u >> 1], u)) + half16(__builtin_amdgcn_readlane(in2[u >> 1], 63), u) * 4u + 127u) >> 7;
        const char* vg = pk + (size_t)(g[u] >> 1) * PAIRB + VMID + ((g[u] & 1u) ? 16 * (int)x16[u] : -16 * (int)(x16[u] + c16));
        if ((unsigned)lane < c16) st_wt<0>(vg, voff, vv[u]);
        if (__builtin_expect(c16 > 64u, 0) && 64u + lane < c16) st_wt<1024>(vg, voff, *(LAS const u32x4*)(scr + u * SCR_UNIT + 1152 + voff));
        info[u] |= (x16[u] + c16) << 8;
    }
    LDS_FENCE();
}

template <class Mid>
__device__ __forceinline__ void unpack2(const u32x4 (&p)[2][2], const u32x2 (&cc)[2], const unsigned (&info)[2], unsigned la, LAS char* lds,
                                        LAS char* T, LAS char* M, int lane, u32x4 (&o)[2][4], Mid mid) {
    u32x4 X[2] = {{0u, 0u, 0u, 0u}, {0u, 0u, 0u, 0u}};
    LAS char* A[2] = {M - 16u * la, M};
    const bool anyx = ((info[0] | info[1]) >> 16) != 0u;
    if (__builtin_expect(anyx, 0)) {
#pragma unroll
        for (int u = 0; u < 2; ++u) {
            if (info[u] & 0x10000u) {
                const u32x2 mv = *(LAS const u32x2*)(u ? M : M - 16);
                const u64 em = (u64)rfl(mv[0]) | (u64)rfl(mv[1]) << 32;
                const unsigned k = 1u + lane_rank(em);
                if ((em >> lane) & 1ull) X[u] = *(LAS const u32x4*)(u ? M + 16u * k : M - 16 - 16u * k);
                if (u) A[1] = M + 16u * (1u + __builtin_popcountll(em));
            } else {
                const unsigned xv = (info[u] >> 28) << (((info[u] >> 25) & 7u) * 4u), xa = (info[u] >> 23) & 3u;
                const bool me = (unsigned)lane == ((info[u] >> 17) & 63u);
#pragma unroll
                for (unsigned a = 0; a < 4; ++a) X[u][a] = me && xa == a ? xv : 0u;
            }
        }
    }
    unsigned z[2], rr[2];
#pragma unroll
    for (int u = 0; u < 2; ++u) {
        z[u] = *(LAS const unsigned short*)(A[u] + lane * 2);
        rr[u] = *(LAS const unsigned*)(A[u] + 128 + lane * 4);
    }
    unsigned pc[2][3], cnt = 0;
    u32x2 sel[2][4];
#pragma unroll
    for (int u = 0; u < 2; ++u) {
        unsigned c = 0;
#pragma unroll
        for (int k = 0; k < 2; ++k) {
            const unsigned h = cc[u][k] >> 1;
            const unsigned f0 = ~(cc[u][k] | h) & 0x11111111u, f1 = ~(cc[u][k] | h) & 0x44444444u;
            c += __builtin_popcount(f0);
            pc[u][2 * k] = c;
            c += __builtin_popcount(f1);
            if (k == 0) pc[u][1] = c;
            sel[u][2 * k] = lut_at(lds, f0);
            sel[u][2 * k + 1] = *(LAS const u32x2*)(lds + __builtin_amdgcn_udot4(f1, 0x10080402u, (unsigned)LDS_LUT, false));
        }
        cnt |= c << (16 * u);
    }
    unsigned c2[2], f2[2], c2n = 0;
    u32x2 s2[2];
#pragma unroll
    for (int u = 0; u < 2; ++u) {
        c2[u] = (z[u] | (z[u] << 15)) & 0x55555555u;
        asm("" : "+v"(c2[u]));
        f2[u] = c2[u] & ~(c2[u] >> 2) & 0x11111111u;
        s2[u] = lut_at(lds, f2[u]);
        c2n |= (unsigned)__builtin_popcount(f2[u]) << (16 * u);
        *(LAS unsigned*)(T + u * SCR_T + 256 + lane * 4) = rr[u];
    }
    const unsigned incl = wave_scan(cnt), in2 = wave_scan(c2n), tot = __builtin_amdgcn_readlane(incl, 63);
    unsigned wv[2];
#pragma unroll
    for (int u = 0; u < 2; ++u) wv[u] = get_bits(bit_addr(A[u]) + b_start(half16(tot, u)) + half16(in2 - c2n, u) * 4u);
#pragma unroll
    for (int u = 0; u < 2; ++u) {
        *(LAS unsigned*)(T + u * SCR_T + lane * 4) = (c2[u] ^ f2[u]) | shuffle8(wv[u], wv[u] >> 4, s2[u]);
        if (__builtin_expect(half16(tot, u) > 1024u, 0)) *(LAS u32x2*)(T + u * SCR_T + 512 + lane * 8) = *(LAS const u32x2*)(A[u] + 384 + lane * 8);
    }
    LDS_FENCE();
    unsigned w[2][4];
#pragma unroll
    for (int u = 0; u < 2; ++u) {
        unsigned pos = half16(incl - cnt, u) * 4u + bit_addr(T + u * SCR_T);
        asm("" : "+v"(pos));
#pragma unroll
        for (int a = 0; a < 4; ++a) w[u][a] = get_bits(a ? pos + pc[u][a - 1] * 4u : pos);
    }
    LDS_FENCE();
    mid();
    unsigned e[2][8];
#pragma unroll
    for (int u = 0; u < 2; ++u) {
        const unsigned top4 = (info[u] & 0xffu) * 0x01010101u;
#pragma unroll
        for (int a = 0; a < 4; ++a) {
            const unsigned lo = w[u][a] & 0x0f0f0f0fu, hi = (w[u][a] >> 4) & 0x0f0f0f0fu;
            e[u][a] = top4 - (perm(hi, lo, sel[u][a][0]) | ((cc[u][a >> 1] >> (2 * (a & 1))) & 0x03030303u));
            e[u][4 + a] = top4 - (perm(hi, lo, sel[u][a][1]) | ((cc[u][a >> 1] >> (2 * (a & 1) + 4)) & 0x03030303u));
        }
    }
    if (__builtin_expect(anyx, 0)) {
#pragma unroll
        for (int u = 0; u < 2; ++u)
#pragma unroll
            for (int a = 0; a < 4; ++a) { e[u][a] -= X[u][a] & 0xf0f0f0f0u; e[u][4 + a] -= (X[u][a] << 4) & 0xf0f0f0f0u; }
    }
#pragma unroll
    for (int u = 0; u < 2; ++u)
#pragma unroll
        for (int a = 0; a < 4; ++a) {
            const unsigned q0 = perm(e[u][a], p[u][0][a], 0x05010400u), q1 = perm(e[u][a], p[u][0][a], 0x07030602u);
            const unsigned q2 = perm(e[u][4 + a], p[u][1][a], 0x05010400u), q3 = perm(e[u][4 + a], p[u][1][a], 0x07030602u);
            o[u][0][a] = __builtin_amdgcn_alignbit(q0, q0, 1); o[u][1][a] = __builtin_amdgcn_alignbit(q1, q1, 1);
            o[u][2][a] = __builtin_amdgcn_alignbit(q2, q2, 1); o[u][3][a] = __builtin_amdgcn_alignbit(q3, q3, 1);
        }
}
__device__ __forceinline__ void store_stage(const char* dst, unsigned voff, const u32x4 (&o)[4]) {
    st_wt<0>(dst, voff, o[0]); st_wt<1024>(dst, voff, o[1]); st_wt<2048>(dst, voff, o[2]); st_wt<3072>(dst, voff, o[3]);
}
__device__ __forceinline__ void rows_of(LAS char* buf, int lane, u32x4 (&o)[4]) {
    LAS char* wp = buf + (lane & 15) * 128;
    const unsigned x = (((lane >> 3) & 6) ^ (lane & 7)) * 16;
    LDS_FENCE();
    *(LAS u32x4*)(wp + x) = o[0]; *(LAS u32x4*)(wp + (x ^ 16)) = o[1];
    *(LAS u32x4*)(wp + 2048 + x) = o[2]; *(LAS u32x4*)(wp + 2048 + (x ^ 16)) = o[3];
    LDS_FENCE();
    LAS const char* rp = buf + (lane >> 3) * 128 + (((lane >> 3) ^ lane) & 7) * 16;
#pragma unroll
    for (int c = 0; c < 4; ++c) o[c] = *(LAS const u32x4*)(rp + c * 1024);
    LDS_FENCE();
}
__device__ __forceinline__ void store_out(const char* dst, int lane, const u32x4 (&o)[4]) {
    const unsigned voff = (lane >> 3) * 65536 + (lane & 7) * 16;
#pragma unroll
    for (int c = 0; c < 4; ++c) st_wt<0>(dst + c * (8 * 65536), voff, o[c]);
}

__device__ __forceinline__ void pack_piece(const Params& P, char* ws, const char* px, const char* pw, LAS char* lds, int round, unsigned e,
                                           unsigned inbox, unsigned cntv, int lane, int wid) {
    LAS unsigned* lrec = (LAS unsigned*)(lds + LDS_REC) + 32 + (round % 3) * 8;
    const int li = lane & 15, lg = lane >> 4;
    unsigned* ready = (unsigned*)(ws + OFF_READY);
    const int h = (e >> 3) & 15, idx = (e & 7) * 8 + wid, id = h * 64 + idx;
    const unsigned kind = e >> 7;
    const bool isx = kind == 0u || kind == 9u;
    const unsigned dest = (kind - 1u) & 7u;
    const bool own = kind == 9u || kind == (unsigned)P.rank + 1u;
    const unsigned rdest = isx ? (unsigned)lane >> 3 : dest;
    char* peer = P.ws[rdest];
    u32x4 d[4];
    u32x2 v[8];
    if (isx) {
        const char* src = px + (size_t)(64 * idx + (lane >> 2)) * 1024 + h * 64 + ((lane & 3) ^ a_swz(lg)) * 16;
#pragma unroll
        for (int c = 0; c < 4; ++c) {
            const u32x4 t = *(const u32x4*)(src + 16 * c * 1024);
            v[2 * c] = (u32x2){t[0], t[1]}; v[2 * c + 1] = (u32x2){t[2], t[3]};
        }
    } else {
        const char* src = pw + (size_t)(32 * h + 8 * lg) * 65536 + (size_t)(dest * 4096 + 64 * idx + 4 * li) * 2;
#pragma unroll
        for (int j = 0; j < 8; ++j) v[j] = *(const u32x2*)(src + (size_t)j * 65536);
    }
    __builtin_amdgcn_sched_barrier(0);
    if (round == 0) {
        build_lut(lds, false, threadIdx.x);
        if (threadIdx.x >= 448) ((LAS unsigned*)(lds + LDS_FLG))[(threadIdx.x - 448) * 32 + 16] = 0u;
        asm volatile("s_waitcnt lgkmcnt(0)" ::: "memory");
        __builtin_amdgcn_s_barrier();
        asm volatile("" ::: "memory");
    }
    if (isx) {
#pragma unroll
        for (int c = 0; c < 4; ++c) d[c] = (u32x4){v[2 * c][0], v[2 * c][1], v[2 * c + 1][0], v[2 * c + 1][1]};
    } else {
#pragma unroll
        for (int hf = 0; hf < 2; ++hf)
#pragma unroll
            for (int a = 0; a < 4; ++a) {
                d[2 * hf][a] = perm(v[2 * a + 1][hf], v[2 * a][hf], 0x05040100u);
                d[2 * hf + 1][a] = perm(v[2 * a + 1][hf], v[2 * a][hf], 0x07060302u);
            }
    }
    asm volatile("" :: "v"(peer));
    const unsigned call = rfl(cntv) + 1u, par = call & 1u;
    if (own) {
        store_stage(ws + (isx ? OFF_A : OFF_B) + (size_t)(h * 256 + 4 * idx) * 1024, lane * 16, d);
        wait_vm0();
        if (lane == 0) __hip_atomic_store(ready + ((isx ? 0 : 64) + idx) * NSLOT + h, call, __ATOMIC_RELAXED, __HIP_MEMORY_SCOPE_SYSTEM);
        return;
    }
    const unsigned g[1] = {isx ? id : 1024u + dest * 1024u + (inbox ? P.rank * 128u + inbox * 8u - 8u + wid : id)};
    unsigned r[1];
    pack_store<1>([&](int, u32x4 (&o)[4]) { o[0] = d[0]; o[1] = d[1]; o[2] = d[2]; o[3] = d[3]; }, (inbox ? P.ws[dest] : ws) + OFF_PK + par * PK_PAR, g, lds, lds + wid * SCR_WAVE, lane, r, [] {});
    if (lane == 0) lrec[wid] = r[0];
    wait_vm0();
    asm volatile("s_waitcnt lgkmcnt(0)" ::: "memory");
    __builtin_amdgcn_s_barrier();
    asm volatile("" ::: "memory");
    if (wid == 0 && (isx || lane < 8) && rdest != (unsigned)P.rank) {
        u64* rp = (u64*)(peer + OFF_REC + par * REC_PAR) + (P.rank * 3 + (isx ? 0 : 1)) * 1024 + h * 64 + (e & 7) * 8 + (lane & 7);
        __hip_atomic_store(rp, ((u64)call << 42) | lrec[lane & 7], __ATOMIC_RELAXED, __HIP_MEMORY_SCOPE_SYSTEM);
    }
}

__device__ __forceinline__ void pack_bulk(const Params& P, char* ws, const char* pw, LAS char* lds, unsigned rank, unsigned call, int lane, int wid) {
    unsigned i = P.dstart + 2u * rank, round = 0;
    const int li = lane & 15, lg = lane >> 4;
    const unsigned par = call & 1u;
    const unsigned lsrc = (unsigned)(8 * lg) * 65536u + (unsigned)(64 * wid + 4 * li) * 2u;
    u32x2 v[2][8];
    unsigned en[2];
    auto load = [&](unsigned i, int u) {
        const unsigned k = (i + u) / 7u, dest = (P.rank + 1u + (i + u) % 7u) & 7u, hp = (P.worder[k >> 2] >> ((k & 3u) * 8u)) & 127u;
        en[u] = dest << 7 | hp;
        const char* src = pw + (size_t)(hp >> 3) * (32 * 65536) + (dest * 4096u + (hp & 7u) * 512u) * 2u;
#pragma unroll
        for (int j = 0; j < 8; ++j) v[u][j] = *(const u32x2*)(src + (size_t)j * 65536 + lsrc);
    };
    load(i, 0);
    load(i, 1);
    __builtin_amdgcn_s_waitcnt(0x0F70);
    for (LAS unsigned* lrec = (LAS unsigned*)(lds + LDS_FLG);; lrec += 32) {
        const unsigned el = (lane & 8) ? en[1] : en[0];
        char* peer = P.ws[el >> 7];
        const unsigned g[2] = {1024u + en[0] * 8u + wid, 1024u + en[1] * 8u + wid};
        u32x4 d[2][4];
#pragma unroll
        for (int u = 0; u < 2; ++u)
#pragma unroll
            for (int hf = 0; hf < 2; ++hf)
#pragma unroll
                for (int a = 0; a < 4; ++a) {
                    d[u][2 * hf][a] = perm(v[u][2 * a + 1][hf], v[u][2 * a][hf], 0x05040100u);
                    d[u][2 * hf + 1][a] = perm(v[u][2 * a + 1][hf], v[u][2 * a][hf], 0x07060302u);
                }
        asm volatile("" : "+v"(d[0][0]), "+v"(d[0][1]), "+v"(d[0][2]), "+v"(d[0][3]), "+v"(d[1][0]), "+v"(d[1][1]), "+v"(d[1][2]), "+v"(d[1][3]));
        i = rank < P.dcount[round + 1u] ? i + 2u * P.dcount[round] : 896u;
        ++round;
        if (i < 896u) load(i, 0);
        __builtin_amdgcn_sched_barrier(0);
        unsigned r[2];
        pack_store<2>([&](int u, u32x4 (&o)[4]) { o[0] = d[u][0]; o[1] = d[u][1]; o[2] = d[u][2]; o[3] = d[u][3]; },
                      ws + OFF_PK + par * PK_PAR, g, lds, lds + wid * SCR_WAVE, lane, r, [&] { if (i < 896u) load(i, 1); });
        if (lane < 2) lrec[lane * 8 + wid] = lane ? r[1] : r[0];
        wait_vm0();
        __builtin_amdgcn_s_waitcnt(0x0F70);
        unsigned arrived = 0;
        if (lane == 0) arrived = __hip_atomic_fetch_add(lrec + 16, 1u, __ATOMIC_RELAXED, __HIP_MEMORY_SCOPE_WORKGROUP);
        LDS_FENCE();
        if (rfl(arrived) == 7u && lane < 16) {
            u64* rp = (u64*)(peer + OFF_REC + par * REC_PAR) + (P.rank * 3 + 1) * 1024 + (el & 127u) * 8u + (lane & 7);
            __hip_atomic_store(rp, ((u64)call << 42) | lrec[lane], __ATOMIC_RELAXED, __HIP_MEMORY_SCOPE_SYSTEM);
        }
        if (i >= 896u) break;
    }
}

constexpr int PSLOT = 15232;
constexpr int PSLOT_M = 2 * FIXB + VCAP;
constexpr int PSLOT_T = PSLOT - 4096;
constexpr int LDS_Q = LDS_REC;
constexpr int LDS_QTAIL = LDS_Q + 256;
constexpr int LDS_FULL = LDS_QTAIL + 16;

__device__ __forceinline__ unsigned lds_poll(LAS char* p) {
    return rfl(__hip_atomic_load((LAS unsigned*)p, __ATOMIC_RELAXED, __HIP_MEMORY_SCOPE_WORKGROUP));
}

template <int N>
__device__ __forceinline__ void fetch_pair(u32x4& next, unsigned naddr, unsigned v0, unsigned v1, unsigned v2, const char* pk, u64 vmask,
                                           unsigned slot, unsigned da, unsigned dv) {
    static_assert(N >= 7 && N <= 12, "window");
#define FW(j) "s_waitcnt vmcnt(%12)\n\t.if %13 == " #j "\n\tds_write_b32 %10, %11\n\t.endif\n\t"
    asm volatile("s_mov_b32 m0, %7\n\t"
                 FW(0) "global_load_lds_dwordx4 %2, %5 offset:2048 sc0 sc1\n\t"
                 FW(1) "global_load_lds_dwordx4 %2, %5 sc0 sc1\n\t"
                 FW(2) "global_load_lds_dwordx4 %2, %5 offset:1024 sc0 sc1\n\t"
                 FW(3) "global_load_lds_dwordx4 %2, %5 offset:3072 sc0 sc1\n\t"
                 "s_mov_b32 m0, %9\n\t"
                 FW(4) "s_mov_b64 exec, %6\n\t"
                 "global_load_lds_dwordx4 %4, %5 sc0 sc1\n\t"
                 "s_mov_b64 exec, -1\n\t"
                 "s_mov_b32 m0, %8\n\t"
                 "ds_read_b128 %0, %1\n\t"
                 FW(5) "global_load_lds_dwordx4 %3, %5 sc0 sc1\n\t"
                 "s_waitcnt lgkmcnt(0)"
                 : "=&v"(next)
                 : "v"(naddr), "v"(v0), "v"(v1), "v"(v2), "s"(pk), "s"(vmask), "s"(slot), "s"(slot + 4096u), "s"(slot + (PSLOT_M - 512)),
                   "v"(da), "v"(dv), "i"(N - 1), "i"((N - 1) % 6)
                 : "memory");
#undef FW
}

__device__ __forceinline__ void fetch_link(const Params& P, LAS char* lds, int q, unsigned call, int lane) {
    const unsigned p = (unsigned)(P.rank + 1 + q) & 7u;
    const char* pkb = P.ws[p] + OFF_PK + (call & 1u) * PK_PAR;
    const unsigned ldsbase = (unsigned)(size_t)lds;
    const unsigned v0 = lane * 16, v1 = v0 + 4096, v2 = v0 + (VMID - 512);
    unsigned fa[2] = {0, 0}, fv[2] = {0, 0};
    if (blockIdx.x < 14) {
        const unsigned k = lane + (blockIdx.x / 7u) * 64u;
        __hip_atomic_store((unsigned*)(P.me + OFF_READY) + (8u * p + (k & 7u)) * NSLOT + 16u + 7u * (k >> 3) + q, call, __ATOMIC_RELAXED,
                           __HIP_MEMORY_SCOPE_SYSTEM);
    }
    __builtin_amdgcn_s_setprio(3);
    unsigned ended = 0;
    u32x4 ev = {0u, 0u, 0u, 0u};
    for (unsigned i = 0;; ++i) {
        while (__builtin_expect(rfl(ev[0]) != i + 1u, 0)) {
            ev = *(volatile LAS u32x4*)(lds + LDS_Q + (i & 15u) * 16u);
            if (rfl(ev[0]) == i + 1u) break;
            if (fa[1]) {
                asm volatile("s_waitcnt vmcnt(6)" ::: "memory"); *(volatile LAS unsigned*)(lds + fa[1]) = fv[1]; fa[1] = 0;
            } else if (fa[0]) {
                asm volatile("s_waitcnt vmcnt(0)" ::: "memory"); *(volatile LAS unsigned*)(lds + fa[0]) = fv[0]; fa[0] = 0;
            } else {
                if (ended == NDEC) return;
                __builtin_amdgcn_s_sleep(1);
            }
        }
        const unsigned w1 = rfl(ev[1]);
        if (__builtin_expect(w1 >> 31, 0)) { ++ended; continue; }
        const unsigned owner = w1 >> 24, slot = ldsbase + (owner - 1u) * PSLOT;
        const char* pk = pkb + (size_t)(w1 & 0x3fffu) * PAIRB;
        const u64 vmask = (u64)rfl(ev[2]) | (u64)rfl(ev[3]) << 32;
        const unsigned da = ldsbase + (fa[1] ? fa[1] : (unsigned)LDS_FULL), dv = fv[1];
        fetch_pair<NFLY>(ev, ldsbase + LDS_Q + ((i + 1u) & 15u) * 16u, v0, v1, v2, pk, vmask, slot, da, dv);
        fa[1] = fa[0]; fv[1] = fv[0];
        fa[0] = LDS_FULL + owner * 4u; fv[0] = i + 1u;
    }
}

struct Desc { unsigned g, dst, rdy, rec; };

__device__ __forceinline__ void decode_link(const Params& P, LAS char* lds, int q, int w, unsigned call, int lane, int wid) {
    char* ws = P.me;
    const unsigned par = call & 1u;
    const unsigned p = (unsigned)(P.rank + 1 + q) & 7u;
    unsigned* tmo = (unsigned*)(ws + OFF_TMO);
    const unsigned rec_base = (unsigned)(OFF_REC + par * REC_PAR) + p * 3072u * 8u;
    auto describe = [&](unsigned n) {
        Desc d;
        if (n < NIN) {
            unsigned e = rfl(*(LAS const unsigned short*)(lds + LDS_FLG + 2u * n));
            const unsigned c = (e >> 2) & 7u;
            if (e < 512u) e = (e & ~28u) | ((P.rank + c + (c > (unsigned)q)) & 7u) << 2;
            const unsigned type = e >> 9, h = (e >> 5) & 15u, idx = (e & 31u) * 2u, id = h * 64u + idx, slot = 16u + 7u * h + q;
            d.g = type ? 1024u + P.rank * 1024u + id : id;
            d.dst = (unsigned)(type ? OFF_B : OFF_A) + (slot * 256u + 4u * idx) * 1024u;
            d.rdy = (unsigned)OFF_READY + ((type * 64u + idx) * NSLOT + slot) * 4u;
            d.rec = rec_base + (type * 1024u + id) * 8u;
        } else {
            d.g = 9216u; d.dst = P.rank * 8192u; d.rdy = 0;
            d.rec = rec_base + (2048u + 2u * ((n - NIN) & 511u)) * 8u;
        }
        return d;
    };
    auto accept = [&](u64 v, Desc& d, unsigned& ia, unsigned& ib) {
        if (__builtin_amdgcn_ballot_w64((unsigned)(v >> 42) != (call & 0x3fffffu)) != 0ull) return false;
        ia = __builtin_amdgcn_readlane((unsigned)v, 0); ib = __builtin_amdgcn_readlane((unsigned)v, 1);
        if (!d.rdy) {
            const unsigned id = __builtin_amdgcn_readlane((unsigned)(v >> 32), 0) & 1022u;
            d.g = 9216u + P.rank * 1024u + id;
            d.dst = (id & 15u) * (32u * 65536u) + (p * 4096u + 64u * (id >> 4)) * 2u;
        }
        return true;
    };
    const char* pkb = P.ws[p] + OFF_PK + par * PK_PAR;
    const unsigned voff = lane * 16;
    const unsigned tk_base = (call - 1u) * (NTK - PW) - 2u * PW;
    LAS char* S = lds + (wid - 1) * PSLOT;
    LAS char* M = S + PSLOT_M;
    bool dead = false;
    unsigned dmask = 0;
    const unsigned vrows = (lane >> 3) * 65536u + (lane & 7) * 16u;
    const __amdgpu_buffer_rsrc_t orsrc = __builtin_amdgcn_make_buffer_rsrc(P.out, 0, 512 * 65536, 0x00020000);
    auto vlen = [](unsigned i) { return min((i >> 8) & 255u, (unsigned)VCAP / 16u); };
    auto post = [&](unsigned w1, unsigned ia, unsigned ib) {
        const unsigned la = vlen(ia), lb = vlen(ib);
        const unsigned mlo = la >= 32u ? ~0u : ~(~0u >> la), mhi = lb >= 32u ? ~0u : ((1u << lb) - 1u) | 1u;
        unsigned i = 0;
        asm volatile("s_waitcnt lgkmcnt(0)" ::: "memory");
        if (lane == 0) {
            i = __hip_atomic_fetch_add((LAS unsigned*)(lds + LDS_QTAIL), 1u, __ATOMIC_RELAXED, __HIP_MEMORY_SCOPE_WORKGROUP);
            *(volatile LAS u32x4*)(lds + LDS_Q + (i & 15u) * 16u) = (u32x4){i + 1u, w1, mlo, mhi};
        }
        return rfl(i) + 1u;
    };
    auto poll = [&](Desc& d, unsigned& ia, unsigned& ib) {
        const u64 t0 = __builtin_amdgcn_s_memrealtime();
        unsigned spins = 0;
        ia = ib = 0;
        while (!dead) {
            const u64 v = __hip_atomic_load((const u64*)(ws + d.rec) + (lane & 1), __ATOMIC_RELAXED, __HIP_MEMORY_SCOPE_SYSTEM);
            if (accept(v, d, ia, ib)) break;
            if (timed_out(tmo, call, t0, spins)) dead = true;
            __builtin_amdgcn_s_sleep(1);
        }
    };
    unsigned n1 = PW + w, ia, ib;
    Desc d0 = describe(w), d1 = describe(n1);
    poll(d0, ia, ib);
    unsigned want = post(d0.g >> 1 | wid << 24, ia, ib);
    for (;;) {
        const bool more = n1 < NTK;
        u32x4 own[8];
        const bool late = !d0.rdy && ((d0.dst >> 9) & 15u) >= LATE_COL;
        if (late) {
            const unsigned bit = ((d0.dst >> 8) & 30u) | (d0.dst >> 24);
            if (!((dmask >> bit) & 1u)) {
                const u64 t0 = __builtin_amdgcn_s_memrealtime();
                unsigned spins = 0;
                while (!dead) {
                    const unsigned v = __hip_atomic_load((const unsigned*)(ws + OFF_DFL) + q * 32 + (lane & 31), __ATOMIC_RELAXED, __HIP_MEMORY_SCOPE_SYSTEM);
                    dmask = (unsigned)__builtin_amdgcn_ballot_w64(v >= call);
                    if ((dmask >> bit) & 1u) break;
                    if (timed_out(tmo, call, t0, spins)) dead = true;
                    __builtin_amdgcn_s_sleep(1);
                }
            }
            asm volatile("" ::: "memory");
#pragma unroll
            for (int c = 0; c < 8; ++c) own[c] = __builtin_amdgcn_raw_buffer_load_b128(orsrc, vrows, d0.dst + c * (8u * 65536u), 17);
        }
        unsigned dv = 0;
        if (n1 + 2u * PW >= NIN + 32u * LATE_COL && dmask == 0u)
            dv = __hip_atomic_load((const unsigned*)(ws + OFF_DFL) + q * 32 + (lane & 31), __ATOMIC_RELAXED, __HIP_MEMORY_SCOPE_SYSTEM);
        const u64 rv = __hip_atomic_load((const u64*)(ws + d1.rec) + (lane & 1), __ATOMIC_RELAXED, __HIP_MEMORY_SCOPE_SYSTEM);
        while (lds_poll(lds + LDS_FULL + wid * 4) != want) __builtin_amdgcn_s_sleep(1);
        LDS_FENCE();
        const unsigned la = vlen(ia), lb = vlen(ib);
        if (la > 32u || lb > 32u) {
            const char* mid = pkb + (size_t)(d0.g >> 1) * PAIRB + VMID;
            for (int c = -3; c <= 3; ++c) {
                const int off = c * 1024 - 512 + (int)voff;
                if (c != 0 && off >= -16 * (int)la && off < 16 * (int)lb) {
                    u32x4 t;
                    asm volatile("global_load_dwordx4 %0, %1, off sc0 sc1\n\ts_waitcnt vmcnt(0)" : "=&v"(t) : "v"(mid + off) : "memory");
                    *(LAS u32x4*)(M + off) = t;
                }
            }
            LDS_FENCE();
        }
        u32x4 o[2][4];
        unsigned ja = 0, jb = 0;
        bool posted = false;
        {
            u32x4 pp[2][2];
            u32x2 cc[2];
#pragma unroll
            for (int u = 0; u < 2; ++u) {
                cc[u] = *(LAS const u32x2*)(S + u * FIXB + 2048 + lane * 8);
                pp[u][0] = *(LAS const u32x4*)(S + u * FIXB + voff);
                pp[u][1] = *(LAS const u32x4*)(S + u * FIXB + 1024 + voff);
            }
            const unsigned inf[2] = {ia, ib};
            unpack2(pp, cc, inf, la, lds, S + PSLOT_T, M, lane, o, [&] {
                asm volatile("" : "+v"(pp[0][0]), "+v"(pp[0][1]), "+v"(pp[1][0]), "+v"(pp[1][1]));
                if (more && accept(rv, d1, ja, jb)) {
                    want = post(d1.g >> 1 | wid << 24, ja, jb);
                    posted = true;
                }
            });
        }
        dmask |= (unsigned)__builtin_amdgcn_ballot_w64(dv >= call);
        if (!d0.rdy) {
            if (dead) {
#pragma unroll
                for (int c = 0; c < 8; ++c) o[c >> 2][c & 3] = (u32x4){0x7fc07fc0u, 0x7fc07fc0u, 0x7fc07fc0u, 0x7fc07fc0u};
            }
            rows_of(S + PSLOT_T, lane, o[0]);
            rows_of(S + PSLOT_T, lane, o[1]);
            if (late) {
#pragma unroll
                for (int c = 0; c < 8; ++c) o[c >> 2][c & 3] = add_bf16(o[c >> 2][c & 3], own[c]);
            }
        }
        unsigned tk;
        u64 rv2;
        {
            const unsigned step = d0.rdy ? 1024u : 8u * 65536u, v0 = d0.rdy ? voff : vrows;
            const char* b0 = (d0.rdy ? ws : P.out) + d0.dst;
            asm volatile("global_load_dwordx2 %1, %2, %5 sc0 sc1\n\t"
                         "s_mov_b64 exec, 1\n\tglobal_atomic_add %0, %3, %4, %5 sc0\n\ts_mov_b64 exec, -1\n\t"
                         "global_store_dwordx4 %8, %12, %6 sc0 sc1\n\tglobal_store_dwordx4 %9, %13, %6 sc0 sc1\n\t"
                         "global_store_dwordx4 %10, %14, %6 sc0 sc1\n\tglobal_store_dwordx4 %11, %15, %6 sc0 sc1\n\t"
                         "global_store_dwordx4 %8, %16, %7 sc0 sc1\n\tglobal_store_dwordx4 %9, %17, %7 sc0 sc1\n\t"
                         "global_store_dwordx4 %10, %18, %7 sc0 sc1\n\tglobal_store_dwordx4 %11, %19, %7 sc0 sc1\n\t"
                         "s_nop 1\n\ts_waitcnt vmcnt(8)"
                         : "=&v"(tk), "=&v"(rv2)
                         : "v"(d1.rec + (lane & 1) * 8u), "v"((unsigned)OFF_TKT + q * 128u), "v"(more ? 1u : 0u), "s"(ws), "s"(b0), "s"(b0 + 4u * step),
                           "v"(v0), "v"(v0 + step), "v"(v0 + 2u * step), "v"(v0 + 3u * step),
                           "v"(o[0][0]), "v"(o[0][1]), "v"(o[0][2]), "v"(o[0][3]), "v"(o[1][0]), "v"(o[1][1]), "v"(o[1][2]), "v"(o[1][3])
                         : "memory");
        }
        if (more && !posted && accept(rv2, d1, ja, jb)) {
            want = post(d1.g >> 1 | wid << 24, ja, jb);
            posted = true;
        }
        if (d0.rdy) {
            wait_vm0();
            if (lane < 2) __hip_atomic_store((unsigned*)(ws + d0.rdy) + lane * NSLOT, call, __ATOMIC_RELAXED, __HIP_MEMORY_SCOPE_SYSTEM);
        } else if (!late) {
            wait_vm0();
            if (lane == 0 && !dead)
                __hip_atomic_store((unsigned*)(ws + OFF_OFL) + q * 512 + ((d0.g >> 1) & 511u), call, __ATOMIC_RELAXED, __HIP_MEMORY_SCOPE_SYSTEM);
        }
        const unsigned n2 = rfl(tk) - tk_base;
        if (!more) break;
        if (!posted) {
            poll(d1, ja, jb);
            want = post(d1.g >> 1 | wid << 24, ja, jb);
        }
        ia = ja; ib = jb;
        d0 = d1;
        n1 = n2;
        d1 = describe(n1);
    }
    post(0x80000000u, 0u, 0u);
}

__device__ __forceinline__ void unbox(const Params& P, LAS char* lds, unsigned first, unsigned call, int lane, int wid) {
    char* ws = P.me;
    build_lut(lds + (LDS_LUT2 - LDS_LUT), true, threadIdx.x);
    asm volatile("s_waitcnt lgkmcnt(0)" ::: "memory");
    __builtin_amdgcn_s_barrier();
    asm volatile("" ::: "memory");
    const unsigned piece = first + (wid >> 2);
    if (piece >= 7u * NPUSH) return;
    const unsigned k = piece / 7u, q = piece % 7u, p = (P.rank + 1u + q) & 7u, par = call & 1u;
    const unsigned hp = (P.worder[(PUSHP + k) >> 2] >> (((PUSHP + k) & 3u) * 8u)) & 127u, idx = (hp & 7u) * 8u + (wid & 3) * 2u, slot = 16u + 7u * (hp >> 3) + q;
    const u64* rec = (const u64*)(ws + OFF_REC + par * REC_PAR) + (p * 3u + 1u) * 1024u + (hp >> 3) * 64u + idx + (lane & 1);
    unsigned inf[2] = {0u, 0u}, spins = 0;
    const u64 t0 = __builtin_amdgcn_s_memrealtime();
    for (;;) {
        const u64 v = __hip_atomic_load(rec, __ATOMIC_RELAXED, __HIP_MEMORY_SCOPE_SYSTEM);
        if (__builtin_amdgcn_ballot_w64((unsigned)(v >> 42) != (call & 0x3fffffu)) == 0ull) {
            inf[0] = __builtin_amdgcn_readlane((unsigned)v, 0); inf[1] = __builtin_amdgcn_readlane((unsigned)v, 1);
            break;
        }
        if (timed_out((unsigned*)(ws + OFF_TMO), call, t0, spins)) break;
        __builtin_amdgcn_s_sleep(2);
    }
    LAS char* S = lds + wid * PSLOT;
    const char* pk = ws + OFF_PK + par * PK_PAR + (size_t)(512u + P.rank * 512u + p * 64u + k * 4u + (wid & 3)) * PAIRB;
    const unsigned voff = lane * 16;
    glds4((unsigned)(size_t)S, voff, pk);
    glds4((unsigned)(size_t)S + 4096u, voff, pk + 4096);
    glds1((unsigned)(size_t)S + 8192u, voff, pk + 8192);
    glds1((unsigned)(size_t)S + 9216u, voff, pk + 9216);
    wait_vm0();
    u32x4 pp[2][2], o[2][4];
    u32x2 cc[2];
#pragma unroll
    for (int u = 0; u < 2; ++u) {
        cc[u] = *(LAS const u32x2*)(S + u * FIXB + 2048 + lane * 8);
        pp[u][0] = *(LAS const u32x4*)(S + u * FIXB + voff);
        pp[u][1] = *(LAS const u32x4*)(S + u * FIXB + 1024 + voff);
    }
    unpack2(pp, cc, inf, min((inf[0] >> 8) & 255u, (unsigned)VCAP / 16u), lds + (LDS_LUT2 - LDS_LUT), S + PSLOT_T, S + PSLOT_M, lane, o, [] {});
    store_stage(ws + OFF_B + (size_t)(slot * 256u + 4u * idx) * 1024, voff, o[0]);
    store_stage(ws + OFF_B + (size_t)(slot * 256u + 4u * idx) * 1024 + 4096, voff, o[1]);
    wait_vm0();
    if (lane < 2) __hip_atomic_store((unsigned*)(ws + OFF_READY) + (64u + idx + lane) * NSLOT + slot, call, __ATOMIC_RELAXED, __HIP_MEMORY_SCOPE_SYSTEM);
}

__device__ __forceinline__ void glds4w(unsigned m0v, unsigned voff, const char* sb) {
    constexpr int R = 8 * 65536 - 1024;
    asm volatile("s_mov_b32 m0, %0\n\ts_nop 0\n\t"
                 "global_load_lds_dwordx4 %1, %2\n\t"
                 "global_load_lds_dwordx4 %1, %3 offset:1024\n\t"
                 "global_load_lds_dwordx4 %1, %4 offset:2048\n\t"
                 "global_load_lds_dwordx4 %1, %5 offset:3072" :: "s"(m0v), "v"(voff), "s"(sb), "s"(sb + R), "s"(sb + 2 * R), "s"(sb + 3 * R) : "memory");
}
__device__ __forceinline__ bf16x8 read_tr(LAS const char* p) {
    const bf16x4 lo = __builtin_amdgcn_ds_read_tr16_b64_v4bf16((LAS bf16x4*)p), hi = __builtin_amdgcn_ds_read_tr16_b64_v4bf16((LAS bf16x4*)(p + 512));
    return __builtin_shufflevector(lo, hi, 0, 1, 2, 3, 4, 5, 6, 7);
}

template <int WR, bool BIG, bool OWN>
__device__ __forceinline__ void k_loop(f32x4 (&acc)[BIG ? 10 : 8][4], char* ws, LAS char* lds, int u0, int c0, const char* wsrc, int rank,
                                       unsigned call, int lane, int wid) {
    constexpr int NL = BIG ? 12 : 8;
    constexpr unsigned ns = OWN ? 16 : NSLOT;
    const unsigned q4 = ((u0 >> 3) - rank - 1) & 7;
    const unsigned steps = q4 < 7 ? 112 : ns, last = q4 == 6 ? ns - 2 : ns - 1;
    unsigned t = 0, tskip = q4 < 7 ? 16 + q4 : 999;
    constexpr bool has5 = BIG && !WR, isx = OWN ? !WR : WR;
    constexpr int RA = OWN ? RING_A2 : 4, RB = OWN ? 9 - RING_A2 : 4, RW = WR ? RB : RA;
    constexpr int RS = OWN ? 16384 : LSLOT, B0 = OWN ? RA * RS : 20480;
    unsigned* tmo = (unsigned*)(ws + OFF_TMO);
    const int wc = wid & 3;
    const unsigned ldsbase = (unsigned)(size_t)lds;
    const unsigned u1 = (isx ? u0 : c0) + wc;
    const char* g1 = OWN && WR ? wsrc + wc * 128 : ws + (isx ? OFF_A : OFF_B) + (size_t)u1 * 4096;
    const unsigned vw = (lane >> 3) * 65536 + ((lane & 7) ^ ((lane >> 4) & 1)) * 16;
    const char* g2 = ws + OFF_B + (size_t)(4 * c0 + 16 + wc) * 1024;
    const unsigned l1 = ldsbase + WR * B0 + wc * 4096, l2 = ldsbase + (16 + wc) * 1024;
    const unsigned voff = lane * 16;
    const unsigned vrow = (unsigned)OFF_READY + ((isx ? 0 : 64) + (lane >= 32 && has5 ? c0 + 4 : u1)) * (NSLOT * 4);
    const unsigned vsnap = vrow + (lane & 31) * 16;
    LAS char* snap = lds + LDS_FLG + wid * 1024;
    *(LAS u32x4*)(snap + voff) = (u32x4){0u, 0u, 0u, 0u};
    asm volatile("s_waitcnt lgkmcnt(0)" ::: "memory");
    unsigned hz = OWN && WR ? 4096u : 0u, look = 0;
#define ADVANCE(v) ({ const u64 m_ = __builtin_amdgcn_ballot_w64((v) >= call); \
    (unsigned)__builtin_ctzll(~(u64)((unsigned)m_ & (unsigned)(m_ >> 32))); })
#define ENSURE(t) do { if (__builtin_expect(hz <= (unsigned)(t), 0)) { const u64 t0_ = __builtin_amdgcn_s_memrealtime(); unsigned sp_ = 0; \
        __builtin_amdgcn_s_waitcnt(0xC07F); \
        for (;;) { \
            const unsigned a_ = __hip_atomic_load((const unsigned*)(ws + vrow) + min(hz + (lane & 31u), ns - 1u), __ATOMIC_RELAXED, __HIP_MEMORY_SCOPE_SYSTEM); \
            hz += ADVANCE(a_); \
            if (hz > (unsigned)(t)) break; \
            if (timed_out(tmo, call, t0_, sp_)) { hz = NSLOT; break; } \
            __builtin_amdgcn_s_sleep(2); } } } while (0)
#define STAGE(t, lo_) do { const size_t go_ = (size_t)(t) * 262144; \
    if (OWN && WR) glds4w(l1 + lo_, vw, g1 + 8 * go_); else glds4(l1 + lo_, voff, g1 + go_); if (has5) glds1(l2 + lo_, voff, g2 + go_); } while (0)
#define WAITV() asm volatile("s_waitcnt vmcnt(%0)" :: "i"((4 + has5) * (RW - 2)) : "memory")
#pragma unroll 1
    for (int i = 0; i < RW - 1; ++i, ++t) { ENSURE(t); STAGE(t, i * RS); }
    WAITV();
    __builtin_amdgcn_s_barrier();
    if (WR == 1) __builtin_amdgcn_s_barrier();
    const unsigned xoff = ((lane & 15) * 4 + ((lane >> 4) ^ a_swz((lane >> 2) & 3))) * 16;
    const unsigned bsw = ((lane >> 3) & 1) * 16;
    const unsigned loff = OWN ? xoff : voff;
    const unsigned soff = B0 + wc * 4096 + (OWN ? (lane >> 4) * 1024 + ((lane >> 2) & 3) * 128 + (lane & 3) * 32 : xoff);
    unsigned so = 0, sb = 0, sw = (RW - 1) * RS;
    for (unsigned s = 0; s < steps; ++s) {
        ENSURE(t);
        STAGE(t, sw);
        __builtin_amdgcn_sched_barrier(0);
        bf16x8 L[NL], S[4];
        LAS const char* pl = lds + loff + so;
        LAS const char* ps = lds + soff + (RA == RB ? so : sb);
#pragma unroll
        for (int n = 0; n < 4; ++n) S[n] = OWN ? read_tr(ps + (((n >> 1) * 16) ^ bsw) + (n & 1) * 8) : *(LAS const bf16x8*)(ps + n * 1024);
#pragma unroll
        for (int m = 0; m < NL; ++m) L[m] = *(LAS const bf16x8*)(pl + (m < 8 ? 8 * WR + m : 8 + m) * 1024);
        __builtin_amdgcn_sched_barrier(0);
        sw = RW == RA ? so : sb;
        so = so == (RA - 1) * RS ? 0u : so + RS;
        if (RA != RB) sb = sb == (RB - 1) * RS ? 0u : sb + RS;
        t += 1u;
        if (!OWN) { const bool hit = t == tskip; t += hit ? 1u : 0u; tskip += hit ? 7u : 0u; }
        t = min(t, last);
        asm volatile("" : "+s"(t), "+s"(tskip), "+s"(so));
        if (!OWN && __builtin_expect(t >= look, 0)) {
            const unsigned fv = *(volatile LAS const unsigned*)(snap + (lane >> 5) * 512 + min(hz + (lane & 31u), ns - 1u) * 4u);
            hz += ADVANCE(fv);
            glds1(ldsbase + LDS_FLG + wid * 1024, vsnap, ws);
            look = max(t + 4u + LOOKAHEAD, hz) - LOOKAHEAD;
        }
        WAITV();
        __builtin_amdgcn_s_waitcnt(0xC07F);
        __builtin_amdgcn_sched_barrier(0);
        __builtin_amdgcn_s_barrier();
        __builtin_amdgcn_sched_barrier(0);
#pragma unroll
        for (int m = 0; m < 8; ++m)
#pragma unroll
            for (int k = 0; k < 4; ++k) {
                const int n = ((m ^ WR) & 1) ? 3 - k : k;
                acc[m][n] = OWN ? __builtin_amdgcn_mfma_f32_16x16x32_bf16(S[n], L[m], acc[m][n], 0, 0, 0)
                                : __builtin_amdgcn_mfma_f32_16x16x32_bf16(L[m], S[n], acc[m][n], 0, 0, 0);
                __builtin_amdgcn_sched_barrier(0);
            }
        if (BIG) {
#pragma unroll
            for (int k = 0; k < 8; ++k) {
                const int r = (k >> 2) ^ WR, n = k < 4 ? k : 7 - k;
                acc[BIG ? 8 + r : 0][n] = __builtin_amdgcn_mfma_f32_16x16x32_bf16(L[BIG ? 8 + n : 0], S[2 * WR + r], acc[BIG ? 8 + r : 0][n], 0, 0, 0);
                __builtin_amdgcn_sched_barrier(0);
            }
        }
        __builtin_amdgcn_s_barrier();
        __builtin_amdgcn_sched_barrier(0);
    }
    if (WR == 0) __builtin_amdgcn_s_barrier();
#undef ENSURE
#undef ADVANCE
#undef STAGE
#undef WAITV
}

template <bool BIG>
__device__ __forceinline__ void gemm_tile(const Params& P, LAS char* lds, int u0, int c0, unsigned call, int lane, int wid) {
    constexpr int NU = BIG ? 5 : 4;
    char* ws = P.me;
    unsigned* tmo = (unsigned*)(ws + OFF_TMO);
    const int wr = wid >> 2, wc = wid & 3;
    f32x4 acc[2 * NU][4];
#pragma unroll
    for (int m = 0; m < 2 * NU; ++m)
#pragma unroll
        for (int n = 0; n < 4; ++n) acc[m][n] = (f32x4){0.f, 0.f, 0.f, 0.f};
    if (wr == 0) k_loop<0, BIG, false>(acc, ws, lds, u0, c0, nullptr, P.rank, call, lane, wid);
    else k_loop<1, BIG, false>(acc, ws, lds, u0, c0, nullptr, P.rank, call, lane, wid);
    const unsigned tmov = __hip_atomic_load(tmo, __ATOMIC_RELAXED, __HIP_MEMORY_SCOPE_AGENT);
    LAS unsigned* meet = (LAS unsigned*)(lds + LDS_REC + 128);
    if (wid == 0 && lane < 5) meet[lane * 16 + 8] = 0u;
    asm volatile("s_waitcnt lgkmcnt(0)" ::: "memory");
    asm volatile("" :: "v"(tmov));
    wait_vm0();
    __builtin_amdgcn_s_barrier();
    asm volatile("" ::: "memory");
    const unsigned par = call & 1u;
    if (__builtin_expect(rfl(tmov) == call, 0)) {
#pragma unroll
        for (int m = 0; m < 2 * NU; ++m)
#pragma unroll
            for (int n = 0; n < 4; ++n) acc[m][n] = (f32x4){__builtin_nanf(""), __builtin_nanf(""), __builtin_nanf(""), __builtin_nanf("")};
    }
    const unsigned dest = u0 >> 3, rb0 = (2 * (u0 + wc)) & 15;
    unsigned id[NU];
#pragma unroll
    for (int u = 0; u < NU; ++u) id[u] = u < 4 ? (c0 + 2 * wr + (u >> 1)) * 16u + rb0 + (u & 1) : (c0 + 4) * 16u + rb0 + wr;
    auto get = [&](int u, u32x4 (&d)[4]) {
#pragma unroll
        for (int c = 0; c < 4; ++c) {
            const int b = c & 1;
            f32x4 v[4];
#pragma unroll
            for (int n = 0; n < 4; ++n) v[n] = u < 4 ? acc[4 * (u >> 1) + n][2 * (u & 1) + (c >> 1)] : acc[2 * NU - 2 + (c >> 1)][n];
            d[c][0] = cvt_pk(v[0][2 * b], v[1][2 * b]);
            d[c][1] = cvt_pk(v[2][2 * b], v[3][2 * b]);
            d[c][2] = cvt_pk(v[0][2 * b + 1], v[1][2 * b + 1]);
            d[c][3] = cvt_pk(v[2][2 * b + 1], v[3][2 * b + 1]);
        }
    };
    LAS char* scr = lds + wid * SCR_WAVE;
    if (dest == (unsigned)P.rank) {
#pragma unroll
        for (int u = 0; u < NU; ++u) {
            u32x4 d[4];
            get(u, d);
            rows_of(scr, lane, d);
            store_out(P.out + (size_t)(id[u] & 15u) * (32 * 65536) + P.rank * 8192u + (id[u] >> 4) * 128u, lane, d);
        }
        return;
    }
    char* peer = P.ws[dest & 7u];
    auto draw = [&](unsigned k) {
        const bool late = c0 + k >= 4u * LATE_COL;
        if (lane == 0)
            meet[k * 16 + 9] = __hip_atomic_fetch_add((unsigned*)(ws + OFF_OPOS) + dest * 32 + (late ? 256 : 0), 4u, __ATOMIC_RELAXED, __HIP_MEMORY_SCOPE_AGENT) +
                               (late ? 32u * LATE_COL - (call - 1u) * (512u - 32u * LATE_COL) : (1u - call) * (32u * LATE_COL));
    };
    auto publish = [&](unsigned k, unsigned n) {
        LAS unsigned* m = meet + k * 16;
        unsigned before = 0;
        if (lane == 0) before = __hip_atomic_fetch_add(m + 8, 1u, __ATOMIC_RELAXED, __HIP_MEMORY_SCOPE_WORKGROUP);
        LDS_FENCE();
        if (rfl(before) == n - 1u && lane < 8) {
            u64* ring = (u64*)(peer + OFF_REC + par * REC_PAR) + (P.rank * 3 + 2) * 1024 + 2u * (m[9] & 508u) + lane;
            __hip_atomic_store(ring, (u64)call << 42 | (u64)((c0 + k) * 16u + ((2u * u0) & 15u) + lane) << 32 | m[lane], __ATOMIC_RELAXED,
                               __HIP_MEMORY_SCOPE_SYSTEM);
        }
    };
    char* pk = ws + OFF_PK + par * PK_PAR;
    const unsigned g0 = 9216u + dest * 1024u;
    {
        const unsigned ga[2] = {g0 + id[0], g0 + id[1]};
        unsigned r[2];
        pack_store<2>(get, pk, ga, lds, scr, lane, r, [] {});
        if (lane < 2) meet[2 * wr * 16 + 2 * wc + lane] = lane ? r[1] : r[0];
        if (wc == 0) draw(2 * wr);
        wait_vm0();
        publish(2 * wr, 4u);
    }
    {
        unsigned ga[NU - 2], r[NU - 2];
#pragma unroll
        for (int u = 2; u < NU; ++u) ga[u - 2] = g0 + id[u];
        pack_store<NU - 2>([&](int u, u32x4 (&d)[4]) { get(u + 2, d); }, pk, ga, lds, scr, lane, r, [] {});
        if (lane < 2) meet[(2 * wr + 1) * 16 + 2 * wc + lane] = lane ? r[1] : r[0];
        if (BIG && lane == 0) meet[64 + 2 * wc + wr] = r[NU - 3];
        if (wc == 0) draw(2 * wr + 1);
        if (BIG && wid == 5) draw(4);
        wait_vm0();
        publish(2 * wr + 1, 4u);
        if (BIG) publish(4, 8u);
    }
}

__device__ __forceinline__ void own_part(const Params& P, LAS char* lds, unsigned j, unsigned call, int lane, int wid) {
    char* ws = P.me;
    unsigned* tmo = (unsigned*)(ws + OFF_TMO);
    const int wr = wid >> 2, wc = wid & 3;
    const unsigned half = j & 1u, pn = (j >> 1) / 7u, q = (j >> 1) % 7u, p = (P.rank + 1u + q) & 7u;
    f32x4 acc[8][4];
#pragma unroll
    for (int m = 0; m < 8; ++m)
#pragma unroll
        for (int n = 0; n < 4; ++n) acc[m][n] = (f32x4){0.f, 0.f, 0.f, 0.f};
    const char* wsrc = P.w + p * 8192u + pn * 512u;
    if (wr == 0) k_loop<0, false, true>(acc, ws, lds, 8 * P.rank + 4 * half, 0, wsrc, P.rank, call, lane, wid);
    else k_loop<1, false, true>(acc, ws, lds, 8 * P.rank + 4 * half, 0, wsrc, P.rank, call, lane, wid);
    wait_vm0();
    __builtin_amdgcn_s_barrier();
    asm volatile("" ::: "memory");
    u32x4 d[4][4];
#pragma unroll
    for (int u = 0; u < 4; ++u) {
#pragma unroll
        for (int c = 0; c < 4; ++c) {
            const int mi = 2 * u + (c >> 1), n = 2 * (c & 1);
            d[u][c][0] = cvt_pk(acc[mi][n][0], acc[mi][n][1]);
            d[u][c][1] = cvt_pk(acc[mi][n][2], acc[mi][n][3]);
            d[u][c][2] = cvt_pk(acc[mi][n + 1][0], acc[mi][n + 1][1]);
            d[u][c][3] = cvt_pk(acc[mi][n + 1][2], acc[mi][n + 1][3]);
        }
        rows_of(lds + wid * SCR_WAVE, lane, d[u]);
    }
    const unsigned dst = (8u * half + 4u * wr) * (32u * 65536u) + p * 8192u + (4u * pn + wc) * 128u;
    if (pn < LATE_COL) {
        const unsigned* fl = (const unsigned*)(ws + OFF_OFL) + q * 512u + (4u * pn + wc) * 8u + 4u * half + 2u * wr + (lane & 1);
        const u64 t0 = __builtin_amdgcn_s_memrealtime();
        unsigned spins = 0;
        while (__builtin_amdgcn_ballot_w64(__hip_atomic_load(fl, __ATOMIC_RELAXED, __HIP_MEMORY_SCOPE_SYSTEM) < call) != 0ull) {
            if (timed_out(tmo, call, t0, spins)) break;
            __builtin_amdgcn_s_sleep(8);
        }
        asm volatile("" ::: "memory");
        const __amdgpu_buffer_rsrc_t orsrc = __builtin_amdgcn_make_buffer_rsrc(P.out, 0, 512 * 65536, 0x00020000);
        u32x4 y[16];
#pragma unroll
        for (int c = 0; c < 16; ++c) y[c] = __builtin_amdgcn_raw_buffer_load_b128(orsrc, (lane >> 3) * 65536 + (lane & 7) * 16, dst + c * (8u * 65536u), 17);
#pragma unroll
        for (int c = 0; c < 16; ++c) d[c >> 2][c & 3] = add_bf16(d[c >> 2][c & 3], y[c]);
    }
    if (rfl(__hip_atomic_load(tmo, __ATOMIC_RELAXED, __HIP_MEMORY_SCOPE_AGENT)) == call)
        d[0][0] = (u32x4){0x7fc07fc0u, 0x7fc07fc0u, 0x7fc07fc0u, 0x7fc07fc0u};
#pragma unroll
    for (int u = 0; u < 4; ++u) store_out(P.out + dst + u * (32 * 65536), lane, d[u]);
    wait_vm0();
    __builtin_amdgcn_s_barrier();
    asm volatile("" ::: "memory");
    if (threadIdx.x == 0)
        __hip_atomic_store((unsigned*)(ws + OFF_DFL) + q * 32u + pn * 2u + half, call, __ATOMIC_RELAXED, __HIP_MEMORY_SCOPE_SYSTEM);
}

__global__ void __launch_bounds__(512, 2) fused_gemm_rs(const Params P) {
    __shared__ __attribute__((aligned(16))) char smem[LDS_BYTES];
    const int bid = blockIdx.x;
    const int lane = threadIdx.x & 63, wid = rfl(threadIdx.x >> 6);
    char* ws = P.me; const char* px = P.x; const char* pw = P.w; const u64 job = P.job[max(bid - 7 * PWG, 0)];
    asm volatile("" :: "s"(ws), "s"(px), "s"(pw), "s"(job), "s"(P.rank));
    unsigned* cnt = (unsigned*)(ws + OFF_CNT) + bid * 8 + wid;
    const unsigned cntv = __hip_atomic_load(cnt, __ATOMIC_RELAXED, __HIP_MEMORY_SCOPE_AGENT);
    if (bid >= 7 * PWG) {
#pragma unroll 1
        for (int r = 0;; ++r) {
            unsigned e = r < 3 ? (unsigned)(job >> (11 + 11 * r)) & 0x7ffu : 0x7ffu;
            if (bid < 48 && r) {
                const unsigned i = 7u * XH0 + 6u * (r - 1) + bid - 7 * PWG, c = i % 7u;
                e = i < 112u ? (i / 7u) << 3 | (c + (c >= (unsigned)P.rank)) : 0x7ffu;
            }
            if ((e >> 7) == 15u) break;
            pack_piece(P, ws, px, pw, (LAS char*)smem, r, e, r ? 0u : (unsigned)(job >> 53) & 15u, cntv, lane, wid);
        }
        const unsigned late = (unsigned)(job >> 44) & 511u;
        if (late < P.dcount[0]) pack_bulk(P, ws, pw, (LAS char*)smem, late, rfl(cntv) + 1u, lane, wid);
        const unsigned ub = (unsigned)(job >> 57);
        if (ub) unbox(P, (LAS char*)smem, 2u * ub - 2u, rfl(cntv) + 1u, lane, wid);
        asm volatile("s_waitcnt lgkmcnt(0)" ::: "memory");
        __builtin_amdgcn_s_barrier();
    } else {
        build_lut((LAS char*)smem, true, threadIdx.x);
        if (threadIdx.x >= 384) ((LAS unsigned*)(smem + LDS_Q))[threadIdx.x - 384] = 0u;
        if (threadIdx.x < NIN / 2) ((LAS unsigned*)(smem + LDS_FLG))[threadIdx.x] = P.order[threadIdx.x];
        asm volatile("s_waitcnt lgkmcnt(0)" ::: "memory");
        __builtin_amdgcn_s_barrier();
        asm volatile("" ::: "memory");
    }
    const unsigned call = rfl(cntv) + 1u;
    if (lane == 0) __hip_atomic_store(cnt, call, __ATOMIC_RELAXED, __HIP_MEMORY_SCOPE_AGENT);
    if (bid < 7 * PWG) {
        if (wid == 0) fetch_link(P, (LAS char*)smem, bid % 7, call, lane);
        else if (wid <= NDEC) decode_link(P, (LAS char*)smem, bid % 7, (wid - 1) * PWG + bid / 7, call, lane, wid);
    } else {
        const unsigned u0 = ((unsigned)job & 15u) * 4u, col = ((unsigned)job >> 4) & 15u;
        const unsigned dj = (P.djob[(bid - 7 * PWG) >> 2] >> (((bid - 7 * PWG) & 3) * 8)) & 255u;
        if (dj < 224u) own_part(P, (LAS char*)smem, dj, call, lane, wid);
        if (col == 0u) gemm_tile<false>(P, (LAS char*)smem, u0, 0, call, lane, wid);
        else if (col < 13u) gemm_tile<true>(P, (LAS char*)smem, u0, 5u * col - 1u, call, lane, wid);
        if (!(job & 256u)) for (LAS unsigned* drawn = (LAS unsigned*)(smem + LDS_REC);; drawn = (LAS unsigned*)((size_t)drawn ^ 4)) {
            if (threadIdx.x == 0) *drawn = __hip_atomic_fetch_add((unsigned*)(ws + OFF_TKT + 1024), 1u, __ATOMIC_RELAXED, __HIP_MEMORY_SCOPE_AGENT);
            asm volatile("s_waitcnt lgkmcnt(0)" ::: "memory");
            __builtin_amdgcn_s_barrier();
            const unsigned k = lds_poll((LAS char*)drawn) - (call - 1u) * (224u - NPRE + P.drawers) + NPRE;
            if (k >= 224u) break;
            own_part(P, (LAS char*)smem, k < NLATE ? 223u - k : k - NLATE, call, lane, wid);
        }
    }
}

#define LAGS {0, 0, 2, 10, 18, 26, 34, 41, 49, 56, 64, 71, 78}
#define LAG_OWN 90
#define RND1 4
#define RND2 8
#define UNB 5
#define NODRAW 11
#define DCOUNT {110, 110, 82, 54, 40}

void make_plan(Params& P, int rank) {
    constexpr int NT = 256 - 7 * PWG;
    static_assert(NT == 214 && PUSHP + NPUSH <= 12 && PUSHP <= PUSH0, "208 tiles and 6 spare workgroups; pushed pieces: first round");
    const int lag[13] = LAGS;
    unsigned short full[960];
    unsigned char* worder = (unsigned char*)P.worder;
    int nf = 0, nw = 0;
    bool seen[128] = {};
    for (int pos = 0; pos < 512; ++pos) {
        for (int pr = 0; pr < 32; ++pr) {
            const int h8 = pos - lag[(2 * pr + 1) / 5];
            if (h8 >= 0 && h8 < 128 && h8 % 8 == 0) {
                full[nf++] = (unsigned short)(512 | h8 << 2 | pr);
                if (!seen[h8 | pr >> 2]) { seen[h8 | pr >> 2] = true; worder[nw++] = (unsigned char)(h8 | pr >> 2); }
            }
        }
        const int o8 = pos - LAG_OWN;
        for (int i = 0; i < 4; ++i) {
            if (pos < 128 && pos % 8 >= 1 && pos % 8 <= 6) full[nf++] = (unsigned short)(pos << 2 | i);
            if (o8 >= 0 && o8 < 128 && o8 % 8 == 0) full[nf++] = (unsigned short)(o8 << 2 | i);
        }
    }
    for (int k = 0; k < NPUSH; ++k) {
        const unsigned char v = worder[PUSH0 + k * PUSHS];
        for (int i = PUSH0 + k * PUSHS; i > PUSHP + k; --i) worder[i] = worder[i - 1];
        worder[PUSHP + k] = v;
    }
    unsigned short* order = (unsigned short*)P.order;
    for (int i = 0, n = 0; i < 960; ++i) {
        bool pushed = false;
        for (int k = 0; k < NPUSH; ++k) pushed |= full[i] >> 2 == (128 | worder[PUSHP + k]);
        if (!pushed) order[n++] = full[i];
    }
    int seq[256], nseq = 0;
    const int mix[8] = MIX;
    for (int qt = 0; qt < NBAND; ++qt)
        for (int i = 0; i < 8; ++i) {
            const int g = mix[i], band = (qt + g) % NBAND, n = g < 7 ? 14 : 13;
            for (int r = (n * band + NBAND / 2) / NBAND; r < (n * (band + 1) + NBAND / 2) / NBAND; ++r)
                for (int c = 0; c < (g ? 2 : 1); ++c) seq[nseq++] = g < 7 ? ((2 * rank + 2 + r) & 15) | (g ? 2 * g - 1 + c : 0) << 4 : (2 * rank + c) | r << 4;
        }
    int tile[NT], byp[NT], first[16], nby = 0;
    for (int x = 0, k = 0; x < 8; ++x)
        for (int b = 248 + x; b >= 7 * PWG; b -= 8) tile[b - 7 * PWG] = b < 48 ? 15 << 4 : seq[k++];
    for (int cls = 0; cls < 16; ++cls) {
        first[cls] = nby;
        for (int x = 0; x < 8; ++x)
            for (int b = 0; b < NT; ++b)
                if ((b + 7 * PWG) % 8 == x && (tile[b] >> 4 == 15 ? 14 : (tile[b] & 15) >> 1 == rank ? 13 : tile[b] >> 4) == cls) byp[nby++] = b;
    }
    P.drawers = P.pad = 0;
    for (int i = 0; i < NT; ++i) {
        const bool nodraw = i >= first[NODRAW] && i < first[14];
        P.drawers += !nodraw;
        P.job[byp[i]] = (u64)(tile[byp[i]] | nodraw << 8) | 0x1ffffffffull << 11 | (u64)(i < first[14] ? first[14] - 1 - i : 511) << 44;
    }
    int nr = 0;
    for (int r = 0; r < 3; ++r) {
        int list[256], n = 0;
        const int own = (rank + 1) << 7, from = first[r == 0 ? 0 : r == 1 ? RND1 : RND2];
        if (r < 2) {
            for (int h = 8 * r; h < 8 * r + 8; ++h) {
                for (int q = 0; q < 8; ++q) list[n++] = 9 << 7 | h << 3 | q;
                list[n++] = own | h << 3; list[n++] = own | h << 3 | 1;
            }
            for (int h = 2 * r; h < (r ? 16 : 2); ++h) { list[n++] = own | h << 3 | 2; list[n++] = own | h << 3 | 3; }
        } else {
            for (int h = 0; h < 16; ++h)
                for (int q = 4; q < 8; ++q) list[n++] = own | h << 3 | q;
        }
        for (int i = 0; i < (int)(8 * XH0) && r == 0; ++i)
            if (i % 8 != rank) list[n++] = i;
        const int room = (r ? first[14] : NT) - from;
        while (n < room - (r == 2 ? (nr & 1) != ((room - n) & 1) : 0)) {
            if (nr >= 7 * PUSHP && nr < 7 * (PUSHP + NPUSH)) P.job[byp[from + n]] |= (u64)(nr / 7 - PUSHP + 1) << 53;
            list[n++] = (((rank + 1 + nr % 7) & 7) + 1) << 7 | worder[nr / 7];
            ++nr;
        }
        for (int i = 0; i < n; ++i) {
            u64& j = P.job[byp[from + i]];
            j = (j & ~(0x7ffull << (11 + 11 * r))) | (u64)list[i] << (11 + 11 * r);
        }
    }
    P.dstart = nr;
    const int dcount[] = DCOUNT;
    for (int r = 0, left = (896 - nr) / 2; r < 12; ++r) {
        const int n = dcount[r < (int)(sizeof dcount / sizeof dcount[0]) ? r : sizeof dcount / sizeof dcount[0] - 1];
        P.dcount[r] = r == 11 ? 0 : n < left ? n : left;
        left -= P.dcount[r];
    }
    for (int i = 0; 2 * i < 7 * NPUSH; ++i) P.job[byp[first[UNB] + i]] |= (u64)(1 + i) << 57;
    unsigned char* djob = (unsigned char*)P.djob;
    for (int b = 0; b < 216; ++b) djob[b] = 255;
    for (unsigned j = 0; j < NPRE; ++j) djob[byp[first[13] + j]] = (unsigned char)(223 - j);
}

}

extern "C" void kernel_launch(int rank, int nranks, ncclComm_t comm, hipStream_t stream,
                              void* const* d_in, const int* in_sizes, int n_in,
                              void* d_out, int out_size, const int* in_dtypes, int out_dtype,
                              void* const* d_out_all, void* const* d_ws_all, size_t ws_bytes) {
    (void)comm; (void)in_dtypes; (void)out_dtype; (void)d_out_all;
    if (nranks != NR || (unsigned)rank >= (unsigned)NR || n_in < 2 || ws_bytes < WS_NEED || in_sizes[0] != 4096 * 512 ||
        in_sizes[1] != 512 * 32768 || out_size != 512 * 32768) {
        fprintf(stderr, "kernel_launch: unexpected problem shape\n");
        return;
    }
    Params P;
    P.x = (const char*)d_in[0];
    P.w = (const char*)d_in[1];
    P.out = (char*)d_out;
    for (int p = 0; p < NR; ++p) P.ws[p] = (char*)d_ws_all[p];
    P.me = P.ws[rank];
    P.rank = rank;
    make_plan(P, rank);
    hipLaunchKernelGGL(fused_gemm_rs, dim3(256), dim3(512), 0, stream, P);
    hipError_t e = hipGetLastError();
    if (e != hipSuccess) fprintf(stderr, "launch failed: %s\n", hipGetErrorString(e));
}
```
